```python
import numpy as np

kernel_name = "gpu_a2a_gemm_m8192_k8192_n8192_f32_gelu"


ABSMAX_RTOL = 0.001

COMPUTE_DTYPE = "float32"
EPILOGUE = "gelu"
IN_QUANT = None
OUT_QUANT = None
IN_DTYPES = {"x": "float32", "w_mat": "float32"}
WORLD_SIZE = 8

def setup_inputs():

    x = np.random.randn(8192, 8192).astype(np.float32)
    w_mat = (np.random.randn(8192, 8192) / np.sqrt(8192)).astype(np.float32)
    return {"x": x, "w_mat": w_mat}

def _epilogue(y):
    c = 0.7978845608028654
    return (0.5 * y * (1.0 + np.tanh(c * (y + 0.044715 * y ** 3)))).astype(np.float32)

def reference(x, w_mat):
    y = (x @ w_mat).astype(np.float32)
    return _epilogue(y)
```

```cpp
#include <hip/hip_runtime.h>
#include <rccl/rccl.h>
#include <cstdio>

#define HIP_CHECK(expr) do { hipError_t _e = (expr); if (_e != hipSuccess) { \
    fprintf(stderr, "HIP error %s at %s:%d\n", hipGetErrorString(_e), __FILE__, __LINE__); \
    return; } } while (0)
#define PXOR 5
#define KPRE \
  "s_nop 4\n" \
  "buffer_load_dword v95, off, s[16:19], s57 sc1\n" \
  "s_mov_b32 s60, 0x0\n" \
  "s_add_u32 m0, s65, 0x0\n" \
  "s_nop 0\n" \
  "buffer_load_dwordx4 v20, s[20:23], s60 offen nt lds\n" \
  "s_mov_b32 s60, 0x8000\n" \
  "s_add_u32 m0, s65, 0x400\n" \
  "s_nop 0\n" \
  "buffer_load_dwordx4 v20, s[20:23], s60 offen nt lds\n" \
  "s_mov_b32 s60, 0x400\n" \
  "s_add_u32 m0, s65, 0x800\n" \
  "s_nop 0\n" \
  "buffer_load_dwordx4 v20, s[20:23], s60 offen nt lds\n" \
  "s_mov_b32 s60, 0x8400\n" \
  "s_add_u32 m0, s65, 0xc00\n" \
  "s_nop 0\n" \
  "buffer_load_dwordx4 v20, s[20:23], s60 offen nt lds\n" \
  "v_add_u32 v17, 0x8000, v16\n" \
  "v_add_u32 v18, 0x10000, v16\n" \
  "v_add_u32 v19, 0x18000, v16\n" \
  "buffer_load_dwordx4 v[96:99], v16, s[8:11], s41 offen\n" \
  "buffer_load_dwordx4 v[100:103], v17, s[8:11], s41 offen\n" \
  "buffer_load_dwordx4 v[104:107], v18, s[8:11], s41 offen\n" \
  "buffer_load_dwordx4 v[108:111], v19, s[8:11], s41 offen\n" \
  ""
#define KBODY \
  "s_nop 4\n" \
  "v_readlane_b32 s39, v9, 0\n" \
  "v_readlane_b32 s64, v9, 1\n" \
  "s_mov_b32 s38, 2\n" \
  "s_mov_b32 s30, s18\n" \
  "s_mov_b32 s31, s19\n" \
  "v_mov_b32 v27, v10\n" \
  "s_add_u32 s24, s16, 0x80000\n" \
  "s_addc_u32 s25, s17, 0\n" \
  "v_accvgpr_write_b32 a0, 0\n" \
  "v_accvgpr_write_b32 a1, 0\n" \
  "v_accvgpr_write_b32 a2, 0\n" \
  "v_accvgpr_write_b32 a3, 0\n" \
  "v_accvgpr_write_b32 a4, 0\n" \
  "v_accvgpr_write_b32 a5, 0\n" \
  "v_accvgpr_write_b32 a6, 0\n" \
  "v_accvgpr_write_b32 a7, 0\n" \
  "v_accvgpr_write_b32 a8, 0\n" \
  "v_accvgpr_write_b32 a9, 0\n" \
  "v_accvgpr_write_b32 a10, 0\n" \
  "v_accvgpr_write_b32 a11, 0\n" \
  "v_accvgpr_write_b32 a12, 0\n" \
  "v_accvgpr_write_b32 a13, 0\n" \
  "v_accvgpr_write_b32 a14, 0\n" \
  "v_accvgpr_write_b32 a15, 0\n" \
  "v_accvgpr_write_b32 a16, 0\n" \
  "v_accvgpr_write_b32 a17, 0\n" \
  "v_accvgpr_write_b32 a18, 0\n" \
  "v_accvgpr_write_b32 a19, 0\n" \
  "v_accvgpr_write_b32 a20, 0\n" \
  "v_accvgpr_write_b32 a21, 0\n" \
  "v_accvgpr_write_b32 a22, 0\n" \
  "v_accvgpr_write_b32 a23, 0\n" \
  "v_accvgpr_write_b32 a24, 0\n" \
  "v_accvgpr_write_b32 a25, 0\n" \
  "v_accvgpr_write_b32 a26, 0\n" \
  "v_accvgpr_write_b32 a27, 0\n" \
  "v_accvgpr_write_b32 a28, 0\n" \
  "v_accvgpr_write_b32 a29, 0\n" \
  "v_accvgpr_write_b32 a30, 0\n" \
  "v_accvgpr_write_b32 a31, 0\n" \
  "v_accvgpr_write_b32 a32, 0\n" \
  "v_accvgpr_write_b32 a33, 0\n" \
  "v_accvgpr_write_b32 a34, 0\n" \
  "v_accvgpr_write_b32 a35, 0\n" \
  "v_accvgpr_write_b32 a36, 0\n" \
  "v_accvgpr_write_b32 a37, 0\n" \
  "v_accvgpr_write_b32 a38, 0\n" \
  "v_accvgpr_write_b32 a39, 0\n" \
  "v_accvgpr_write_b32 a40, 0\n" \
  "v_accvgpr_write_b32 a41, 0\n" \
  "v_accvgpr_write_b32 a42, 0\n" \
  "v_accvgpr_write_b32 a43, 0\n" \
  "v_accvgpr_write_b32 a44, 0\n" \
  "v_accvgpr_write_b32 a45, 0\n" \
  "v_accvgpr_write_b32 a46, 0\n" \
  "v_accvgpr_write_b32 a47, 0\n" \
  "v_accvgpr_write_b32 a48, 0\n" \
  "v_accvgpr_write_b32 a49, 0\n" \
  "v_accvgpr_write_b32 a50, 0\n" \
  "v_accvgpr_write_b32 a51, 0\n" \
  "v_accvgpr_write_b32 a52, 0\n" \
  "v_accvgpr_write_b32 a53, 0\n" \
  "v_accvgpr_write_b32 a54, 0\n" \
  "v_accvgpr_write_b32 a55, 0\n" \
  "v_accvgpr_write_b32 a56, 0\n" \
  "v_accvgpr_write_b32 a57, 0\n" \
  "v_accvgpr_write_b32 a58, 0\n" \
  "v_accvgpr_write_b32 a59, 0\n" \
  "v_accvgpr_write_b32 a60, 0\n" \
  "v_accvgpr_write_b32 a61, 0\n" \
  "v_accvgpr_write_b32 a62, 0\n" \
  "v_accvgpr_write_b32 a63, 0\n" \
  "s_waitcnt vmcnt(4)\n" \
  "v_xor_b32 v30, 64, v26\n" \
  "ds_read_b128 v[224:227], v26 offset:0\n" \
  "ds_read_b128 v[228:231], v30 offset:0\n" \
  "ds_read_b128 v[232:235], v26 offset:2048\n" \
  "ds_read_b128 v[236:239], v30 offset:2048\n" \
  "v_add_u32 v15, 1, v95\n" \
  "s_nop 0\n" \
  "v_readfirstlane_b32 s43, v15\n" \
  "s_nop 1\n" \
  "s_and_b32 s48, s43, 1\n" \
  "s_lshl_b32 s48, s48, 24\n" \
  "v_add_u32 v21, s48, v21\n" \
  "v_add_u32 v1, s48, v1\n" \
  "s_add_u32 s12, s12, s48\n" \
  "s_addc_u32 s13, s13, 0\n" \
  "s_waitcnt lgkmcnt(2)\n" \
  "v_cvt_pk_f16_f32 v224, v224, v225\n" \
  "v_cvt_pk_f16_f32 v225, v226, v227\n" \
  "v_cvt_pk_f16_f32 v226, v228, v229\n" \
  "v_cvt_pk_f16_f32 v227, v230, v231\n" \
  "global_store_dwordx4 v21, v[224:227], s[16:17] sc0 sc1\n" \
  "s_waitcnt lgkmcnt(0)\n" \
  "v_cvt_pk_f16_f32 v232, v232, v233\n" \
  "v_cvt_pk_f16_f32 v233, v234, v235\n" \
  "v_cvt_pk_f16_f32 v234, v236, v237\n" \
  "v_cvt_pk_f16_f32 v235, v238, v239\n" \
  "global_store_dwordx4 v21, v[232:235], s[24:25] sc0 sc1\n" \
  "s_bfe_u32 s40, s39, 0x80000\n" \
  "s_lshl_b32 s41, s40, 20\n" \
  "s_add_u32 s46, s41, 0x80000\n" \
  "buffer_load_dwordx4 v[112:115], v16, s[8:11], s46 offen\n" \
  "buffer_load_dwordx4 v[116:119], v17, s[8:11], s46 offen\n" \
  "buffer_load_dwordx4 v[120:123], v18, s[8:11], s46 offen\n" \
  "buffer_load_dwordx4 v[124:127], v19, s[8:11], s46 offen\n" \
  "s_bfe_u32 s40, s39, 0x80008\n" \
  "s_lshl_b32 s41, s40, 20\n" \
  "s_add_u32 s46, s41, 0x80000\n" \
  "buffer_load_dwordx4 v[128:131], v16, s[8:11], s41 offen\n" \
  "buffer_load_dwordx4 v[132:135], v17, s[8:11], s41 offen\n" \
  "buffer_load_dwordx4 v[136:139], v18, s[8:11], s41 offen\n" \
  "buffer_load_dwordx4 v[140:143], v19, s[8:11], s41 offen\n" \
  "s_lshr_b32 s60, s48, 24\n" \
  "s_mul_i32 s61, s60, 0x3c\n" \
  "s_mul_i32 s62, s60, 0x780\n" \
  "s_mul_i32 s60, s60, 0x0f0f0f0f\n" \
  "v_and_b32 v30, s60, v20\n" \
  "v_xor_b32 v9, v30, v9\n" \
  "v_ashrrev_i32 v30, 31, v4\n" \
  "v_bfi_b32 v30, v30, 0, s61\n" \
  "v_xor_b32 v4, v30, v4\n" \
  "v_ashrrev_i32 v30, 31, v3\n" \
  "v_bfi_b32 v30, v30, 0, s62\n" \
  "v_xor_b32 v3, v30, v3\n" \
  "s_nop 0\n" \
  "v_readlane_b32 s45, v3, 0\n" \
  "v_accvgpr_write_b32 a64, 0\n" \
  "v_accvgpr_write_b32 a65, 0\n" \
  "v_accvgpr_write_b32 a66, 0\n" \
  "v_accvgpr_write_b32 a67, 0\n" \
  "v_accvgpr_write_b32 a68, 0\n" \
  "v_accvgpr_write_b32 a69, 0\n" \
  "v_accvgpr_write_b32 a70, 0\n" \
  "v_accvgpr_write_b32 a71, 0\n" \
  "v_accvgpr_write_b32 a72, 0\n" \
  "v_accvgpr_write_b32 a73, 0\n" \
  "v_accvgpr_write_b32 a74, 0\n" \
  "v_accvgpr_write_b32 a75, 0\n" \
  "v_accvgpr_write_b32 a76, 0\n" \
  "v_accvgpr_write_b32 a77, 0\n" \
  "v_accvgpr_write_b32 a78, 0\n" \
  "v_accvgpr_write_b32 a79, 0\n" \
  "v_accvgpr_write_b32 a80, 0\n" \
  "v_accvgpr_write_b32 a81, 0\n" \
  "v_accvgpr_write_b32 a82, 0\n" \
  "v_accvgpr_write_b32 a83, 0\n" \
  "v_accvgpr_write_b32 a84, 0\n" \
  "v_accvgpr_write_b32 a85, 0\n" \
  "v_accvgpr_write_b32 a86, 0\n" \
  "v_accvgpr_write_b32 a87, 0\n" \
  "v_accvgpr_write_b32 a88, 0\n" \
  "v_accvgpr_write_b32 a89, 0\n" \
  "v_accvgpr_write_b32 a90, 0\n" \
  "v_accvgpr_write_b32 a91, 0\n" \
  "v_accvgpr_write_b32 a92, 0\n" \
  "v_accvgpr_write_b32 a93, 0\n" \
  "v_accvgpr_write_b32 a94, 0\n" \
  "v_accvgpr_write_b32 a95, 0\n" \
  "v_accvgpr_write_b32 a96, 0\n" \
  "v_accvgpr_write_b32 a97, 0\n" \
  "v_accvgpr_write_b32 a98, 0\n" \
  "v_accvgpr_write_b32 a99, 0\n" \
  "v_accvgpr_write_b32 a100, 0\n" \
  "v_accvgpr_write_b32 a101, 0\n" \
  "v_accvgpr_write_b32 a102, 0\n" \
  "v_accvgpr_write_b32 a103, 0\n" \
  "v_accvgpr_write_b32 a104, 0\n" \
  "v_accvgpr_write_b32 a105, 0\n" \
  "v_accvgpr_write_b32 a106, 0\n" \
  "v_accvgpr_write_b32 a107, 0\n" \
  "v_accvgpr_write_b32 a108, 0\n" \
  "v_accvgpr_write_b32 a109, 0\n" \
  "v_accvgpr_write_b32 a110, 0\n" \
  "v_accvgpr_write_b32 a111, 0\n" \
  "v_accvgpr_write_b32 a112, 0\n" \
  "v_accvgpr_write_b32 a113, 0\n" \
  "v_accvgpr_write_b32 a114, 0\n" \
  "v_accvgpr_write_b32 a115, 0\n" \
  "v_accvgpr_write_b32 a116, 0\n" \
  "v_accvgpr_write_b32 a117, 0\n" \
  "v_accvgpr_write_b32 a118, 0\n" \
  "v_accvgpr_write_b32 a119, 0\n" \
  "v_accvgpr_write_b32 a120, 0\n" \
  "v_accvgpr_write_b32 a121, 0\n" \
  "v_accvgpr_write_b32 a122, 0\n" \
  "v_accvgpr_write_b32 a123, 0\n" \
  "v_accvgpr_write_b32 a124, 0\n" \
  "v_accvgpr_write_b32 a125, 0\n" \
  "v_accvgpr_write_b32 a126, 0\n" \
  "v_accvgpr_write_b32 a127, 0\n" \
  "v_accvgpr_write_b32 a128, 0\n" \
  "v_accvgpr_write_b32 a129, 0\n" \
  "v_accvgpr_write_b32 a130, 0\n" \
  "v_accvgpr_write_b32 a131, 0\n" \
  "v_accvgpr_write_b32 a132, 0\n" \
  "v_accvgpr_write_b32 a133, 0\n" \
  "v_accvgpr_write_b32 a134, 0\n" \
  "v_accvgpr_write_b32 a135, 0\n" \
  "v_accvgpr_write_b32 a136, 0\n" \
  "v_accvgpr_write_b32 a137, 0\n" \
  "v_accvgpr_write_b32 a138, 0\n" \
  "v_accvgpr_write_b32 a139, 0\n" \
  "v_accvgpr_write_b32 a140, 0\n" \
  "v_accvgpr_write_b32 a141, 0\n" \
  "v_accvgpr_write_b32 a142, 0\n" \
  "v_accvgpr_write_b32 a143, 0\n" \
  "v_accvgpr_write_b32 a144, 0\n" \
  "v_accvgpr_write_b32 a145, 0\n" \
  "v_accvgpr_write_b32 a146, 0\n" \
  "v_accvgpr_write_b32 a147, 0\n" \
  "v_accvgpr_write_b32 a148, 0\n" \
  "v_accvgpr_write_b32 a149, 0\n" \
  "v_accvgpr_write_b32 a150, 0\n" \
  "v_accvgpr_write_b32 a151, 0\n" \
  "v_accvgpr_write_b32 a152, 0\n" \
  "v_accvgpr_write_b32 a153, 0\n" \
  "v_accvgpr_write_b32 a154, 0\n" \
  "v_accvgpr_write_b32 a155, 0\n" \
  "v_accvgpr_write_b32 a156, 0\n" \
  "v_accvgpr_write_b32 a157, 0\n" \
  "v_accvgpr_write_b32 a158, 0\n" \
  "v_accvgpr_write_b32 a159, 0\n" \
  "v_accvgpr_write_b32 a160, 0\n" \
  "v_accvgpr_write_b32 a161, 0\n" \
  "v_accvgpr_write_b32 a162, 0\n" \
  "v_accvgpr_write_b32 a163, 0\n" \
  "v_accvgpr_write_b32 a164, 0\n" \
  "v_accvgpr_write_b32 a165, 0\n" \
  "v_accvgpr_write_b32 a166, 0\n" \
  "v_accvgpr_write_b32 a167, 0\n" \
  "v_accvgpr_write_b32 a168, 0\n" \
  "v_accvgpr_write_b32 a169, 0\n" \
  "v_accvgpr_write_b32 a170, 0\n" \
  "v_accvgpr_write_b32 a171, 0\n" \
  "v_accvgpr_write_b32 a172, 0\n" \
  "v_accvgpr_write_b32 a173, 0\n" \
  "v_accvgpr_write_b32 a174, 0\n" \
  "v_accvgpr_write_b32 a175, 0\n" \
  "v_accvgpr_write_b32 a176, 0\n" \
  "v_accvgpr_write_b32 a177, 0\n" \
  "v_accvgpr_write_b32 a178, 0\n" \
  "v_accvgpr_write_b32 a179, 0\n" \
  "v_accvgpr_write_b32 a180, 0\n" \
  "v_accvgpr_write_b32 a181, 0\n" \
  "v_accvgpr_write_b32 a182, 0\n" \
  "v_accvgpr_write_b32 a183, 0\n" \
  "v_accvgpr_write_b32 a184, 0\n" \
  "v_accvgpr_write_b32 a185, 0\n" \
  "v_accvgpr_write_b32 a186, 0\n" \
  "v_accvgpr_write_b32 a187, 0\n" \
  "v_accvgpr_write_b32 a188, 0\n" \
  "v_accvgpr_write_b32 a189, 0\n" \
  "v_accvgpr_write_b32 a190, 0\n" \
  "v_accvgpr_write_b32 a191, 0\n" \
  "v_accvgpr_write_b32 a192, 0\n" \
  "v_accvgpr_write_b32 a193, 0\n" \
  "v_accvgpr_write_b32 a194, 0\n" \
  "v_accvgpr_write_b32 a195, 0\n" \
  "v_accvgpr_write_b32 a196, 0\n" \
  "v_accvgpr_write_b32 a197, 0\n" \
  "v_accvgpr_write_b32 a198, 0\n" \
  "v_accvgpr_write_b32 a199, 0\n" \
  "v_accvgpr_write_b32 a200, 0\n" \
  "v_accvgpr_write_b32 a201, 0\n" \
  "v_accvgpr_write_b32 a202, 0\n" \
  "v_accvgpr_write_b32 a203, 0\n" \
  "v_accvgpr_write_b32 a204, 0\n" \
  "v_accvgpr_write_b32 a205, 0\n" \
  "v_accvgpr_write_b32 a206, 0\n" \
  "v_accvgpr_write_b32 a207, 0\n" \
  "v_accvgpr_write_b32 a208, 0\n" \
  "v_accvgpr_write_b32 a209, 0\n" \
  "v_accvgpr_write_b32 a210, 0\n" \
  "v_accvgpr_write_b32 a211, 0\n" \
  "v_accvgpr_write_b32 a212, 0\n" \
  "v_accvgpr_write_b32 a213, 0\n" \
  "v_accvgpr_write_b32 a214, 0\n" \
  "v_accvgpr_write_b32 a215, 0\n" \
  "v_accvgpr_write_b32 a216, 0\n" \
  "v_accvgpr_write_b32 a217, 0\n" \
  "v_accvgpr_write_b32 a218, 0\n" \
  "v_accvgpr_write_b32 a219, 0\n" \
  "v_accvgpr_write_b32 a220, 0\n" \
  "v_accvgpr_write_b32 a221, 0\n" \
  "v_accvgpr_write_b32 a222, 0\n" \
  "v_accvgpr_write_b32 a223, 0\n" \
  "v_accvgpr_write_b32 a224, 0\n" \
  "v_accvgpr_write_b32 a225, 0\n" \
  "v_accvgpr_write_b32 a226, 0\n" \
  "v_accvgpr_write_b32 a227, 0\n" \
  "v_accvgpr_write_b32 a228, 0\n" \
  "v_accvgpr_write_b32 a229, 0\n" \
  "v_accvgpr_write_b32 a230, 0\n" \
  "v_accvgpr_write_b32 a231, 0\n" \
  "v_accvgpr_write_b32 a232, 0\n" \
  "v_accvgpr_write_b32 a233, 0\n" \
  "v_accvgpr_write_b32 a234, 0\n" \
  "v_accvgpr_write_b32 a235, 0\n" \
  "v_accvgpr_write_b32 a236, 0\n" \
  "v_accvgpr_write_b32 a237, 0\n" \
  "v_accvgpr_write_b32 a238, 0\n" \
  "v_accvgpr_write_b32 a239, 0\n" \
  "v_accvgpr_write_b32 a240, 0\n" \
  "v_accvgpr_write_b32 a241, 0\n" \
  "v_accvgpr_write_b32 a242, 0\n" \
  "v_accvgpr_write_b32 a243, 0\n" \
  "v_accvgpr_write_b32 a244, 0\n" \
  "v_accvgpr_write_b32 a245, 0\n" \
  "v_accvgpr_write_b32 a246, 0\n" \
  "v_accvgpr_write_b32 a247, 0\n" \
  "v_accvgpr_write_b32 a248, 0\n" \
  "v_accvgpr_write_b32 a249, 0\n" \
  "v_accvgpr_write_b32 a250, 0\n" \
  "v_accvgpr_write_b32 a251, 0\n" \
  "v_accvgpr_write_b32 a252, 0\n" \
  "v_accvgpr_write_b32 a253, 0\n" \
  "v_accvgpr_write_b32 a254, 0\n" \
  "v_accvgpr_write_b32 a255, 0\n" \
  "s_waitcnt vmcnt(8)\n" \
  "s_barrier\n" \
  "buffer_store_dword v15, v22, s[16:19], s58 offen sc0 sc1\n" \
  "buffer_store_dword v15, v22, s[16:19], s58 offen offset:512 sc0 sc1\n" \
  "buffer_store_dword v15, v22, s[16:19], s57 offen sc1\n" \
  "buffer_load_dwordx4 v[144:147], v16, s[8:11], s46 offen\n" \
  "buffer_load_dwordx4 v[148:151], v17, s[8:11], s46 offen\n" \
  "buffer_load_dwordx4 v[152:155], v18, s[8:11], s46 offen\n" \
  "buffer_load_dwordx4 v[156:159], v19, s[8:11], s46 offen\n" \
  "s_bfe_u32 s40, s39, 0x80010\n" \
  "s_lshl_b32 s41, s40, 20\n" \
  "s_add_u32 s46, s41, 0x80000\n" \
  "buffer_load_dwordx4 v[160:163], v16, s[8:11], s41 offen\n" \
  "buffer_load_dwordx4 v[164:167], v17, s[8:11], s41 offen\n" \
  "buffer_load_dwordx4 v[168:171], v18, s[8:11], s41 offen\n" \
  "buffer_load_dwordx4 v[172:175], v19, s[8:11], s41 offen\n" \
  "s_call_b64 s[54:55], Lpoll_%=\n" \
  "s_bfe_u32 s40, s39, 0x80000\n" \
  "s_lshl_b32 s42, s40, 16\n" \
  "s_add_u32 m0, s36, 0x0\n" \
  "s_nop 0\n" \
  "buffer_load_dwordx4 v24, s[12:15], s42 offen offset:0 sc0 sc1 lds\n" \
  "buffer_load_dwordx4 v24, s[12:15], s42 offen offset:1024 sc0 sc1 lds\n" \
  "buffer_load_dwordx4 v24, s[12:15], s42 offen offset:2048 sc0 sc1 lds\n" \
  "buffer_load_dwordx4 v24, s[12:15], s42 offen offset:3072 sc0 sc1 lds\n" \
  "s_bfe_u32 s40, s39, 0x80008\n" \
  "s_lshl_b32 s42, s40, 16\n" \
  "s_add_u32 m0, s36, 0x4000\n" \
  "s_nop 0\n" \
  "buffer_load_dwordx4 v24, s[12:15], s42 offen offset:0 sc0 sc1 lds\n" \
  "buffer_load_dwordx4 v24, s[12:15], s42 offen offset:1024 sc0 sc1 lds\n" \
  "buffer_load_dwordx4 v24, s[12:15], s42 offen offset:2048 sc0 sc1 lds\n" \
  "buffer_load_dwordx4 v24, s[12:15], s42 offen offset:3072 sc0 sc1 lds\n" \
  "v_cvt_pk_f16_f32 v64, v96, v100\n" \
  "v_cvt_pk_f16_f32 v68, v97, v101\n" \
  "v_cvt_pk_f16_f32 v72, v98, v102\n" \
  "v_cvt_pk_f16_f32 v76, v99, v103\n" \
  "v_cvt_pk_f16_f32 v65, v104, v108\n" \
  "v_cvt_pk_f16_f32 v69, v105, v109\n" \
  "v_cvt_pk_f16_f32 v73, v106, v110\n" \
  "v_cvt_pk_f16_f32 v77, v107, v111\n" \
  "v_cvt_pk_f16_f32 v66, v112, v116\n" \
  "v_cvt_pk_f16_f32 v70, v113, v117\n" \
  "v_cvt_pk_f16_f32 v74, v114, v118\n" \
  "v_cvt_pk_f16_f32 v78, v115, v119\n" \
  "v_cvt_pk_f16_f32 v67, v120, v124\n" \
  "v_cvt_pk_f16_f32 v71, v121, v125\n" \
  "v_cvt_pk_f16_f32 v75, v122, v126\n" \
  "v_cvt_pk_f16_f32 v79, v123, v127\n" \
  "s_mov_b32 s44, 1\n" \
  "s_mov_b32 s63, -1\n" \
  "s_bfe_u32 s40, s39, 0x80010\n" \
  "s_lshl_b32 s41, s40, 20\n" \
  "s_add_u32 s46, s41, 0x80000\n" \
  "buffer_load_dwordx4 v[176:179], v16, s[8:11], s46 offen\n" \
  "buffer_load_dwordx4 v[180:183], v17, s[8:11], s46 offen\n" \
  "buffer_load_dwordx4 v[184:187], v18, s[8:11], s46 offen\n" \
  "buffer_load_dwordx4 v[188:191], v19, s[8:11], s46 offen\n" \
  "s_bfe_u32 s40, s39, 0x80010\n" \
  "s_lshl_b32 s42, s40, 16\n" \
  "s_add_u32 m0, s36, 0x8000\n" \
  "s_nop 0\n" \
  "buffer_load_dwordx4 v24, s[12:15], s42 offen offset:0 sc0 sc1 lds\n" \
  "buffer_load_dwordx4 v24, s[12:15], s42 offen offset:1024 sc0 sc1 lds\n" \
  "buffer_load_dwordx4 v24, s[12:15], s42 offen offset:2048 sc0 sc1 lds\n" \
  "buffer_load_dwordx4 v24, s[12:15], s42 offen offset:3072 sc0 sc1 lds\n" \
  "s_waitcnt vmcnt(12)\n" \
  "s_barrier\n" \
  "ds_read_b128 v[32:35], v25 offset:0\n" \
  "ds_read_b128 v[36:39], v25 offset:1024\n" \
  "ds_read_b128 v[40:43], v25 offset:2048\n" \
  "ds_read_b128 v[44:47], v25 offset:3072\n" \
  "ds_read_b128 v[48:51], v25 offset:4096\n" \
  "ds_read_b128 v[52:55], v25 offset:5120\n" \
  "ds_read_b128 v[56:59], v25 offset:6144\n" \
  "ds_read_b128 v[60:63], v25 offset:7168\n" \
  "s_cmp_lt_i32 s45, 0\n" \
  "s_cbranch_scc1 3f\n" \
  "s_add_u32 m0, s65, 0x0\n" \
  "s_nop 0\n" \
  "buffer_load_dwordx4 v0, s[20:23], s45 offen nt lds\n" \
  "s_add_u32 m0, s65, 0x400\n" \
  "s_add_u32 s45, s45, 0x8000\n" \
  "buffer_load_dwordx4 v0, s[20:23], s45 offen nt lds\n" \
  "s_add_u32 m0, s65, 0x800\n" \
  "s_add_u32 s45, s45, 0x8000\n" \
  "buffer_load_dwordx4 v0, s[20:23], s45 offen nt lds\n" \
  "s_add_u32 m0, s65, 0xc00\n" \
  "s_add_u32 s45, s45, 0x8000\n" \
  "buffer_load_dwordx4 v0, s[20:23], s45 offen nt lds\n" \
  "s_add_u32 m0, s65, 0x1000\n" \
  "s_add_u32 s45, s45, 0x8000\n" \
  "buffer_load_dwordx4 v0, s[20:23], s45 offen nt lds\n" \
  "s_add_u32 m0, s65, 0x1400\n" \
  "s_add_u32 s45, s45, 0x8000\n" \
  "buffer_load_dwordx4 v0, s[20:23], s45 offen nt lds\n" \
  "s_add_u32 m0, s65, 0x1800\n" \
  "s_add_u32 s45, s45, 0x8000\n" \
  "buffer_load_dwordx4 v0, s[20:23], s45 offen nt lds\n" \
  "s_add_u32 m0, s65, 0x1c00\n" \
  "s_add_u32 s45, s45, 0x8000\n" \
  "buffer_load_dwordx4 v0, s[20:23], s45 offen nt lds\n" \
  "3:\n" \
  "s_waitcnt lgkmcnt(6)\n" \
  "v_mfma_f32_16x16x32_f16 a[0:3], v[32:35], v[64:67], a[0:3]\n" \
  "s_bfe_u32 s40, s39, 0x80018\n" \
  "s_lshl_b32 s41, s40, 20\n" \
  "buffer_load_dwordx4 v[192:195], v16, s[8:11], s41 offen\n" \
  "v_mfma_f32_16x16x32_f16 a[4:7], v[32:35], v[68:71], a[4:7]\n" \
  "v_cvt_pk_f16_f32 v80, v128, v132\n" \
  "v_mfma_f32_16x16x32_f16 a[8:11], v[32:35], v[72:75], a[8:11]\n" \
  "v_cvt_pk_f16_f32 v84, v129, v133\n" \
  "v_mfma_f32_16x16x32_f16 a[12:15], v[32:35], v[76:79], a[12:15]\n" \
  "ds_read_b128 v[32:35], v25 offset:8192\n" \
  "v_mfma_f32_16x16x32_f16 a[28:31], v[36:39], v[76:79], a[28:31]\n" \
  "buffer_load_dwordx4 v[196:199], v17, s[8:11], s41 offen\n" \
  "v_mfma_f32_16x16x32_f16 a[24:27], v[36:39], v[72:75], a[24:27]\n" \
  "v_cvt_pk_f16_f32 v88, v130, v134\n" \
  "v_mfma_f32_16x16x32_f16 a[20:23], v[36:39], v[68:71], a[20:23]\n" \
  "s_waitcnt lgkmcnt(5)\n" \
  "buffer_load_dwordx4 v[200:203], v18, s[8:11], s41 offen\n" \
  "v_mfma_f32_16x16x32_f16 a[16:19], v[36:39], v[64:67], a[16:19]\n" \
  "ds_read_b128 v[36:39], v25 offset:9216\n" \
  "v_mfma_f32_16x16x32_f16 a[32:35], v[40:43], v[64:67], a[32:35]\n" \
  "v_cvt_pk_f16_f32 v92, v131, v135\n" \
  "v_mfma_f32_16x16x32_f16 a[36:39], v[40:43], v[68:71], a[36:39]\n" \
  "v_cvt_pk_f16_f32 v81, v136, v140\n" \
  "v_mfma_f32_16x16x32_f16 a[40:43], v[40:43], v[72:75], a[40:43]\n" \
  "buffer_load_dwordx4 v[204:207], v19, s[8:11], s41 offen\n" \
  "s_add_u32 s46, s41, 0x80000\n" \
  "v_mfma_f32_16x16x32_f16 a[44:47], v[40:43], v[76:79], a[44:47]\n" \
  "ds_read_b128 v[40:43], v25 offset:10240\n" \
  "v_mfma_f32_16x16x32_f16 a[60:63], v[44:47], v[76:79], a[60:63]\n" \
  "v_cvt_pk_f16_f32 v85, v137, v141\n" \
  "v_mfma_f32_16x16x32_f16 a[56:59], v[44:47], v[72:75], a[56:59]\n" \
  "v_cvt_pk_f16_f32 v89, v138, v142\n" \
  "v_mfma_f32_16x16x32_f16 a[52:55], v[44:47], v[68:71], a[52:55]\n" \
  "s_waitcnt lgkmcnt(5)\n" \
  "buffer_load_dwordx4 v[208:211], v16, s[8:11], s46 offen\n" \
  "v_mfma_f32_16x16x32_f16 a[48:51], v[44:47], v[64:67], a[48:51]\n" \
  "ds_read_b128 v[44:47], v25 offset:11264\n" \
  "v_mfma_f32_16x16x32_f16 a[64:67], v[48:51], v[64:67], a[64:67]\n" \
  "v_cvt_pk_f16_f32 v93, v139, v143\n" \
  "v_mfma_f32_16x16x32_f16 a[68:71], v[48:51], v[68:71], a[68:71]\n" \
  "v_cvt_pk_f16_f32 v82, v144, v148\n" \
  "v_mfma_f32_16x16x32_f16 a[72:75], v[48:51], v[72:75], a[72:75]\n" \
  "buffer_load_dwordx4 v[212:215], v17, s[8:11], s46 offen\n" \
  "s_lshl_b32 s42, s40, 16\n" \
  "v_mfma_f32_16x16x32_f16 a[76:79], v[48:51], v[76:79], a[76:79]\n" \
  "ds_read_b128 v[48:51], v25 offset:12288\n" \
  "v_mfma_f32_16x16x32_f16 a[92:95], v[52:55], v[76:79], a[92:95]\n" \
  "v_cvt_pk_f16_f32 v86, v145, v149\n" \
  "v_mfma_f32_16x16x32_f16 a[88:91], v[52:55], v[72:75], a[88:91]\n" \
  "v_cvt_pk_f16_f32 v90, v146, v150\n" \
  "v_mfma_f32_16x16x32_f16 a[84:87], v[52:55], v[68:71], a[84:87]\n" \
  "s_waitcnt lgkmcnt(5)\n" \
  "buffer_load_dwordx4 v[216:219], v18, s[8:11], s46 offen\n" \
  "v_mfma_f32_16x16x32_f16 a[80:83], v[52:55], v[64:67], a[80:83]\n" \
  "ds_read_b128 v[52:55], v25 offset:13312\n" \
  "v_mfma_f32_16x16x32_f16 a[96:99], v[56:59], v[64:67], a[96:99]\n" \
  "v_cvt_pk_f16_f32 v94, v147, v151\n" \
  "v_mfma_f32_16x16x32_f16 a[100:103], v[56:59], v[68:71], a[100:103]\n" \
  "buffer_load_dwordx4 v[220:223], v19, s[8:11], s46 offen\n" \
  "s_add_u32 m0, s36, 0xc000\n" \
  "v_mfma_f32_16x16x32_f16 a[104:107], v[56:59], v[72:75], a[104:107]\n" \
  "v_cvt_pk_f16_f32 v83, v152, v156\n" \
  "v_mfma_f32_16x16x32_f16 a[108:111], v[56:59], v[76:79], a[108:111]\n" \
  "ds_read_b128 v[56:59], v25 offset:14336\n" \
  "v_mfma_f32_16x16x32_f16 a[124:127], v[60:63], v[76:79], a[124:127]\n" \
  "buffer_load_dwordx4 v24, s[12:15], s42 offen offset:0 sc0 sc1 lds\n" \
  "v_mfma_f32_16x16x32_f16 a[120:123], v[60:63], v[72:75], a[120:123]\n" \
  "v_cvt_pk_f16_f32 v87, v153, v157\n" \
  "s_waitcnt vmcnt(17)\n" \
  "v_mfma_f32_16x16x32_f16 a[116:119], v[60:63], v[68:71], a[116:119]\n" \
  "s_waitcnt lgkmcnt(5)\n" \
  "s_barrier\n" \
  "v_mfma_f32_16x16x32_f16 a[112:115], v[60:63], v[64:67], a[112:115]\n" \
  "ds_read_b128 v[60:63], v25 offset:15360\n" \
  "v_mfma_f32_16x16x32_f16 a[128:131], v[32:35], v[64:67], a[128:131]\n" \
  "buffer_load_dwordx4 v24, s[12:15], s42 offen offset:1024 sc0 sc1 lds\n" \
  "v_mfma_f32_16x16x32_f16 a[132:135], v[32:35], v[68:71], a[132:135]\n" \
  "v_cvt_pk_f16_f32 v91, v154, v158\n" \
  "v_mfma_f32_16x16x32_f16 a[136:139], v[32:35], v[72:75], a[136:139]\n" \
  "v_cvt_pk_f16_f32 v95, v155, v159\n" \
  "v_mfma_f32_16x16x32_f16 a[140:143], v[32:35], v[76:79], a[140:143]\n" \
  "ds_read_b128 v[32:35], v25 offset:16384\n" \
  "v_mfma_f32_16x16x32_f16 a[156:159], v[36:39], v[76:79], a[156:159]\n" \
  "buffer_load_dwordx4 v24, s[12:15], s42 offen offset:2048 sc0 sc1 lds\n" \
  "v_mfma_f32_16x16x32_f16 a[152:155], v[36:39], v[72:75], a[152:155]\n" \
  "v_mfma_f32_16x16x32_f16 a[148:151], v[36:39], v[68:71], a[148:151]\n" \
  "s_waitcnt lgkmcnt(5)\n" \
  "buffer_load_dwordx4 v24, s[12:15], s42 offen offset:3072 sc0 sc1 lds\n" \
  "v_mfma_f32_16x16x32_f16 a[144:147], v[36:39], v[64:67], a[144:147]\n" \
  "ds_read_b128 v[36:39], v25 offset:17408\n" \
  "v_mfma_f32_16x16x32_f16 a[160:163], v[40:43], v[64:67], a[160:163]\n" \
  "s_sub_u32 s56, s56, 4\n" \
  "v_mfma_f32_16x16x32_f16 a[164:167], v[40:43], v[68:71], a[164:167]\n" \
  "v_mfma_f32_16x16x32_f16 a[168:171], v[40:43], v[72:75], a[168:171]\n" \
  "s_mov_b32 s39, s64\n" \
  "s_bfe_u32 s40, s39, 0x80000\n" \
  "s_lshl_b32 s41, s40, 20\n" \
  "buffer_load_dwordx4 v[96:99], v16, s[8:11], s41 offen\n" \
  "v_mfma_f32_16x16x32_f16 a[172:175], v[40:43], v[76:79], a[172:175]\n" \
  "ds_read_b128 v[40:43], v25 offset:18432\n" \
  "v_mfma_f32_16x16x32_f16 a[188:191], v[44:47], v[76:79], a[188:191]\n" \
  "v_mfma_f32_16x16x32_f16 a[184:187], v[44:47], v[72:75], a[184:187]\n" \
  "v_mfma_f32_16x16x32_f16 a[180:183], v[44:47], v[68:71], a[180:183]\n" \
  "s_waitcnt lgkmcnt(5)\n" \
  "buffer_load_dwordx4 v[100:103], v17, s[8:11], s41 offen\n" \
  "v_mfma_f32_16x16x32_f16 a[176:179], v[44:47], v[64:67], a[176:179]\n" \
  "ds_read_b128 v[44:47], v25 offset:19456\n" \
  "v_mfma_f32_16x16x32_f16 a[192:195], v[48:51], v[64:67], a[192:195]\n" \
  "v_mfma_f32_16x16x32_f16 a[196:199], v[48:51], v[68:71], a[196:199]\n" \
  "v_mfma_f32_16x16x32_f16 a[200:203], v[48:51], v[72:75], a[200:203]\n" \
  "buffer_load_dwordx4 v[104:107], v18, s[8:11], s41 offen\n" \
  "v_mfma_f32_16x16x32_f16 a[204:207], v[48:51], v[76:79], a[204:207]\n" \
  "ds_read_b128 v[48:51], v25 offset:20480\n" \
  "v_mfma_f32_16x16x32_f16 a[220:223], v[52:55], v[76:79], a[220:223]\n" \
  "v_mfma_f32_16x16x32_f16 a[216:219], v[52:55], v[72:75], a[216:219]\n" \
  "v_mfma_f32_16x16x32_f16 a[212:215], v[52:55], v[68:71], a[212:215]\n" \
  "s_waitcnt lgkmcnt(5)\n" \
  "buffer_load_dwordx4 v[108:111], v19, s[8:11], s41 offen\n" \
  "s_add_u32 s46, s41, 0x80000\n" \
  "v_mfma_f32_16x16x32_f16 a[208:211], v[52:55], v[64:67], a[208:211]\n" \
  "ds_read_b128 v[52:55], v25 offset:21504\n" \
  "v_mfma_f32_16x16x32_f16 a[224:227], v[56:59], v[64:67], a[224:227]\n" \
  "v_mfma_f32_16x16x32_f16 a[228:231], v[56:59], v[68:71], a[228:231]\n" \
  "buffer_load_dwordx4 v[112:115], v16, s[8:11], s46 offen\n" \
  "v_mfma_f32_16x16x32_f16 a[232:235], v[56:59], v[72:75], a[232:235]\n" \
  "v_mfma_f32_16x16x32_f16 a[236:239], v[56:59], v[76:79], a[236:239]\n" \
  "ds_read_b128 v[56:59], v25 offset:22528\n" \
  "v_mfma_f32_16x16x32_f16 a[252:255], v[60:63], v[76:79], a[252:255]\n" \
  "buffer_load_dwordx4 v[116:119], v17, s[8:11], s46 offen\n" \
  "s_lshl_b32 s42, s40, 16\n" \
  "v_mfma_f32_16x16x32_f16 a[248:251], v[60:63], v[72:75], a[248:251]\n" \
  "s_cmp_le_i32 s56, 0\n" \
  "v_mfma_f32_16x16x32_f16 a[244:247], v[60:63], v[68:71], a[244:247]\n" \
  "s_waitcnt lgkmcnt(5)\n" \
  "s_cbranch_scc1 Lpact0_%=\n" \
  "Lpactb0_%=:\n" \
  "v_mfma_f32_16x16x32_f16 a[240:243], v[60:63], v[64:67], a[240:243]\n" \
  "ds_read_b128 v[60:63], v25 offset:23552\n" \
  "v_mfma_f32_16x16x32_f16 a[0:3], v[32:35], v[80:83], a[0:3]\n" \
  "buffer_load_dwordx4 v[120:123], v18, s[8:11], s46 offen\n" \
  "s_xor_b32 s36, s36, 0x10000\n" \
  "v_mfma_f32_16x16x32_f16 a[4:7], v[32:35], v[84:87], a[4:7]\n" \
  "v_cvt_pk_f16_f32 v64, v160, v164\n" \
  "v_mfma_f32_16x16x32_f16 a[8:11], v[32:35], v[88:91], a[8:11]\n" \
  "v_cvt_pk_f16_f32 v68, v161, v165\n" \
  "v_mfma_f32_16x16x32_f16 a[12:15], v[32:35], v[92:95], a[12:15]\n" \
  "ds_read_b128 v[32:35], v25 offset:24576\n" \
  "v_mfma_f32_16x16x32_f16 a[28:31], v[36:39], v[92:95], a[28:31]\n" \
  "buffer_load_dwordx4 v[124:127], v19, s[8:11], s46 offen\n" \
  "s_add_u32 m0, s36, 0x0\n" \
  "v_mfma_f32_16x16x32_f16 a[24:27], v[36:39], v[88:91], a[24:27]\n" \
  "v_cvt_pk_f16_f32 v72, v162, v166\n" \
  "v_mfma_f32_16x16x32_f16 a[20:23], v[36:39], v[84:87], a[20:23]\n" \
  "s_waitcnt lgkmcnt(5)\n" \
  "buffer_load_dwordx4 v24, s[12:15], s42 offen offset:0 sc0 sc1 lds\n" \
  "v_mfma_f32_16x16x32_f16 a[16:19], v[36:39], v[80:83], a[16:19]\n" \
  "ds_read_b128 v[36:39], v25 offset:25600\n" \
  "v_mfma_f32_16x16x32_f16 a[32:35], v[40:43], v[80:83], a[32:35]\n" \
  "v_cvt_pk_f16_f32 v76, v163, v167\n" \
  "v_mfma_f32_16x16x32_f16 a[36:39], v[40:43], v[84:87], a[36:39]\n" \
  "v_cvt_pk_f16_f32 v65, v168, v172\n" \
  "v_mfma_f32_16x16x32_f16 a[40:43], v[40:43], v[88:91], a[40:43]\n" \
  "buffer_load_dwordx4 v24, s[12:15], s42 offen offset:1024 sc0 sc1 lds\n" \
  "v_mfma_f32_16x16x32_f16 a[44:47], v[40:43], v[92:95], a[44:47]\n" \
  "ds_read_b128 v[40:43], v25 offset:26624\n" \
  "v_mfma_f32_16x16x32_f16 a[60:63], v[44:47], v[92:95], a[60:63]\n" \
  "v_cvt_pk_f16_f32 v69, v169, v173\n" \
  "v_mfma_f32_16x16x32_f16 a[56:59], v[44:47], v[88:91], a[56:59]\n" \
  "v_cvt_pk_f16_f32 v73, v170, v174\n" \
  "v_mfma_f32_16x16x32_f16 a[52:55], v[44:47], v[84:87], a[52:55]\n" \
  "s_waitcnt lgkmcnt(5)\n" \
  "buffer_load_dwordx4 v24, s[12:15], s42 offen offset:2048 sc0 sc1 lds\n" \
  "v_mfma_f32_16x16x32_f16 a[48:51], v[44:47], v[80:83], a[48:51]\n" \
  "ds_read_b128 v[44:47], v25 offset:27648\n" \
  "v_mfma_f32_16x16x32_f16 a[64:67], v[48:51], v[80:83], a[64:67]\n" \
  "v_cvt_pk_f16_f32 v77, v171, v175\n" \
  "v_mfma_f32_16x16x32_f16 a[68:71], v[48:51], v[84:87], a[68:71]\n" \
  "s_waitcnt vmcnt(29)\n" \
  "v_cvt_pk_f16_f32 v66, v176, v180\n" \
  "v_mfma_f32_16x16x32_f16 a[72:75], v[48:51], v[88:91], a[72:75]\n" \
  "buffer_load_dwordx4 v24, s[12:15], s42 offen offset:3072 sc0 sc1 lds\n" \
  "v_mfma_f32_16x16x32_f16 a[76:79], v[48:51], v[92:95], a[76:79]\n" \
  "ds_read_b128 v[48:51], v25 offset:28672\n" \
  "v_mfma_f32_16x16x32_f16 a[92:95], v[52:55], v[92:95], a[92:95]\n" \
  "v_cvt_pk_f16_f32 v70, v177, v181\n" \
  "v_mfma_f32_16x16x32_f16 a[88:91], v[52:55], v[88:91], a[88:91]\n" \
  "v_cvt_pk_f16_f32 v74, v178, v182\n" \
  "v_mfma_f32_16x16x32_f16 a[84:87], v[52:55], v[84:87], a[84:87]\n" \
  "s_waitcnt lgkmcnt(5)\n" \
  "s_bfe_u32 s40, s39, 0x80008\n" \
  "s_lshl_b32 s41, s40, 20\n" \
  "buffer_load_dwordx4 v[128:131], v16, s[8:11], s41 offen\n" \
  "v_mfma_f32_16x16x32_f16 a[80:83], v[52:55], v[80:83], a[80:83]\n" \
  "ds_read_b128 v[52:55], v25 offset:29696\n" \
  "v_mfma_f32_16x16x32_f16 a[96:99], v[56:59], v[80:83], a[96:99]\n" \
  "v_cvt_pk_f16_f32 v78, v179, v183\n" \
  "v_mfma_f32_16x16x32_f16 a[100:103], v[56:59], v[84:87], a[100:103]\n" \
  "buffer_load_dwordx4 v[132:135], v17, s[8:11], s41 offen\n" \
  "v_mfma_f32_16x16x32_f16 a[104:107], v[56:59], v[88:91], a[104:107]\n" \
  "s_waitcnt vmcnt(30)\n" \
  "v_cvt_pk_f16_f32 v67, v184, v188\n" \
  "v_mfma_f32_16x16x32_f16 a[108:111], v[56:59], v[92:95], a[108:111]\n" \
  "ds_read_b128 v[56:59], v25 offset:30720\n" \
  "v_mfma_f32_16x16x32_f16 a[124:127], v[60:63], v[92:95], a[124:127]\n" \
  "buffer_load_dwordx4 v[136:139], v18, s[8:11], s41 offen\n" \
  "v_mfma_f32_16x16x32_f16 a[120:123], v[60:63], v[88:91], a[120:123]\n" \
  "v_cvt_pk_f16_f32 v71, v185, v189\n" \
  "s_waitcnt vmcnt(27)\n" \
  "v_mfma_f32_16x16x32_f16 a[116:119], v[60:63], v[84:87], a[116:119]\n" \
  "s_waitcnt lgkmcnt(5)\n" \
  "s_barrier\n" \
  "v_mfma_f32_16x16x32_f16 a[112:115], v[60:63], v[80:83], a[112:115]\n" \
  "ds_read_b128 v[60:63], v25 offset:31744\n" \
  "v_mfma_f32_16x16x32_f16 a[128:131], v[32:35], v[80:83], a[128:131]\n" \
  "buffer_load_dwordx4 v[140:143], v19, s[8:11], s41 offen\n" \
  "s_add_u32 s46, s41, 0x80000\n" \
  "v_mfma_f32_16x16x32_f16 a[132:135], v[32:35], v[84:87], a[132:135]\n" \
  "v_readlane_b32 s64, v9, s38\n" \
  "s_add_u32 s38, s38, 1\n" \
  "v_mfma_f32_16x16x32_f16 a[136:139], v[32:35], v[88:91], a[136:139]\n" \
  "v_cvt_pk_f16_f32 v75, v186, v190\n" \
  "v_mfma_f32_16x16x32_f16 a[140:143], v[32:35], v[92:95], a[140:143]\n" \
  "ds_read_b128 v[32:35], v25 offset:32768\n" \
  "v_mfma_f32_16x16x32_f16 a[156:159], v[36:39], v[92:95], a[156:159]\n" \
  "buffer_load_dwordx4 v[144:147], v16, s[8:11], s46 offen\n" \
  "v_mfma_f32_16x16x32_f16 a[152:155], v[36:39], v[88:91], a[152:155]\n" \
  "v_bfe_u32 v27, s64, v7, 8\n" \
  "v_mfma_f32_16x16x32_f16 a[148:151], v[36:39], v[84:87], a[148:151]\n" \
  "s_waitcnt lgkmcnt(5)\n" \
  "buffer_load_dwordx4 v[148:151], v17, s[8:11], s46 offen\n" \
  "s_lshl_b32 s42, s40, 16\n" \
  "v_mfma_f32_16x16x32_f16 a[144:147], v[36:39], v[80:83], a[144:147]\n" \
  "ds_read_b128 v[36:39], v25 offset:33792\n" \
  "v_mfma_f32_16x16x32_f16 a[160:163], v[40:43], v[80:83], a[160:163]\n" \
  "v_lshl_add_u32 v27, v27, 6, v8\n" \
  "v_mfma_f32_16x16x32_f16 a[164:167], v[40:43], v[84:87], a[164:167]\n" \
  "buffer_load_dword v28, v27, s[16:19], 0 offen sc0 sc1\n" \
  "v_mfma_f32_16x16x32_f16 a[168:171], v[40:43], v[88:91], a[168:171]\n" \
  "buffer_load_dwordx4 v[152:155], v18, s[8:11], s46 offen\n" \
  "v_mfma_f32_16x16x32_f16 a[172:175], v[40:43], v[92:95], a[172:175]\n" \
  "ds_read_b128 v[40:43], v25 offset:34816\n" \
  "v_mfma_f32_16x16x32_f16 a[188:191], v[44:47], v[92:95], a[188:191]\n" \
  "v_cvt_pk_f16_f32 v79, v187, v191\n" \
  "v_mfma_f32_16x16x32_f16 a[184:187], v[44:47], v[88:91], a[184:187]\n" \
  "v_mfma_f32_16x16x32_f16 a[180:183], v[44:47], v[84:87], a[180:183]\n" \
  "s_waitcnt lgkmcnt(5)\n" \
  "buffer_load_dwordx4 v[156:159], v19, s[8:11], s46 offen\n" \
  "s_add_u32 m0, s36, 0x4000\n" \
  "v_mfma_f32_16x16x32_f16 a[176:179], v[44:47], v[80:83], a[176:179]\n" \
  "ds_read_b128 v[44:47], v25 offset:35840\n" \
  "v_mfma_f32_16x16x32_f16 a[192:195], v[48:51], v[80:83], a[192:195]\n" \
  "v_mfma_f32_16x16x32_f16 a[196:199], v[48:51], v[84:87], a[196:199]\n" \
  "v_mfma_f32_16x16x32_f16 a[200:203], v[48:51], v[88:91], a[200:203]\n" \
  "buffer_load_dwordx4 v24, s[12:15], s42 offen offset:0 sc0 sc1 lds\n" \
  "v_mfma_f32_16x16x32_f16 a[204:207], v[48:51], v[92:95], a[204:207]\n" \
  "ds_read_b128 v[48:51], v25 offset:36864\n" \
  "v_mfma_f32_16x16x32_f16 a[220:223], v[52:55], v[92:95], a[220:223]\n" \
  "v_mfma_f32_16x16x32_f16 a[216:219], v[52:55], v[88:91], a[216:219]\n" \
  "v_mfma_f32_16x16x32_f16 a[212:215], v[52:55], v[84:87], a[212:215]\n" \
  "s_waitcnt lgkmcnt(5)\n" \
  "buffer_load_dwordx4 v24, s[12:15], s42 offen offset:1024 sc0 sc1 lds\n" \
  "v_mfma_f32_16x16x32_f16 a[208:211], v[52:55], v[80:83], a[208:211]\n" \
  "ds_read_b128 v[52:55], v25 offset:37888\n" \
  "v_mfma_f32_16x16x32_f16 a[224:227], v[56:59], v[80:83], a[224:227]\n" \
  "v_mfma_f32_16x16x32_f16 a[228:231], v[56:59], v[84:87], a[228:231]\n" \
  "buffer_load_dwordx4 v24, s[12:15], s42 offen offset:2048 sc0 sc1 lds\n" \
  "v_mfma_f32_16x16x32_f16 a[232:235], v[56:59], v[88:91], a[232:235]\n" \
  "v_mfma_f32_16x16x32_f16 a[236:239], v[56:59], v[92:95], a[236:239]\n" \
  "ds_read_b128 v[56:59], v25 offset:38912\n" \
  "v_mfma_f32_16x16x32_f16 a[252:255], v[60:63], v[92:95], a[252:255]\n" \
  "buffer_load_dwordx4 v24, s[12:15], s42 offen offset:3072 sc0 sc1 lds\n" \
  "v_mfma_f32_16x16x32_f16 a[248:251], v[60:63], v[88:91], a[248:251]\n" \
  "s_cmp_le_i32 s56, 1\n" \
  "v_mfma_f32_16x16x32_f16 a[244:247], v[60:63], v[84:87], a[244:247]\n" \
  "s_waitcnt lgkmcnt(5)\n" \
  "s_cbranch_scc1 Lpact1_%=\n" \
  "Lpactb1_%=:\n" \
  "v_mfma_f32_16x16x32_f16 a[240:243], v[60:63], v[80:83], a[240:243]\n" \
  "ds_read_b128 v[60:63], v25 offset:39936\n" \
  "v_mfma_f32_16x16x32_f16 a[0:3], v[32:35], v[64:67], a[0:3]\n" \
  "s_bfe_u32 s40, s39, 0x80010\n" \
  "s_lshl_b32 s41, s40, 20\n" \
  "buffer_load_dwordx4 v[160:163], v16, s[8:11], s41 offen\n" \
  "v_mfma_f32_16x16x32_f16 a[4:7], v[32:35], v[68:71], a[4:7]\n" \
  "s_waitcnt vmcnt(36)\n" \
  "v_cvt_pk_f16_f32 v80, v192, v196\n" \
  "v_mfma_f32_16x16x32_f16 a[8:11], v[32:35], v[72:75], a[8:11]\n" \
  "v_cvt_pk_f16_f32 v84, v193, v197\n" \
  "v_mfma_f32_16x16x32_f16 a[12:15], v[32:35], v[76:79], a[12:15]\n" \
  "ds_read_b128 v[32:35], v25 offset:40960\n" \
  "v_mfma_f32_16x16x32_f16 a[28:31], v[36:39], v[76:79], a[28:31]\n" \
  "buffer_load_dwordx4 v[164:167], v17, s[8:11], s41 offen\n" \
  "v_mfma_f32_16x16x32_f16 a[24:27], v[36:39], v[72:75], a[24:27]\n" \
  "v_cvt_pk_f16_f32 v88, v194, v198\n" \
  "v_mfma_f32_16x16x32_f16 a[20:23], v[36:39], v[68:71], a[20:23]\n" \
  "s_waitcnt lgkmcnt(5)\n" \
  "buffer_load_dwordx4 v[168:171], v18, s[8:11], s41 offen\n" \
  "v_mfma_f32_16x16x32_f16 a[16:19], v[36:39], v[64:67], a[16:19]\n" \
  "ds_read_b128 v[36:39], v25 offset:41984\n" \
  "v_mfma_f32_16x16x32_f16 a[32:35], v[40:43], v[64:67], a[32:35]\n" \
  "v_cvt_pk_f16_f32 v92, v195, v199\n" \
  "v_mfma_f32_16x16x32_f16 a[36:39], v[40:43], v[68:71], a[36:39]\n" \
  "s_waitcnt vmcnt(36)\n" \
  "v_cvt_pk_f16_f32 v81, v200, v204\n" \
  "v_mfma_f32_16x16x32_f16 a[40:43], v[40:43], v[72:75], a[40:43]\n" \
  "buffer_load_dwordx4 v[172:175], v19, s[8:11], s41 offen\n" \
  "s_add_u32 s46, s41, 0x80000\n" \
  "v_mfma_f32_16x16x32_f16 a[44:47], v[40:43], v[76:79], a[44:47]\n" \
  "ds_read_b128 v[40:43], v25 offset:43008\n" \
  "v_mfma_f32_16x16x32_f16 a[60:63], v[44:47], v[76:79], a[60:63]\n" \
  "v_cvt_pk_f16_f32 v85, v201, v205\n" \
  "v_mfma_f32_16x16x32_f16 a[56:59], v[44:47], v[72:75], a[56:59]\n" \
  "v_cvt_pk_f16_f32 v89, v202, v206\n" \
  "v_mfma_f32_16x16x32_f16 a[52:55], v[44:47], v[68:71], a[52:55]\n" \
  "s_waitcnt lgkmcnt(5)\n" \
  "buffer_load_dwordx4 v[176:179], v16, s[8:11], s46 offen\n" \
  "v_mfma_f32_16x16x32_f16 a[48:51], v[44:47], v[64:67], a[48:51]\n" \
  "ds_read_b128 v[44:47], v25 offset:44032\n" \
  "v_mfma_f32_16x16x32_f16 a[64:67], v[48:51], v[64:67], a[64:67]\n" \
  "v_cvt_pk_f16_f32 v93, v203, v207\n" \
  "v_mfma_f32_16x16x32_f16 a[68:71], v[48:51], v[68:71], a[68:71]\n" \
  "s_waitcnt vmcnt(36)\n" \
  "v_cvt_pk_f16_f32 v82, v208, v212\n" \
  "v_mfma_f32_16x16x32_f16 a[72:75], v[48:51], v[72:75], a[72:75]\n" \
  "buffer_load_dwordx4 v[180:183], v17, s[8:11], s46 offen\n" \
  "s_lshl_b32 s42, s40, 16\n" \
  "v_mfma_f32_16x16x32_f16 a[76:79], v[48:51], v[76:79], a[76:79]\n" \
  "ds_read_b128 v[48:51], v25 offset:45056\n" \
  "v_mfma_f32_16x16x32_f16 a[92:95], v[52:55], v[76:79], a[92:95]\n" \
  "v_cvt_pk_f16_f32 v86, v209, v213\n" \
  "v_mfma_f32_16x16x32_f16 a[88:91], v[52:55], v[72:75], a[88:91]\n" \
  "v_cvt_pk_f16_f32 v90, v210, v214\n" \
  "v_mfma_f32_16x16x32_f16 a[84:87], v[52:55], v[68:71], a[84:87]\n" \
  "s_waitcnt lgkmcnt(5)\n" \
  "buffer_load_dwordx4 v[184:187], v18, s[8:11], s46 offen\n" \
  "v_mfma_f32_16x16x32_f16 a[80:83], v[52:55], v[64:67], a[80:83]\n" \
  "ds_read_b128 v[52:55], v25 offset:46080\n" \
  "v_mfma_f32_16x16x32_f16 a[96:99], v[56:59], v[64:67], a[96:99]\n" \
  "v_cvt_pk_f16_f32 v94, v211, v215\n" \
  "v_mfma_f32_16x16x32_f16 a[100:103], v[56:59], v[68:71], a[100:103]\n" \
  "buffer_load_dwordx4 v[188:191], v19, s[8:11], s46 offen\n" \
  "s_add_u32 m0, s36, 0x8000\n" \
  "v_mfma_f32_16x16x32_f16 a[104:107], v[56:59], v[72:75], a[104:107]\n" \
  "s_waitcnt vmcnt(37)\n" \
  "v_cvt_pk_f16_f32 v83, v216, v220\n" \
  "v_mfma_f32_16x16x32_f16 a[108:111], v[56:59], v[76:79], a[108:111]\n" \
  "ds_read_b128 v[56:59], v25 offset:47104\n" \
  "v_mfma_f32_16x16x32_f16 a[124:127], v[60:63], v[76:79], a[124:127]\n" \
  "buffer_load_dwordx4 v24, s[12:15], s42 offen offset:0 sc0 sc1 lds\n" \
  "v_mfma_f32_16x16x32_f16 a[120:123], v[60:63], v[72:75], a[120:123]\n" \
  "v_cvt_pk_f16_f32 v87, v217, v221\n" \
  "s_waitcnt vmcnt(34)\n" \
  "v_mfma_f32_16x16x32_f16 a[116:119], v[60:63], v[68:71], a[116:119]\n" \
  "s_waitcnt lgkmcnt(5)\n" \
  "s_barrier\n" \
  "v_mfma_f32_16x16x32_f16 a[112:115], v[60:63], v[64:67], a[112:115]\n" \
  "ds_read_b128 v[60:63], v25 offset:48128\n" \
  "v_mfma_f32_16x16x32_f16 a[128:131], v[32:35], v[64:67], a[128:131]\n" \
  "buffer_load_dwordx4 v24, s[12:15], s42 offen offset:1024 sc0 sc1 lds\n" \
  "v_mfma_f32_16x16x32_f16 a[132:135], v[32:35], v[68:71], a[132:135]\n" \
  "v_cvt_pk_f16_f32 v91, v218, v222\n" \
  "v_mfma_f32_16x16x32_f16 a[136:139], v[32:35], v[72:75], a[136:139]\n" \
  "v_cvt_pk_f16_f32 v95, v219, v223\n" \
  "v_mfma_f32_16x16x32_f16 a[140:143], v[32:35], v[76:79], a[140:143]\n" \
  "ds_read_b128 v[32:35], v25 offset:49152\n" \
  "v_mfma_f32_16x16x32_f16 a[156:159], v[36:39], v[76:79], a[156:159]\n" \
  "buffer_load_dwordx4 v24, s[12:15], s42 offen offset:2048 sc0 sc1 lds\n" \
  "v_mfma_f32_16x16x32_f16 a[152:155], v[36:39], v[72:75], a[152:155]\n" \
  "v_mfma_f32_16x16x32_f16 a[148:151], v[36:39], v[68:71], a[148:151]\n" \
  "s_waitcnt lgkmcnt(5)\n" \
  "buffer_load_dwordx4 v24, s[12:15], s42 offen offset:3072 sc0 sc1 lds\n" \
  "v_mfma_f32_16x16x32_f16 a[144:147], v[36:39], v[64:67], a[144:147]\n" \
  "ds_read_b128 v[36:39], v25 offset:50176\n" \
  "v_mfma_f32_16x16x32_f16 a[160:163], v[40:43], v[64:67], a[160:163]\n" \
  "v_mfma_f32_16x16x32_f16 a[164:167], v[40:43], v[68:71], a[164:167]\n" \
  "v_mfma_f32_16x16x32_f16 a[168:171], v[40:43], v[72:75], a[168:171]\n" \
  "s_bfe_u32 s40, s39, 0x80018\n" \
  "s_lshl_b32 s41, s40, 20\n" \
  "buffer_load_dwordx4 v[192:195], v16, s[8:11], s41 offen\n" \
  "v_mfma_f32_16x16x32_f16 a[172:175], v[40:43], v[76:79], a[172:175]\n" \
  "ds_read_b128 v[40:43], v25 offset:51200\n" \
  "v_mfma_f32_16x16x32_f16 a[188:191], v[44:47], v[76:79], a[188:191]\n" \
  "v_mfma_f32_16x16x32_f16 a[184:187], v[44:47], v[72:75], a[184:187]\n" \
  "v_mfma_f32_16x16x32_f16 a[180:183], v[44:47], v[68:71], a[180:183]\n" \
  "s_waitcnt lgkmcnt(5)\n" \
  "buffer_load_dwordx4 v[196:199], v17, s[8:11], s41 offen\n" \
  "v_mfma_f32_16x16x32_f16 a[176:179], v[44:47], v[64:67], a[176:179]\n" \
  "ds_read_b128 v[44:47], v25 offset:52224\n" \
  "v_mfma_f32_16x16x32_f16 a[192:195], v[48:51], v[64:67], a[192:195]\n" \
  "v_mfma_f32_16x16x32_f16 a[196:199], v[48:51], v[68:71], a[196:199]\n" \
  "v_mfma_f32_16x16x32_f16 a[200:203], v[48:51], v[72:75], a[200:203]\n" \
  "buffer_load_dwordx4 v[200:203], v18, s[8:11], s41 offen\n" \
  "v_mfma_f32_16x16x32_f16 a[204:207], v[48:51], v[76:79], a[204:207]\n" \
  "ds_read_b128 v[48:51], v25 offset:53248\n" \
  "v_mfma_f32_16x16x32_f16 a[220:223], v[52:55], v[76:79], a[220:223]\n" \
  "v_mfma_f32_16x16x32_f16 a[216:219], v[52:55], v[72:75], a[216:219]\n" \
  "v_mfma_f32_16x16x32_f16 a[212:215], v[52:55], v[68:71], a[212:215]\n" \
  "s_waitcnt lgkmcnt(5)\n" \
  "buffer_load_dwordx4 v[204:207], v19, s[8:11], s41 offen\n" \
  "s_add_u32 s46, s41, 0x80000\n" \
  "v_mfma_f32_16x16x32_f16 a[208:211], v[52:55], v[64:67], a[208:211]\n" \
  "ds_read_b128 v[52:55], v25 offset:54272\n" \
  "v_mfma_f32_16x16x32_f16 a[224:227], v[56:59], v[64:67], a[224:227]\n" \
  "v_mfma_f32_16x16x32_f16 a[228:231], v[56:59], v[68:71], a[228:231]\n" \
  "buffer_load_dwordx4 v[208:211], v16, s[8:11], s46 offen\n" \
  "v_mfma_f32_16x16x32_f16 a[232:235], v[56:59], v[72:75], a[232:235]\n" \
  "v_mfma_f32_16x16x32_f16 a[236:239], v[56:59], v[76:79], a[236:239]\n" \
  "ds_read_b128 v[56:59], v25 offset:55296\n" \
  "v_mfma_f32_16x16x32_f16 a[252:255], v[60:63], v[76:79], a[252:255]\n" \
  "buffer_load_dwordx4 v[212:215], v17, s[8:11], s46 offen\n" \
  "s_lshl_b32 s42, s40, 16\n" \
  "v_mfma_f32_16x16x32_f16 a[248:251], v[60:63], v[72:75], a[248:251]\n" \
  "s_cmp_le_i32 s56, 2\n" \
  "v_mfma_f32_16x16x32_f16 a[244:247], v[60:63], v[68:71], a[244:247]\n" \
  "s_waitcnt lgkmcnt(5)\n" \
  "s_cbranch_scc1 Lpact2_%=\n" \
  "Lpactb2_%=:\n" \
  "v_mfma_f32_16x16x32_f16 a[240:243], v[60:63], v[64:67], a[240:243]\n" \
  "ds_read_b128 v[60:63], v25 offset:56320\n" \
  "v_mfma_f32_16x16x32_f16 a[0:3], v[32:35], v[80:83], a[0:3]\n" \
  "buffer_load_dwordx4 v[216:219], v18, s[8:11], s46 offen\n" \
  "v_mfma_f32_16x16x32_f16 a[4:7], v[32:35], v[84:87], a[4:7]\n" \
  "s_waitcnt vmcnt(42)\n" \
  "v_cvt_pk_f16_f32 v64, v96, v100\n" \
  "v_mfma_f32_16x16x32_f16 a[8:11], v[32:35], v[88:91], a[8:11]\n" \
  "v_cvt_pk_f16_f32 v68, v97, v101\n" \
  "v_mfma_f32_16x16x32_f16 a[12:15], v[32:35], v[92:95], a[12:15]\n" \
  "ds_read_b128 v[32:35], v25 offset:57344\n" \
  "v_mfma_f32_16x16x32_f16 a[28:31], v[36:39], v[92:95], a[28:31]\n" \
  "buffer_load_dwordx4 v[220:223], v19, s[8:11], s46 offen\n" \
  "s_add_u32 m0, s36, 0xc000\n" \
  "v_mfma_f32_16x16x32_f16 a[24:27], v[36:39], v[88:91], a[24:27]\n" \
  "v_cvt_pk_f16_f32 v72, v98, v102\n" \
  "v_mfma_f32_16x16x32_f16 a[20:23], v[36:39], v[84:87], a[20:23]\n" \
  "s_waitcnt lgkmcnt(5)\n" \
  "buffer_load_dwordx4 v24, s[12:15], s42 offen offset:0 sc0 sc1 lds\n" \
  "v_mfma_f32_16x16x32_f16 a[16:19], v[36:39], v[80:83], a[16:19]\n" \
  "ds_read_b128 v[36:39], v25 offset:58368\n" \
  "v_mfma_f32_16x16x32_f16 a[32:35], v[40:43], v[80:83], a[32:35]\n" \
  "v_cvt_pk_f16_f32 v76, v99, v103\n" \
  "v_mfma_f32_16x16x32_f16 a[36:39], v[40:43], v[84:87], a[36:39]\n" \
  "s_waitcnt vmcnt(42)\n" \
  "v_cvt_pk_f16_f32 v65, v104, v108\n" \
  "v_mfma_f32_16x16x32_f16 a[40:43], v[40:43], v[88:91], a[40:43]\n" \
  "buffer_load_dwordx4 v24, s[12:15], s42 offen offset:1024 sc0 sc1 lds\n" \
  "v_mfma_f32_16x16x32_f16 a[44:47], v[40:43], v[92:95], a[44:47]\n" \
  "ds_read_b128 v[40:43], v25 offset:59392\n" \
  "v_mfma_f32_16x16x32_f16 a[60:63], v[44:47], v[92:95], a[60:63]\n" \
  "v_cvt_pk_f16_f32 v69, v105, v109\n" \
  "v_mfma_f32_16x16x32_f16 a[56:59], v[44:47], v[88:91], a[56:59]\n" \
  "v_cvt_pk_f16_f32 v73, v106, v110\n" \
  "v_mfma_f32_16x16x32_f16 a[52:55], v[44:47], v[84:87], a[52:55]\n" \
  "s_waitcnt lgkmcnt(5)\n" \
  "buffer_load_dwordx4 v24, s[12:15], s42 offen offset:2048 sc0 sc1 lds\n" \
  "v_mfma_f32_16x16x32_f16 a[48:51], v[44:47], v[80:83], a[48:51]\n" \
  "ds_read_b128 v[44:47], v25 offset:60416\n" \
  "v_mfma_f32_16x16x32_f16 a[64:67], v[48:51], v[80:83], a[64:67]\n" \
  "v_cvt_pk_f16_f32 v77, v107, v111\n" \
  "v_mfma_f32_16x16x32_f16 a[68:71], v[48:51], v[84:87], a[68:71]\n" \
  "s_waitcnt vmcnt(42)\n" \
  "v_cvt_pk_f16_f32 v66, v112, v116\n" \
  "v_mfma_f32_16x16x32_f16 a[72:75], v[48:51], v[88:91], a[72:75]\n" \
  "buffer_load_dwordx4 v24, s[12:15], s42 offen offset:3072 sc0 sc1 lds\n" \
  "v_mfma_f32_16x16x32_f16 a[76:79], v[48:51], v[92:95], a[76:79]\n" \
  "ds_read_b128 v[48:51], v25 offset:61440\n" \
  "v_mfma_f32_16x16x32_f16 a[92:95], v[52:55], v[92:95], a[92:95]\n" \
  "v_cvt_pk_f16_f32 v70, v113, v117\n" \
  "v_mfma_f32_16x16x32_f16 a[88:91], v[52:55], v[88:91], a[88:91]\n" \
  "v_cvt_pk_f16_f32 v74, v114, v118\n" \
  "v_mfma_f32_16x16x32_f16 a[84:87], v[52:55], v[84:87], a[84:87]\n" \
  "s_waitcnt lgkmcnt(5)\n" \
  "s_mov_b32 s39, s64\n" \
  "s_bfe_u32 s40, s39, 0x80000\n" \
  "s_lshl_b32 s41, s40, 20\n" \
  "buffer_load_dwordx4 v[96:99], v16, s[8:11], s41 offen\n" \
  "v_mfma_f32_16x16x32_f16 a[80:83], v[52:55], v[80:83], a[80:83]\n" \
  "ds_read_b128 v[52:55], v25 offset:62464\n" \
  "v_mfma_f32_16x16x32_f16 a[96:99], v[56:59], v[80:83], a[96:99]\n" \
  "v_cvt_pk_f16_f32 v78, v115, v119\n" \
  "v_mfma_f32_16x16x32_f16 a[100:103], v[56:59], v[84:87], a[100:103]\n" \
  "buffer_load_dwordx4 v[100:103], v17, s[8:11], s41 offen\n" \
  "v_mfma_f32_16x16x32_f16 a[104:107], v[56:59], v[88:91], a[104:107]\n" \
  "s_waitcnt vmcnt(43)\n" \
  "v_cvt_pk_f16_f32 v67, v120, v124\n" \
  "v_mfma_f32_16x16x32_f16 a[108:111], v[56:59], v[92:95], a[108:111]\n" \
  "ds_read_b128 v[56:59], v25 offset:63488\n" \
  "v_mfma_f32_16x16x32_f16 a[124:127], v[60:63], v[92:95], a[124:127]\n" \
  "buffer_load_dwordx4 v[104:107], v18, s[8:11], s41 offen\n" \
  "v_mfma_f32_16x16x32_f16 a[120:123], v[60:63], v[88:91], a[120:123]\n" \
  "v_cvt_pk_f16_f32 v71, v121, v125\n" \
  "s_waitcnt vmcnt(40)\n" \
  "v_mfma_f32_16x16x32_f16 a[116:119], v[60:63], v[84:87], a[116:119]\n" \
  "s_waitcnt lgkmcnt(5)\n" \
  "s_barrier\n" \
  "v_mfma_f32_16x16x32_f16 a[112:115], v[60:63], v[80:83], a[112:115]\n" \
  "ds_read_b128 v[60:63], v25 offset:64512\n" \
  "v_mfma_f32_16x16x32_f16 a[128:131], v[32:35], v[80:83], a[128:131]\n" \
  "buffer_load_dwordx4 v[108:111], v19, s[8:11], s41 offen\n" \
  "s_add_u32 s46, s41, 0x80000\n" \
  "v_mfma_f32_16x16x32_f16 a[132:135], v[32:35], v[84:87], a[132:135]\n" \
  "v_cvt_pk_f16_f32 v75, v122, v126\n" \
  "v_mfma_f32_16x16x32_f16 a[136:139], v[32:35], v[88:91], a[136:139]\n" \
  "v_xor_b32 v25, 0x10000, v25\n" \
  "v_mfma_f32_16x16x32_f16 a[140:143], v[32:35], v[92:95], a[140:143]\n" \
  "ds_read_b128 v[32:35], v25 offset:0\n" \
  "v_mfma_f32_16x16x32_f16 a[156:159], v[36:39], v[92:95], a[156:159]\n" \
  "buffer_load_dwordx4 v[112:115], v16, s[8:11], s46 offen\n" \
  "v_mfma_f32_16x16x32_f16 a[152:155], v[36:39], v[88:91], a[152:155]\n" \
  "v_cvt_pk_f16_f32 v79, v123, v127\n" \
  "v_mfma_f32_16x16x32_f16 a[148:151], v[36:39], v[84:87], a[148:151]\n" \
  "s_waitcnt lgkmcnt(5)\n" \
  "buffer_load_dwordx4 v[116:119], v17, s[8:11], s46 offen\n" \
  "s_lshl_b32 s42, s40, 16\n" \
  "v_mfma_f32_16x16x32_f16 a[144:147], v[36:39], v[80:83], a[144:147]\n" \
  "ds_read_b128 v[36:39], v25 offset:1024\n" \
  "v_mfma_f32_16x16x32_f16 a[160:163], v[40:43], v[80:83], a[160:163]\n" \
  "v_mfma_f32_16x16x32_f16 a[164:167], v[40:43], v[84:87], a[164:167]\n" \
  "v_mfma_f32_16x16x32_f16 a[168:171], v[40:43], v[88:91], a[168:171]\n" \
  "buffer_load_dwordx4 v[120:123], v18, s[8:11], s46 offen\n" \
  "s_xor_b32 s36, s36, 0x10000\n" \
  "v_mfma_f32_16x16x32_f16 a[172:175], v[40:43], v[92:95], a[172:175]\n" \
  "ds_read_b128 v[40:43], v25 offset:2048\n" \
  "v_mfma_f32_16x16x32_f16 a[188:191], v[44:47], v[92:95], a[188:191]\n" \
  "v_mfma_f32_16x16x32_f16 a[184:187], v[44:47], v[88:91], a[184:187]\n" \
  "v_mfma_f32_16x16x32_f16 a[180:183], v[44:47], v[84:87], a[180:183]\n" \
  "s_waitcnt lgkmcnt(5)\n" \
  "buffer_load_dwordx4 v[124:127], v19, s[8:11], s46 offen\n" \
  "s_add_u32 m0, s36, 0x0\n" \
  "v_mfma_f32_16x16x32_f16 a[176:179], v[44:47], v[80:83], a[176:179]\n" \
  "ds_read_b128 v[44:47], v25 offset:3072\n" \
  "v_mfma_f32_16x16x32_f16 a[192:195], v[48:51], v[80:83], a[192:195]\n" \
  "s_waitcnt vmcnt(38)\n" \
  "v_cmp_le_u32 vcc, s43, v28\n" \
  "v_mfma_f32_16x16x32_f16 a[196:199], v[48:51], v[84:87], a[196:199]\n" \
  "s_cmp_eq_u64 vcc, -1\n" \
  "s_cbranch_scc0 Lstub9_%=\n" \
  "Lback9_%=:\n" \
  "v_mfma_f32_16x16x32_f16 a[200:203], v[48:51], v[88:91], a[200:203]\n" \
  "buffer_load_dwordx4 v24, s[12:15], s42 offen offset:0 sc0 sc1 lds\n" \
  "v_mfma_f32_16x16x32_f16 a[204:207], v[48:51], v[92:95], a[204:207]\n" \
  "ds_read_b128 v[48:51], v25 offset:4096\n" \
  "v_mfma_f32_16x16x32_f16 a[220:223], v[52:55], v[92:95], a[220:223]\n" \
  "v_mfma_f32_16x16x32_f16 a[216:219], v[52:55], v[88:91], a[216:219]\n" \
  "v_mfma_f32_16x16x32_f16 a[212:215], v[52:55], v[84:87], a[212:215]\n" \
  "s_waitcnt lgkmcnt(5)\n" \
  "buffer_load_dwordx4 v24, s[12:15], s42 offen offset:1024 sc0 sc1 lds\n" \
  "v_mfma_f32_16x16x32_f16 a[208:211], v[52:55], v[80:83], a[208:211]\n" \
  "ds_read_b128 v[52:55], v25 offset:5120\n" \
  "v_mfma_f32_16x16x32_f16 a[224:227], v[56:59], v[80:83], a[224:227]\n" \
  "v_mfma_f32_16x16x32_f16 a[228:231], v[56:59], v[84:87], a[228:231]\n" \
  "buffer_load_dwordx4 v24, s[12:15], s42 offen offset:2048 sc0 sc1 lds\n" \
  "v_mfma_f32_16x16x32_f16 a[232:235], v[56:59], v[88:91], a[232:235]\n" \
  "v_mfma_f32_16x16x32_f16 a[236:239], v[56:59], v[92:95], a[236:239]\n" \
  "ds_read_b128 v[56:59], v25 offset:6144\n" \
  "v_mfma_f32_16x16x32_f16 a[252:255], v[60:63], v[92:95], a[252:255]\n" \
  "buffer_load_dwordx4 v24, s[12:15], s42 offen offset:3072 sc0 sc1 lds\n" \
  "v_mfma_f32_16x16x32_f16 a[248:251], v[60:63], v[88:91], a[248:251]\n" \
  "s_cmp_le_i32 s56, 3\n" \
  "v_mfma_f32_16x16x32_f16 a[244:247], v[60:63], v[84:87], a[244:247]\n" \
  "s_waitcnt lgkmcnt(5)\n" \
  "s_cbranch_scc1 Lpact3_%=\n" \
  "Lpactb3_%=:\n" \
  ".p2align 6\n" \
  "Lloop_%=:\n" \
  "v_mfma_f32_16x16x32_f16 a[240:243], v[60:63], v[80:83], a[240:243]\n" \
  "ds_read_b128 v[60:63], v25 offset:7168\n" \
  "v_mfma_f32_16x16x32_f16 a[0:3], v[32:35], v[64:67], a[0:3]\n" \
  "s_waitcnt vmcnt(47)\n" \
  "v_cvt_pk_f16_f32 v80, v128, v132\n" \
  "v_mfma_f32_16x16x32_f16 a[4:7], v[32:35], v[68:71], a[4:7]\n" \
  "v_cvt_pk_f16_f32 v84, v129, v133\n" \
  "v_mfma_f32_16x16x32_f16 a[8:11], v[32:35], v[72:75], a[8:11]\n" \
  "v_cvt_pk_f16_f32 v88, v130, v134\n" \
  "s_bfe_u32 s40, s39, 0x80008\n" \
  "v_mfma_f32_16x16x32_f16 a[12:15], v[32:35], v[76:79], a[12:15]\n" \
  "ds_read_b128 v[32:35], v25 offset:8192\n" \
  "v_mfma_f32_16x16x32_f16 a[28:31], v[36:39], v[76:79], a[28:31]\n" \
  "v_cvt_pk_f16_f32 v92, v131, v135\n" \
  "s_lshl_b32 s41, s40, 20\n" \
  "v_mfma_f32_16x16x32_f16 a[24:27], v[36:39], v[72:75], a[24:27]\n" \
  "buffer_load_dwordx4 v[128:131], v16, s[8:11], s41 offen\n" \
  "v_mfma_f32_16x16x32_f16 a[20:23], v[36:39], v[68:71], a[20:23]\n" \
  "s_waitcnt lgkmcnt(5)\n" \
  "v_mfma_f32_16x16x32_f16 a[16:19], v[36:39], v[64:67], a[16:19]\n" \
  "ds_read_b128 v[36:39], v25 offset:9216\n" \
  "v_mfma_f32_16x16x32_f16 a[32:35], v[40:43], v[64:67], a[32:35]\n" \
  "s_waitcnt vmcnt(46)\n" \
  "v_cvt_pk_f16_f32 v81, v136, v140\n" \
  "v_mfma_f32_16x16x32_f16 a[36:39], v[40:43], v[68:71], a[36:39]\n" \
  "v_cvt_pk_f16_f32 v85, v137, v141\n" \
  "v_mfma_f32_16x16x32_f16 a[40:43], v[40:43], v[72:75], a[40:43]\n" \
  "buffer_load_dwordx4 v[132:135], v17, s[8:11], s41 offen\n" \
  "v_mfma_f32_16x16x32_f16 a[44:47], v[40:43], v[76:79], a[44:47]\n" \
  "ds_read_b128 v[40:43], v25 offset:10240\n" \
  "v_mfma_f32_16x16x32_f16 a[60:63], v[44:47], v[76:79], a[60:63]\n" \
  "v_cvt_pk_f16_f32 v89, v138, v142\n" \
  "v_mfma_f32_16x16x32_f16 a[56:59], v[44:47], v[72:75], a[56:59]\n" \
  "v_cvt_pk_f16_f32 v93, v139, v143\n" \
  "v_mfma_f32_16x16x32_f16 a[52:55], v[44:47], v[68:71], a[52:55]\n" \
  "s_waitcnt lgkmcnt(5)\n" \
  "v_mfma_f32_16x16x32_f16 a[48:51], v[44:47], v[64:67], a[48:51]\n" \
  "ds_read_b128 v[44:47], v25 offset:11264\n" \
  "v_mfma_f32_16x16x32_f16 a[64:67], v[48:51], v[64:67], a[64:67]\n" \
  "buffer_load_dwordx4 v[136:139], v18, s[8:11], s41 offen\n" \
  "v_mfma_f32_16x16x32_f16 a[68:71], v[48:51], v[68:71], a[68:71]\n" \
  "s_waitcnt vmcnt(46)\n" \
  "v_cvt_pk_f16_f32 v82, v144, v148\n" \
  "v_mfma_f32_16x16x32_f16 a[72:75], v[48:51], v[72:75], a[72:75]\n" \
  "v_cvt_pk_f16_f32 v86, v145, v149\n" \
  "v_mfma_f32_16x16x32_f16 a[76:79], v[48:51], v[76:79], a[76:79]\n" \
  "ds_read_b128 v[48:51], v25 offset:12288\n" \
  "v_mfma_f32_16x16x32_f16 a[92:95], v[52:55], v[76:79], a[92:95]\n" \
  "v_cvt_pk_f16_f32 v90, v146, v150\n" \
  "v_mfma_f32_16x16x32_f16 a[88:91], v[52:55], v[72:75], a[88:91]\n" \
  "buffer_load_dwordx4 v[140:143], v19, s[8:11], s41 offen\n" \
  "v_mfma_f32_16x16x32_f16 a[84:87], v[52:55], v[68:71], a[84:87]\n" \
  "s_waitcnt lgkmcnt(5)\n" \
  "s_add_u32 s46, s41, 0x80000\n" \
  "v_mfma_f32_16x16x32_f16 a[80:83], v[52:55], v[64:67], a[80:83]\n" \
  "ds_read_b128 v[52:55], v25 offset:13312\n" \
  "v_mfma_f32_16x16x32_f16 a[96:99], v[56:59], v[64:67], a[96:99]\n" \
  "v_cvt_pk_f16_f32 v94, v147, v151\n" \
  "v_mfma_f32_16x16x32_f16 a[100:103], v[56:59], v[68:71], a[100:103]\n" \
  "s_waitcnt vmcnt(44)\n" \
  "v_cvt_pk_f16_f32 v83, v152, v156\n" \
  "v_mfma_f32_16x16x32_f16 a[104:107], v[56:59], v[72:75], a[104:107]\n" \
  "buffer_load_dwordx4 v[144:147], v16, s[8:11], s46 offen\n" \
  "v_mfma_f32_16x16x32_f16 a[108:111], v[56:59], v[76:79], a[108:111]\n" \
  "ds_read_b128 v[56:59], v25 offset:14336\n" \
  "v_mfma_f32_16x16x32_f16 a[124:127], v[60:63], v[76:79], a[124:127]\n" \
  "v_cvt_pk_f16_f32 v87, v153, v157\n" \
  "v_mfma_f32_16x16x32_f16 a[120:123], v[60:63], v[72:75], a[120:123]\n" \
  "v_cvt_pk_f16_f32 v91, v154, v158\n" \
  "s_waitcnt vmcnt(41)\n" \
  "v_mfma_f32_16x16x32_f16 a[116:119], v[60:63], v[68:71], a[116:119]\n" \
  "s_waitcnt lgkmcnt(5)\n" \
  "s_barrier\n" \
  "v_mfma_f32_16x16x32_f16 a[112:115], v[60:63], v[64:67], a[112:115]\n" \
  "ds_read_b128 v[60:63], v25 offset:15360\n" \
  "v_mfma_f32_16x16x32_f16 a[128:131], v[32:35], v[64:67], a[128:131]\n" \
  "buffer_load_dwordx4 v[148:151], v17, s[8:11], s46 offen\n" \
  "v_mfma_f32_16x16x32_f16 a[132:135], v[32:35], v[68:71], a[132:135]\n" \
  "v_cvt_pk_f16_f32 v95, v155, v159\n" \
  "v_mfma_f32_16x16x32_f16 a[136:139], v[32:35], v[72:75], a[136:139]\n" \
  "v_mfma_f32_16x16x32_f16 a[140:143], v[32:35], v[76:79], a[140:143]\n" \
  "ds_read_b128 v[32:35], v25 offset:16384\n" \
  "v_mfma_f32_16x16x32_f16 a[156:159], v[36:39], v[76:79], a[156:159]\n" \
  "v_mfma_f32_16x16x32_f16 a[152:155], v[36:39], v[72:75], a[152:155]\n" \
  "buffer_load_dwordx4 v[152:155], v18, s[8:11], s46 offen\n" \
  "v_mfma_f32_16x16x32_f16 a[148:151], v[36:39], v[68:71], a[148:151]\n" \
  "s_waitcnt lgkmcnt(5)\n" \
  "s_lshl_b32 s42, s40, 16\n" \
  "v_mfma_f32_16x16x32_f16 a[144:147], v[36:39], v[64:67], a[144:147]\n" \
  "ds_read_b128 v[36:39], v25 offset:17408\n" \
  "v_mfma_f32_16x16x32_f16 a[160:163], v[40:43], v[64:67], a[160:163]\n" \
  "s_sub_u32 s56, s56, 4\n" \
  "v_mfma_f32_16x16x32_f16 a[164:167], v[40:43], v[68:71], a[164:167]\n" \
  "v_mfma_f32_16x16x32_f16 a[168:171], v[40:43], v[72:75], a[168:171]\n" \
  "buffer_load_dwordx4 v[156:159], v19, s[8:11], s46 offen\n" \
  "v_mfma_f32_16x16x32_f16 a[172:175], v[40:43], v[76:79], a[172:175]\n" \
  "ds_read_b128 v[40:43], v25 offset:18432\n" \
  "v_mfma_f32_16x16x32_f16 a[188:191], v[44:47], v[76:79], a[188:191]\n" \
  "v_mfma_f32_16x16x32_f16 a[184:187], v[44:47], v[72:75], a[184:187]\n" \
  "v_mfma_f32_16x16x32_f16 a[180:183], v[44:47], v[68:71], a[180:183]\n" \
  "s_waitcnt lgkmcnt(5)\n" \
  "s_add_u32 m0, s36, 0x4000\n" \
  "v_mfma_f32_16x16x32_f16 a[176:179], v[44:47], v[64:67], a[176:179]\n" \
  "ds_read_b128 v[44:47], v25 offset:19456\n" \
  "v_mfma_f32_16x16x32_f16 a[192:195], v[48:51], v[64:67], a[192:195]\n" \
  "buffer_load_dwordx4 v24, s[12:15], s42 offen offset:0 sc0 sc1 lds\n" \
  "v_mfma_f32_16x16x32_f16 a[196:199], v[48:51], v[68:71], a[196:199]\n" \
  "v_mfma_f32_16x16x32_f16 a[200:203], v[48:51], v[72:75], a[200:203]\n" \
  "v_mfma_f32_16x16x32_f16 a[204:207], v[48:51], v[76:79], a[204:207]\n" \
  "ds_read_b128 v[48:51], v25 offset:20480\n" \
  "v_mfma_f32_16x16x32_f16 a[220:223], v[52:55], v[76:79], a[220:223]\n" \
  "v_mfma_f32_16x16x32_f16 a[216:219], v[52:55], v[72:75], a[216:219]\n" \
  "buffer_load_dwordx4 v24, s[12:15], s42 offen offset:1024 sc0 sc1 lds\n" \
  "v_mfma_f32_16x16x32_f16 a[212:215], v[52:55], v[68:71], a[212:215]\n" \
  "s_waitcnt lgkmcnt(5)\n" \
  "v_mfma_f32_16x16x32_f16 a[208:211], v[52:55], v[64:67], a[208:211]\n" \
  "ds_read_b128 v[52:55], v25 offset:21504\n" \
  "v_mfma_f32_16x16x32_f16 a[224:227], v[56:59], v[64:67], a[224:227]\n" \
  "v_mfma_f32_16x16x32_f16 a[228:231], v[56:59], v[68:71], a[228:231]\n" \
  "buffer_load_dwordx4 v24, s[12:15], s42 offen offset:2048 sc0 sc1 lds\n" \
  "v_mfma_f32_16x16x32_f16 a[232:235], v[56:59], v[72:75], a[232:235]\n" \
  "v_mfma_f32_16x16x32_f16 a[236:239], v[56:59], v[76:79], a[236:239]\n" \
  "ds_read_b128 v[56:59], v25 offset:22528\n" \
  "v_mfma_f32_16x16x32_f16 a[252:255], v[60:63], v[76:79], a[252:255]\n" \
  "v_mfma_f32_16x16x32_f16 a[248:251], v[60:63], v[72:75], a[248:251]\n" \
  "buffer_load_dwordx4 v24, s[12:15], s42 offen offset:3072 sc0 sc1 lds\n" \
  "s_cmp_le_i32 s56, 0\n" \
  "v_mfma_f32_16x16x32_f16 a[244:247], v[60:63], v[68:71], a[244:247]\n" \
  "s_waitcnt lgkmcnt(5)\n" \
  "s_cbranch_scc1 Lact0_%=\n" \
  "Lactb0_%=:\n" \
  "v_mfma_f32_16x16x32_f16 a[240:243], v[60:63], v[64:67], a[240:243]\n" \
  "ds_read_b128 v[60:63], v25 offset:23552\n" \
  "v_mfma_f32_16x16x32_f16 a[0:3], v[32:35], v[80:83], a[0:3]\n" \
  "s_waitcnt vmcnt(46)\n" \
  "v_cvt_pk_f16_f32 v64, v160, v164\n" \
  "v_mfma_f32_16x16x32_f16 a[4:7], v[32:35], v[84:87], a[4:7]\n" \
  "v_cvt_pk_f16_f32 v68, v161, v165\n" \
  "v_mfma_f32_16x16x32_f16 a[8:11], v[32:35], v[88:91], a[8:11]\n" \
  "v_cvt_pk_f16_f32 v72, v162, v166\n" \
  "s_bfe_u32 s40, s39, 0x80010\n" \
  "v_mfma_f32_16x16x32_f16 a[12:15], v[32:35], v[92:95], a[12:15]\n" \
  "ds_read_b128 v[32:35], v25 offset:24576\n" \
  "v_mfma_f32_16x16x32_f16 a[28:31], v[36:39], v[92:95], a[28:31]\n" \
  "v_cvt_pk_f16_f32 v76, v163, v167\n" \
  "s_lshl_b32 s41, s40, 20\n" \
  "v_mfma_f32_16x16x32_f16 a[24:27], v[36:39], v[88:91], a[24:27]\n" \
  "buffer_load_dwordx4 v[160:163], v16, s[8:11], s41 offen\n" \
  "v_mfma_f32_16x16x32_f16 a[20:23], v[36:39], v[84:87], a[20:23]\n" \
  "s_waitcnt lgkmcnt(5)\n" \
  "v_mfma_f32_16x16x32_f16 a[16:19], v[36:39], v[80:83], a[16:19]\n" \
  "ds_read_b128 v[36:39], v25 offset:25600\n" \
  "v_mfma_f32_16x16x32_f16 a[32:35], v[40:43], v[80:83], a[32:35]\n" \
  "s_waitcnt vmcnt(45)\n" \
  "v_cvt_pk_f16_f32 v65, v168, v172\n" \
  "v_mfma_f32_16x16x32_f16 a[36:39], v[40:43], v[84:87], a[36:39]\n" \
  "v_cvt_pk_f16_f32 v69, v169, v173\n" \
  "v_mfma_f32_16x16x32_f16 a[40:43], v[40:43], v[88:91], a[40:43]\n" \
  "buffer_load_dwordx4 v[164:167], v17, s[8:11], s41 offen\n" \
  "v_mfma_f32_16x16x32_f16 a[44:47], v[40:43], v[92:95], a[44:47]\n" \
  "ds_read_b128 v[40:43], v25 offset:26624\n" \
  "v_mfma_f32_16x16x32_f16 a[60:63], v[44:47], v[92:95], a[60:63]\n" \
  "v_cvt_pk_f16_f32 v73, v170, v174\n" \
  "v_mfma_f32_16x16x32_f16 a[56:59], v[44:47], v[88:91], a[56:59]\n" \
  "v_cvt_pk_f16_f32 v77, v171, v175\n" \
  "v_mfma_f32_16x16x32_f16 a[52:55], v[44:47], v[84:87], a[52:55]\n" \
  "s_waitcnt lgkmcnt(5)\n" \
  "v_mfma_f32_16x16x32_f16 a[48:51], v[44:47], v[80:83], a[48:51]\n" \
  "ds_read_b128 v[44:47], v25 offset:27648\n" \
  "v_mfma_f32_16x16x32_f16 a[64:67], v[48:51], v[80:83], a[64:67]\n" \
  "buffer_load_dwordx4 v[168:171], v18, s[8:11], s41 offen\n" \
  "v_mfma_f32_16x16x32_f16 a[68:71], v[48:51], v[84:87], a[68:71]\n" \
  "s_waitcnt vmcnt(45)\n" \
  "v_cvt_pk_f16_f32 v66, v176, v180\n" \
  "v_mfma_f32_16x16x32_f16 a[72:75], v[48:51], v[88:91], a[72:75]\n" \
  "v_cvt_pk_f16_f32 v70, v177, v181\n" \
  "v_mfma_f32_16x16x32_f16 a[76:79], v[48:51], v[92:95], a[76:79]\n" \
  "ds_read_b128 v[48:51], v25 offset:28672\n" \
  "v_mfma_f32_16x16x32_f16 a[92:95], v[52:55], v[92:95], a[92:95]\n" \
  "v_cvt_pk_f16_f32 v74, v178, v182\n" \
  "v_mfma_f32_16x16x32_f16 a[88:91], v[52:55], v[88:91], a[88:91]\n" \
  "buffer_load_dwordx4 v[172:175], v19, s[8:11], s41 offen\n" \
  "v_mfma_f32_16x16x32_f16 a[84:87], v[52:55], v[84:87], a[84:87]\n" \
  "s_waitcnt lgkmcnt(5)\n" \
  "s_add_u32 s46, s41, 0x80000\n" \
  "v_mfma_f32_16x16x32_f16 a[80:83], v[52:55], v[80:83], a[80:83]\n" \
  "ds_read_b128 v[52:55], v25 offset:29696\n" \
  "v_mfma_f32_16x16x32_f16 a[96:99], v[56:59], v[80:83], a[96:99]\n" \
  "v_cvt_pk_f16_f32 v78, v179, v183\n" \
  "v_mfma_f32_16x16x32_f16 a[100:103], v[56:59], v[84:87], a[100:103]\n" \
  "s_waitcnt vmcnt(44)\n" \
  "v_cvt_pk_f16_f32 v67, v184, v188\n" \
  "v_mfma_f32_16x16x32_f16 a[104:107], v[56:59], v[88:91], a[104:107]\n" \
  "buffer_load_dwordx4 v[176:179], v16, s[8:11], s46 offen\n" \
  "v_mfma_f32_16x16x32_f16 a[108:111], v[56:59], v[92:95], a[108:111]\n" \
  "ds_read_b128 v[56:59], v25 offset:30720\n" \
  "v_mfma_f32_16x16x32_f16 a[124:127], v[60:63], v[92:95], a[124:127]\n" \
  "v_cvt_pk_f16_f32 v71, v185, v189\n" \
  "v_mfma_f32_16x16x32_f16 a[120:123], v[60:63], v[88:91], a[120:123]\n" \
  "v_cvt_pk_f16_f32 v75, v186, v190\n" \
  "s_waitcnt vmcnt(41)\n" \
  "v_mfma_f32_16x16x32_f16 a[116:119], v[60:63], v[84:87], a[116:119]\n" \
  "s_waitcnt lgkmcnt(5)\n" \
  "s_barrier\n" \
  "v_mfma_f32_16x16x32_f16 a[112:115], v[60:63], v[80:83], a[112:115]\n" \
  "ds_read_b128 v[60:63], v25 offset:31744\n" \
  "v_mfma_f32_16x16x32_f16 a[128:131], v[32:35], v[80:83], a[128:131]\n" \
  "buffer_load_dwordx4 v[180:183], v17, s[8:11], s46 offen\n" \
  "v_mfma_f32_16x16x32_f16 a[132:135], v[32:35], v[84:87], a[132:135]\n" \
  "v_cvt_pk_f16_f32 v79, v187, v191\n" \
  "v_mfma_f32_16x16x32_f16 a[136:139], v[32:35], v[88:91], a[136:139]\n" \
  "v_readlane_b32 s64, v9, s38\n" \
  "s_add_u32 s38, s38, 1\n" \
  "v_mfma_f32_16x16x32_f16 a[140:143], v[32:35], v[92:95], a[140:143]\n" \
  "ds_read_b128 v[32:35], v25 offset:32768\n" \
  "v_mfma_f32_16x16x32_f16 a[156:159], v[36:39], v[92:95], a[156:159]\n" \
  "v_bfe_u32 v27, s64, v7, 8\n" \
  "v_mfma_f32_16x16x32_f16 a[152:155], v[36:39], v[88:91], a[152:155]\n" \
  "buffer_load_dwordx4 v[184:187], v18, s[8:11], s46 offen\n" \
  "v_mfma_f32_16x16x32_f16 a[148:151], v[36:39], v[84:87], a[148:151]\n" \
  "s_waitcnt lgkmcnt(5)\n" \
  "s_lshl_b32 s42, s40, 16\n" \
  "v_mfma_f32_16x16x32_f16 a[144:147], v[36:39], v[80:83], a[144:147]\n" \
  "ds_read_b128 v[36:39], v25 offset:33792\n" \
  "v_mfma_f32_16x16x32_f16 a[160:163], v[40:43], v[80:83], a[160:163]\n" \
  "v_lshl_add_u32 v27, v27, 6, v8\n" \
  "v_mfma_f32_16x16x32_f16 a[164:167], v[40:43], v[84:87], a[164:167]\n" \
  "buffer_load_dword v28, v27, s[16:19], 0 offen sc0 sc1\n" \
  "v_mfma_f32_16x16x32_f16 a[168:171], v[40:43], v[88:91], a[168:171]\n" \
  "buffer_load_dwordx4 v[188:191], v19, s[8:11], s46 offen\n" \
  "v_mfma_f32_16x16x32_f16 a[172:175], v[40:43], v[92:95], a[172:175]\n" \
  "ds_read_b128 v[40:43], v25 offset:34816\n" \
  "v_mfma_f32_16x16x32_f16 a[188:191], v[44:47], v[92:95], a[188:191]\n" \
  "v_mfma_f32_16x16x32_f16 a[184:187], v[44:47], v[88:91], a[184:187]\n" \
  "v_mfma_f32_16x16x32_f16 a[180:183], v[44:47], v[84:87], a[180:183]\n" \
  "s_waitcnt lgkmcnt(5)\n" \
  "s_add_u32 m0, s36, 0x8000\n" \
  "v_mfma_f32_16x16x32_f16 a[176:179], v[44:47], v[80:83], a[176:179]\n" \
  "ds_read_b128 v[44:47], v25 offset:35840\n" \
  "v_mfma_f32_16x16x32_f16 a[192:195], v[48:51], v[80:83], a[192:195]\n" \
  "buffer_load_dwordx4 v24, s[12:15], s42 offen offset:0 sc0 sc1 lds\n" \
  "v_mfma_f32_16x16x32_f16 a[196:199], v[48:51], v[84:87], a[196:199]\n" \
  "v_mfma_f32_16x16x32_f16 a[200:203], v[48:51], v[88:91], a[200:203]\n" \
  "v_mfma_f32_16x16x32_f16 a[204:207], v[48:51], v[92:95], a[204:207]\n" \
  "ds_read_b128 v[48:51], v25 offset:36864\n" \
  "v_mfma_f32_16x16x32_f16 a[220:223], v[52:55], v[92:95], a[220:223]\n" \
  "v_mfma_f32_16x16x32_f16 a[216:219], v[52:55], v[88:91], a[216:219]\n" \
  "buffer_load_dwordx4 v24, s[12:15], s42 offen offset:1024 sc0 sc1 lds\n" \
  "v_mfma_f32_16x16x32_f16 a[212:215], v[52:55], v[84:87], a[212:215]\n" \
  "s_waitcnt lgkmcnt(5)\n" \
  "v_mfma_f32_16x16x32_f16 a[208:211], v[52:55], v[80:83], a[208:211]\n" \
  "ds_read_b128 v[52:55], v25 offset:37888\n" \
  "v_mfma_f32_16x16x32_f16 a[224:227], v[56:59], v[80:83], a[224:227]\n" \
  "v_mfma_f32_16x16x32_f16 a[228:231], v[56:59], v[84:87], a[228:231]\n" \
  "buffer_load_dwordx4 v24, s[12:15], s42 offen offset:2048 sc0 sc1 lds\n" \
  "v_mfma_f32_16x16x32_f16 a[232:235], v[56:59], v[88:91], a[232:235]\n" \
  "v_mfma_f32_16x16x32_f16 a[236:239], v[56:59], v[92:95], a[236:239]\n" \
  "ds_read_b128 v[56:59], v25 offset:38912\n" \
  "v_mfma_f32_16x16x32_f16 a[252:255], v[60:63], v[92:95], a[252:255]\n" \
  "v_mfma_f32_16x16x32_f16 a[248:251], v[60:63], v[88:91], a[248:251]\n" \
  "buffer_load_dwordx4 v24, s[12:15], s42 offen offset:3072 sc0 sc1 lds\n" \
  "s_cmp_le_i32 s56, 1\n" \
  "v_mfma_f32_16x16x32_f16 a[244:247], v[60:63], v[84:87], a[244:247]\n" \
  "s_waitcnt lgkmcnt(5)\n" \
  "s_cbranch_scc1 Lact1_%=\n" \
  "Lactb1_%=:\n" \
  "v_mfma_f32_16x16x32_f16 a[240:243], v[60:63], v[80:83], a[240:243]\n" \
  "ds_read_b128 v[60:63], v25 offset:39936\n" \
  "v_mfma_f32_16x16x32_f16 a[0:3], v[32:35], v[64:67], a[0:3]\n" \
  "s_waitcnt vmcnt(47)\n" \
  "v_cvt_pk_f16_f32 v80, v192, v196\n" \
  "v_mfma_f32_16x16x32_f16 a[4:7], v[32:35], v[68:71], a[4:7]\n" \
  "v_cvt_pk_f16_f32 v84, v193, v197\n" \
  "v_mfma_f32_16x16x32_f16 a[8:11], v[32:35], v[72:75], a[8:11]\n" \
  "v_cvt_pk_f16_f32 v88, v194, v198\n" \
  "s_bfe_u32 s40, s39, 0x80018\n" \
  "v_mfma_f32_16x16x32_f16 a[12:15], v[32:35], v[76:79], a[12:15]\n" \
  "ds_read_b128 v[32:35], v25 offset:40960\n" \
  "v_mfma_f32_16x16x32_f16 a[28:31], v[36:39], v[76:79], a[28:31]\n" \
  "v_cvt_pk_f16_f32 v92, v195, v199\n" \
  "s_lshl_b32 s41, s40, 20\n" \
  "v_mfma_f32_16x16x32_f16 a[24:27], v[36:39], v[72:75], a[24:27]\n" \
  "buffer_load_dwordx4 v[192:195], v16, s[8:11], s41 offen\n" \
  "v_mfma_f32_16x16x32_f16 a[20:23], v[36:39], v[68:71], a[20:23]\n" \
  "s_waitcnt lgkmcnt(5)\n" \
  "v_mfma_f32_16x16x32_f16 a[16:19], v[36:39], v[64:67], a[16:19]\n" \
  "ds_read_b128 v[36:39], v25 offset:41984\n" \
  "v_mfma_f32_16x16x32_f16 a[32:35], v[40:43], v[64:67], a[32:35]\n" \
  "s_waitcnt vmcnt(46)\n" \
  "v_cvt_pk_f16_f32 v81, v200, v204\n" \
  "v_mfma_f32_16x16x32_f16 a[36:39], v[40:43], v[68:71], a[36:39]\n" \
  "v_cvt_pk_f16_f32 v85, v201, v205\n" \
  "v_mfma_f32_16x16x32_f16 a[40:43], v[40:43], v[72:75], a[40:43]\n" \
  "buffer_load_dwordx4 v[196:199], v17, s[8:11], s41 offen\n" \
  "v_mfma_f32_16x16x32_f16 a[44:47], v[40:43], v[76:79], a[44:47]\n" \
  "ds_read_b128 v[40:43], v25 offset:43008\n" \
  "v_mfma_f32_16x16x32_f16 a[60:63], v[44:47], v[76:79], a[60:63]\n" \
  "v_cvt_pk_f16_f32 v89, v202, v206\n" \
  "v_mfma_f32_16x16x32_f16 a[56:59], v[44:47], v[72:75], a[56:59]\n" \
  "v_cvt_pk_f16_f32 v93, v203, v207\n" \
  "v_mfma_f32_16x16x32_f16 a[52:55], v[44:47], v[68:71], a[52:55]\n" \
  "s_waitcnt lgkmcnt(5)\n" \
  "v_mfma_f32_16x16x32_f16 a[48:51], v[44:47], v[64:67], a[48:51]\n" \
  "ds_read_b128 v[44:47], v25 offset:44032\n" \
  "v_mfma_f32_16x16x32_f16 a[64:67], v[48:51], v[64:67], a[64:67]\n" \
  "buffer_load_dwordx4 v[200:203], v18, s[8:11], s41 offen\n" \
  "v_mfma_f32_16x16x32_f16 a[68:71], v[48:51], v[68:71], a[68:71]\n" \
  "s_waitcnt vmcnt(46)\n" \
  "v_cvt_pk_f16_f32 v82, v208, v212\n" \
  "v_mfma_f32_16x16x32_f16 a[72:75], v[48:51], v[72:75], a[72:75]\n" \
  "v_cvt_pk_f16_f32 v86, v209, v213\n" \
  "v_mfma_f32_16x16x32_f16 a[76:79], v[48:51], v[76:79], a[76:79]\n" \
  "ds_read_b128 v[48:51], v25 offset:45056\n" \
  "v_mfma_f32_16x16x32_f16 a[92:95], v[52:55], v[76:79], a[92:95]\n" \
  "v_cvt_pk_f16_f32 v90, v210, v214\n" \
  "v_mfma_f32_16x16x32_f16 a[88:91], v[52:55], v[72:75], a[88:91]\n" \
  "buffer_load_dwordx4 v[204:207], v19, s[8:11], s41 offen\n" \
  "v_mfma_f32_16x16x32_f16 a[84:87], v[52:55], v[68:71], a[84:87]\n" \
  "s_waitcnt lgkmcnt(5)\n" \
  "s_add_u32 s46, s41, 0x80000\n" \
  "v_mfma_f32_16x16x32_f16 a[80:83], v[52:55], v[64:67], a[80:83]\n" \
  "ds_read_b128 v[52:55], v25 offset:46080\n" \
  "v_mfma_f32_16x16x32_f16 a[96:99], v[56:59], v[64:67], a[96:99]\n" \
  "v_cvt_pk_f16_f32 v94, v211, v215\n" \
  "v_mfma_f32_16x16x32_f16 a[100:103], v[56:59], v[68:71], a[100:103]\n" \
  "s_waitcnt vmcnt(45)\n" \
  "v_cvt_pk_f16_f32 v83, v216, v220\n" \
  "v_mfma_f32_16x16x32_f16 a[104:107], v[56:59], v[72:75], a[104:107]\n" \
  "buffer_load_dwordx4 v[208:211], v16, s[8:11], s46 offen\n" \
  "v_mfma_f32_16x16x32_f16 a[108:111], v[56:59], v[76:79], a[108:111]\n" \
  "ds_read_b128 v[56:59], v25 offset:47104\n" \
  "v_mfma_f32_16x16x32_f16 a[124:127], v[60:63], v[76:79], a[124:127]\n" \
  "v_cvt_pk_f16_f32 v87, v217, v221\n" \
  "v_mfma_f32_16x16x32_f16 a[120:123], v[60:63], v[72:75], a[120:123]\n" \
  "v_cvt_pk_f16_f32 v91, v218, v222\n" \
  "s_waitcnt vmcnt(42)\n" \
  "v_mfma_f32_16x16x32_f16 a[116:119], v[60:63], v[68:71], a[116:119]\n" \
  "s_waitcnt lgkmcnt(5)\n" \
  "s_barrier\n" \
  "v_mfma_f32_16x16x32_f16 a[112:115], v[60:63], v[64:67], a[112:115]\n" \
  "ds_read_b128 v[60:63], v25 offset:48128\n" \
  "v_mfma_f32_16x16x32_f16 a[128:131], v[32:35], v[64:67], a[128:131]\n" \
  "buffer_load_dwordx4 v[212:215], v17, s[8:11], s46 offen\n" \
  "v_mfma_f32_16x16x32_f16 a[132:135], v[32:35], v[68:71], a[132:135]\n" \
  "v_cvt_pk_f16_f32 v95, v219, v223\n" \
  "v_mfma_f32_16x16x32_f16 a[136:139], v[32:35], v[72:75], a[136:139]\n" \
  "v_mfma_f32_16x16x32_f16 a[140:143], v[32:35], v[76:79], a[140:143]\n" \
  "ds_read_b128 v[32:35], v25 offset:49152\n" \
  "v_mfma_f32_16x16x32_f16 a[156:159], v[36:39], v[76:79], a[156:159]\n" \
  "v_mfma_f32_16x16x32_f16 a[152:155], v[36:39], v[72:75], a[152:155]\n" \
  "buffer_load_dwordx4 v[216:219], v18, s[8:11], s46 offen\n" \
  "v_mfma_f32_16x16x32_f16 a[148:151], v[36:39], v[68:71], a[148:151]\n" \
  "s_waitcnt lgkmcnt(5)\n" \
  "s_lshl_b32 s42, s40, 16\n" \
  "v_mfma_f32_16x16x32_f16 a[144:147], v[36:39], v[64:67], a[144:147]\n" \
  "ds_read_b128 v[36:39], v25 offset:50176\n" \
  "v_mfma_f32_16x16x32_f16 a[160:163], v[40:43], v[64:67], a[160:163]\n" \
  "v_mfma_f32_16x16x32_f16 a[164:167], v[40:43], v[68:71], a[164:167]\n" \
  "v_mfma_f32_16x16x32_f16 a[168:171], v[40:43], v[72:75], a[168:171]\n" \
  "buffer_load_dwordx4 v[220:223], v19, s[8:11], s46 offen\n" \
  "v_mfma_f32_16x16x32_f16 a[172:175], v[40:43], v[76:79], a[172:175]\n" \
  "ds_read_b128 v[40:43], v25 offset:51200\n" \
  "v_mfma_f32_16x16x32_f16 a[188:191], v[44:47], v[76:79], a[188:191]\n" \
  "v_mfma_f32_16x16x32_f16 a[184:187], v[44:47], v[72:75], a[184:187]\n" \
  "v_mfma_f32_16x16x32_f16 a[180:183], v[44:47], v[68:71], a[180:183]\n" \
  "s_waitcnt lgkmcnt(5)\n" \
  "s_add_u32 m0, s36, 0xc000\n" \
  "v_mfma_f32_16x16x32_f16 a[176:179], v[44:47], v[64:67], a[176:179]\n" \
  "ds_read_b128 v[44:47], v25 offset:52224\n" \
  "v_mfma_f32_16x16x32_f16 a[192:195], v[48:51], v[64:67], a[192:195]\n" \
  "buffer_load_dwordx4 v24, s[12:15], s42 offen offset:0 sc0 sc1 lds\n" \
  "v_mfma_f32_16x16x32_f16 a[196:199], v[48:51], v[68:71], a[196:199]\n" \
  "v_mfma_f32_16x16x32_f16 a[200:203], v[48:51], v[72:75], a[200:203]\n" \
  "v_mfma_f32_16x16x32_f16 a[204:207], v[48:51], v[76:79], a[204:207]\n" \
  "ds_read_b128 v[48:51], v25 offset:53248\n" \
  "v_mfma_f32_16x16x32_f16 a[220:223], v[52:55], v[76:79], a[220:223]\n" \
  "v_mfma_f32_16x16x32_f16 a[216:219], v[52:55], v[72:75], a[216:219]\n" \
  "buffer_load_dwordx4 v24, s[12:15], s42 offen offset:1024 sc0 sc1 lds\n" \
  "v_mfma_f32_16x16x32_f16 a[212:215], v[52:55], v[68:71], a[212:215]\n" \
  "s_waitcnt lgkmcnt(5)\n" \
  "v_mfma_f32_16x16x32_f16 a[208:211], v[52:55], v[64:67], a[208:211]\n" \
  "ds_read_b128 v[52:55], v25 offset:54272\n" \
  "v_mfma_f32_16x16x32_f16 a[224:227], v[56:59], v[64:67], a[224:227]\n" \
  "v_mfma_f32_16x16x32_f16 a[228:231], v[56:59], v[68:71], a[228:231]\n" \
  "buffer_load_dwordx4 v24, s[12:15], s42 offen offset:2048 sc0 sc1 lds\n" \
  "v_mfma_f32_16x16x32_f16 a[232:235], v[56:59], v[72:75], a[232:235]\n" \
  "v_mfma_f32_16x16x32_f16 a[236:239], v[56:59], v[76:79], a[236:239]\n" \
  "ds_read_b128 v[56:59], v25 offset:55296\n" \
  "v_mfma_f32_16x16x32_f16 a[252:255], v[60:63], v[76:79], a[252:255]\n" \
  "v_mfma_f32_16x16x32_f16 a[248:251], v[60:63], v[72:75], a[248:251]\n" \
  "buffer_load_dwordx4 v24, s[12:15], s42 offen offset:3072 sc0 sc1 lds\n" \
  "s_cmp_le_i32 s56, 2\n" \
  "v_mfma_f32_16x16x32_f16 a[244:247], v[60:63], v[68:71], a[244:247]\n" \
  "s_waitcnt lgkmcnt(5)\n" \
  "s_cbranch_scc1 Lact2_%=\n" \
  "Lactb2_%=:\n" \
  "v_mfma_f32_16x16x32_f16 a[240:243], v[60:63], v[64:67], a[240:243]\n" \
  "ds_read_b128 v[60:63], v25 offset:56320\n" \
  "v_mfma_f32_16x16x32_f16 a[0:3], v[32:35], v[80:83], a[0:3]\n" \
  "s_waitcnt vmcnt(47)\n" \
  "v_cvt_pk_f16_f32 v64, v96, v100\n" \
  "v_mfma_f32_16x16x32_f16 a[4:7], v[32:35], v[84:87], a[4:7]\n" \
  "v_cvt_pk_f16_f32 v68, v97, v101\n" \
  "v_mfma_f32_16x16x32_f16 a[8:11], v[32:35], v[88:91], a[8:11]\n" \
  "v_cvt_pk_f16_f32 v72, v98, v102\n" \
  "s_bfe_u32 s40, s64, 0x80000\n" \
  "v_mfma_f32_16x16x32_f16 a[12:15], v[32:35], v[92:95], a[12:15]\n" \
  "ds_read_b128 v[32:35], v25 offset:57344\n" \
  "v_mfma_f32_16x16x32_f16 a[28:31], v[36:39], v[92:95], a[28:31]\n" \
  "v_cvt_pk_f16_f32 v76, v99, v103\n" \
  "s_lshl_b32 s41, s40, 20\n" \
  "v_mfma_f32_16x16x32_f16 a[24:27], v[36:39], v[88:91], a[24:27]\n" \
  "buffer_load_dwordx4 v[96:99], v16, s[8:11], s41 offen\n" \
  "v_mfma_f32_16x16x32_f16 a[20:23], v[36:39], v[84:87], a[20:23]\n" \
  "s_waitcnt lgkmcnt(5)\n" \
  "v_mfma_f32_16x16x32_f16 a[16:19], v[36:39], v[80:83], a[16:19]\n" \
  "ds_read_b128 v[36:39], v25 offset:58368\n" \
  "v_mfma_f32_16x16x32_f16 a[32:35], v[40:43], v[80:83], a[32:35]\n" \
  "s_waitcnt vmcnt(46)\n" \
  "v_cvt_pk_f16_f32 v65, v104, v108\n" \
  "v_mfma_f32_16x16x32_f16 a[36:39], v[40:43], v[84:87], a[36:39]\n" \
  "v_cvt_pk_f16_f32 v69, v105, v109\n" \
  "v_mfma_f32_16x16x32_f16 a[40:43], v[40:43], v[88:91], a[40:43]\n" \
  "buffer_load_dwordx4 v[100:103], v17, s[8:11], s41 offen\n" \
  "v_mfma_f32_16x16x32_f16 a[44:47], v[40:43], v[92:95], a[44:47]\n" \
  "ds_read_b128 v[40:43], v25 offset:59392\n" \
  "v_mfma_f32_16x16x32_f16 a[60:63], v[44:47], v[92:95], a[60:63]\n" \
  "v_cvt_pk_f16_f32 v73, v106, v110\n" \
  "v_mfma_f32_16x16x32_f16 a[56:59], v[44:47], v[88:91], a[56:59]\n" \
  "v_cvt_pk_f16_f32 v77, v107, v111\n" \
  "v_mfma_f32_16x16x32_f16 a[52:55], v[44:47], v[84:87], a[52:55]\n" \
  "s_waitcnt lgkmcnt(5)\n" \
  "s_mov_b32 s39, s64\n" \
  "v_mfma_f32_16x16x32_f16 a[48:51], v[44:47], v[80:83], a[48:51]\n" \
  "ds_read_b128 v[44:47], v25 offset:60416\n" \
  "v_mfma_f32_16x16x32_f16 a[64:67], v[48:51], v[80:83], a[64:67]\n" \
  "buffer_load_dwordx4 v[104:107], v18, s[8:11], s41 offen\n" \
  "v_mfma_f32_16x16x32_f16 a[68:71], v[48:51], v[84:87], a[68:71]\n" \
  "s_waitcnt vmcnt(46)\n" \
  "v_cvt_pk_f16_f32 v66, v112, v116\n" \
  "v_mfma_f32_16x16x32_f16 a[72:75], v[48:51], v[88:91], a[72:75]\n" \
  "v_cvt_pk_f16_f32 v70, v113, v117\n" \
  "v_mfma_f32_16x16x32_f16 a[76:79], v[48:51], v[92:95], a[76:79]\n" \
  "ds_read_b128 v[48:51], v25 offset:61440\n" \
  "v_mfma_f32_16x16x32_f16 a[92:95], v[52:55], v[92:95], a[92:95]\n" \
  "v_cvt_pk_f16_f32 v74, v114, v118\n" \
  "v_mfma_f32_16x16x32_f16 a[88:91], v[52:55], v[88:91], a[88:91]\n" \
  "buffer_load_dwordx4 v[108:111], v19, s[8:11], s41 offen\n" \
  "v_mfma_f32_16x16x32_f16 a[84:87], v[52:55], v[84:87], a[84:87]\n" \
  "s_waitcnt lgkmcnt(5)\n" \
  "s_add_u32 s46, s41, 0x80000\n" \
  "v_mfma_f32_16x16x32_f16 a[80:83], v[52:55], v[80:83], a[80:83]\n" \
  "ds_read_b128 v[52:55], v25 offset:62464\n" \
  "v_mfma_f32_16x16x32_f16 a[96:99], v[56:59], v[80:83], a[96:99]\n" \
  "v_cvt_pk_f16_f32 v78, v115, v119\n" \
  "v_mfma_f32_16x16x32_f16 a[100:103], v[56:59], v[84:87], a[100:103]\n" \
  "s_waitcnt vmcnt(45)\n" \
  "v_cvt_pk_f16_f32 v67, v120, v124\n" \
  "v_mfma_f32_16x16x32_f16 a[104:107], v[56:59], v[88:91], a[104:107]\n" \
  "buffer_load_dwordx4 v[112:115], v16, s[8:11], s46 offen\n" \
  "v_mfma_f32_16x16x32_f16 a[108:111], v[56:59], v[92:95], a[108:111]\n" \
  "ds_read_b128 v[56:59], v25 offset:63488\n" \
  "v_mfma_f32_16x16x32_f16 a[124:127], v[60:63], v[92:95], a[124:127]\n" \
  "v_cvt_pk_f16_f32 v71, v121, v125\n" \
  "v_mfma_f32_16x16x32_f16 a[120:123], v[60:63], v[88:91], a[120:123]\n" \
  "v_cvt_pk_f16_f32 v75, v122, v126\n" \
  "s_waitcnt vmcnt(42)\n" \
  "v_mfma_f32_16x16x32_f16 a[116:119], v[60:63], v[84:87], a[116:119]\n" \
  "s_waitcnt lgkmcnt(5)\n" \
  "s_barrier\n" \
  "v_mfma_f32_16x16x32_f16 a[112:115], v[60:63], v[80:83], a[112:115]\n" \
  "ds_read_b128 v[60:63], v25 offset:64512\n" \
  "v_mfma_f32_16x16x32_f16 a[128:131], v[32:35], v[80:83], a[128:131]\n" \
  "buffer_load_dwordx4 v[116:119], v17, s[8:11], s46 offen\n" \
  "v_mfma_f32_16x16x32_f16 a[132:135], v[32:35], v[84:87], a[132:135]\n" \
  "v_cvt_pk_f16_f32 v79, v123, v127\n" \
  "v_mfma_f32_16x16x32_f16 a[136:139], v[32:35], v[88:91], a[136:139]\n" \
  "v_xor_b32 v25, 0x10000, v25\n" \
  "v_mfma_f32_16x16x32_f16 a[140:143], v[32:35], v[92:95], a[140:143]\n" \
  "ds_read_b128 v[32:35], v25 offset:0\n" \
  "v_mfma_f32_16x16x32_f16 a[156:159], v[36:39], v[92:95], a[156:159]\n" \
  "v_mfma_f32_16x16x32_f16 a[152:155], v[36:39], v[88:91], a[152:155]\n" \
  "buffer_load_dwordx4 v[120:123], v18, s[8:11], s46 offen\n" \
  "v_mfma_f32_16x16x32_f16 a[148:151], v[36:39], v[84:87], a[148:151]\n" \
  "s_waitcnt lgkmcnt(5)\n" \
  "s_lshl_b32 s42, s40, 16\n" \
  "v_mfma_f32_16x16x32_f16 a[144:147], v[36:39], v[80:83], a[144:147]\n" \
  "ds_read_b128 v[36:39], v25 offset:1024\n" \
  "v_mfma_f32_16x16x32_f16 a[160:163], v[40:43], v[80:83], a[160:163]\n" \
  "s_xor_b32 s36, s36, 0x10000\n" \
  "v_mfma_f32_16x16x32_f16 a[164:167], v[40:43], v[84:87], a[164:167]\n" \
  "v_mfma_f32_16x16x32_f16 a[168:171], v[40:43], v[88:91], a[168:171]\n" \
  "buffer_load_dwordx4 v[124:127], v19, s[8:11], s46 offen\n" \
  "v_mfma_f32_16x16x32_f16 a[172:175], v[40:43], v[92:95], a[172:175]\n" \
  "ds_read_b128 v[40:43], v25 offset:2048\n" \
  "v_mfma_f32_16x16x32_f16 a[188:191], v[44:47], v[92:95], a[188:191]\n" \
  "s_waitcnt vmcnt(25)\n" \
  "v_cmp_le_u32 vcc, s43, v28\n" \
  "v_mfma_f32_16x16x32_f16 a[184:187], v[44:47], v[88:91], a[184:187]\n" \
  "s_cmp_eq_u64 vcc, -1\n" \
  "s_cbranch_scc0 Lstub0_%=\n" \
  "Lback0_%=:\n" \
  "v_mfma_f32_16x16x32_f16 a[180:183], v[44:47], v[84:87], a[180:183]\n" \
  "s_waitcnt lgkmcnt(5)\n" \
  "s_add_u32 m0, s36, 0x0\n" \
  "v_mfma_f32_16x16x32_f16 a[176:179], v[44:47], v[80:83], a[176:179]\n" \
  "ds_read_b128 v[44:47], v25 offset:3072\n" \
  "v_mfma_f32_16x16x32_f16 a[192:195], v[48:51], v[80:83], a[192:195]\n" \
  "buffer_load_dwordx4 v24, s[12:15], s42 offen offset:0 sc0 sc1 lds\n" \
  "v_mfma_f32_16x16x32_f16 a[196:199], v[48:51], v[84:87], a[196:199]\n" \
  "v_mfma_f32_16x16x32_f16 a[200:203], v[48:51], v[88:91], a[200:203]\n" \
  "v_mfma_f32_16x16x32_f16 a[204:207], v[48:51], v[92:95], a[204:207]\n" \
  "ds_read_b128 v[48:51], v25 offset:4096\n" \
  "v_mfma_f32_16x16x32_f16 a[220:223], v[52:55], v[92:95], a[220:223]\n" \
  "v_mfma_f32_16x16x32_f16 a[216:219], v[52:55], v[88:91], a[216:219]\n" \
  "buffer_load_dwordx4 v24, s[12:15], s42 offen offset:1024 sc0 sc1 lds\n" \
  "v_mfma_f32_16x16x32_f16 a[212:215], v[52:55], v[84:87], a[212:215]\n" \
  "s_waitcnt lgkmcnt(5)\n" \
  "v_mfma_f32_16x16x32_f16 a[208:211], v[52:55], v[80:83], a[208:211]\n" \
  "ds_read_b128 v[52:55], v25 offset:5120\n" \
  "v_mfma_f32_16x16x32_f16 a[224:227], v[56:59], v[80:83], a[224:227]\n" \
  "v_mfma_f32_16x16x32_f16 a[228:231], v[56:59], v[84:87], a[228:231]\n" \
  "buffer_load_dwordx4 v24, s[12:15], s42 offen offset:2048 sc0 sc1 lds\n" \
  "v_mfma_f32_16x16x32_f16 a[232:235], v[56:59], v[88:91], a[232:235]\n" \
  "v_mfma_f32_16x16x32_f16 a[236:239], v[56:59], v[92:95], a[236:239]\n" \
  "ds_read_b128 v[56:59], v25 offset:6144\n" \
  "v_mfma_f32_16x16x32_f16 a[252:255], v[60:63], v[92:95], a[252:255]\n" \
  "v_mfma_f32_16x16x32_f16 a[248:251], v[60:63], v[88:91], a[248:251]\n" \
  "buffer_load_dwordx4 v24, s[12:15], s42 offen offset:3072 sc0 sc1 lds\n" \
  "s_cmp_le_i32 s56, 3\n" \
  "v_mfma_f32_16x16x32_f16 a[244:247], v[60:63], v[84:87], a[244:247]\n" \
  "s_waitcnt lgkmcnt(5)\n" \
  "s_cbranch_scc1 Lact3_%=\n" \
  "Lactb3_%=:\n" \
  "s_branch Lloop_%=\n" \
  "Lact0_%=:\n" \
  "s_call_b64 s[54:55], Laction_%=\n" \
  "s_cbranch_scc0 Lactb0_%=\n" \
  "s_branch Ltail1_%=\n" \
  "Lact1_%=:\n" \
  "s_call_b64 s[54:55], Laction_%=\n" \
  "s_cbranch_scc0 Lactb1_%=\n" \
  "s_branch Ltail2_%=\n" \
  "Lact2_%=:\n" \
  "s_call_b64 s[54:55], Laction_%=\n" \
  "s_cbranch_scc0 Lactb2_%=\n" \
  "s_branch Ltail3_%=\n" \
  "Lact3_%=:\n" \
  "s_call_b64 s[54:55], Laction_%=\n" \
  "s_cbranch_scc0 Lactb3_%=\n" \
  "s_branch Ltail0_%=\n" \
  "Lpact0_%=:\n" \
  "s_waitcnt vmcnt(18)\n" \
  "s_call_b64 s[54:55], Laction_%=\n" \
  "s_branch Lpactb0_%=\n" \
  "Lpact1_%=:\n" \
  "s_waitcnt vmcnt(37)\n" \
  "s_call_b64 s[54:55], Laction_%=\n" \
  "s_branch Lpactb1_%=\n" \
  "Lpact2_%=:\n" \
  "s_waitcnt vmcnt(55)\n" \
  "s_call_b64 s[54:55], Laction_%=\n" \
  "s_branch Lpactb2_%=\n" \
  "Lpact3_%=:\n" \
  "s_waitcnt vmcnt(63)\n" \
  "s_call_b64 s[54:55], Laction_%=\n" \
  "s_branch Lpactb3_%=\n" \
  "Laction_%=:\n" \
  "v_readlane_b32 s62, v4, s44\n" \
  "v_readlane_b32 s60, v5, s44\n" \
  "v_readlane_b32 s61, v6, s44\n" \
  "v_readlane_b32 s45, v3, s44\n" \
  "v_readlane_b32 s26, v11, s44\n" \
  "s_add_u32 s44, s44, 1\n" \
  "s_cmp_lt_i32 s63, 0\n" \
  "s_cbranch_scc1 3f\n" \
  "buffer_store_dword v15, v2, s[28:31], s47 offen sc0 sc1\n" \
  "3:\n" \
  "s_add_u32 s56, s56, s26\n" \
  "s_mov_b32 s63, s62\n" \
  "s_cmp_lt_i32 s62, -1\n" \
  "s_cbranch_scc1 Lspecial_%=\n" \
  "s_cmp_lt_i32 s62, 0\n" \
  "s_cbranch_scc1 3f\n" \
  "s_lshl_b32 s26, s62, 14\n" \
  "s_add_u32 s24, s60, s26\n" \
  "s_addc_u32 s25, s61, 0\n" \
  "s_mov_b32 s28, s60\n" \
  "s_mov_b32 s29, s61\n" \
  "s_lshl_b32 s47, s62, 4\n" \
  "s_waitcnt vmcnt(44)\n" \
  "v_xor_b32 v30, 64, v26\n" \
  "ds_read_b128 v[224:227], v26 offset:0\n" \
  "ds_read_b128 v[228:231], v30 offset:0\n" \
  "ds_read_b128 v[232:235], v26 offset:2048\n" \
  "ds_read_b128 v[236:239], v30 offset:2048\n" \
  "ds_read_b128 v[240:243], v26 offset:4096\n" \
  "ds_read_b128 v[244:247], v30 offset:4096\n" \
  "ds_read_b128 v[248:251], v26 offset:6144\n" \
  "ds_read_b128 v[252:255], v30 offset:6144\n" \
  "s_waitcnt lgkmcnt(6)\n" \
  "v_cvt_pk_f16_f32 v224, v224, v225\n" \
  "v_cvt_pk_f16_f32 v225, v226, v227\n" \
  "v_cvt_pk_f16_f32 v226, v228, v229\n" \
  "v_cvt_pk_f16_f32 v227, v230, v231\n" \
  "global_store_dwordx4 v1, v[224:227], s[24:25] offset:0 sc0 sc1\n" \
  "s_waitcnt lgkmcnt(4)\n" \
  "v_cvt_pk_f16_f32 v232, v232, v233\n" \
  "v_cvt_pk_f16_f32 v233, v234, v235\n" \
  "v_cvt_pk_f16_f32 v234, v236, v237\n" \
  "v_cvt_pk_f16_f32 v235, v238, v239\n" \
  "global_store_dwordx4 v1, v[232:235], s[24:25] offset:1024 sc0 sc1\n" \
  "s_waitcnt lgkmcnt(2)\n" \
  "v_cvt_pk_f16_f32 v240, v240, v241\n" \
  "v_cvt_pk_f16_f32 v241, v242, v243\n" \
  "v_cvt_pk_f16_f32 v242, v244, v245\n" \
  "v_cvt_pk_f16_f32 v243, v246, v247\n" \
  "global_store_dwordx4 v1, v[240:243], s[24:25] offset:2048 sc0 sc1\n" \
  "s_waitcnt lgkmcnt(0)\n" \
  "v_cvt_pk_f16_f32 v248, v248, v249\n" \
  "v_cvt_pk_f16_f32 v249, v250, v251\n" \
  "v_cvt_pk_f16_f32 v250, v252, v253\n" \
  "v_cvt_pk_f16_f32 v251, v254, v255\n" \
  "global_store_dwordx4 v1, v[248:251], s[24:25] offset:3072 sc0 sc1\n" \
  "3:\n" \
  "s_cmp_lt_i32 s45, 0\n" \
  "s_cbranch_scc1 Lactret_%=\n" \
  "s_add_u32 m0, s65, 0x0\n" \
  "s_nop 0\n" \
  "buffer_load_dwordx4 v0, s[20:23], s45 offen nt lds\n" \
  "s_add_u32 m0, s65, 0x400\n" \
  "s_add_u32 s45, s45, 0x8000\n" \
  "buffer_load_dwordx4 v0, s[20:23], s45 offen nt lds\n" \
  "s_add_u32 m0, s65, 0x800\n" \
  "s_add_u32 s45, s45, 0x8000\n" \
  "buffer_load_dwordx4 v0, s[20:23], s45 offen nt lds\n" \
  "s_add_u32 m0, s65, 0xc00\n" \
  "s_add_u32 s45, s45, 0x8000\n" \
  "buffer_load_dwordx4 v0, s[20:23], s45 offen nt lds\n" \
  "s_add_u32 m0, s65, 0x1000\n" \
  "s_add_u32 s45, s45, 0x8000\n" \
  "buffer_load_dwordx4 v0, s[20:23], s45 offen nt lds\n" \
  "s_add_u32 m0, s65, 0x1400\n" \
  "s_add_u32 s45, s45, 0x8000\n" \
  "buffer_load_dwordx4 v0, s[20:23], s45 offen nt lds\n" \
  "s_add_u32 m0, s65, 0x1800\n" \
  "s_add_u32 s45, s45, 0x8000\n" \
  "buffer_load_dwordx4 v0, s[20:23], s45 offen nt lds\n" \
  "s_add_u32 m0, s65, 0x1c00\n" \
  "s_add_u32 s45, s45, 0x8000\n" \
  "buffer_load_dwordx4 v0, s[20:23], s45 offen nt lds\n" \
  "Lactret_%=:\n" \
  "s_cmp_eq_u32 0, 1\n" \
  "s_setpc_b64 s[54:55]\n" \
  "Lspecial_%=:\n" \
  "s_cmp_eq_i32 s62, -3\n" \
  "s_cbranch_scc1 Ldecide_%=\n" \
  "s_cmp_eq_i32 s62, -2\n" \
  "s_cbranch_scc1 Lts_%=\n" \
  "s_cmp_eq_u32 0, 0\n" \
  "s_setpc_b64 s[54:55]\n" \
  "Lts_%=:\n" \
  "s_memrealtime s[52:53]\n" \
  "v_mov_b32 v31, v15\n" \
  "s_waitcnt lgkmcnt(0)\n" \
  "v_mov_b32 v30, s52\n" \
  "buffer_store_dwordx2 v[30:31], v22, s[16:19], s57 offen offset:8 sc1\n" \
  "s_branch Lactret_%=\n" \
  "Ldecide_%=:\n" \
  "s_memrealtime s[50:51]\n" \
  "s_mov_b32 s49, 0x8000\n" \
  "Ldec1_%=:\n" \
  "buffer_load_dwordx2 v[30:31], v23, s[16:19], 0 offen sc1\n" \
  "buffer_load_dword v29, off, s[16:19], s49 sc1\n" \
  "s_waitcnt vmcnt(0) lgkmcnt(0)\n" \
  "v_cmp_le_u32 vcc, s43, v31\n" \
  "s_cmp_eq_u64 vcc, -1\n" \
  "s_cbranch_scc1 Ldec3_%=\n" \
  "v_cmp_eq_u32 vcc, s43, v29\n" \
  "s_cmp_lg_u64 vcc, 0\n" \
  "s_cbranch_scc1 Ldec3_%=\n" \
  "s_memrealtime s[52:53]\n" \
  "s_waitcnt lgkmcnt(0)\n" \
  "s_sub_u32 s52, s52, s50\n" \
  "s_cmp_gt_u32 s52, 0x2faf080\n" \
  "s_cbranch_scc1 Ldec2_%=\n" \
  "s_sleep 1\n" \
  "s_branch Ldec1_%=\n" \
  "Ldec2_%=:\n" \
  "buffer_store_dword v15, v14, s[16:19], s49 offen sc1\n" \
  "v_mov_b32 v29, v15\n" \
  "s_waitcnt vmcnt(0)\n" \
  "Ldec3_%=:\n" \
  "v_readlane_b32 s52, v30, 0\n" \
  "v_readlane_b32 s53, v30, 1\n" \
  "s_nop 1\n" \
  "s_sub_u32 s52, s53, s52\n" \
  "s_min_i32 s52, s52, 0x4000\n" \
  "s_max_i32 s52, s52, 0xffffc000\n" \
  "s_mul_i32 s52, s52, 596\n" \
  "s_add_i32 s52, s52, 0x8000\n" \
  "s_ashr_i32 s52, s52, 16\n" \
  "s_min_i32 s52, s52, 8\n" \
  "s_max_i32 s52, s52, -8\n" \
  "s_mul_i32 s52, s52, s37\n" \
  "s_add_i32 s56, s56, s52\n" \
  "s_branch Lactret_%=\n" \
  "Lstub0_%=:\n" \
  "s_call_b64 s[54:55], Lpoll_%=\n" \
  "s_branch Lback0_%=\n" \
  "Lstub2_%=:\n" \
  "s_call_b64 s[54:55], Lpoll_%=\n" \
  "s_branch Lback2_%=\n" \
  "Lstub3_%=:\n" \
  "s_call_b64 s[54:55], Lpoll_%=\n" \
  "s_branch Lback3_%=\n" \
  "Lstub4_%=:\n" \
  "s_call_b64 s[54:55], Lpoll_%=\n" \
  "s_branch Lback4_%=\n" \
  "Lstub9_%=:\n" \
  "s_call_b64 s[54:55], Lpoll_%=\n" \
  "s_branch Lback9_%=\n" \
  "Lpoll_%=:\n" \
  "s_memrealtime s[50:51]\n" \
  "s_mov_b32 s49, 0x8000\n" \
  "Lpoll1_%=:\n" \
  "buffer_load_dword v28, v27, s[16:19], 0 offen sc0 sc1\n" \
  "buffer_load_dword v29, off, s[16:19], s49 sc1\n" \
  "s_waitcnt vmcnt(0) lgkmcnt(0)\n" \
  "v_cmp_le_u32 vcc, s43, v28\n" \
  "s_cmp_eq_u64 vcc, -1\n" \
  "s_cbranch_scc1 Lpoll3_%=\n" \
  "v_cmp_eq_u32 vcc, s43, v29\n" \
  "s_cmp_lg_u64 vcc, 0\n" \
  "s_cbranch_scc1 Lpoll3_%=\n" \
  "s_memrealtime s[52:53]\n" \
  "s_waitcnt lgkmcnt(0)\n" \
  "s_sub_u32 s52, s52, s50\n" \
  "s_cmp_gt_u32 s52, 0x2faf080\n" \
  "s_cbranch_scc1 Lpoll2_%=\n" \
  "s_sleep 1\n" \
  "s_branch Lpoll1_%=\n" \
  "Lpoll2_%=:\n" \
  "buffer_store_dword v15, v14, s[16:19], s49 offen sc1\n" \
  "v_mov_b32 v29, v15\n" \
  "s_waitcnt vmcnt(0)\n" \
  "Lpoll3_%=:\n" \
  "s_setpc_b64 s[54:55]\n" \
  ".p2align 6\n" \
  "Ltail1_%=:\n" \
  "v_mfma_f32_16x16x32_f16 a[240:243], v[60:63], v[64:67], a[240:243]\n" \
  "ds_read_b128 v[60:63], v25 offset:23552\n" \
  "v_mfma_f32_16x16x32_f16 a[0:3], v[32:35], v[80:83], a[0:3]\n" \
  "v_xor_b32 v26, 0x10000, v25\n" \
  "s_waitcnt vmcnt(47)\n" \
  "s_waitcnt vmcnt(36)\n" \
  "v_mfma_f32_16x16x32_f16 a[4:7], v[32:35], v[84:87], a[4:7]\n" \
  "v_cvt_pk_f16_f32 v64, v160, v164\n" \
  "v_cvt_pk_f16_f32 v68, v161, v165\n" \
  "s_barrier\n" \
  "v_mfma_f32_16x16x32_f16 a[8:11], v[32:35], v[88:91], a[8:11]\n" \
  "v_cvt_pk_f16_f32 v72, v162, v166\n" \
  "v_cvt_pk_f16_f32 v76, v163, v167\n" \
  "v_mfma_f32_16x16x32_f16 a[12:15], v[32:35], v[92:95], a[12:15]\n" \
  "ds_read_b128 v[32:35], v25 offset:32768\n" \
  "s_waitcnt vmcnt(45)\n" \
  "v_mfma_f32_16x16x32_f16 a[28:31], v[36:39], v[92:95], a[28:31]\n" \
  "v_cvt_pk_f16_f32 v65, v168, v172\n" \
  "v_cvt_pk_f16_f32 v69, v169, v173\n" \
  "v_mfma_f32_16x16x32_f16 a[24:27], v[36:39], v[88:91], a[24:27]\n" \
  "v_cvt_pk_f16_f32 v73, v170, v174\n" \
  "v_cvt_pk_f16_f32 v77, v171, v175\n" \
  "v_mfma_f32_16x16x32_f16 a[20:23], v[36:39], v[84:87], a[20:23]\n" \
  "s_waitcnt lgkmcnt(5)\n" \
  "s_bfe_u32 s40, s39, 0x80010\n" \
  "s_lshl_b32 s42, s40, 16\n" \
  "v_mfma_f32_16x16x32_f16 a[16:19], v[36:39], v[80:83], a[16:19]\n" \
  "ds_read_b128 v[36:39], v25 offset:33792\n" \
  "s_add_u32 m0, s36, 0x8000\n" \
  "v_mfma_f32_16x16x32_f16 a[32:35], v[40:43], v[80:83], a[32:35]\n" \
  "buffer_load_dwordx4 v24, s[12:15], s42 offen offset:0 sc0 sc1 lds\n" \
  "s_waitcnt vmcnt(44)\n" \
  "v_mfma_f32_16x16x32_f16 a[36:39], v[40:43], v[84:87], a[36:39]\n" \
  "v_cvt_pk_f16_f32 v66, v176, v180\n" \
  "v_cvt_pk_f16_f32 v70, v177, v181\n" \
  "v_mfma_f32_16x16x32_f16 a[40:43], v[40:43], v[88:91], a[40:43]\n" \
  "v_cvt_pk_f16_f32 v74, v178, v182\n" \
  "v_cvt_pk_f16_f32 v78, v179, v183\n" \
  "v_mfma_f32_16x16x32_f16 a[44:47], v[40:43], v[92:95], a[44:47]\n" \
  "ds_read_b128 v[40:43], v25 offset:34816\n" \
  "v_mfma_f32_16x16x32_f16 a[60:63], v[44:47], v[92:95], a[60:63]\n" \
  "buffer_load_dwordx4 v24, s[12:15], s42 offen offset:1024 sc0 sc1 lds\n" \
  "s_waitcnt vmcnt(42)\n" \
  "v_mfma_f32_16x16x32_f16 a[56:59], v[44:47], v[88:91], a[56:59]\n" \
  "v_cvt_pk_f16_f32 v67, v184, v188\n" \
  "v_cvt_pk_f16_f32 v71, v185, v189\n" \
  "v_mfma_f32_16x16x32_f16 a[52:55], v[44:47], v[84:87], a[52:55]\n" \
  "s_waitcnt lgkmcnt(5)\n" \
  "v_mfma_f32_16x16x32_f16 a[48:51], v[44:47], v[80:83], a[48:51]\n" \
  "ds_read_b128 v[44:47], v25 offset:35840\n" \
  "v_mfma_f32_16x16x32_f16 a[64:67], v[48:51], v[80:83], a[64:67]\n" \
  "buffer_load_dwordx4 v24, s[12:15], s42 offen offset:2048 sc0 sc1 lds\n" \
  "v_mfma_f32_16x16x32_f16 a[68:71], v[48:51], v[84:87], a[68:71]\n" \
  "v_readlane_b32 s64, v9, s38\n" \
  "v_mfma_f32_16x16x32_f16 a[72:75], v[48:51], v[88:91], a[72:75]\n" \
  "v_cvt_pk_f16_f32 v75, v186, v190\n" \
  "v_cvt_pk_f16_f32 v79, v187, v191\n" \
  "v_mfma_f32_16x16x32_f16 a[76:79], v[48:51], v[92:95], a[76:79]\n" \
  "ds_read_b128 v[48:51], v25 offset:36864\n" \
  "v_mfma_f32_16x16x32_f16 a[92:95], v[52:55], v[92:95], a[92:95]\n" \
  "buffer_load_dwordx4 v24, s[12:15], s42 offen offset:3072 sc0 sc1 lds\n" \
  "v_mfma_f32_16x16x32_f16 a[88:91], v[52:55], v[88:91], a[88:91]\n" \
  "v_bfe_u32 v27, s64, v7, 8\n" \
  "v_mfma_f32_16x16x32_f16 a[84:87], v[52:55], v[84:87], a[84:87]\n" \
  "s_waitcnt lgkmcnt(5)\n" \
  "s_lshl_b32 s41, s40, 20\n" \
  "v_mfma_f32_16x16x32_f16 a[80:83], v[52:55], v[80:83], a[80:83]\n" \
  "ds_read_b128 v[52:55], v25 offset:37888\n" \
  "v_mfma_f32_16x16x32_f16 a[96:99], v[56:59], v[80:83], a[96:99]\n" \
  "buffer_load_dwordx4 v[160:163], v16, s[8:11], s41 offen\n" \
  "v_mfma_f32_16x16x32_f16 a[100:103], v[56:59], v[84:87], a[100:103]\n" \
  "v_lshl_add_u32 v27, v27, 6, v8\n" \
  "v_mfma_f32_16x16x32_f16 a[104:107], v[56:59], v[88:91], a[104:107]\n" \
  "buffer_load_dword v28, v27, s[16:19], 0 offen sc0 sc1\n" \
  "v_mfma_f32_16x16x32_f16 a[108:111], v[56:59], v[92:95], a[108:111]\n" \
  "ds_read_b128 v[56:59], v25 offset:38912\n" \
  "v_mfma_f32_16x16x32_f16 a[124:127], v[60:63], v[92:95], a[124:127]\n" \
  "buffer_load_dwordx4 v[164:167], v17, s[8:11], s41 offen\n" \
  "v_mfma_f32_16x16x32_f16 a[120:123], v[60:63], v[88:91], a[120:123]\n" \
  "s_waitcnt vmcnt(31)\n" \
  "v_mfma_f32_16x16x32_f16 a[116:119], v[60:63], v[84:87], a[116:119]\n" \
  "s_waitcnt lgkmcnt(5)\n" \
  "s_barrier\n" \
  "v_mfma_f32_16x16x32_f16 a[112:115], v[60:63], v[80:83], a[112:115]\n" \
  "ds_read_b128 v[60:63], v25 offset:39936\n" \
  "v_mfma_f32_16x16x32_f16 a[0:3], v[32:35], v[64:67], a[0:3]\n" \
  "buffer_load_dwordx4 v[168:171], v18, s[8:11], s41 offen\n" \
  "s_waitcnt vmcnt(42)\n" \
  "v_mfma_f32_16x16x32_f16 a[4:7], v[32:35], v[68:71], a[4:7]\n" \
  "v_cvt_pk_f16_f32 v224, v192, v196\n" \
  "v_cvt_pk_f16_f32 v228, v193, v197\n" \
  "v_mfma_f32_16x16x32_f16 a[8:11], v[32:35], v[72:75], a[8:11]\n" \
  "v_cvt_pk_f16_f32 v232, v194, v198\n" \
  "v_cvt_pk_f16_f32 v236, v195, v199\n" \
  "v_mfma_f32_16x16x32_f16 a[12:15], v[32:35], v[76:79], a[12:15]\n" \
  "ds_read_b128 v[32:35], v25 offset:49152\n" \
  "s_add_u32 s46, s41, 0x80000\n" \
  "v_mfma_f32_16x16x32_f16 a[28:31], v[36:39], v[76:79], a[28:31]\n" \
  "buffer_load_dwordx4 v[172:175], v19, s[8:11], s41 offen\n" \
  "s_waitcnt vmcnt(41)\n" \
  "v_mfma_f32_16x16x32_f16 a[24:27], v[36:39], v[72:75], a[24:27]\n" \
  "v_cvt_pk_f16_f32 v225, v200, v204\n" \
  "v_cvt_pk_f16_f32 v229, v201, v205\n" \
  "v_mfma_f32_16x16x32_f16 a[20:23], v[36:39], v[68:71], a[20:23]\n" \
  "s_waitcnt lgkmcnt(5)\n" \
  "v_mfma_f32_16x16x32_f16 a[16:19], v[36:39], v[64:67], a[16:19]\n" \
  "ds_read_b128 v[36:39], v25 offset:50176\n" \
  "v_mfma_f32_16x16x32_f16 a[32:35], v[40:43], v[64:67], a[32:35]\n" \
  "buffer_load_dwordx4 v[176:179], v16, s[8:11], s46 offen\n" \
  "v_mfma_f32_16x16x32_f16 a[36:39], v[40:43], v[68:71], a[36:39]\n" \
  "v_cvt_pk_f16_f32 v233, v202, v206\n" \
  "v_cvt_pk_f16_f32 v237, v203, v207\n" \
  "s_waitcnt vmcnt(40)\n" \
  "v_mfma_f32_16x16x32_f16 a[40:43], v[40:43], v[72:75], a[40:43]\n" \
  "v_cvt_pk_f16_f32 v226, v208, v212\n" \
  "v_cvt_pk_f16_f32 v230, v209, v213\n" \
  "v_mfma_f32_16x16x32_f16 a[44:47], v[40:43], v[76:79], a[44:47]\n" \
  "ds_read_b128 v[40:43], v25 offset:51200\n" \
  "v_mfma_f32_16x16x32_f16 a[60:63], v[44:47], v[76:79], a[60:63]\n" \
  "buffer_load_dwordx4 v[180:183], v17, s[8:11], s46 offen\n" \
  "v_mfma_f32_16x16x32_f16 a[56:59], v[44:47], v[72:75], a[56:59]\n" \
  "v_cvt_pk_f16_f32 v234, v210, v214\n" \
  "v_cvt_pk_f16_f32 v238, v211, v215\n" \
  "v_mfma_f32_16x16x32_f16 a[52:55], v[44:47], v[68:71], a[52:55]\n" \
  "s_waitcnt lgkmcnt(5)\n" \
  "v_mfma_f32_16x16x32_f16 a[48:51], v[44:47], v[64:67], a[48:51]\n" \
  "ds_read_b128 v[44:47], v25 offset:52224\n" \
  "v_mfma_f32_16x16x32_f16 a[64:67], v[48:51], v[64:67], a[64:67]\n" \
  "buffer_load_dwordx4 v[184:187], v18, s[8:11], s46 offen\n" \
  "s_waitcnt vmcnt(40)\n" \
  "v_mfma_f32_16x16x32_f16 a[68:71], v[48:51], v[68:71], a[68:71]\n" \
  "v_cvt_pk_f16_f32 v227, v216, v220\n" \
  "v_cvt_pk_f16_f32 v231, v217, v221\n" \
  "v_mfma_f32_16x16x32_f16 a[72:75], v[48:51], v[72:75], a[72:75]\n" \
  "v_cvt_pk_f16_f32 v235, v218, v222\n" \
  "v_cvt_pk_f16_f32 v239, v219, v223\n" \
  "v_mfma_f32_16x16x32_f16 a[76:79], v[48:51], v[76:79], a[76:79]\n" \
  "ds_read_b128 v[48:51], v25 offset:53248\n" \
  "v_mfma_f32_16x16x32_f16 a[92:95], v[52:55], v[76:79], a[92:95]\n" \
  "buffer_load_dwordx4 v[188:191], v19, s[8:11], s46 offen\n" \
  "v_mfma_f32_16x16x32_f16 a[88:91], v[52:55], v[72:75], a[88:91]\n" \
  "v_mfma_f32_16x16x32_f16 a[84:87], v[52:55], v[68:71], a[84:87]\n" \
  "s_waitcnt lgkmcnt(5)\n" \
  "s_bfe_u32 s40, s39, 0x80018\n" \
  "s_lshl_b32 s42, s40, 16\n" \
  "v_mfma_f32_16x16x32_f16 a[80:83], v[52:55], v[64:67], a[80:83]\n" \
  "ds_read_b128 v[52:55], v25 offset:54272\n" \
  "s_add_u32 m0, s36, 0xc000\n" \
  "v_mfma_f32_16x16x32_f16 a[96:99], v[56:59], v[64:67], a[96:99]\n" \
  "buffer_load_dwordx4 v24, s[12:15], s42 offen offset:0 sc0 sc1 lds\n" \
  "v_mfma_f32_16x16x32_f16 a[100:103], v[56:59], v[68:71], a[100:103]\n" \
  "v_mfma_f32_16x16x32_f16 a[104:107], v[56:59], v[72:75], a[104:107]\n" \
  "v_mfma_f32_16x16x32_f16 a[108:111], v[56:59], v[76:79], a[108:111]\n" \
  "ds_read_b128 v[56:59], v25 offset:55296\n" \
  "v_mfma_f32_16x16x32_f16 a[124:127], v[60:63], v[76:79], a[124:127]\n" \
  "buffer_load_dwordx4 v24, s[12:15], s42 offen offset:1024 sc0 sc1 lds\n" \
  "v_mfma_f32_16x16x32_f16 a[120:123], v[60:63], v[72:75], a[120:123]\n" \
  "s_waitcnt vmcnt(27)\n" \
  "v_mfma_f32_16x16x32_f16 a[116:119], v[60:63], v[68:71], a[116:119]\n" \
  "s_waitcnt lgkmcnt(5)\n" \
  "s_barrier\n" \
  "v_mfma_f32_16x16x32_f16 a[112:115], v[60:63], v[64:67], a[112:115]\n" \
  "ds_read_b128 v[60:63], v25 offset:56320\n" \
  "v_mfma_f32_16x16x32_f16 a[0:3], v[32:35], v[224:227], a[0:3]\n" \
  "buffer_load_dwordx4 v24, s[12:15], s42 offen offset:2048 sc0 sc1 lds\n" \
  "s_waitcnt vmcnt(38)\n" \
  "v_mfma_f32_16x16x32_f16 a[4:7], v[32:35], v[228:231], a[4:7]\n" \
  "v_cvt_pk_f16_f32 v240, v96, v100\n" \
  "v_cvt_pk_f16_f32 v244, v97, v101\n" \
  "v_mfma_f32_16x16x32_f16 a[8:11], v[32:35], v[232:235], a[8:11]\n" \
  "v_cvt_pk_f16_f32 v248, v98, v102\n" \
  "v_cvt_pk_f16_f32 v252, v99, v103\n" \
  "v_mfma_f32_16x16x32_f16 a[12:15], v[32:35], v[236:239], a[12:15]\n" \
  "ds_read_b128 v[32:35], v26 offset:0\n" \
  "v_mfma_f32_16x16x32_f16 a[28:31], v[36:39], v[236:239], a[28:31]\n" \
  "buffer_load_dwordx4 v24, s[12:15], s42 offen offset:3072 sc0 sc1 lds\n" \
  "s_waitcnt vmcnt(37)\n" \
  "v_mfma_f32_16x16x32_f16 a[24:27], v[36:39], v[232:235], a[24:27]\n" \
  "v_cvt_pk_f16_f32 v241, v104, v108\n" \
  "v_cvt_pk_f16_f32 v245, v105, v109\n" \
  "v_mfma_f32_16x16x32_f16 a[20:23], v[36:39], v[228:231], a[20:23]\n" \
  "s_waitcnt lgkmcnt(5)\n" \
  "s_lshl_b32 s41, s40, 20\n" \
  "v_mfma_f32_16x16x32_f16 a[16:19], v[36:39], v[224:227], a[16:19]\n" \
  "ds_read_b128 v[36:39], v26 offset:1024\n" \
  "v_mfma_f32_16x16x32_f16 a[32:35], v[40:43], v[224:227], a[32:35]\n" \
  "buffer_load_dwordx4 v[192:195], v16, s[8:11], s41 offen\n" \
  "v_mfma_f32_16x16x32_f16 a[36:39], v[40:43], v[228:231], a[36:39]\n" \
  "v_cvt_pk_f16_f32 v249, v106, v110\n" \
  "v_cvt_pk_f16_f32 v253, v107, v111\n" \
  "s_waitcnt vmcnt(36)\n" \
  "v_mfma_f32_16x16x32_f16 a[40:43], v[40:43], v[232:235], a[40:43]\n" \
  "v_cvt_pk_f16_f32 v242, v112, v116\n" \
  "v_cvt_pk_f16_f32 v246, v113, v117\n" \
  "v_mfma_f32_16x16x32_f16 a[44:47], v[40:43], v[236:239], a[44:47]\n" \
  "ds_read_b128 v[40:43], v26 offset:2048\n" \
  "v_mfma_f32_16x16x32_f16 a[60:63], v[44:47], v[236:239], a[60:63]\n" \
  "buffer_load_dwordx4 v[196:199], v17, s[8:11], s41 offen\n" \
  "v_mfma_f32_16x16x32_f16 a[56:59], v[44:47], v[232:235], a[56:59]\n" \
  "v_cvt_pk_f16_f32 v250, v114, v118\n" \
  "v_cvt_pk_f16_f32 v254, v115, v119\n" \
  "v_mfma_f32_16x16x32_f16 a[52:55], v[44:47], v[228:231], a[52:55]\n" \
  "s_waitcnt lgkmcnt(5)\n" \
  "v_mfma_f32_16x16x32_f16 a[48:51], v[44:47], v[224:227], a[48:51]\n" \
  "ds_read_b128 v[44:47], v26 offset:3072\n" \
  "v_mfma_f32_16x16x32_f16 a[64:67], v[48:51], v[224:227], a[64:67]\n" \
  "buffer_load_dwordx4 v[200:203], v18, s[8:11], s41 offen\n" \
  "s_waitcnt vmcnt(36)\n" \
  "v_mfma_f32_16x16x32_f16 a[68:71], v[48:51], v[228:231], a[68:71]\n" \
  "v_cvt_pk_f16_f32 v243, v120, v124\n" \
  "v_cvt_pk_f16_f32 v247, v121, v125\n" \
  "v_mfma_f32_16x16x32_f16 a[72:75], v[48:51], v[232:235], a[72:75]\n" \
  "v_cvt_pk_f16_f32 v251, v122, v126\n" \
  "v_cvt_pk_f16_f32 v255, v123, v127\n" \
  "v_mfma_f32_16x16x32_f16 a[76:79], v[48:51], v[236:239], a[76:79]\n" \
  "ds_read_b128 v[48:51], v26 offset:4096\n" \
  "s_add_u32 s46, s41, 0x80000\n" \
  "v_mfma_f32_16x16x32_f16 a[92:95], v[52:55], v[236:239], a[92:95]\n" \
  "buffer_load_dwordx4 v[204:207], v19, s[8:11], s41 offen\n" \
  "v_mfma_f32_16x16x32_f16 a[88:91], v[52:55], v[232:235], a[88:91]\n" \
  "v_mfma_f32_16x16x32_f16 a[84:87], v[52:55], v[228:231], a[84:87]\n" \
  "s_waitcnt lgkmcnt(5)\n" \
  "v_mfma_f32_16x16x32_f16 a[80:83], v[52:55], v[224:227], a[80:83]\n" \
  "ds_read_b128 v[52:55], v26 offset:5120\n" \
  "v_mfma_f32_16x16x32_f16 a[96:99], v[56:59], v[224:227], a[96:99]\n" \
  "buffer_load_dwordx4 v[208:211], v16, s[8:11], s46 offen\n" \
  "v_mfma_f32_16x16x32_f16 a[100:103], v[56:59], v[228:231], a[100:103]\n" \
  "v_mfma_f32_16x16x32_f16 a[104:107], v[56:59], v[232:235], a[104:107]\n" \
  "v_mfma_f32_16x16x32_f16 a[108:111], v[56:59], v[236:239], a[108:111]\n" \
  "ds_read_b128 v[56:59], v26 offset:6144\n" \
  "v_mfma_f32_16x16x32_f16 a[124:127], v[60:63], v[236:239], a[124:127]\n" \
  "buffer_load_dwordx4 v[212:215], v17, s[8:11], s46 offen\n" \
  "v_mfma_f32_16x16x32_f16 a[120:123], v[60:63], v[232:235], a[120:123]\n" \
  "s_waitcnt vmcnt(23)\n" \
  "v_mfma_f32_16x16x32_f16 a[116:119], v[60:63], v[228:231], a[116:119]\n" \
  "s_waitcnt lgkmcnt(5)\n" \
  "s_barrier\n" \
  "v_mfma_f32_16x16x32_f16 a[112:115], v[60:63], v[224:227], a[112:115]\n" \
  "ds_read_b128 v[60:63], v26 offset:7168\n" \
  "v_mfma_f32_16x16x32_f16 a[0:3], v[32:35], v[240:243], a[0:3]\n" \
  "buffer_load_dwordx4 v[216:219], v18, s[8:11], s46 offen\n" \
  "s_waitcnt vmcnt(34)\n" \
  "v_mfma_f32_16x16x32_f16 a[4:7], v[32:35], v[244:247], a[4:7]\n" \
  "v_cvt_pk_f16_f32 v0, v128, v132\n" \
  "v_cvt_pk_f16_f32 v4, v129, v133\n" \
  "v_mfma_f32_16x16x32_f16 a[8:11], v[32:35], v[248:251], a[8:11]\n" \
  "v_cvt_pk_f16_f32 v8, v130, v134\n" \
  "v_cvt_pk_f16_f32 v20, v131, v135\n" \
  "v_mfma_f32_16x16x32_f16 a[12:15], v[32:35], v[252:255], a[12:15]\n" \
  "ds_read_b128 v[32:35], v26 offset:16384\n" \
  "v_mfma_f32_16x16x32_f16 a[28:31], v[36:39], v[252:255], a[28:31]\n" \
  "s_waitcnt vmcnt(18)\n" \
  "v_cmp_le_u32 vcc, s43, v28\n" \
  "buffer_load_dwordx4 v[220:223], v19, s[8:11], s46 offen\n" \
  "v_mfma_f32_16x16x32_f16 a[24:27], v[36:39], v[248:251], a[24:27]\n" \
  "s_cmp_eq_u64 vcc, -1\n" \
  "s_cbranch_scc0 Lstub2_%=\n" \
  "Lback2_%=:\n" \
  "s_mov_b32 s39, s64\n" \
  "s_xor_b32 s36, s36, 0x10000\n" \
  "v_mfma_f32_16x16x32_f16 a[20:23], v[36:39], v[244:247], a[20:23]\n" \
  "s_waitcnt lgkmcnt(5)\n" \
  "s_bfe_u32 s40, s39, 0x80000\n" \
  "s_lshl_b32 s42, s40, 16\n" \
  "v_mfma_f32_16x16x32_f16 a[16:19], v[36:39], v[240:243], a[16:19]\n" \
  "ds_read_b128 v[36:39], v26 offset:17408\n" \
  "s_add_u32 m0, s36, 0x0\n" \
  "v_mfma_f32_16x16x32_f16 a[32:35], v[40:43], v[240:243], a[32:35]\n" \
  "buffer_load_dwordx4 v24, s[12:15], s42 offen offset:0 sc0 sc1 lds\n" \
  "s_waitcnt vmcnt(34)\n" \
  "v_mfma_f32_16x16x32_f16 a[36:39], v[40:43], v[244:247], a[36:39]\n" \
  "v_cvt_pk_f16_f32 v1, v136, v140\n" \
  "v_cvt_pk_f16_f32 v5, v137, v141\n" \
  "v_mfma_f32_16x16x32_f16 a[40:43], v[40:43], v[248:251], a[40:43]\n" \
  "v_cvt_pk_f16_f32 v9, v138, v142\n" \
  "v_cvt_pk_f16_f32 v21, v139, v143\n" \
  "v_mfma_f32_16x16x32_f16 a[44:47], v[40:43], v[252:255], a[44:47]\n" \
  "ds_read_b128 v[40:43], v26 offset:18432\n" \
  "v_mfma_f32_16x16x32_f16 a[60:63], v[44:47], v[252:255], a[60:63]\n" \
  "buffer_load_dwordx4 v24, s[12:15], s42 offen offset:1024 sc0 sc1 lds\n" \
  "s_waitcnt vmcnt(33)\n" \
  "v_mfma_f32_16x16x32_f16 a[56:59], v[44:47], v[248:251], a[56:59]\n" \
  "v_cvt_pk_f16_f32 v2, v144, v148\n" \
  "v_cvt_pk_f16_f32 v6, v145, v149\n" \
  "v_mfma_f32_16x16x32_f16 a[52:55], v[44:47], v[244:247], a[52:55]\n" \
  "s_waitcnt lgkmcnt(5)\n" \
  "v_mfma_f32_16x16x32_f16 a[48:51], v[44:47], v[240:243], a[48:51]\n" \
  "ds_read_b128 v[44:47], v26 offset:19456\n" \
  "v_mfma_f32_16x16x32_f16 a[64:67], v[48:51], v[240:243], a[64:67]\n" \
  "buffer_load_dwordx4 v24, s[12:15], s42 offen offset:2048 sc0 sc1 lds\n" \
  "v_mfma_f32_16x16x32_f16 a[68:71], v[48:51], v[244:247], a[68:71]\n" \
  "v_cvt_pk_f16_f32 v10, v146, v150\n" \
  "v_cvt_pk_f16_f32 v22, v147, v151\n" \
  "s_waitcnt vmcnt(32)\n" \
  "v_mfma_f32_16x16x32_f16 a[72:75], v[48:51], v[248:251], a[72:75]\n" \
  "v_cvt_pk_f16_f32 v3, v152, v156\n" \
  "v_cvt_pk_f16_f32 v7, v153, v157\n" \
  "v_mfma_f32_16x16x32_f16 a[76:79], v[48:51], v[252:255], a[76:79]\n" \
  "ds_read_b128 v[48:51], v26 offset:20480\n" \
  "v_mfma_f32_16x16x32_f16 a[92:95], v[52:55], v[252:255], a[92:95]\n" \
  "buffer_load_dwordx4 v24, s[12:15], s42 offen offset:3072 sc0 sc1 lds\n" \
  "v_mfma_f32_16x16x32_f16 a[88:91], v[52:55], v[248:251], a[88:91]\n" \
  "v_cvt_pk_f16_f32 v11, v154, v158\n" \
  "v_cvt_pk_f16_f32 v23, v155, v159\n" \
  "v_mfma_f32_16x16x32_f16 a[84:87], v[52:55], v[244:247], a[84:87]\n" \
  "s_waitcnt lgkmcnt(5)\n" \
  "s_lshl_b32 s41, s40, 20\n" \
  "v_mfma_f32_16x16x32_f16 a[80:83], v[52:55], v[240:243], a[80:83]\n" \
  "ds_read_b128 v[52:55], v26 offset:21504\n" \
  "v_mfma_f32_16x16x32_f16 a[96:99], v[56:59], v[240:243], a[96:99]\n" \
  "buffer_load_dwordx4 v[96:99], v16, s[8:11], s41 offen\n" \
  "v_mfma_f32_16x16x32_f16 a[100:103], v[56:59], v[244:247], a[100:103]\n" \
  "v_mfma_f32_16x16x32_f16 a[104:107], v[56:59], v[248:251], a[104:107]\n" \
  "v_mfma_f32_16x16x32_f16 a[108:111], v[56:59], v[252:255], a[108:111]\n" \
  "ds_read_b128 v[56:59], v26 offset:22528\n" \
  "v_mfma_f32_16x16x32_f16 a[124:127], v[60:63], v[252:255], a[124:127]\n" \
  "buffer_load_dwordx4 v[100:103], v17, s[8:11], s41 offen\n" \
  "v_mfma_f32_16x16x32_f16 a[120:123], v[60:63], v[248:251], a[120:123]\n" \
  "s_waitcnt vmcnt(27)\n" \
  "v_mfma_f32_16x16x32_f16 a[116:119], v[60:63], v[244:247], a[116:119]\n" \
  "s_waitcnt lgkmcnt(5)\n" \
  "s_barrier\n" \
  "v_mfma_f32_16x16x32_f16 a[112:115], v[60:63], v[240:243], a[112:115]\n" \
  "ds_read_b128 v[60:63], v26 offset:23552\n" \
  "v_mfma_f32_16x16x32_f16 a[0:3], v[32:35], v[0:3], a[0:3]\n" \
  "buffer_load_dwordx4 v[104:107], v18, s[8:11], s41 offen\n" \
  "s_waitcnt vmcnt(25)\n" \
  "v_mfma_f32_16x16x32_f16 a[4:7], v[32:35], v[4:7], a[4:7]\n" \
  "v_cvt_pk_f16_f32 v128, v160, v164\n" \
  "v_cvt_pk_f16_f32 v132, v161, v165\n" \
  "v_mfma_f32_16x16x32_f16 a[8:11], v[32:35], v[8:11], a[8:11]\n" \
  "v_cvt_pk_f16_f32 v136, v162, v166\n" \
  "v_cvt_pk_f16_f32 v140, v163, v167\n" \
  "v_mfma_f32_16x16x32_f16 a[12:15], v[32:35], v[20:23], a[12:15]\n" \
  "ds_read_b128 v[32:35], v26 offset:32768\n" \
  "s_add_u32 s46, s41, 0x80000\n" \
  "v_mfma_f32_16x16x32_f16 a[28:31], v[36:39], v[20:23], a[28:31]\n" \
  "buffer_load_dwordx4 v[108:111], v19, s[8:11], s41 offen\n" \
  "s_waitcnt vmcnt(24)\n" \
  "v_mfma_f32_16x16x32_f16 a[24:27], v[36:39], v[8:11], a[24:27]\n" \
  "v_cvt_pk_f16_f32 v129, v168, v172\n" \
  "v_cvt_pk_f16_f32 v133, v169, v173\n" \
  "v_mfma_f32_16x16x32_f16 a[20:23], v[36:39], v[4:7], a[20:23]\n" \
  "s_waitcnt lgkmcnt(5)\n" \
  "v_mfma_f32_16x16x32_f16 a[16:19], v[36:39], v[0:3], a[16:19]\n" \
  "ds_read_b128 v[36:39], v26 offset:33792\n" \
  "v_mfma_f32_16x16x32_f16 a[32:35], v[40:43], v[0:3], a[32:35]\n" \
  "buffer_load_dwordx4 v[112:115], v16, s[8:11], s46 offen\n" \
  "v_mfma_f32_16x16x32_f16 a[36:39], v[40:43], v[4:7], a[36:39]\n" \
  "v_cvt_pk_f16_f32 v137, v170, v174\n" \
  "v_cvt_pk_f16_f32 v141, v171, v175\n" \
  "s_waitcnt vmcnt(23)\n" \
  "v_mfma_f32_16x16x32_f16 a[40:43], v[40:43], v[8:11], a[40:43]\n" \
  "v_cvt_pk_f16_f32 v130, v176, v180\n" \
  "v_cvt_pk_f16_f32 v134, v177, v181\n" \
  "v_mfma_f32_16x16x32_f16 a[44:47], v[40:43], v[20:23], a[44:47]\n" \
  "ds_read_b128 v[40:43], v26 offset:34816\n" \
  "v_mfma_f32_16x16x32_f16 a[60:63], v[44:47], v[20:23], a[60:63]\n" \
  "buffer_load_dwordx4 v[116:119], v17, s[8:11], s46 offen\n" \
  "v_mfma_f32_16x16x32_f16 a[56:59], v[44:47], v[8:11], a[56:59]\n" \
  "v_cvt_pk_f16_f32 v138, v178, v182\n" \
  "v_cvt_pk_f16_f32 v142, v179, v183\n" \
  "v_mfma_f32_16x16x32_f16 a[52:55], v[44:47], v[4:7], a[52:55]\n" \
  "s_waitcnt lgkmcnt(5)\n" \
  "v_mfma_f32_16x16x32_f16 a[48:51], v[44:47], v[0:3], a[48:51]\n" \
  "ds_read_b128 v[44:47], v26 offset:35840\n" \
  "v_mfma_f32_16x16x32_f16 a[64:67], v[48:51], v[0:3], a[64:67]\n" \
  "buffer_load_dwordx4 v[120:123], v18, s[8:11], s46 offen\n" \
  "s_waitcnt vmcnt(23)\n" \
  "v_mfma_f32_16x16x32_f16 a[68:71], v[48:51], v[4:7], a[68:71]\n" \
  "v_cvt_pk_f16_f32 v131, v184, v188\n" \
  "v_cvt_pk_f16_f32 v135, v185, v189\n" \
  "v_mfma_f32_16x16x32_f16 a[72:75], v[48:51], v[8:11], a[72:75]\n" \
  "v_cvt_pk_f16_f32 v139, v186, v190\n" \
  "v_cvt_pk_f16_f32 v143, v187, v191\n" \
  "v_mfma_f32_16x16x32_f16 a[76:79], v[48:51], v[20:23], a[76:79]\n" \
  "ds_read_b128 v[48:51], v26 offset:36864\n" \
  "v_mfma_f32_16x16x32_f16 a[92:95], v[52:55], v[20:23], a[92:95]\n" \
  "buffer_load_dwordx4 v[124:127], v19, s[8:11], s46 offen\n" \
  "v_mfma_f32_16x16x32_f16 a[88:91], v[52:55], v[8:11], a[88:91]\n" \
  "v_mfma_f32_16x16x32_f16 a[84:87], v[52:55], v[4:7], a[84:87]\n" \
  "s_waitcnt lgkmcnt(5)\n" \
  "v_mfma_f32_16x16x32_f16 a[80:83], v[52:55], v[0:3], a[80:83]\n" \
  "ds_read_b128 v[52:55], v26 offset:37888\n" \
  "v_mfma_f32_16x16x32_f16 a[96:99], v[56:59], v[0:3], a[96:99]\n" \
  "v_mfma_f32_16x16x32_f16 a[100:103], v[56:59], v[4:7], a[100:103]\n" \
  "v_mfma_f32_16x16x32_f16 a[104:107], v[56:59], v[8:11], a[104:107]\n" \
  "v_mfma_f32_16x16x32_f16 a[108:111], v[56:59], v[20:23], a[108:111]\n" \
  "ds_read_b128 v[56:59], v26 offset:38912\n" \
  "v_mfma_f32_16x16x32_f16 a[124:127], v[60:63], v[20:23], a[124:127]\n" \
  "v_mfma_f32_16x16x32_f16 a[120:123], v[60:63], v[8:11], a[120:123]\n" \
  "s_waitcnt vmcnt(20)\n" \
  "v_mfma_f32_16x16x32_f16 a[116:119], v[60:63], v[4:7], a[116:119]\n" \
  "s_waitcnt lgkmcnt(5)\n" \
  "s_barrier\n" \
  "v_mfma_f32_16x16x32_f16 a[112:115], v[60:63], v[0:3], a[112:115]\n" \
  "ds_read_b128 v[60:63], v26 offset:39936\n" \
  "v_mfma_f32_16x16x32_f16 a[0:3], v[32:35], v[128:131], a[0:3]\n" \
  "s_waitcnt vmcnt(18)\n" \
  "v_mfma_f32_16x16x32_f16 a[4:7], v[32:35], v[132:135], a[4:7]\n" \
  "v_cvt_pk_f16_f32 v144, v192, v196\n" \
  "v_cvt_pk_f16_f32 v148, v193, v197\n" \
  "v_mfma_f32_16x16x32_f16 a[8:11], v[32:35], v[136:139], a[8:11]\n" \
  "v_cvt_pk_f16_f32 v152, v194, v198\n" \
  "v_cvt_pk_f16_f32 v156, v195, v199\n" \
  "v_mfma_f32_16x16x32_f16 a[12:15], v[32:35], v[140:143], a[12:15]\n" \
  "ds_read_b128 v[32:35], v26 offset:49152\n" \
  "s_waitcnt vmcnt(16)\n" \
  "v_mfma_f32_16x16x32_f16 a[28:31], v[36:39], v[140:143], a[28:31]\n" \
  "v_cvt_pk_f16_f32 v145, v200, v204\n" \
  "v_cvt_pk_f16_f32 v149, v201, v205\n" \
  "v_mfma_f32_16x16x32_f16 a[24:27], v[36:39], v[136:139], a[24:27]\n" \
  "v_cvt_pk_f16_f32 v153, v202, v206\n" \
  "v_cvt_pk_f16_f32 v157, v203, v207\n" \
  "v_mfma_f32_16x16x32_f16 a[20:23], v[36:39], v[132:135], a[20:23]\n" \
  "s_waitcnt lgkmcnt(5)\n" \
  "v_mfma_f32_16x16x32_f16 a[16:19], v[36:39], v[128:131], a[16:19]\n" \
  "ds_read_b128 v[36:39], v26 offset:50176\n" \
  "s_waitcnt vmcnt(14)\n" \
  "v_mfma_f32_16x16x32_f16 a[32:35], v[40:43], v[128:131], a[32:35]\n" \
  "v_cvt_pk_f16_f32 v146, v208, v212\n" \
  "v_cvt_pk_f16_f32 v150, v209, v213\n" \
  "v_mfma_f32_16x16x32_f16 a[36:39], v[40:43], v[132:135], a[36:39]\n" \
  "v_cvt_pk_f16_f32 v154, v210, v214\n" \
  "v_cvt_pk_f16_f32 v158, v211, v215\n" \
  "s_waitcnt vmcnt(12)\n" \
  "v_mfma_f32_16x16x32_f16 a[40:43], v[40:43], v[136:139], a[40:43]\n" \
  "v_cvt_pk_f16_f32 v147, v216, v220\n" \
  "v_cvt_pk_f16_f32 v151, v217, v221\n" \
  "v_mfma_f32_16x16x32_f16 a[44:47], v[40:43], v[140:143], a[44:47]\n" \
  "ds_read_b128 v[40:43], v26 offset:51200\n" \
  "v_mfma_f32_16x16x32_f16 a[60:63], v[44:47], v[140:143], a[60:63]\n" \
  "v_cvt_pk_f16_f32 v155, v218, v222\n" \
  "v_cvt_pk_f16_f32 v159, v219, v223\n" \
  "v_mfma_f32_16x16x32_f16 a[56:59], v[44:47], v[136:139], a[56:59]\n" \
  "v_mfma_f32_16x16x32_f16 a[52:55], v[44:47], v[132:135], a[52:55]\n" \
  "s_waitcnt lgkmcnt(5)\n" \
  "v_mfma_f32_16x16x32_f16 a[48:51], v[44:47], v[128:131], a[48:51]\n" \
  "ds_read_b128 v[44:47], v26 offset:52224\n" \
  "v_mfma_f32_16x16x32_f16 a[64:67], v[48:51], v[128:131], a[64:67]\n" \
  "v_mfma_f32_16x16x32_f16 a[68:71], v[48:51], v[132:135], a[68:71]\n" \
  "v_mfma_f32_16x16x32_f16 a[72:75], v[48:51], v[136:139], a[72:75]\n" \
  "v_mfma_f32_16x16x32_f16 a[76:79], v[48:51], v[140:143], a[76:79]\n" \
  "ds_read_b128 v[48:51], v26 offset:53248\n" \
  "v_mfma_f32_16x16x32_f16 a[92:95], v[52:55], v[140:143], a[92:95]\n" \
  "v_mfma_f32_16x16x32_f16 a[88:91], v[52:55], v[136:139], a[88:91]\n" \
  "v_mfma_f32_16x16x32_f16 a[84:87], v[52:55], v[132:135], a[84:87]\n" \
  "s_waitcnt lgkmcnt(5)\n" \
  "v_mfma_f32_16x16x32_f16 a[80:83], v[52:55], v[128:131], a[80:83]\n" \
  "ds_read_b128 v[52:55], v26 offset:54272\n" \
  "v_mfma_f32_16x16x32_f16 a[96:99], v[56:59], v[128:131], a[96:99]\n" \
  "v_mfma_f32_16x16x32_f16 a[100:103], v[56:59], v[132:135], a[100:103]\n" \
  "v_mfma_f32_16x16x32_f16 a[104:107], v[56:59], v[136:139], a[104:107]\n" \
  "v_mfma_f32_16x16x32_f16 a[108:111], v[56:59], v[140:143], a[108:111]\n" \
  "ds_read_b128 v[56:59], v26 offset:55296\n" \
  "v_mfma_f32_16x16x32_f16 a[124:127], v[60:63], v[140:143], a[124:127]\n" \
  "v_mfma_f32_16x16x32_f16 a[120:123], v[60:63], v[136:139], a[120:123]\n" \
  "s_waitcnt vmcnt(8)\n" \
  "v_mfma_f32_16x16x32_f16 a[116:119], v[60:63], v[132:135], a[116:119]\n" \
  "s_waitcnt lgkmcnt(5)\n" \
  "s_barrier\n" \
  "v_mfma_f32_16x16x32_f16 a[112:115], v[60:63], v[128:131], a[112:115]\n" \
  "ds_read_b128 v[60:63], v26 offset:56320\n" \
  "v_mfma_f32_16x16x32_f16 a[0:3], v[32:35], v[144:147], a[0:3]\n" \
  "s_waitcnt vmcnt(6)\n" \
  "v_mfma_f32_16x16x32_f16 a[4:7], v[32:35], v[148:151], a[4:7]\n" \
  "v_cvt_pk_f16_f32 v160, v96, v100\n" \
  "v_cvt_pk_f16_f32 v164, v97, v101\n" \
  "v_mfma_f32_16x16x32_f16 a[8:11], v[32:35], v[152:155], a[8:11]\n" \
  "v_cvt_pk_f16_f32 v168, v98, v102\n" \
  "v_cvt_pk_f16_f32 v172, v99, v103\n" \
  "v_mfma_f32_16x16x32_f16 a[12:15], v[32:35], v[156:159], a[12:15]\n" \
  "ds_read_b128 v[32:35], v25 offset:0\n" \
  "s_waitcnt vmcnt(4)\n" \
  "v_mfma_f32_16x16x32_f16 a[28:31], v[36:39], v[156:159], a[28:31]\n" \
  "v_cvt_pk_f16_f32 v161, v104, v108\n" \
  "v_cvt_pk_f16_f32 v165, v105, v109\n" \
  "v_mfma_f32_16x16x32_f16 a[24:27], v[36:39], v[152:155], a[24:27]\n" \
  "v_cvt_pk_f16_f32 v169, v106, v110\n" \
  "v_cvt_pk_f16_f32 v173, v107, v111\n" \
  "v_mfma_f32_16x16x32_f16 a[20:23], v[36:39], v[148:151], a[20:23]\n" \
  "s_waitcnt lgkmcnt(5)\n" \
  "v_mfma_f32_16x16x32_f16 a[16:19], v[36:39], v[144:147], a[16:19]\n" \
  "ds_read_b128 v[36:39], v25 offset:1024\n" \
  "s_waitcnt vmcnt(2)\n" \
  "v_mfma_f32_16x16x32_f16 a[32:35], v[40:43], v[144:147], a[32:35]\n" \
  "v_cvt_pk_f16_f32 v162, v112, v116\n" \
  "v_cvt_pk_f16_f32 v166, v113, v117\n" \
  "v_mfma_f32_16x16x32_f16 a[36:39], v[40:43], v[148:151], a[36:39]\n" \
  "v_cvt_pk_f16_f32 v170, v114, v118\n" \
  "v_cvt_pk_f16_f32 v174, v115, v119\n" \
  "s_waitcnt vmcnt(0)\n" \
  "v_mfma_f32_16x16x32_f16 a[40:43], v[40:43], v[152:155], a[40:43]\n" \
  "v_cvt_pk_f16_f32 v163, v120, v124\n" \
  "v_cvt_pk_f16_f32 v167, v121, v125\n" \
  "v_mfma_f32_16x16x32_f16 a[44:47], v[40:43], v[156:159], a[44:47]\n" \
  "ds_read_b128 v[40:43], v25 offset:2048\n" \
  "v_mfma_f32_16x16x32_f16 a[60:63], v[44:47], v[156:159], a[60:63]\n" \
  "v_cvt_pk_f16_f32 v171, v122, v126\n" \
  "v_cvt_pk_f16_f32 v175, v123, v127\n" \
  "v_mfma_f32_16x16x32_f16 a[56:59], v[44:47], v[152:155], a[56:59]\n" \
  "v_mfma_f32_16x16x32_f16 a[52:55], v[44:47], v[148:151], a[52:55]\n" \
  "s_waitcnt lgkmcnt(5)\n" \
  "v_mfma_f32_16x16x32_f16 a[48:51], v[44:47], v[144:147], a[48:51]\n" \
  "ds_read_b128 v[44:47], v25 offset:3072\n" \
  "v_mfma_f32_16x16x32_f16 a[64:67], v[48:51], v[144:147], a[64:67]\n" \
  "v_mfma_f32_16x16x32_f16 a[68:71], v[48:51], v[148:151], a[68:71]\n" \
  "v_mfma_f32_16x16x32_f16 a[72:75], v[48:51], v[152:155], a[72:75]\n" \
  "v_mfma_f32_16x16x32_f16 a[76:79], v[48:51], v[156:159], a[76:79]\n" \
  "ds_read_b128 v[48:51], v25 offset:4096\n" \
  "v_mfma_f32_16x16x32_f16 a[92:95], v[52:55], v[156:159], a[92:95]\n" \
  "v_mfma_f32_16x16x32_f16 a[88:91], v[52:55], v[152:155], a[88:91]\n" \
  "v_mfma_f32_16x16x32_f16 a[84:87], v[52:55], v[148:151], a[84:87]\n" \
  "s_waitcnt lgkmcnt(5)\n" \
  "v_mfma_f32_16x16x32_f16 a[80:83], v[52:55], v[144:147], a[80:83]\n" \
  "ds_read_b128 v[52:55], v25 offset:5120\n" \
  "v_mfma_f32_16x16x32_f16 a[96:99], v[56:59], v[144:147], a[96:99]\n" \
  "v_mfma_f32_16x16x32_f16 a[100:103], v[56:59], v[148:151], a[100:103]\n" \
  "v_mfma_f32_16x16x32_f16 a[104:107], v[56:59], v[152:155], a[104:107]\n" \
  "v_mfma_f32_16x16x32_f16 a[108:111], v[56:59], v[156:159], a[108:111]\n" \
  "ds_read_b128 v[56:59], v25 offset:6144\n" \
  "v_mfma_f32_16x16x32_f16 a[124:127], v[60:63], v[156:159], a[124:127]\n" \
  "v_mfma_f32_16x16x32_f16 a[120:123], v[60:63], v[152:155], a[120:123]\n" \
  "v_mfma_f32_16x16x32_f16 a[116:119], v[60:63], v[148:151], a[116:119]\n" \
  "s_waitcnt lgkmcnt(5)\n" \
  "v_mfma_f32_16x16x32_f16 a[112:115], v[60:63], v[144:147], a[112:115]\n" \
  "ds_read_b128 v[60:63], v25 offset:7168\n" \
  "v_mfma_f32_16x16x32_f16 a[0:3], v[32:35], v[160:163], a[0:3]\n" \
  "v_mfma_f32_16x16x32_f16 a[4:7], v[32:35], v[164:167], a[4:7]\n" \
  "v_mfma_f32_16x16x32_f16 a[8:11], v[32:35], v[168:171], a[8:11]\n" \
  "v_mfma_f32_16x16x32_f16 a[12:15], v[32:35], v[172:175], a[12:15]\n" \
  "ds_read_b128 v[32:35], v25 offset:24576\n" \
  "v_mfma_f32_16x16x32_f16 a[28:31], v[36:39], v[172:175], a[28:31]\n" \
  "v_mfma_f32_16x16x32_f16 a[24:27], v[36:39], v[168:171], a[24:27]\n" \
  "v_mfma_f32_16x16x32_f16 a[20:23], v[36:39], v[164:167], a[20:23]\n" \
  "s_waitcnt lgkmcnt(5)\n" \
  "v_mfma_f32_16x16x32_f16 a[16:19], v[36:39], v[160:163], a[16:19]\n" \
  "ds_read_b128 v[36:39], v25 offset:25600\n" \
  "v_mfma_f32_16x16x32_f16 a[32:35], v[40:43], v[160:163], a[32:35]\n" \
  "v_mfma_f32_16x16x32_f16 a[36:39], v[40:43], v[164:167], a[36:39]\n" \
  "v_mfma_f32_16x16x32_f16 a[40:43], v[40:43], v[168:171], a[40:43]\n" \
  "s_mov_b32 s40, 0x0\n" \
  "v_mfma_f32_16x16x32_f16 a[44:47], v[40:43], v[172:175], a[44:47]\n" \
  "ds_read_b128 v[40:43], v25 offset:26624\n" \
  "v_mfma_f32_16x16x32_f16 a[60:63], v[44:47], v[172:175], a[60:63]\n" \
  "buffer_store_dwordx4 a[0:3], v12, s[16:19], s40 offen offset:0 sc1\n" \
  "v_mfma_f32_16x16x32_f16 a[56:59], v[44:47], v[168:171], a[56:59]\n" \
  "v_mfma_f32_16x16x32_f16 a[52:55], v[44:47], v[164:167], a[52:55]\n" \
  "s_waitcnt lgkmcnt(5)\n" \
  "v_mfma_f32_16x16x32_f16 a[48:51], v[44:47], v[160:163], a[48:51]\n" \
  "ds_read_b128 v[44:47], v25 offset:27648\n" \
  "v_mfma_f32_16x16x32_f16 a[64:67], v[48:51], v[160:163], a[64:67]\n" \
  "v_mfma_f32_16x16x32_f16 a[68:71], v[48:51], v[164:167], a[68:71]\n" \
  "v_mfma_f32_16x16x32_f16 a[72:75], v[48:51], v[168:171], a[72:75]\n" \
  "v_mfma_f32_16x16x32_f16 a[76:79], v[48:51], v[172:175], a[76:79]\n" \
  "ds_read_b128 v[48:51], v25 offset:28672\n" \
  "v_mfma_f32_16x16x32_f16 a[92:95], v[52:55], v[172:175], a[92:95]\n" \
  "buffer_store_dwordx4 a[4:7], v12, s[16:19], s40 offen offset:1024 sc1\n" \
  "v_mfma_f32_16x16x32_f16 a[88:91], v[52:55], v[168:171], a[88:91]\n" \
  "v_mfma_f32_16x16x32_f16 a[84:87], v[52:55], v[164:167], a[84:87]\n" \
  "s_waitcnt lgkmcnt(5)\n" \
  "v_mfma_f32_16x16x32_f16 a[80:83], v[52:55], v[160:163], a[80:83]\n" \
  "ds_read_b128 v[52:55], v25 offset:29696\n" \
  "v_mfma_f32_16x16x32_f16 a[96:99], v[56:59], v[160:163], a[96:99]\n" \
  "v_mfma_f32_16x16x32_f16 a[100:103], v[56:59], v[164:167], a[100:103]\n" \
  "v_mfma_f32_16x16x32_f16 a[104:107], v[56:59], v[168:171], a[104:107]\n" \
  "v_mfma_f32_16x16x32_f16 a[108:111], v[56:59], v[172:175], a[108:111]\n" \
  "ds_read_b128 v[56:59], v25 offset:30720\n" \
  "v_mfma_f32_16x16x32_f16 a[124:127], v[60:63], v[172:175], a[124:127]\n" \
  "buffer_store_dwordx4 a[8:11], v12, s[16:19], s40 offen offset:2048 sc1\n" \
  "v_mfma_f32_16x16x32_f16 a[120:123], v[60:63], v[168:171], a[120:123]\n" \
  "v_mfma_f32_16x16x32_f16 a[116:119], v[60:63], v[164:167], a[116:119]\n" \
  "s_waitcnt lgkmcnt(5)\n" \
  "v_mfma_f32_16x16x32_f16 a[112:115], v[60:63], v[160:163], a[112:115]\n" \
  "ds_read_b128 v[60:63], v25 offset:31744\n" \
  "v_mfma_f32_16x16x32_f16 a[128:131], v[32:35], v[80:83], a[128:131]\n" \
  "v_mfma_f32_16x16x32_f16 a[132:135], v[32:35], v[84:87], a[132:135]\n" \
  "v_mfma_f32_16x16x32_f16 a[136:139], v[32:35], v[88:91], a[136:139]\n" \
  "v_mfma_f32_16x16x32_f16 a[140:143], v[32:35], v[92:95], a[140:143]\n" \
  "ds_read_b128 v[32:35], v25 offset:40960\n" \
  "v_mfma_f32_16x16x32_f16 a[156:159], v[36:39], v[92:95], a[156:159]\n" \
  "buffer_store_dwordx4 a[12:15], v12, s[16:19], s40 offen offset:3072 sc1\n" \
  "v_mfma_f32_16x16x32_f16 a[152:155], v[36:39], v[88:91], a[152:155]\n" \
  "v_mfma_f32_16x16x32_f16 a[148:151], v[36:39], v[84:87], a[148:151]\n" \
  "s_waitcnt lgkmcnt(5)\n" \
  "v_mfma_f32_16x16x32_f16 a[144:147], v[36:39], v[80:83], a[144:147]\n" \
  "ds_read_b128 v[36:39], v25 offset:41984\n" \
  "v_mfma_f32_16x16x32_f16 a[160:163], v[40:43], v[80:83], a[160:163]\n" \
  "v_mfma_f32_16x16x32_f16 a[164:167], v[40:43], v[84:87], a[164:167]\n" \
  "v_mfma_f32_16x16x32_f16 a[168:171], v[40:43], v[88:91], a[168:171]\n" \
  "s_mov_b32 s40, 0x1000\n" \
  "v_mfma_f32_16x16x32_f16 a[172:175], v[40:43], v[92:95], a[172:175]\n" \
  "ds_read_b128 v[40:43], v25 offset:43008\n" \
  "v_mfma_f32_16x16x32_f16 a[188:191], v[44:47], v[92:95], a[188:191]\n" \
  "buffer_store_dwordx4 a[16:19], v12, s[16:19], s40 offen offset:0 sc1\n" \
  "v_mfma_f32_16x16x32_f16 a[184:187], v[44:47], v[88:91], a[184:187]\n" \
  "v_mfma_f32_16x16x32_f16 a[180:183], v[44:47], v[84:87], a[180:183]\n" \
  "s_waitcnt lgkmcnt(5)\n" \
  "v_mfma_f32_16x16x32_f16 a[176:179], v[44:47], v[80:83], a[176:179]\n" \
  "ds_read_b128 v[44:47], v25 offset:44032\n" \
  "v_mfma_f32_16x16x32_f16 a[192:195], v[48:51], v[80:83], a[192:195]\n" \
  "v_mfma_f32_16x16x32_f16 a[196:199], v[48:51], v[84:87], a[196:199]\n" \
  "v_mfma_f32_16x16x32_f16 a[200:203], v[48:51], v[88:91], a[200:203]\n" \
  "v_mfma_f32_16x16x32_f16 a[204:207], v[48:51], v[92:95], a[204:207]\n" \
  "ds_read_b128 v[48:51], v25 offset:45056\n" \
  "v_mfma_f32_16x16x32_f16 a[220:223], v[52:55], v[92:95], a[220:223]\n" \
  "buffer_store_dwordx4 a[20:23], v12, s[16:19], s40 offen offset:1024 sc1\n" \
  "v_mfma_f32_16x16x32_f16 a[216:219], v[52:55], v[88:91], a[216:219]\n" \
  "v_mfma_f32_16x16x32_f16 a[212:215], v[52:55], v[84:87], a[212:215]\n" \
  "s_waitcnt lgkmcnt(5)\n" \
  "v_mfma_f32_16x16x32_f16 a[208:211], v[52:55], v[80:83], a[208:211]\n" \
  "ds_read_b128 v[52:55], v25 offset:46080\n" \
  "v_mfma_f32_16x16x32_f16 a[224:227], v[56:59], v[80:83], a[224:227]\n" \
  "v_mfma_f32_16x16x32_f16 a[228:231], v[56:59], v[84:87], a[228:231]\n" \
  "v_mfma_f32_16x16x32_f16 a[232:235], v[56:59], v[88:91], a[232:235]\n" \
  "v_mfma_f32_16x16x32_f16 a[236:239], v[56:59], v[92:95], a[236:239]\n" \
  "ds_read_b128 v[56:59], v25 offset:47104\n" \
  "v_mfma_f32_16x16x32_f16 a[252:255], v[60:63], v[92:95], a[252:255]\n" \
  "buffer_store_dwordx4 a[24:27], v12, s[16:19], s40 offen offset:2048 sc1\n" \
  "v_mfma_f32_16x16x32_f16 a[248:251], v[60:63], v[88:91], a[248:251]\n" \
  "v_mfma_f32_16x16x32_f16 a[244:247], v[60:63], v[84:87], a[244:247]\n" \
  "s_waitcnt lgkmcnt(5)\n" \
  "v_mfma_f32_16x16x32_f16 a[240:243], v[60:63], v[80:83], a[240:243]\n" \
  "ds_read_b128 v[60:63], v25 offset:48128\n" \
  "v_mfma_f32_16x16x32_f16 a[128:131], v[32:35], v[64:67], a[128:131]\n" \
  "v_mfma_f32_16x16x32_f16 a[132:135], v[32:35], v[68:71], a[132:135]\n" \
  "v_mfma_f32_16x16x32_f16 a[136:139], v[32:35], v[72:75], a[136:139]\n" \
  "v_mfma_f32_16x16x32_f16 a[140:143], v[32:35], v[76:79], a[140:143]\n" \
  "ds_read_b128 v[32:35], v25 offset:57344\n" \
  "v_mfma_f32_16x16x32_f16 a[156:159], v[36:39], v[76:79], a[156:159]\n" \
  "buffer_store_dwordx4 a[28:31], v12, s[16:19], s40 offen offset:3072 sc1\n" \
  "v_mfma_f32_16x16x32_f16 a[152:155], v[36:39], v[72:75], a[152:155]\n" \
  "v_mfma_f32_16x16x32_f16 a[148:151], v[36:39], v[68:71], a[148:151]\n" \
  "s_waitcnt lgkmcnt(5)\n" \
  "v_mfma_f32_16x16x32_f16 a[144:147], v[36:39], v[64:67], a[144:147]\n" \
  "ds_read_b128 v[36:39], v25 offset:58368\n" \
  "v_mfma_f32_16x16x32_f16 a[160:163], v[40:43], v[64:67], a[160:163]\n" \
  "v_mfma_f32_16x16x32_f16 a[164:167], v[40:43], v[68:71], a[164:167]\n" \
  "v_mfma_f32_16x16x32_f16 a[168:171], v[40:43], v[72:75], a[168:171]\n" \
  "s_mov_b32 s40, 0x2000\n" \
  "v_mfma_f32_16x16x32_f16 a[172:175], v[40:43], v[76:79], a[172:175]\n" \
  "ds_read_b128 v[40:43], v25 offset:59392\n" \
  "v_mfma_f32_16x16x32_f16 a[188:191], v[44:47], v[76:79], a[188:191]\n" \
  "buffer_store_dwordx4 a[32:35], v12, s[16:19], s40 offen offset:0 sc1\n" \
  "v_mfma_f32_16x16x32_f16 a[184:187], v[44:47], v[72:75], a[184:187]\n" \
  "v_mfma_f32_16x16x32_f16 a[180:183], v[44:47], v[68:71], a[180:183]\n" \
  "s_waitcnt lgkmcnt(5)\n" \
  "v_mfma_f32_16x16x32_f16 a[176:179], v[44:47], v[64:67], a[176:179]\n" \
  "ds_read_b128 v[44:47], v25 offset:60416\n" \
  "v_mfma_f32_16x16x32_f16 a[192:195], v[48:51], v[64:67], a[192:195]\n" \
  "v_mfma_f32_16x16x32_f16 a[196:199], v[48:51], v[68:71], a[196:199]\n" \
  "v_mfma_f32_16x16x32_f16 a[200:203], v[48:51], v[72:75], a[200:203]\n" \
  "v_mfma_f32_16x16x32_f16 a[204:207], v[48:51], v[76:79], a[204:207]\n" \
  "ds_read_b128 v[48:51], v25 offset:61440\n" \
  "v_mfma_f32_16x16x32_f16 a[220:223], v[52:55], v[76:79], a[220:223]\n" \
  "buffer_store_dwordx4 a[36:39], v12, s[16:19], s40 offen offset:1024 sc1\n" \
  "v_mfma_f32_16x16x32_f16 a[216:219], v[52:55], v[72:75], a[216:219]\n" \
  "v_mfma_f32_16x16x32_f16 a[212:215], v[52:55], v[68:71], a[212:215]\n" \
  "s_waitcnt lgkmcnt(5)\n" \
  "v_mfma_f32_16x16x32_f16 a[208:211], v[52:55], v[64:67], a[208:211]\n" \
  "ds_read_b128 v[52:55], v25 offset:62464\n" \
  "v_mfma_f32_16x16x32_f16 a[224:227], v[56:59], v[64:67], a[224:227]\n" \
  "v_mfma_f32_16x16x32_f16 a[228:231], v[56:59], v[68:71], a[228:231]\n" \
  "v_mfma_f32_16x16x32_f16 a[232:235], v[56:59], v[72:75], a[232:235]\n" \
  "v_mfma_f32_16x16x32_f16 a[236:239], v[56:59], v[76:79], a[236:239]\n" \
  "ds_read_b128 v[56:59], v25 offset:63488\n" \
  "v_mfma_f32_16x16x32_f16 a[252:255], v[60:63], v[76:79], a[252:255]\n" \
  "buffer_store_dwordx4 a[40:43], v12, s[16:19], s40 offen offset:2048 sc1\n" \
  "v_mfma_f32_16x16x32_f16 a[248:251], v[60:63], v[72:75], a[248:251]\n" \
  "v_mfma_f32_16x16x32_f16 a[244:247], v[60:63], v[68:71], a[244:247]\n" \
  "s_waitcnt lgkmcnt(5)\n" \
  "v_mfma_f32_16x16x32_f16 a[240:243], v[60:63], v[64:67], a[240:243]\n" \
  "ds_read_b128 v[60:63], v25 offset:64512\n" \
  "v_mfma_f32_16x16x32_f16 a[128:131], v[32:35], v[224:227], a[128:131]\n" \
  "v_mfma_f32_16x16x32_f16 a[132:135], v[32:35], v[228:231], a[132:135]\n" \
  "v_mfma_f32_16x16x32_f16 a[136:139], v[32:35], v[232:235], a[136:139]\n" \
  "v_mfma_f32_16x16x32_f16 a[140:143], v[32:35], v[236:239], a[140:143]\n" \
  "ds_read_b128 v[32:35], v26 offset:8192\n" \
  "v_mfma_f32_16x16x32_f16 a[156:159], v[36:39], v[236:239], a[156:159]\n" \
  "buffer_store_dwordx4 a[44:47], v12, s[16:19], s40 offen offset:3072 sc1\n" \
  "v_mfma_f32_16x16x32_f16 a[152:155], v[36:39], v[232:235], a[152:155]\n" \
  "v_mfma_f32_16x16x32_f16 a[148:151], v[36:39], v[228:231], a[148:151]\n" \
  "s_waitcnt lgkmcnt(5)\n" \
  "v_mfma_f32_16x16x32_f16 a[144:147], v[36:39], v[224:227], a[144:147]\n" \
  "ds_read_b128 v[36:39], v26 offset:9216\n" \
  "v_mfma_f32_16x16x32_f16 a[160:163], v[40:43], v[224:227], a[160:163]\n" \
  "v_mfma_f32_16x16x32_f16 a[164:167], v[40:43], v[228:231], a[164:167]\n" \
  "v_mfma_f32_16x16x32_f16 a[168:171], v[40:43], v[232:235], a[168:171]\n" \
  "s_mov_b32 s40, 0x3000\n" \
  "v_mfma_f32_16x16x32_f16 a[172:175], v[40:43], v[236:239], a[172:175]\n" \
  "ds_read_b128 v[40:43], v26 offset:10240\n" \
  "v_mfma_f32_16x16x32_f16 a[188:191], v[44:47], v[236:239], a[188:191]\n" \
  "buffer_store_dwordx4 a[48:51], v12, s[16:19], s40 offen offset:0 sc1\n" \
  "v_mfma_f32_16x16x32_f16 a[184:187], v[44:47], v[232:235], a[184:187]\n" \
  "v_mfma_f32_16x16x32_f16 a[180:183], v[44:47], v[228:231], a[180:183]\n" \
  "s_waitcnt lgkmcnt(5)\n" \
  "v_mfma_f32_16x16x32_f16 a[176:179], v[44:47], v[224:227], a[176:179]\n" \
  "ds_read_b128 v[44:47], v26 offset:11264\n" \
  "v_mfma_f32_16x16x32_f16 a[192:195], v[48:51], v[224:227], a[192:195]\n" \
  "v_mfma_f32_16x16x32_f16 a[196:199], v[48:51], v[228:231], a[196:199]\n" \
  "v_mfma_f32_16x16x32_f16 a[200:203], v[48:51], v[232:235], a[200:203]\n" \
  "v_mfma_f32_16x16x32_f16 a[204:207], v[48:51], v[236:239], a[204:207]\n" \
  "ds_read_b128 v[48:51], v26 offset:12288\n" \
  "v_mfma_f32_16x16x32_f16 a[220:223], v[52:55], v[236:239], a[220:223]\n" \
  "buffer_store_dwordx4 a[52:55], v12, s[16:19], s40 offen offset:1024 sc1\n" \
  "v_mfma_f32_16x16x32_f16 a[216:219], v[52:55], v[232:235], a[216:219]\n" \
  "v_mfma_f32_16x16x32_f16 a[212:215], v[52:55], v[228:231], a[212:215]\n" \
  "s_waitcnt lgkmcnt(5)\n" \
  "v_mfma_f32_16x16x32_f16 a[208:211], v[52:55], v[224:227], a[208:211]\n" \
  "ds_read_b128 v[52:55], v26 offset:13312\n" \
  "v_mfma_f32_16x16x32_f16 a[224:227], v[56:59], v[224:227], a[224:227]\n" \
  "v_mfma_f32_16x16x32_f16 a[228:231], v[56:59], v[228:231], a[228:231]\n" \
  "v_mfma_f32_16x16x32_f16 a[232:235], v[56:59], v[232:235], a[232:235]\n" \
  "v_mfma_f32_16x16x32_f16 a[236:239], v[56:59], v[236:239], a[236:239]\n" \
  "ds_read_b128 v[56:59], v26 offset:14336\n" \
  "v_mfma_f32_16x16x32_f16 a[252:255], v[60:63], v[236:239], a[252:255]\n" \
  "buffer_store_dwordx4 a[56:59], v12, s[16:19], s40 offen offset:2048 sc1\n" \
  "v_mfma_f32_16x16x32_f16 a[248:251], v[60:63], v[232:235], a[248:251]\n" \
  "v_mfma_f32_16x16x32_f16 a[244:247], v[60:63], v[228:231], a[244:247]\n" \
  "s_waitcnt lgkmcnt(5)\n" \
  "v_mfma_f32_16x16x32_f16 a[240:243], v[60:63], v[224:227], a[240:243]\n" \
  "ds_read_b128 v[60:63], v26 offset:15360\n" \
  "v_mfma_f32_16x16x32_f16 a[128:131], v[32:35], v[240:243], a[128:131]\n" \
  "v_mfma_f32_16x16x32_f16 a[132:135], v[32:35], v[244:247], a[132:135]\n" \
  "v_mfma_f32_16x16x32_f16 a[136:139], v[32:35], v[248:251], a[136:139]\n" \
  "v_mfma_f32_16x16x32_f16 a[140:143], v[32:35], v[252:255], a[140:143]\n" \
  "ds_read_b128 v[32:35], v26 offset:24576\n" \
  "v_mfma_f32_16x16x32_f16 a[156:159], v[36:39], v[252:255], a[156:159]\n" \
  "buffer_store_dwordx4 a[60:63], v12, s[16:19], s40 offen offset:3072 sc1\n" \
  "v_mfma_f32_16x16x32_f16 a[152:155], v[36:39], v[248:251], a[152:155]\n" \
  "v_mfma_f32_16x16x32_f16 a[148:151], v[36:39], v[244:247], a[148:151]\n" \
  "s_waitcnt lgkmcnt(5)\n" \
  "v_mfma_f32_16x16x32_f16 a[144:147], v[36:39], v[240:243], a[144:147]\n" \
  "ds_read_b128 v[36:39], v26 offset:25600\n" \
  "v_mfma_f32_16x16x32_f16 a[160:163], v[40:43], v[240:243], a[160:163]\n" \
  "v_mfma_f32_16x16x32_f16 a[164:167], v[40:43], v[244:247], a[164:167]\n" \
  "v_mfma_f32_16x16x32_f16 a[168:171], v[40:43], v[248:251], a[168:171]\n" \
  "s_mov_b32 s40, 0x4000\n" \
  "v_mfma_f32_16x16x32_f16 a[172:175], v[40:43], v[252:255], a[172:175]\n" \
  "ds_read_b128 v[40:43], v26 offset:26624\n" \
  "v_mfma_f32_16x16x32_f16 a[188:191], v[44:47], v[252:255], a[188:191]\n" \
  "buffer_store_dwordx4 a[64:67], v12, s[16:19], s40 offen offset:0 sc1\n" \
  "v_mfma_f32_16x16x32_f16 a[184:187], v[44:47], v[248:251], a[184:187]\n" \
  "v_mfma_f32_16x16x32_f16 a[180:183], v[44:47], v[244:247], a[180:183]\n" \
  "s_waitcnt lgkmcnt(5)\n" \
  "v_mfma_f32_16x16x32_f16 a[176:179], v[44:47], v[240:243], a[176:179]\n" \
  "ds_read_b128 v[44:47], v26 offset:27648\n" \
  "v_mfma_f32_16x16x32_f16 a[192:195], v[48:51], v[240:243], a[192:195]\n" \
  "v_mfma_f32_16x16x32_f16 a[196:199], v[48:51], v[244:247], a[196:199]\n" \
  "v_mfma_f32_16x16x32_f16 a[200:203], v[48:51], v[248:251], a[200:203]\n" \
  "v_mfma_f32_16x16x32_f16 a[204:207], v[48:51], v[252:255], a[204:207]\n" \
  "ds_read_b128 v[48:51], v26 offset:28672\n" \
  "v_mfma_f32_16x16x32_f16 a[220:223], v[52:55], v[252:255], a[220:223]\n" \
  "buffer_store_dwordx4 a[68:71], v12, s[16:19], s40 offen offset:1024 sc1\n" \
  "v_mfma_f32_16x16x32_f16 a[216:219], v[52:55], v[248:251], a[216:219]\n" \
  "v_mfma_f32_16x16x32_f16 a[212:215], v[52:55], v[244:247], a[212:215]\n" \
  "s_waitcnt lgkmcnt(5)\n" \
  "v_mfma_f32_16x16x32_f16 a[208:211], v[52:55], v[240:243], a[208:211]\n" \
  "ds_read_b128 v[52:55], v26 offset:29696\n" \
  "v_mfma_f32_16x16x32_f16 a[224:227], v[56:59], v[240:243], a[224:227]\n" \
  "v_mfma_f32_16x16x32_f16 a[228:231], v[56:59], v[244:247], a[228:231]\n" \
  "v_mfma_f32_16x16x32_f16 a[232:235], v[56:59], v[248:251], a[232:235]\n" \
  "v_mfma_f32_16x16x32_f16 a[236:239], v[56:59], v[252:255], a[236:239]\n" \
  "ds_read_b128 v[56:59], v26 offset:30720\n" \
  "v_mfma_f32_16x16x32_f16 a[252:255], v[60:63], v[252:255], a[252:255]\n" \
  "buffer_store_dwordx4 a[72:75], v12, s[16:19], s40 offen offset:2048 sc1\n" \
  "v_mfma_f32_16x16x32_f16 a[248:251], v[60:63], v[248:251], a[248:251]\n" \
  "v_mfma_f32_16x16x32_f16 a[244:247], v[60:63], v[244:247], a[244:247]\n" \
  "s_waitcnt lgkmcnt(5)\n" \
  "v_mfma_f32_16x16x32_f16 a[240:243], v[60:63], v[240:243], a[240:243]\n" \
  "ds_read_b128 v[60:63], v26 offset:31744\n" \
  "v_mfma_f32_16x16x32_f16 a[128:131], v[32:35], v[0:3], a[128:131]\n" \
  "v_mfma_f32_16x16x32_f16 a[132:135], v[32:35], v[4:7], a[132:135]\n" \
  "v_mfma_f32_16x16x32_f16 a[136:139], v[32:35], v[8:11], a[136:139]\n" \
  "v_mfma_f32_16x16x32_f16 a[140:143], v[32:35], v[20:23], a[140:143]\n" \
  "ds_read_b128 v[32:35], v26 offset:40960\n" \
  "v_mfma_f32_16x16x32_f16 a[156:159], v[36:39], v[20:23], a[156:159]\n" \
  "buffer_store_dwordx4 a[76:79], v12, s[16:19], s40 offen offset:3072 sc1\n" \
  "v_mfma_f32_16x16x32_f16 a[152:155], v[36:39], v[8:11], a[152:155]\n" \
  "v_mfma_f32_16x16x32_f16 a[148:151], v[36:39], v[4:7], a[148:151]\n" \
  "s_waitcnt lgkmcnt(5)\n" \
  "v_mfma_f32_16x16x32_f16 a[144:147], v[36:39], v[0:3], a[144:147]\n" \
  "ds_read_b128 v[36:39], v26 offset:41984\n" \
  "v_mfma_f32_16x16x32_f16 a[160:163], v[40:43], v[0:3], a[160:163]\n" \
  "v_mfma_f32_16x16x32_f16 a[164:167], v[40:43], v[4:7], a[164:167]\n" \
  "v_mfma_f32_16x16x32_f16 a[168:171], v[40:43], v[8:11], a[168:171]\n" \
  "s_mov_b32 s40, 0x5000\n" \
  "v_mfma_f32_16x16x32_f16 a[172:175], v[40:43], v[20:23], a[172:175]\n" \
  "ds_read_b128 v[40:43], v26 offset:43008\n" \
  "v_mfma_f32_16x16x32_f16 a[188:191], v[44:47], v[20:23], a[188:191]\n" \
  "buffer_store_dwordx4 a[80:83], v12, s[16:19], s40 offen offset:0 sc1\n" \
  "v_mfma_f32_16x16x32_f16 a[184:187], v[44:47], v[8:11], a[184:187]\n" \
  "v_mfma_f32_16x16x32_f16 a[180:183], v[44:47], v[4:7], a[180:183]\n" \
  "s_waitcnt lgkmcnt(5)\n" \
  "v_mfma_f32_16x16x32_f16 a[176:179], v[44:47], v[0:3], a[176:179]\n" \
  "ds_read_b128 v[44:47], v26 offset:44032\n" \
  "v_mfma_f32_16x16x32_f16 a[192:195], v[48:51], v[0:3], a[192:195]\n" \
  "v_mfma_f32_16x16x32_f16 a[196:199], v[48:51], v[4:7], a[196:199]\n" \
  "v_mfma_f32_16x16x32_f16 a[200:203], v[48:51], v[8:11], a[200:203]\n" \
  "v_mfma_f32_16x16x32_f16 a[204:207], v[48:51], v[20:23], a[204:207]\n" \
  "ds_read_b128 v[48:51], v26 offset:45056\n" \
  "v_mfma_f32_16x16x32_f16 a[220:223], v[52:55], v[20:23], a[220:223]\n" \
  "buffer_store_dwordx4 a[84:87], v12, s[16:19], s40 offen offset:1024 sc1\n" \
  "v_mfma_f32_16x16x32_f16 a[216:219], v[52:55], v[8:11], a[216:219]\n" \
  "v_mfma_f32_16x16x32_f16 a[212:215], v[52:55], v[4:7], a[212:215]\n" \
  "s_waitcnt lgkmcnt(5)\n" \
  "v_mfma_f32_16x16x32_f16 a[208:211], v[52:55], v[0:3], a[208:211]\n" \
  "ds_read_b128 v[52:55], v26 offset:46080\n" \
  "v_mfma_f32_16x16x32_f16 a[224:227], v[56:59], v[0:3], a[224:227]\n" \
  "v_mfma_f32_16x16x32_f16 a[228:231], v[56:59], v[4:7], a[228:231]\n" \
  "v_mfma_f32_16x16x32_f16 a[232:235], v[56:59], v[8:11], a[232:235]\n" \
  "v_mfma_f32_16x16x32_f16 a[236:239], v[56:59], v[20:23], a[236:239]\n" \
  "ds_read_b128 v[56:59], v26 offset:47104\n" \
  "v_mfma_f32_16x16x32_f16 a[252:255], v[60:63], v[20:23], a[252:255]\n" \
  "buffer_store_dwordx4 a[88:91], v12, s[16:19], s40 offen offset:2048 sc1\n" \
  "v_mfma_f32_16x16x32_f16 a[248:251], v[60:63], v[8:11], a[248:251]\n" \
  "v_mfma_f32_16x16x32_f16 a[244:247], v[60:63], v[4:7], a[244:247]\n" \
  "s_waitcnt lgkmcnt(5)\n" \
  "v_mfma_f32_16x16x32_f16 a[240:243], v[60:63], v[0:3], a[240:243]\n" \
  "ds_read_b128 v[60:63], v26 offset:48128\n" \
  "v_mfma_f32_16x16x32_f16 a[128:131], v[32:35], v[128:131], a[128:131]\n" \
  "v_mfma_f32_16x16x32_f16 a[132:135], v[32:35], v[132:135], a[132:135]\n" \
  "v_mfma_f32_16x16x32_f16 a[136:139], v[32:35], v[136:139], a[136:139]\n" \
  "v_mfma_f32_16x16x32_f16 a[140:143], v[32:35], v[140:143], a[140:143]\n" \
  "ds_read_b128 v[32:35], v26 offset:57344\n" \
  "v_mfma_f32_16x16x32_f16 a[156:159], v[36:39], v[140:143], a[156:159]\n" \
  "buffer_store_dwordx4 a[92:95], v12, s[16:19], s40 offen offset:3072 sc1\n" \
  "v_mfma_f32_16x16x32_f16 a[152:155], v[36:39], v[136:139], a[152:155]\n" \
  "v_mfma_f32_16x16x32_f16 a[148:151], v[36:39], v[132:135], a[148:151]\n" \
  "s_waitcnt lgkmcnt(5)\n" \
  "v_mfma_f32_16x16x32_f16 a[144:147], v[36:39], v[128:131], a[144:147]\n" \
  "ds_read_b128 v[36:39], v26 offset:58368\n" \
  "v_mfma_f32_16x16x32_f16 a[160:163], v[40:43], v[128:131], a[160:163]\n" \
  "v_mfma_f32_16x16x32_f16 a[164:167], v[40:43], v[132:135], a[164:167]\n" \
  "v_mfma_f32_16x16x32_f16 a[168:171], v[40:43], v[136:139], a[168:171]\n" \
  "s_mov_b32 s40, 0x6000\n" \
  "v_mfma_f32_16x16x32_f16 a[172:175], v[40:43], v[140:143], a[172:175]\n" \
  "ds_read_b128 v[40:43], v26 offset:59392\n" \
  "v_mfma_f32_16x16x32_f16 a[188:191], v[44:47], v[140:143], a[188:191]\n" \
  "buffer_store_dwordx4 a[96:99], v12, s[16:19], s40 offen offset:0 sc1\n" \
  "v_mfma_f32_16x16x32_f16 a[184:187], v[44:47], v[136:139], a[184:187]\n" \
  "v_mfma_f32_16x16x32_f16 a[180:183], v[44:47], v[132:135], a[180:183]\n" \
  "s_waitcnt lgkmcnt(5)\n" \
  "v_mfma_f32_16x16x32_f16 a[176:179], v[44:47], v[128:131], a[176:179]\n" \
  "ds_read_b128 v[44:47], v26 offset:60416\n" \
  "v_mfma_f32_16x16x32_f16 a[192:195], v[48:51], v[128:131], a[192:195]\n" \
  "v_mfma_f32_16x16x32_f16 a[196:199], v[48:51], v[132:135], a[196:199]\n" \
  "v_mfma_f32_16x16x32_f16 a[200:203], v[48:51], v[136:139], a[200:203]\n" \
  "v_mfma_f32_16x16x32_f16 a[204:207], v[48:51], v[140:143], a[204:207]\n" \
  "ds_read_b128 v[48:51], v26 offset:61440\n" \
  "v_mfma_f32_16x16x32_f16 a[220:223], v[52:55], v[140:143], a[220:223]\n" \
  "buffer_store_dwordx4 a[100:103], v12, s[16:19], s40 offen offset:1024 sc1\n" \
  "v_mfma_f32_16x16x32_f16 a[216:219], v[52:55], v[136:139], a[216:219]\n" \
  "v_mfma_f32_16x16x32_f16 a[212:215], v[52:55], v[132:135], a[212:215]\n" \
  "s_waitcnt lgkmcnt(5)\n" \
  "v_mfma_f32_16x16x32_f16 a[208:211], v[52:55], v[128:131], a[208:211]\n" \
  "ds_read_b128 v[52:55], v26 offset:62464\n" \
  "v_mfma_f32_16x16x32_f16 a[224:227], v[56:59], v[128:131], a[224:227]\n" \
  "v_mfma_f32_16x16x32_f16 a[228:231], v[56:59], v[132:135], a[228:231]\n" \
  "v_mfma_f32_16x16x32_f16 a[232:235], v[56:59], v[136:139], a[232:235]\n" \
  "v_mfma_f32_16x16x32_f16 a[236:239], v[56:59], v[140:143], a[236:239]\n" \
  "ds_read_b128 v[56:59], v26 offset:63488\n" \
  "v_mfma_f32_16x16x32_f16 a[252:255], v[60:63], v[140:143], a[252:255]\n" \
  "buffer_store_dwordx4 a[104:107], v12, s[16:19], s40 offen offset:2048 sc1\n" \
  "v_mfma_f32_16x16x32_f16 a[248:251], v[60:63], v[136:139], a[248:251]\n" \
  "v_mfma_f32_16x16x32_f16 a[244:247], v[60:63], v[132:135], a[244:247]\n" \
  "s_waitcnt lgkmcnt(5)\n" \
  "v_mfma_f32_16x16x32_f16 a[240:243], v[60:63], v[128:131], a[240:243]\n" \
  "ds_read_b128 v[60:63], v26 offset:64512\n" \
  "v_mfma_f32_16x16x32_f16 a[128:131], v[32:35], v[144:147], a[128:131]\n" \
  "v_mfma_f32_16x16x32_f16 a[132:135], v[32:35], v[148:151], a[132:135]\n" \
  "v_mfma_f32_16x16x32_f16 a[136:139], v[32:35], v[152:155], a[136:139]\n" \
  "v_mfma_f32_16x16x32_f16 a[140:143], v[32:35], v[156:159], a[140:143]\n" \
  "ds_read_b128 v[32:35], v25 offset:8192\n" \
  "v_mfma_f32_16x16x32_f16 a[156:159], v[36:39], v[156:159], a[156:159]\n" \
  "buffer_store_dwordx4 a[108:111], v12, s[16:19], s40 offen offset:3072 sc1\n" \
  "v_mfma_f32_16x16x32_f16 a[152:155], v[36:39], v[152:155], a[152:155]\n" \
  "v_mfma_f32_16x16x32_f16 a[148:151], v[36:39], v[148:151], a[148:151]\n" \
  "s_waitcnt lgkmcnt(5)\n" \
  "v_mfma_f32_16x16x32_f16 a[144:147], v[36:39], v[144:147], a[144:147]\n" \
  "ds_read_b128 v[36:39], v25 offset:9216\n" \
  "v_mfma_f32_16x16x32_f16 a[160:163], v[40:43], v[144:147], a[160:163]\n" \
  "v_mfma_f32_16x16x32_f16 a[164:167], v[40:43], v[148:151], a[164:167]\n" \
  "v_mfma_f32_16x16x32_f16 a[168:171], v[40:43], v[152:155], a[168:171]\n" \
  "s_mov_b32 s40, 0x7000\n" \
  "v_mfma_f32_16x16x32_f16 a[172:175], v[40:43], v[156:159], a[172:175]\n" \
  "ds_read_b128 v[40:43], v25 offset:10240\n" \
  "v_mfma_f32_16x16x32_f16 a[188:191], v[44:47], v[156:159], a[188:191]\n" \
  "buffer_store_dwordx4 a[112:115], v12, s[16:19], s40 offen offset:0 sc1\n" \
  "v_mfma_f32_16x16x32_f16 a[184:187], v[44:47], v[152:155], a[184:187]\n" \
  "v_mfma_f32_16x16x32_f16 a[180:183], v[44:47], v[148:151], a[180:183]\n" \
  "s_waitcnt lgkmcnt(5)\n" \
  "v_mfma_f32_16x16x32_f16 a[176:179], v[44:47], v[144:147], a[176:179]\n" \
  "ds_read_b128 v[44:47], v25 offset:11264\n" \
  "v_mfma_f32_16x16x32_f16 a[192:195], v[48:51], v[144:147], a[192:195]\n" \
  "v_mfma_f32_16x16x32_f16 a[196:199], v[48:51], v[148:151], a[196:199]\n" \
  "v_mfma_f32_16x16x32_f16 a[200:203], v[48:51], v[152:155], a[200:203]\n" \
  "v_mfma_f32_16x16x32_f16 a[204:207], v[48:51], v[156:159], a[204:207]\n" \
  "ds_read_b128 v[48:51], v25 offset:12288\n" \
  "v_mfma_f32_16x16x32_f16 a[220:223], v[52:55], v[156:159], a[220:223]\n" \
  "buffer_store_dwordx4 a[116:119], v12, s[16:19], s40 offen offset:1024 sc1\n" \
  "v_mfma_f32_16x16x32_f16 a[216:219], v[52:55], v[152:155], a[216:219]\n" \
  "v_mfma_f32_16x16x32_f16 a[212:215], v[52:55], v[148:151], a[212:215]\n" \
  "s_waitcnt lgkmcnt(5)\n" \
  "v_mfma_f32_16x16x32_f16 a[208:211], v[52:55], v[144:147], a[208:211]\n" \
  "ds_read_b128 v[52:55], v25 offset:13312\n" \
  "v_mfma_f32_16x16x32_f16 a[224:227], v[56:59], v[144:147], a[224:227]\n" \
  "v_mfma_f32_16x16x32_f16 a[228:231], v[56:59], v[148:151], a[228:231]\n" \
  "v_mfma_f32_16x16x32_f16 a[232:235], v[56:59], v[152:155], a[232:235]\n" \
  "v_mfma_f32_16x16x32_f16 a[236:239], v[56:59], v[156:159], a[236:239]\n" \
  "ds_read_b128 v[56:59], v25 offset:14336\n" \
  "v_mfma_f32_16x16x32_f16 a[252:255], v[60:63], v[156:159], a[252:255]\n" \
  "buffer_store_dwordx4 a[120:123], v12, s[16:19], s40 offen offset:2048 sc1\n" \
  "v_mfma_f32_16x16x32_f16 a[248:251], v[60:63], v[152:155], a[248:251]\n" \
  "v_mfma_f32_16x16x32_f16 a[244:247], v[60:63], v[148:151], a[244:247]\n" \
  "s_waitcnt lgkmcnt(5)\n" \
  "v_mfma_f32_16x16x32_f16 a[240:243], v[60:63], v[144:147], a[240:243]\n" \
  "ds_read_b128 v[60:63], v25 offset:15360\n" \
  "v_mfma_f32_16x16x32_f16 a[128:131], v[32:35], v[160:163], a[128:131]\n" \
  "v_mfma_f32_16x16x32_f16 a[132:135], v[32:35], v[164:167], a[132:135]\n" \
  "v_mfma_f32_16x16x32_f16 a[136:139], v[32:35], v[168:171], a[136:139]\n" \
  "v_mfma_f32_16x16x32_f16 a[140:143], v[32:35], v[172:175], a[140:143]\n" \
  "v_mfma_f32_16x16x32_f16 a[156:159], v[36:39], v[172:175], a[156:159]\n" \
  "buffer_store_dwordx4 a[124:127], v12, s[16:19], s40 offen offset:3072 sc1\n" \
  "v_mfma_f32_16x16x32_f16 a[152:155], v[36:39], v[168:171], a[152:155]\n" \
  "v_mfma_f32_16x16x32_f16 a[148:151], v[36:39], v[164:167], a[148:151]\n" \
  "s_waitcnt lgkmcnt(4)\n" \
  "v_mfma_f32_16x16x32_f16 a[144:147], v[36:39], v[160:163], a[144:147]\n" \
  "v_mfma_f32_16x16x32_f16 a[160:163], v[40:43], v[160:163], a[160:163]\n" \
  "v_mfma_f32_16x16x32_f16 a[164:167], v[40:43], v[164:167], a[164:167]\n" \
  "v_mfma_f32_16x16x32_f16 a[168:171], v[40:43], v[168:171], a[168:171]\n" \
  "v_mfma_f32_16x16x32_f16 a[172:175], v[40:43], v[172:175], a[172:175]\n" \
  "v_mfma_f32_16x16x32_f16 a[188:191], v[44:47], v[172:175], a[188:191]\n" \
  "v_mfma_f32_16x16x32_f16 a[184:187], v[44:47], v[168:171], a[184:187]\n" \
  "v_mfma_f32_16x16x32_f16 a[180:183], v[44:47], v[164:167], a[180:183]\n" \
  "s_waitcnt lgkmcnt(2)\n" \
  "v_mfma_f32_16x16x32_f16 a[176:179], v[44:47], v[160:163], a[176:179]\n" \
  "v_mfma_f32_16x16x32_f16 a[192:195], v[48:51], v[160:163], a[192:195]\n" \
  "s_waitcnt vmcnt(16)\n" \
  "buffer_store_dword v15, v14, s[16:19], s59 offen sc1\n" \
  "v_mfma_f32_16x16x32_f16 a[196:199], v[48:51], v[164:167], a[196:199]\n" \
  "v_mfma_f32_16x16x32_f16 a[200:203], v[48:51], v[168:171], a[200:203]\n" \
  "v_mfma_f32_16x16x32_f16 a[204:207], v[48:51], v[172:175], a[204:207]\n" \
  "v_mfma_f32_16x16x32_f16 a[220:223], v[52:55], v[172:175], a[220:223]\n" \
  "v_mfma_f32_16x16x32_f16 a[216:219], v[52:55], v[168:171], a[216:219]\n" \
  "v_mfma_f32_16x16x32_f16 a[212:215], v[52:55], v[164:167], a[212:215]\n" \
  "s_waitcnt lgkmcnt(0)\n" \
  "v_mfma_f32_16x16x32_f16 a[208:211], v[52:55], v[160:163], a[208:211]\n" \
  "v_mfma_f32_16x16x32_f16 a[224:227], v[56:59], v[160:163], a[224:227]\n" \
  "v_mfma_f32_16x16x32_f16 a[228:231], v[56:59], v[164:167], a[228:231]\n" \
  "v_mfma_f32_16x16x32_f16 a[232:235], v[56:59], v[168:171], a[232:235]\n" \
  "v_mfma_f32_16x16x32_f16 a[236:239], v[56:59], v[172:175], a[236:239]\n" \
  "v_mfma_f32_16x16x32_f16 a[252:255], v[60:63], v[172:175], a[252:255]\n" \
  "v_mfma_f32_16x16x32_f16 a[248:251], v[60:63], v[168:171], a[248:251]\n" \
  "v_mfma_f32_16x16x32_f16 a[244:247], v[60:63], v[164:167], a[244:247]\n" \
  "v_mfma_f32_16x16x32_f16 a[240:243], v[60:63], v[160:163], a[240:243]\n" \
  "s_branch Lepi_%=\n" \
  ".p2align 6\n" \
  "Ltail2_%=:\n" \
  "v_mfma_f32_16x16x32_f16 a[240:243], v[60:63], v[80:83], a[240:243]\n" \
  "ds_read_b128 v[60:63], v25 offset:39936\n" \
  "v_mfma_f32_16x16x32_f16 a[0:3], v[32:35], v[64:67], a[0:3]\n" \
  "v_xor_b32 v26, 0x10000, v25\n" \
  "s_waitcnt vmcnt(47)\n" \
  "s_waitcnt vmcnt(37)\n" \
  "v_mfma_f32_16x16x32_f16 a[4:7], v[32:35], v[68:71], a[4:7]\n" \
  "v_cvt_pk_f16_f32 v80, v192, v196\n" \
  "v_cvt_pk_f16_f32 v84, v193, v197\n" \
  "s_barrier\n" \
  "v_mfma_f32_16x16x32_f16 a[8:11], v[32:35], v[72:75], a[8:11]\n" \
  "v_cvt_pk_f16_f32 v88, v194, v198\n" \
  "v_cvt_pk_f16_f32 v92, v195, v199\n" \
  "v_mfma_f32_16x16x32_f16 a[12:15], v[32:35], v[76:79], a[12:15]\n" \
  "ds_read_b128 v[32:35], v25 offset:49152\n" \
  "s_waitcnt vmcnt(45)\n" \
  "v_mfma_f32_16x16x32_f16 a[28:31], v[36:39], v[76:79], a[28:31]\n" \
  "v_cvt_pk_f16_f32 v81, v200, v204\n" \
  "v_cvt_pk_f16_f32 v85, v201, v205\n" \
  "v_mfma_f32_16x16x32_f16 a[24:27], v[36:39], v[72:75], a[24:27]\n" \
  "v_cvt_pk_f16_f32 v89, v202, v206\n" \
  "v_cvt_pk_f16_f32 v93, v203, v207\n" \
  "v_mfma_f32_16x16x32_f16 a[20:23], v[36:39], v[68:71], a[20:23]\n" \
  "s_waitcnt lgkmcnt(5)\n" \
  "s_bfe_u32 s40, s39, 0x80018\n" \
  "s_lshl_b32 s42, s40, 16\n" \
  "v_mfma_f32_16x16x32_f16 a[16:19], v[36:39], v[64:67], a[16:19]\n" \
  "ds_read_b128 v[36:39], v25 offset:50176\n" \
  "s_add_u32 m0, s36, 0xc000\n" \
  "v_mfma_f32_16x16x32_f16 a[32:35], v[40:43], v[64:67], a[32:35]\n" \
  "buffer_load_dwordx4 v24, s[12:15], s42 offen offset:0 sc0 sc1 lds\n" \
  "s_waitcnt vmcnt(44)\n" \
  "v_mfma_f32_16x16x32_f16 a[36:39], v[40:43], v[68:71], a[36:39]\n" \
  "v_cvt_pk_f16_f32 v82, v208, v212\n" \
  "v_cvt_pk_f16_f32 v86, v209, v213\n" \
  "v_mfma_f32_16x16x32_f16 a[40:43], v[40:43], v[72:75], a[40:43]\n" \
  "v_cvt_pk_f16_f32 v90, v210, v214\n" \
  "v_cvt_pk_f16_f32 v94, v211, v215\n" \
  "v_mfma_f32_16x16x32_f16 a[44:47], v[40:43], v[76:79], a[44:47]\n" \
  "ds_read_b128 v[40:43], v25 offset:51200\n" \
  "v_mfma_f32_16x16x32_f16 a[60:63], v[44:47], v[76:79], a[60:63]\n" \
  "buffer_load_dwordx4 v24, s[12:15], s42 offen offset:1024 sc0 sc1 lds\n" \
  "s_waitcnt vmcnt(43)\n" \
  "v_mfma_f32_16x16x32_f16 a[56:59], v[44:47], v[72:75], a[56:59]\n" \
  "v_cvt_pk_f16_f32 v83, v216, v220\n" \
  "v_cvt_pk_f16_f32 v87, v217, v221\n" \
  "v_mfma_f32_16x16x32_f16 a[52:55], v[44:47], v[68:71], a[52:55]\n" \
  "s_waitcnt lgkmcnt(5)\n" \
  "v_mfma_f32_16x16x32_f16 a[48:51], v[44:47], v[64:67], a[48:51]\n" \
  "ds_read_b128 v[44:47], v25 offset:52224\n" \
  "v_mfma_f32_16x16x32_f16 a[64:67], v[48:51], v[64:67], a[64:67]\n" \
  "buffer_load_dwordx4 v24, s[12:15], s42 offen offset:2048 sc0 sc1 lds\n" \
  "v_mfma_f32_16x16x32_f16 a[68:71], v[48:51], v[68:71], a[68:71]\n" \
  "v_cvt_pk_f16_f32 v91, v218, v222\n" \
  "v_cvt_pk_f16_f32 v95, v219, v223\n" \
  "v_mfma_f32_16x16x32_f16 a[72:75], v[48:51], v[72:75], a[72:75]\n" \
  "v_mfma_f32_16x16x32_f16 a[76:79], v[48:51], v[76:79], a[76:79]\n" \
  "ds_read_b128 v[48:51], v25 offset:53248\n" \
  "v_mfma_f32_16x16x32_f16 a[92:95], v[52:55], v[76:79], a[92:95]\n" \
  "buffer_load_dwordx4 v24, s[12:15], s42 offen offset:3072 sc0 sc1 lds\n" \
  "v_mfma_f32_16x16x32_f16 a[88:91], v[52:55], v[72:75], a[88:91]\n" \
  "v_mfma_f32_16x16x32_f16 a[84:87], v[52:55], v[68:71], a[84:87]\n" \
  "s_waitcnt lgkmcnt(5)\n" \
  "s_lshl_b32 s41, s40, 20\n" \
  "v_mfma_f32_16x16x32_f16 a[80:83], v[52:55], v[64:67], a[80:83]\n" \
  "ds_read_b128 v[52:55], v25 offset:54272\n" \
  "v_mfma_f32_16x16x32_f16 a[96:99], v[56:59], v[64:67], a[96:99]\n" \
  "buffer_load_dwordx4 v[192:195], v16, s[8:11], s41 offen\n" \
  "v_mfma_f32_16x16x32_f16 a[100:103], v[56:59], v[68:71], a[100:103]\n" \
  "v_mfma_f32_16x16x32_f16 a[104:107], v[56:59], v[72:75], a[104:107]\n" \
  "v_mfma_f32_16x16x32_f16 a[108:111], v[56:59], v[76:79], a[108:111]\n" \
  "ds_read_b128 v[56:59], v25 offset:55296\n" \
  "v_mfma_f32_16x16x32_f16 a[124:127], v[60:63], v[76:79], a[124:127]\n" \
  "buffer_load_dwordx4 v[196:199], v17, s[8:11], s41 offen\n" \
  "v_mfma_f32_16x16x32_f16 a[120:123], v[60:63], v[72:75], a[120:123]\n" \
  "s_waitcnt vmcnt(31)\n" \
  "v_mfma_f32_16x16x32_f16 a[116:119], v[60:63], v[68:71], a[116:119]\n" \
  "s_waitcnt lgkmcnt(5)\n" \
  "s_barrier\n" \
  "v_mfma_f32_16x16x32_f16 a[112:115], v[60:63], v[64:67], a[112:115]\n" \
  "ds_read_b128 v[60:63], v25 offset:56320\n" \
  "v_mfma_f32_16x16x32_f16 a[0:3], v[32:35], v[80:83], a[0:3]\n" \
  "buffer_load_dwordx4 v[200:203], v18, s[8:11], s41 offen\n" \
  "s_waitcnt vmcnt(42)\n" \
  "v_mfma_f32_16x16x32_f16 a[4:7], v[32:35], v[84:87], a[4:7]\n" \
  "v_cvt_pk_f16_f32 v224, v96, v100\n" \
  "v_cvt_pk_f16_f32 v228, v97, v101\n" \
  "v_mfma_f32_16x16x32_f16 a[8:11], v[32:35], v[88:91], a[8:11]\n" \
  "v_cvt_pk_f16_f32 v232, v98, v102\n" \
  "v_cvt_pk_f16_f32 v236, v99, v103\n" \
  "v_mfma_f32_16x16x32_f16 a[12:15], v[32:35], v[92:95], a[12:15]\n" \
  "ds_read_b128 v[32:35], v26 offset:0\n" \
  "s_add_u32 s46, s41, 0x80000\n" \
  "v_mfma_f32_16x16x32_f16 a[28:31], v[36:39], v[92:95], a[28:31]\n" \
  "buffer_load_dwordx4 v[204:207], v19, s[8:11], s41 offen\n" \
  "s_waitcnt vmcnt(41)\n" \
  "v_mfma_f32_16x16x32_f16 a[24:27], v[36:39], v[88:91], a[24:27]\n" \
  "v_cvt_pk_f16_f32 v225, v104, v108\n" \
  "v_cvt_pk_f16_f32 v229, v105, v109\n" \
  "v_mfma_f32_16x16x32_f16 a[20:23], v[36:39], v[84:87], a[20:23]\n" \
  "s_waitcnt lgkmcnt(5)\n" \
  "v_mfma_f32_16x16x32_f16 a[16:19], v[36:39], v[80:83], a[16:19]\n" \
  "ds_read_b128 v[36:39], v26 offset:1024\n" \
  "v_mfma_f32_16x16x32_f16 a[32:35], v[40:43], v[80:83], a[32:35]\n" \
  "buffer_load_dwordx4 v[208:211], v16, s[8:11], s46 offen\n" \
  "v_mfma_f32_16x16x32_f16 a[36:39], v[40:43], v[84:87], a[36:39]\n" \
  "v_cvt_pk_f16_f32 v233, v106, v110\n" \
  "v_cvt_pk_f16_f32 v237, v107, v111\n" \
  "s_waitcnt vmcnt(40)\n" \
  "v_mfma_f32_16x16x32_f16 a[40:43], v[40:43], v[88:91], a[40:43]\n" \
  "v_cvt_pk_f16_f32 v226, v112, v116\n" \
  "v_cvt_pk_f16_f32 v230, v113, v117\n" \
  "v_mfma_f32_16x16x32_f16 a[44:47], v[40:43], v[92:95], a[44:47]\n" \
  "ds_read_b128 v[40:43], v26 offset:2048\n" \
  "v_mfma_f32_16x16x32_f16 a[60:63], v[44:47], v[92:95], a[60:63]\n" \
  "buffer_load_dwordx4 v[212:215], v17, s[8:11], s46 offen\n" \
  "v_mfma_f32_16x16x32_f16 a[56:59], v[44:47], v[88:91], a[56:59]\n" \
  "v_cvt_pk_f16_f32 v234, v114, v118\n" \
  "v_cvt_pk_f16_f32 v238, v115, v119\n" \
  "v_mfma_f32_16x16x32_f16 a[52:55], v[44:47], v[84:87], a[52:55]\n" \
  "s_waitcnt lgkmcnt(5)\n" \
  "v_mfma_f32_16x16x32_f16 a[48:51], v[44:47], v[80:83], a[48:51]\n" \
  "ds_read_b128 v[44:47], v26 offset:3072\n" \
  "v_mfma_f32_16x16x32_f16 a[64:67], v[48:51], v[80:83], a[64:67]\n" \
  "buffer_load_dwordx4 v[216:219], v18, s[8:11], s46 offen\n" \
  "s_waitcnt vmcnt(40)\n" \
  "v_mfma_f32_16x16x32_f16 a[68:71], v[48:51], v[84:87], a[68:71]\n" \
  "v_cvt_pk_f16_f32 v227, v120, v124\n" \
  "v_cvt_pk_f16_f32 v231, v121, v125\n" \
  "v_mfma_f32_16x16x32_f16 a[72:75], v[48:51], v[88:91], a[72:75]\n" \
  "v_cvt_pk_f16_f32 v235, v122, v126\n" \
  "v_cvt_pk_f16_f32 v239, v123, v127\n" \
  "v_mfma_f32_16x16x32_f16 a[76:79], v[48:51], v[92:95], a[76:79]\n" \
  "ds_read_b128 v[48:51], v26 offset:4096\n" \
  "v_mfma_f32_16x16x32_f16 a[92:95], v[52:55], v[92:95], a[92:95]\n" \
  "s_waitcnt vmcnt(16)\n" \
  "v_cmp_le_u32 vcc, s43, v28\n" \
  "buffer_load_dwordx4 v[220:223], v19, s[8:11], s46 offen\n" \
  "v_mfma_f32_16x16x32_f16 a[88:91], v[52:55], v[88:91], a[88:91]\n" \
  "s_cmp_eq_u64 vcc, -1\n" \
  "s_cbranch_scc0 Lstub3_%=\n" \
  "Lback3_%=:\n" \
  "s_mov_b32 s39, s64\n" \
  "s_xor_b32 s36, s36, 0x10000\n" \
  "v_mfma_f32_16x16x32_f16 a[84:87], v[52:55], v[84:87], a[84:87]\n" \
  "s_waitcnt lgkmcnt(5)\n" \
  "s_bfe_u32 s40, s39, 0x80000\n" \
  "s_lshl_b32 s42, s40, 16\n" \
  "v_mfma_f32_16x16x32_f16 a[80:83], v[52:55], v[80:83], a[80:83]\n" \
  "ds_read_b128 v[52:55], v26 offset:5120\n" \
  "s_add_u32 m0, s36, 0x0\n" \
  "v_mfma_f32_16x16x32_f16 a[96:99], v[56:59], v[80:83], a[96:99]\n" \
  "buffer_load_dwordx4 v24, s[12:15], s42 offen offset:0 sc0 sc1 lds\n" \
  "v_mfma_f32_16x16x32_f16 a[100:103], v[56:59], v[84:87], a[100:103]\n" \
  "v_mfma_f32_16x16x32_f16 a[104:107], v[56:59], v[88:91], a[104:107]\n" \
  "v_mfma_f32_16x16x32_f16 a[108:111], v[56:59], v[92:95], a[108:111]\n" \
  "ds_read_b128 v[56:59], v26 offset:6144\n" \
  "v_mfma_f32_16x16x32_f16 a[124:127], v[60:63], v[92:95], a[124:127]\n" \
  "buffer_load_dwordx4 v24, s[12:15], s42 offen offset:1024 sc0 sc1 lds\n" \
  "v_mfma_f32_16x16x32_f16 a[120:123], v[60:63], v[88:91], a[120:123]\n" \
  "s_waitcnt vmcnt(27)\n" \
  "v_mfma_f32_16x16x32_f16 a[116:119], v[60:63], v[84:87], a[116:119]\n" \
  "s_waitcnt lgkmcnt(5)\n" \
  "s_barrier\n" \
  "v_mfma_f32_16x16x32_f16 a[112:115], v[60:63], v[80:83], a[112:115]\n" \
  "ds_read_b128 v[60:63], v26 offset:7168\n" \
  "v_mfma_f32_16x16x32_f16 a[0:3], v[32:35], v[224:227], a[0:3]\n" \
  "buffer_load_dwordx4 v24, s[12:15], s42 offen offset:2048 sc0 sc1 lds\n" \
  "s_waitcnt vmcnt(38)\n" \
  "v_mfma_f32_16x16x32_f16 a[4:7], v[32:35], v[228:231], a[4:7]\n" \
  "v_cvt_pk_f16_f32 v240, v128, v132\n" \
  "v_cvt_pk_f16_f32 v244, v129, v133\n" \
  "v_mfma_f32_16x16x32_f16 a[8:11], v[32:35], v[232:235], a[8:11]\n" \
  "v_cvt_pk_f16_f32 v248, v130, v134\n" \
  "v_cvt_pk_f16_f32 v252, v131, v135\n" \
  "v_mfma_f32_16x16x32_f16 a[12:15], v[32:35], v[236:239], a[12:15]\n" \
  "ds_read_b128 v[32:35], v26 offset:16384\n" \
  "v_mfma_f32_16x16x32_f16 a[28:31], v[36:39], v[236:239], a[28:31]\n" \
  "buffer_load_dwordx4 v24, s[12:15], s42 offen offset:3072 sc0 sc1 lds\n" \
  "s_waitcnt vmcnt(37)\n" \
  "v_mfma_f32_16x16x32_f16 a[24:27], v[36:39], v[232:235], a[24:27]\n" \
  "v_cvt_pk_f16_f32 v241, v136, v140\n" \
  "v_cvt_pk_f16_f32 v245, v137, v141\n" \
  "v_mfma_f32_16x16x32_f16 a[20:23], v[36:39], v[228:231], a[20:23]\n" \
  "s_waitcnt lgkmcnt(5)\n" \
  "s_lshl_b32 s41, s40, 20\n" \
  "v_mfma_f32_16x16x32_f16 a[16:19], v[36:39], v[224:227], a[16:19]\n" \
  "ds_read_b128 v[36:39], v26 offset:17408\n" \
  "v_mfma_f32_16x16x32_f16 a[32:35], v[40:43], v[224:227], a[32:35]\n" \
  "buffer_load_dwordx4 v[96:99], v16, s[8:11], s41 offen\n" \
  "v_mfma_f32_16x16x32_f16 a[36:39], v[40:43], v[228:231], a[36:39]\n" \
  "v_cvt_pk_f16_f32 v249, v138, v142\n" \
  "v_cvt_pk_f16_f32 v253, v139, v143\n" \
  "s_waitcnt vmcnt(36)\n" \
  "v_mfma_f32_16x16x32_f16 a[40:43], v[40:43], v[232:235], a[40:43]\n" \
  "v_cvt_pk_f16_f32 v242, v144, v148\n" \
  "v_cvt_pk_f16_f32 v246, v145, v149\n" \
  "v_mfma_f32_16x16x32_f16 a[44:47], v[40:43], v[236:239], a[44:47]\n" \
  "ds_read_b128 v[40:43], v26 offset:18432\n" \
  "v_mfma_f32_16x16x32_f16 a[60:63], v[44:47], v[236:239], a[60:63]\n" \
  "buffer_load_dwordx4 v[100:103], v17, s[8:11], s41 offen\n" \
  "v_mfma_f32_16x16x32_f16 a[56:59], v[44:47], v[232:235], a[56:59]\n" \
  "v_cvt_pk_f16_f32 v250, v146, v150\n" \
  "v_cvt_pk_f16_f32 v254, v147, v151\n" \
  "v_mfma_f32_16x16x32_f16 a[52:55], v[44:47], v[228:231], a[52:55]\n" \
  "s_waitcnt lgkmcnt(5)\n" \
  "v_mfma_f32_16x16x32_f16 a[48:51], v[44:47], v[224:227], a[48:51]\n" \
  "ds_read_b128 v[44:47], v26 offset:19456\n" \
  "v_mfma_f32_16x16x32_f16 a[64:67], v[48:51], v[224:227], a[64:67]\n" \
  "buffer_load_dwordx4 v[104:107], v18, s[8:11], s41 offen\n" \
  "s_waitcnt vmcnt(36)\n" \
  "v_mfma_f32_16x16x32_f16 a[68:71], v[48:51], v[228:231], a[68:71]\n" \
  "v_cvt_pk_f16_f32 v243, v152, v156\n" \
  "v_cvt_pk_f16_f32 v247, v153, v157\n" \
  "v_mfma_f32_16x16x32_f16 a[72:75], v[48:51], v[232:235], a[72:75]\n" \
  "v_cvt_pk_f16_f32 v251, v154, v158\n" \
  "v_cvt_pk_f16_f32 v255, v155, v159\n" \
  "v_mfma_f32_16x16x32_f16 a[76:79], v[48:51], v[236:239], a[76:79]\n" \
  "ds_read_b128 v[48:51], v26 offset:20480\n" \
  "s_add_u32 s46, s41, 0x80000\n" \
  "v_mfma_f32_16x16x32_f16 a[92:95], v[52:55], v[236:239], a[92:95]\n" \
  "buffer_load_dwordx4 v[108:111], v19, s[8:11], s41 offen\n" \
  "v_mfma_f32_16x16x32_f16 a[88:91], v[52:55], v[232:235], a[88:91]\n" \
  "v_mfma_f32_16x16x32_f16 a[84:87], v[52:55], v[228:231], a[84:87]\n" \
  "s_waitcnt lgkmcnt(5)\n" \
  "v_mfma_f32_16x16x32_f16 a[80:83], v[52:55], v[224:227], a[80:83]\n" \
  "ds_read_b128 v[52:55], v26 offset:21504\n" \
  "v_mfma_f32_16x16x32_f16 a[96:99], v[56:59], v[224:227], a[96:99]\n" \
  "buffer_load_dwordx4 v[112:115], v16, s[8:11], s46 offen\n" \
  "v_mfma_f32_16x16x32_f16 a[100:103], v[56:59], v[228:231], a[100:103]\n" \
  "v_mfma_f32_16x16x32_f16 a[104:107], v[56:59], v[232:235], a[104:107]\n" \
  "v_mfma_f32_16x16x32_f16 a[108:111], v[56:59], v[236:239], a[108:111]\n" \
  "ds_read_b128 v[56:59], v26 offset:22528\n" \
  "v_mfma_f32_16x16x32_f16 a[124:127], v[60:63], v[236:239], a[124:127]\n" \
  "buffer_load_dwordx4 v[116:119], v17, s[8:11], s46 offen\n" \
  "v_mfma_f32_16x16x32_f16 a[120:123], v[60:63], v[232:235], a[120:123]\n" \
  "s_waitcnt vmcnt(22)\n" \
  "v_mfma_f32_16x16x32_f16 a[116:119], v[60:63], v[228:231], a[116:119]\n" \
  "s_waitcnt lgkmcnt(5)\n" \
  "s_barrier\n" \
  "v_mfma_f32_16x16x32_f16 a[112:115], v[60:63], v[224:227], a[112:115]\n" \
  "ds_read_b128 v[60:63], v26 offset:23552\n" \
  "v_mfma_f32_16x16x32_f16 a[0:3], v[32:35], v[240:243], a[0:3]\n" \
  "buffer_load_dwordx4 v[120:123], v18, s[8:11], s46 offen\n" \
  "s_waitcnt vmcnt(34)\n" \
  "v_mfma_f32_16x16x32_f16 a[4:7], v[32:35], v[244:247], a[4:7]\n" \
  "v_cvt_pk_f16_f32 v0, v160, v164\n" \
  "v_cvt_pk_f16_f32 v4, v161, v165\n" \
  "v_mfma_f32_16x16x32_f16 a[8:11], v[32:35], v[248:251], a[8:11]\n" \
  "v_cvt_pk_f16_f32 v8, v162, v166\n" \
  "v_cvt_pk_f16_f32 v20, v163, v167\n" \
  "v_mfma_f32_16x16x32_f16 a[12:15], v[32:35], v[252:255], a[12:15]\n" \
  "ds_read_b128 v[32:35], v26 offset:32768\n" \
  "v_mfma_f32_16x16x32_f16 a[28:31], v[36:39], v[252:255], a[28:31]\n" \
  "buffer_load_dwordx4 v[124:127], v19, s[8:11], s46 offen\n" \
  "s_waitcnt vmcnt(33)\n" \
  "v_mfma_f32_16x16x32_f16 a[24:27], v[36:39], v[248:251], a[24:27]\n" \
  "v_cvt_pk_f16_f32 v1, v168, v172\n" \
  "v_cvt_pk_f16_f32 v5, v169, v173\n" \
  "v_mfma_f32_16x16x32_f16 a[20:23], v[36:39], v[244:247], a[20:23]\n" \
  "s_waitcnt lgkmcnt(5)\n" \
  "s_bfe_u32 s40, s39, 0x80008\n" \
  "s_lshl_b32 s42, s40, 16\n" \
  "v_mfma_f32_16x16x32_f16 a[16:19], v[36:39], v[240:243], a[16:19]\n" \
  "ds_read_b128 v[36:39], v26 offset:33792\n" \
  "s_add_u32 m0, s36, 0x4000\n" \
  "v_mfma_f32_16x16x32_f16 a[32:35], v[40:43], v[240:243], a[32:35]\n" \
  "buffer_load_dwordx4 v24, s[12:15], s42 offen offset:0 sc0 sc1 lds\n" \
  "v_mfma_f32_16x16x32_f16 a[36:39], v[40:43], v[244:247], a[36:39]\n" \
  "v_cvt_pk_f16_f32 v9, v170, v174\n" \
  "v_cvt_pk_f16_f32 v21, v171, v175\n" \
  "s_waitcnt vmcnt(32)\n" \
  "v_mfma_f32_16x16x32_f16 a[40:43], v[40:43], v[248:251], a[40:43]\n" \
  "v_cvt_pk_f16_f32 v2, v176, v180\n" \
  "v_cvt_pk_f16_f32 v6, v177, v181\n" \
  "v_mfma_f32_16x16x32_f16 a[44:47], v[40:43], v[252:255], a[44:47]\n" \
  "ds_read_b128 v[40:43], v26 offset:34816\n" \
  "v_mfma_f32_16x16x32_f16 a[60:63], v[44:47], v[252:255], a[60:63]\n" \
  "buffer_load_dwordx4 v24, s[12:15], s42 offen offset:1024 sc0 sc1 lds\n" \
  "v_mfma_f32_16x16x32_f16 a[56:59], v[44:47], v[248:251], a[56:59]\n" \
  "v_cvt_pk_f16_f32 v10, v178, v182\n" \
  "v_cvt_pk_f16_f32 v22, v179, v183\n" \
  "v_mfma_f32_16x16x32_f16 a[52:55], v[44:47], v[244:247], a[52:55]\n" \
  "s_waitcnt lgkmcnt(5)\n" \
  "v_mfma_f32_16x16x32_f16 a[48:51], v[44:47], v[240:243], a[48:51]\n" \
  "ds_read_b128 v[44:47], v26 offset:35840\n" \
  "v_mfma_f32_16x16x32_f16 a[64:67], v[48:51], v[240:243], a[64:67]\n" \
  "buffer_load_dwordx4 v24, s[12:15], s42 offen offset:2048 sc0 sc1 lds\n" \
  "s_waitcnt vmcnt(31)\n" \
  "v_mfma_f32_16x16x32_f16 a[68:71], v[48:51], v[244:247], a[68:71]\n" \
  "v_cvt_pk_f16_f32 v3, v184, v188\n" \
  "v_cvt_pk_f16_f32 v7, v185, v189\n" \
  "v_mfma_f32_16x16x32_f16 a[72:75], v[48:51], v[248:251], a[72:75]\n" \
  "v_cvt_pk_f16_f32 v11, v186, v190\n" \
  "v_cvt_pk_f16_f32 v23, v187, v191\n" \
  "v_mfma_f32_16x16x32_f16 a[76:79], v[48:51], v[252:255], a[76:79]\n" \
  "ds_read_b128 v[48:51], v26 offset:36864\n" \
  "v_mfma_f32_16x16x32_f16 a[92:95], v[52:55], v[252:255], a[92:95]\n" \
  "buffer_load_dwordx4 v24, s[12:15], s42 offen offset:3072 sc0 sc1 lds\n" \
  "v_mfma_f32_16x16x32_f16 a[88:91], v[52:55], v[248:251], a[88:91]\n" \
  "v_mfma_f32_16x16x32_f16 a[84:87], v[52:55], v[244:247], a[84:87]\n" \
  "s_waitcnt lgkmcnt(5)\n" \
  "s_lshl_b32 s41, s40, 20\n" \
  "v_mfma_f32_16x16x32_f16 a[80:83], v[52:55], v[240:243], a[80:83]\n" \
  "ds_read_b128 v[52:55], v26 offset:37888\n" \
  "v_mfma_f32_16x16x32_f16 a[96:99], v[56:59], v[240:243], a[96:99]\n" \
  "buffer_load_dwordx4 v[128:131], v16, s[8:11], s41 offen\n" \
  "v_mfma_f32_16x16x32_f16 a[100:103], v[56:59], v[244:247], a[100:103]\n" \
  "v_mfma_f32_16x16x32_f16 a[104:107], v[56:59], v[248:251], a[104:107]\n" \
  "v_mfma_f32_16x16x32_f16 a[108:111], v[56:59], v[252:255], a[108:111]\n" \
  "ds_read_b128 v[56:59], v26 offset:38912\n" \
  "v_mfma_f32_16x16x32_f16 a[124:127], v[60:63], v[252:255], a[124:127]\n" \
  "buffer_load_dwordx4 v[132:135], v17, s[8:11], s41 offen\n" \
  "v_mfma_f32_16x16x32_f16 a[120:123], v[60:63], v[248:251], a[120:123]\n" \
  "s_waitcnt vmcnt(26)\n" \
  "v_mfma_f32_16x16x32_f16 a[116:119], v[60:63], v[244:247], a[116:119]\n" \
  "s_waitcnt lgkmcnt(5)\n" \
  "s_barrier\n" \
  "v_mfma_f32_16x16x32_f16 a[112:115], v[60:63], v[240:243], a[112:115]\n" \
  "ds_read_b128 v[60:63], v26 offset:39936\n" \
  "v_mfma_f32_16x16x32_f16 a[0:3], v[32:35], v[0:3], a[0:3]\n" \
  "buffer_load_dwordx4 v[136:139], v18, s[8:11], s41 offen\n" \
  "s_waitcnt vmcnt(25)\n" \
  "v_mfma_f32_16x16x32_f16 a[4:7], v[32:35], v[4:7], a[4:7]\n" \
  "v_cvt_pk_f16_f32 v160, v192, v196\n" \
  "v_cvt_pk_f16_f32 v164, v193, v197\n" \
  "v_mfma_f32_16x16x32_f16 a[8:11], v[32:35], v[8:11], a[8:11]\n" \
  "v_cvt_pk_f16_f32 v168, v194, v198\n" \
  "v_cvt_pk_f16_f32 v172, v195, v199\n" \
  "v_mfma_f32_16x16x32_f16 a[12:15], v[32:35], v[20:23], a[12:15]\n" \
  "ds_read_b128 v[32:35], v26 offset:49152\n" \
  "s_add_u32 s46, s41, 0x80000\n" \
  "v_mfma_f32_16x16x32_f16 a[28:31], v[36:39], v[20:23], a[28:31]\n" \
  "buffer_load_dwordx4 v[140:143], v19, s[8:11], s41 offen\n" \
  "s_waitcnt vmcnt(24)\n" \
  "v_mfma_f32_16x16x32_f16 a[24:27], v[36:39], v[8:11], a[24:27]\n" \
  "v_cvt_pk_f16_f32 v161, v200, v204\n" \
  "v_cvt_pk_f16_f32 v165, v201, v205\n" \
  "v_mfma_f32_16x16x32_f16 a[20:23], v[36:39], v[4:7], a[20:23]\n" \
  "s_waitcnt lgkmcnt(5)\n" \
  "v_mfma_f32_16x16x32_f16 a[16:19], v[36:39], v[0:3], a[16:19]\n" \
  "ds_read_b128 v[36:39], v26 offset:50176\n" \
  "v_mfma_f32_16x16x32_f16 a[32:35], v[40:43], v[0:3], a[32:35]\n" \
  "buffer_load_dwordx4 v[144:147], v16, s[8:11], s46 offen\n" \
  "v_mfma_f32_16x16x32_f16 a[36:39], v[40:43], v[4:7], a[36:39]\n" \
  "v_cvt_pk_f16_f32 v169, v202, v206\n" \
  "v_cvt_pk_f16_f32 v173, v203, v207\n" \
  "s_waitcnt vmcnt(23)\n" \
  "v_mfma_f32_16x16x32_f16 a[40:43], v[40:43], v[8:11], a[40:43]\n" \
  "v_cvt_pk_f16_f32 v162, v208, v212\n" \
  "v_cvt_pk_f16_f32 v166, v209, v213\n" \
  "v_mfma_f32_16x16x32_f16 a[44:47], v[40:43], v[20:23], a[44:47]\n" \
  "ds_read_b128 v[40:43], v26 offset:51200\n" \
  "v_mfma_f32_16x16x32_f16 a[60:63], v[44:47], v[20:23], a[60:63]\n" \
  "buffer_load_dwordx4 v[148:151], v17, s[8:11], s46 offen\n" \
  "v_mfma_f32_16x16x32_f16 a[56:59], v[44:47], v[8:11], a[56:59]\n" \
  "v_cvt_pk_f16_f32 v170, v210, v214\n" \
  "v_cvt_pk_f16_f32 v174, v211, v215\n" \
  "v_mfma_f32_16x16x32_f16 a[52:55], v[44:47], v[4:7], a[52:55]\n" \
  "s_waitcnt lgkmcnt(5)\n" \
  "v_mfma_f32_16x16x32_f16 a[48:51], v[44:47], v[0:3], a[48:51]\n" \
  "ds_read_b128 v[44:47], v26 offset:52224\n" \
  "v_mfma_f32_16x16x32_f16 a[64:67], v[48:51], v[0:3], a[64:67]\n" \
  "buffer_load_dwordx4 v[152:155], v18, s[8:11], s46 offen\n" \
  "s_waitcnt vmcnt(23)\n" \
  "v_mfma_f32_16x16x32_f16 a[68:71], v[48:51], v[4:7], a[68:71]\n" \
  "v_cvt_pk_f16_f32 v163, v216, v220\n" \
  "v_cvt_pk_f16_f32 v167, v217, v221\n" \
  "v_mfma_f32_16x16x32_f16 a[72:75], v[48:51], v[8:11], a[72:75]\n" \
  "v_cvt_pk_f16_f32 v171, v218, v222\n" \
  "v_cvt_pk_f16_f32 v175, v219, v223\n" \
  "v_mfma_f32_16x16x32_f16 a[76:79], v[48:51], v[20:23], a[76:79]\n" \
  "ds_read_b128 v[48:51], v26 offset:53248\n" \
  "v_mfma_f32_16x16x32_f16 a[92:95], v[52:55], v[20:23], a[92:95]\n" \
  "buffer_load_dwordx4 v[156:159], v19, s[8:11], s46 offen\n" \
  "v_mfma_f32_16x16x32_f16 a[88:91], v[52:55], v[8:11], a[88:91]\n" \
  "v_mfma_f32_16x16x32_f16 a[84:87], v[52:55], v[4:7], a[84:87]\n" \
  "s_waitcnt lgkmcnt(5)\n" \
  "v_mfma_f32_16x16x32_f16 a[80:83], v[52:55], v[0:3], a[80:83]\n" \
  "ds_read_b128 v[52:55], v26 offset:54272\n" \
  "v_mfma_f32_16x16x32_f16 a[96:99], v[56:59], v[0:3], a[96:99]\n" \
  "v_mfma_f32_16x16x32_f16 a[100:103], v[56:59], v[4:7], a[100:103]\n" \
  "v_mfma_f32_16x16x32_f16 a[104:107], v[56:59], v[8:11], a[104:107]\n" \
  "v_mfma_f32_16x16x32_f16 a[108:111], v[56:59], v[20:23], a[108:111]\n" \
  "ds_read_b128 v[56:59], v26 offset:55296\n" \
  "v_mfma_f32_16x16x32_f16 a[124:127], v[60:63], v[20:23], a[124:127]\n" \
  "v_mfma_f32_16x16x32_f16 a[120:123], v[60:63], v[8:11], a[120:123]\n" \
  "s_waitcnt vmcnt(20)\n" \
  "v_mfma_f32_16x16x32_f16 a[116:119], v[60:63], v[4:7], a[116:119]\n" \
  "s_waitcnt lgkmcnt(5)\n" \
  "s_barrier\n" \
  "v_mfma_f32_16x16x32_f16 a[112:115], v[60:63], v[0:3], a[112:115]\n" \
  "ds_read_b128 v[60:63], v26 offset:56320\n" \
  "v_mfma_f32_16x16x32_f16 a[0:3], v[32:35], v[160:163], a[0:3]\n" \
  "s_waitcnt vmcnt(18)\n" \
  "v_mfma_f32_16x16x32_f16 a[4:7], v[32:35], v[164:167], a[4:7]\n" \
  "v_cvt_pk_f16_f32 v176, v96, v100\n" \
  "v_cvt_pk_f16_f32 v180, v97, v101\n" \
  "v_mfma_f32_16x16x32_f16 a[8:11], v[32:35], v[168:171], a[8:11]\n" \
  "v_cvt_pk_f16_f32 v184, v98, v102\n" \
  "v_cvt_pk_f16_f32 v188, v99, v103\n" \
  "v_mfma_f32_16x16x32_f16 a[12:15], v[32:35], v[172:175], a[12:15]\n" \
  "ds_read_b128 v[32:35], v25 offset:0\n" \
  "s_waitcnt vmcnt(16)\n" \
  "v_mfma_f32_16x16x32_f16 a[28:31], v[36:39], v[172:175], a[28:31]\n" \
  "v_cvt_pk_f16_f32 v177, v104, v108\n" \
  "v_cvt_pk_f16_f32 v181, v105, v109\n" \
  "v_mfma_f32_16x16x32_f16 a[24:27], v[36:39], v[168:171], a[24:27]\n" \
  "v_cvt_pk_f16_f32 v185, v106, v110\n" \
  "v_cvt_pk_f16_f32 v189, v107, v111\n" \
  "v_mfma_f32_16x16x32_f16 a[20:23], v[36:39], v[164:167], a[20:23]\n" \
  "s_waitcnt lgkmcnt(5)\n" \
  "v_mfma_f32_16x16x32_f16 a[16:19], v[36:39], v[160:163], a[16:19]\n" \
  "ds_read_b128 v[36:39], v25 offset:1024\n" \
  "s_waitcnt vmcnt(14)\n" \
  "v_mfma_f32_16x16x32_f16 a[32:35], v[40:43], v[160:163], a[32:35]\n" \
  "v_cvt_pk_f16_f32 v178, v112, v116\n" \
  "v_cvt_pk_f16_f32 v182, v113, v117\n" \
  "v_mfma_f32_16x16x32_f16 a[36:39], v[40:43], v[164:167], a[36:39]\n" \
  "v_cvt_pk_f16_f32 v186, v114, v118\n" \
  "v_cvt_pk_f16_f32 v190, v115, v119\n" \
  "s_waitcnt vmcnt(12)\n" \
  "v_mfma_f32_16x16x32_f16 a[40:43], v[40:43], v[168:171], a[40:43]\n" \
  "v_cvt_pk_f16_f32 v179, v120, v124\n" \
  "v_cvt_pk_f16_f32 v183, v121, v125\n" \
  "v_mfma_f32_16x16x32_f16 a[44:47], v[40:43], v[172:175], a[44:47]\n" \
  "ds_read_b128 v[40:43], v25 offset:2048\n" \
  "v_mfma_f32_16x16x32_f16 a[60:63], v[44:47], v[172:175], a[60:63]\n" \
  "v_cvt_pk_f16_f32 v187, v122, v126\n" \
  "v_cvt_pk_f16_f32 v191, v123, v127\n" \
  "v_mfma_f32_16x16x32_f16 a[56:59], v[44:47], v[168:171], a[56:59]\n" \
  "v_mfma_f32_16x16x32_f16 a[52:55], v[44:47], v[164:167], a[52:55]\n" \
  "s_waitcnt lgkmcnt(5)\n" \
  "v_mfma_f32_16x16x32_f16 a[48:51], v[44:47], v[160:163], a[48:51]\n" \
  "ds_read_b128 v[44:47], v25 offset:3072\n" \
  "v_mfma_f32_16x16x32_f16 a[64:67], v[48:51], v[160:163], a[64:67]\n" \
  "v_mfma_f32_16x16x32_f16 a[68:71], v[48:51], v[164:167], a[68:71]\n" \
  "v_mfma_f32_16x16x32_f16 a[72:75], v[48:51], v[168:171], a[72:75]\n" \
  "v_mfma_f32_16x16x32_f16 a[76:79], v[48:51], v[172:175], a[76:79]\n" \
  "ds_read_b128 v[48:51], v25 offset:4096\n" \
  "v_mfma_f32_16x16x32_f16 a[92:95], v[52:55], v[172:175], a[92:95]\n" \
  "v_mfma_f32_16x16x32_f16 a[88:91], v[52:55], v[168:171], a[88:91]\n" \
  "v_mfma_f32_16x16x32_f16 a[84:87], v[52:55], v[164:167], a[84:87]\n" \
  "s_waitcnt lgkmcnt(5)\n" \
  "v_mfma_f32_16x16x32_f16 a[80:83], v[52:55], v[160:163], a[80:83]\n" \
  "ds_read_b128 v[52:55], v25 offset:5120\n" \
  "v_mfma_f32_16x16x32_f16 a[96:99], v[56:59], v[160:163], a[96:99]\n" \
  "v_mfma_f32_16x16x32_f16 a[100:103], v[56:59], v[164:167], a[100:103]\n" \
  "v_mfma_f32_16x16x32_f16 a[104:107], v[56:59], v[168:171], a[104:107]\n" \
  "v_mfma_f32_16x16x32_f16 a[108:111], v[56:59], v[172:175], a[108:111]\n" \
  "ds_read_b128 v[56:59], v25 offset:6144\n" \
  "v_mfma_f32_16x16x32_f16 a[124:127], v[60:63], v[172:175], a[124:127]\n" \
  "v_mfma_f32_16x16x32_f16 a[120:123], v[60:63], v[168:171], a[120:123]\n" \
  "s_waitcnt vmcnt(8)\n" \
  "v_mfma_f32_16x16x32_f16 a[116:119], v[60:63], v[164:167], a[116:119]\n" \
  "s_waitcnt lgkmcnt(5)\n" \
  "s_barrier\n" \
  "v_mfma_f32_16x16x32_f16 a[112:115], v[60:63], v[160:163], a[112:115]\n" \
  "ds_read_b128 v[60:63], v25 offset:7168\n" \
  "v_mfma_f32_16x16x32_f16 a[0:3], v[32:35], v[176:179], a[0:3]\n" \
  "s_waitcnt vmcnt(6)\n" \
  "v_mfma_f32_16x16x32_f16 a[4:7], v[32:35], v[180:183], a[4:7]\n" \
  "v_cvt_pk_f16_f32 v192, v128, v132\n" \
  "v_cvt_pk_f16_f32 v196, v129, v133\n" \
  "v_mfma_f32_16x16x32_f16 a[8:11], v[32:35], v[184:187], a[8:11]\n" \
  "v_cvt_pk_f16_f32 v200, v130, v134\n" \
  "v_cvt_pk_f16_f32 v204, v131, v135\n" \
  "v_mfma_f32_16x16x32_f16 a[12:15], v[32:35], v[188:191], a[12:15]\n" \
  "ds_read_b128 v[32:35], v25 offset:16384\n" \
  "s_waitcnt vmcnt(4)\n" \
  "v_mfma_f32_16x16x32_f16 a[28:31], v[36:39], v[188:191], a[28:31]\n" \
  "v_cvt_pk_f16_f32 v193, v136, v140\n" \
  "v_cvt_pk_f16_f32 v197, v137, v141\n" \
  "v_mfma_f32_16x16x32_f16 a[24:27], v[36:39], v[184:187], a[24:27]\n" \
  "v_cvt_pk_f16_f32 v201, v138, v142\n" \
  "v_cvt_pk_f16_f32 v205, v139, v143\n" \
  "v_mfma_f32_16x16x32_f16 a[20:23], v[36:39], v[180:183], a[20:23]\n" \
  "s_waitcnt lgkmcnt(5)\n" \
  "v_mfma_f32_16x16x32_f16 a[16:19], v[36:39], v[176:179], a[16:19]\n" \
  "ds_read_b128 v[36:39], v25 offset:17408\n" \
  "s_waitcnt vmcnt(2)\n" \
  "v_mfma_f32_16x16x32_f16 a[32:35], v[40:43], v[176:179], a[32:35]\n" \
  "v_cvt_pk_f16_f32 v194, v144, v148\n" \
  "v_cvt_pk_f16_f32 v198, v145, v149\n" \
  "v_mfma_f32_16x16x32_f16 a[36:39], v[40:43], v[180:183], a[36:39]\n" \
  "v_cvt_pk_f16_f32 v202, v146, v150\n" \
  "v_cvt_pk_f16_f32 v206, v147, v151\n" \
  "s_waitcnt vmcnt(0)\n" \
  "v_mfma_f32_16x16x32_f16 a[40:43], v[40:43], v[184:187], a[40:43]\n" \
  "v_cvt_pk_f16_f32 v195, v152, v156\n" \
  "v_cvt_pk_f16_f32 v199, v153, v157\n" \
  "v_mfma_f32_16x16x32_f16 a[44:47], v[40:43], v[188:191], a[44:47]\n" \
  "ds_read_b128 v[40:43], v25 offset:18432\n" \
  "v_mfma_f32_16x16x32_f16 a[60:63], v[44:47], v[188:191], a[60:63]\n" \
  "v_cvt_pk_f16_f32 v203, v154, v158\n" \
  "v_cvt_pk_f16_f32 v207, v155, v159\n" \
  "v_mfma_f32_16x16x32_f16 a[56:59], v[44:47], v[184:187], a[56:59]\n" \
  "v_mfma_f32_16x16x32_f16 a[52:55], v[44:47], v[180:183], a[52:55]\n" \
  "s_waitcnt lgkmcnt(5)\n" \
  "v_mfma_f32_16x16x32_f16 a[48:51], v[44:47], v[176:179], a[48:51]\n" \
  "ds_read_b128 v[44:47], v25 offset:19456\n" \
  "v_mfma_f32_16x16x32_f16 a[64:67], v[48:51], v[176:179], a[64:67]\n" \
  "v_mfma_f32_16x16x32_f16 a[68:71], v[48:51], v[180:183], a[68:71]\n" \
  "v_mfma_f32_16x16x32_f16 a[72:75], v[48:51], v[184:187], a[72:75]\n" \
  "v_mfma_f32_16x16x32_f16 a[76:79], v[48:51], v[188:191], a[76:79]\n" \
  "ds_read_b128 v[48:51], v25 offset:20480\n" \
  "v_mfma_f32_16x16x32_f16 a[92:95], v[52:55], v[188:191], a[92:95]\n" \
  "v_mfma_f32_16x16x32_f16 a[88:91], v[52:55], v[184:187], a[88:91]\n" \
  "v_mfma_f32_16x16x32_f16 a[84:87], v[52:55], v[180:183], a[84:87]\n" \
  "s_waitcnt lgkmcnt(5)\n" \
  "v_mfma_f32_16x16x32_f16 a[80:83], v[52:55], v[176:179], a[80:83]\n" \
  "ds_read_b128 v[52:55], v25 offset:21504\n" \
  "v_mfma_f32_16x16x32_f16 a[96:99], v[56:59], v[176:179], a[96:99]\n" \
  "v_mfma_f32_16x16x32_f16 a[100:103], v[56:59], v[180:183], a[100:103]\n" \
  "v_mfma_f32_16x16x32_f16 a[104:107], v[56:59], v[184:187], a[104:107]\n" \
  "v_mfma_f32_16x16x32_f16 a[108:111], v[56:59], v[188:191], a[108:111]\n" \
  "ds_read_b128 v[56:59], v25 offset:22528\n" \
  "v_mfma_f32_16x16x32_f16 a[124:127], v[60:63], v[188:191], a[124:127]\n" \
  "v_mfma_f32_16x16x32_f16 a[120:123], v[60:63], v[184:187], a[120:123]\n" \
  "v_mfma_f32_16x16x32_f16 a[116:119], v[60:63], v[180:183], a[116:119]\n" \
  "s_waitcnt lgkmcnt(5)\n" \
  "v_mfma_f32_16x16x32_f16 a[112:115], v[60:63], v[176:179], a[112:115]\n" \
  "ds_read_b128 v[60:63], v25 offset:23552\n" \
  "v_mfma_f32_16x16x32_f16 a[0:3], v[32:35], v[192:195], a[0:3]\n" \
  "v_mfma_f32_16x16x32_f16 a[4:7], v[32:35], v[196:199], a[4:7]\n" \
  "v_mfma_f32_16x16x32_f16 a[8:11], v[32:35], v[200:203], a[8:11]\n" \
  "v_mfma_f32_16x16x32_f16 a[12:15], v[32:35], v[204:207], a[12:15]\n" \
  "ds_read_b128 v[32:35], v25 offset:40960\n" \
  "v_mfma_f32_16x16x32_f16 a[28:31], v[36:39], v[204:207], a[28:31]\n" \
  "v_mfma_f32_16x16x32_f16 a[24:27], v[36:39], v[200:203], a[24:27]\n" \
  "v_mfma_f32_16x16x32_f16 a[20:23], v[36:39], v[196:199], a[20:23]\n" \
  "s_waitcnt lgkmcnt(5)\n" \
  "v_mfma_f32_16x16x32_f16 a[16:19], v[36:39], v[192:195], a[16:19]\n" \
  "ds_read_b128 v[36:39], v25 offset:41984\n" \
  "v_mfma_f32_16x16x32_f16 a[32:35], v[40:43], v[192:195], a[32:35]\n" \
  "v_mfma_f32_16x16x32_f16 a[36:39], v[40:43], v[196:199], a[36:39]\n" \
  "v_mfma_f32_16x16x32_f16 a[40:43], v[40:43], v[200:203], a[40:43]\n" \
  "s_mov_b32 s40, 0x0\n" \
  "v_mfma_f32_16x16x32_f16 a[44:47], v[40:43], v[204:207], a[44:47]\n" \
  "ds_read_b128 v[40:43], v25 offset:43008\n" \
  "v_mfma_f32_16x16x32_f16 a[60:63], v[44:47], v[204:207], a[60:63]\n" \
  "buffer_store_dwordx4 a[0:3], v12, s[16:19], s40 offen offset:0 sc1\n" \
  "v_mfma_f32_16x16x32_f16 a[56:59], v[44:47], v[200:203], a[56:59]\n" \
  "v_mfma_f32_16x16x32_f16 a[52:55], v[44:47], v[196:199], a[52:55]\n" \
  "s_waitcnt lgkmcnt(5)\n" \
  "v_mfma_f32_16x16x32_f16 a[48:51], v[44:47], v[192:195], a[48:51]\n" \
  "ds_read_b128 v[44:47], v25 offset:44032\n" \
  "v_mfma_f32_16x16x32_f16 a[64:67], v[48:51], v[192:195], a[64:67]\n" \
  "v_mfma_f32_16x16x32_f16 a[68:71], v[48:51], v[196:199], a[68:71]\n" \
  "v_mfma_f32_16x16x32_f16 a[72:75], v[48:51], v[200:203], a[72:75]\n" \
  "v_mfma_f32_16x16x32_f16 a[76:79], v[48:51], v[204:207], a[76:79]\n" \
  "ds_read_b128 v[48:51], v25 offset:45056\n" \
  "v_mfma_f32_16x16x32_f16 a[92:95], v[52:55], v[204:207], a[92:95]\n" \
  "buffer_store_dwordx4 a[4:7], v12, s[16:19], s40 offen offset:1024 sc1\n" \
  "v_mfma_f32_16x16x32_f16 a[88:91], v[52:55], v[200:203], a[88:91]\n" \
  "v_mfma_f32_16x16x32_f16 a[84:87], v[52:55], v[196:199], a[84:87]\n" \
  "s_waitcnt lgkmcnt(5)\n" \
  "v_mfma_f32_16x16x32_f16 a[80:83], v[52:55], v[192:195], a[80:83]\n" \
  "ds_read_b128 v[52:55], v25 offset:46080\n" \
  "v_mfma_f32_16x16x32_f16 a[96:99], v[56:59], v[192:195], a[96:99]\n" \
  "v_mfma_f32_16x16x32_f16 a[100:103], v[56:59], v[196:199], a[100:103]\n" \
  "v_mfma_f32_16x16x32_f16 a[104:107], v[56:59], v[200:203], a[104:107]\n" \
  "v_mfma_f32_16x16x32_f16 a[108:111], v[56:59], v[204:207], a[108:111]\n" \
  "ds_read_b128 v[56:59], v25 offset:47104\n" \
  "v_mfma_f32_16x16x32_f16 a[124:127], v[60:63], v[204:207], a[124:127]\n" \
  "buffer_store_dwordx4 a[8:11], v12, s[16:19], s40 offen offset:2048 sc1\n" \
  "v_mfma_f32_16x16x32_f16 a[120:123], v[60:63], v[200:203], a[120:123]\n" \
  "v_mfma_f32_16x16x32_f16 a[116:119], v[60:63], v[196:199], a[116:119]\n" \
  "s_waitcnt lgkmcnt(5)\n" \
  "v_mfma_f32_16x16x32_f16 a[112:115], v[60:63], v[192:195], a[112:115]\n" \
  "ds_read_b128 v[60:63], v25 offset:48128\n" \
  "v_mfma_f32_16x16x32_f16 a[128:131], v[32:35], v[64:67], a[128:131]\n" \
  "v_mfma_f32_16x16x32_f16 a[132:135], v[32:35], v[68:71], a[132:135]\n" \
  "v_mfma_f32_16x16x32_f16 a[136:139], v[32:35], v[72:75], a[136:139]\n" \
  "v_mfma_f32_16x16x32_f16 a[140:143], v[32:35], v[76:79], a[140:143]\n" \
  "ds_read_b128 v[32:35], v25 offset:57344\n" \
  "v_mfma_f32_16x16x32_f16 a[156:159], v[36:39], v[76:79], a[156:159]\n" \
  "buffer_store_dwordx4 a[12:15], v12, s[16:19], s40 offen offset:3072 sc1\n" \
  "v_mfma_f32_16x16x32_f16 a[152:155], v[36:39], v[72:75], a[152:155]\n" \
  "v_mfma_f32_16x16x32_f16 a[148:151], v[36:39], v[68:71], a[148:151]\n" \
  "s_waitcnt lgkmcnt(5)\n" \
  "v_mfma_f32_16x16x32_f16 a[144:147], v[36:39], v[64:67], a[144:147]\n" \
  "ds_read_b128 v[36:39], v25 offset:58368\n" \
  "v_mfma_f32_16x16x32_f16 a[160:163], v[40:43], v[64:67], a[160:163]\n" \
  "v_mfma_f32_16x16x32_f16 a[164:167], v[40:43], v[68:71], a[164:167]\n" \
  "v_mfma_f32_16x16x32_f16 a[168:171], v[40:43], v[72:75], a[168:171]\n" \
  "s_mov_b32 s40, 0x1000\n" \
  "v_mfma_f32_16x16x32_f16 a[172:175], v[40:43], v[76:79], a[172:175]\n" \
  "ds_read_b128 v[40:43], v25 offset:59392\n" \
  "v_mfma_f32_16x16x32_f16 a[188:191], v[44:47], v[76:79], a[188:191]\n" \
  "buffer_store_dwordx4 a[16:19], v12, s[16:19], s40 offen offset:0 sc1\n" \
  "v_mfma_f32_16x16x32_f16 a[184:187], v[44:47], v[72:75], a[184:187]\n" \
  "v_mfma_f32_16x16x32_f16 a[180:183], v[44:47], v[68:71], a[180:183]\n" \
  "s_waitcnt lgkmcnt(5)\n" \
  "v_mfma_f32_16x16x32_f16 a[176:179], v[44:47], v[64:67], a[176:179]\n" \
  "ds_read_b128 v[44:47], v25 offset:60416\n" \
  "v_mfma_f32_16x16x32_f16 a[192:195], v[48:51], v[64:67], a[192:195]\n" \
  "v_mfma_f32_16x16x32_f16 a[196:199], v[48:51], v[68:71], a[196:199]\n" \
  "v_mfma_f32_16x16x32_f16 a[200:203], v[48:51], v[72:75], a[200:203]\n" \
  "v_mfma_f32_16x16x32_f16 a[204:207], v[48:51], v[76:79], a[204:207]\n" \
  "ds_read_b128 v[48:51], v25 offset:61440\n" \
  "v_mfma_f32_16x16x32_f16 a[220:223], v[52:55], v[76:79], a[220:223]\n" \
  "buffer_store_dwordx4 a[20:23], v12, s[16:19], s40 offen offset:1024 sc1\n" \
  "v_mfma_f32_16x16x32_f16 a[216:219], v[52:55], v[72:75], a[216:219]\n" \
  "v_mfma_f32_16x16x32_f16 a[212:215], v[52:55], v[68:71], a[212:215]\n" \
  "s_waitcnt lgkmcnt(5)\n" \
  "v_mfma_f32_16x16x32_f16 a[208:211], v[52:55], v[64:67], a[208:211]\n" \
  "ds_read_b128 v[52:55], v25 offset:62464\n" \
  "v_mfma_f32_16x16x32_f16 a[224:227], v[56:59], v[64:67], a[224:227]\n" \
  "v_mfma_f32_16x16x32_f16 a[228:231], v[56:59], v[68:71], a[228:231]\n" \
  "v_mfma_f32_16x16x32_f16 a[232:235], v[56:59], v[72:75], a[232:235]\n" \
  "v_mfma_f32_16x16x32_f16 a[236:239], v[56:59], v[76:79], a[236:239]\n" \
  "ds_read_b128 v[56:59], v25 offset:63488\n" \
  "v_mfma_f32_16x16x32_f16 a[252:255], v[60:63], v[76:79], a[252:255]\n" \
  "buffer_store_dwordx4 a[24:27], v12, s[16:19], s40 offen offset:2048 sc1\n" \
  "v_mfma_f32_16x16x32_f16 a[248:251], v[60:63], v[72:75], a[248:251]\n" \
  "v_mfma_f32_16x16x32_f16 a[244:247], v[60:63], v[68:71], a[244:247]\n" \
  "s_waitcnt lgkmcnt(5)\n" \
  "v_mfma_f32_16x16x32_f16 a[240:243], v[60:63], v[64:67], a[240:243]\n" \
  "ds_read_b128 v[60:63], v25 offset:64512\n" \
  "v_mfma_f32_16x16x32_f16 a[128:131], v[32:35], v[80:83], a[128:131]\n" \
  "v_mfma_f32_16x16x32_f16 a[132:135], v[32:35], v[84:87], a[132:135]\n" \
  "v_mfma_f32_16x16x32_f16 a[136:139], v[32:35], v[88:91], a[136:139]\n" \
  "v_mfma_f32_16x16x32_f16 a[140:143], v[32:35], v[92:95], a[140:143]\n" \
  "ds_read_b128 v[32:35], v26 offset:8192\n" \
  "v_mfma_f32_16x16x32_f16 a[156:159], v[36:39], v[92:95], a[156:159]\n" \
  "buffer_store_dwordx4 a[28:31], v12, s[16:19], s40 offen offset:3072 sc1\n" \
  "v_mfma_f32_16x16x32_f16 a[152:155], v[36:39], v[88:91], a[152:155]\n" \
  "v_mfma_f32_16x16x32_f16 a[148:151], v[36:39], v[84:87], a[148:151]\n" \
  "s_waitcnt lgkmcnt(5)\n" \
  "v_mfma_f32_16x16x32_f16 a[144:147], v[36:39], v[80:83], a[144:147]\n" \
  "ds_read_b128 v[36:39], v26 offset:9216\n" \
  "v_mfma_f32_16x16x32_f16 a[160:163], v[40:43], v[80:83], a[160:163]\n" \
  "v_mfma_f32_16x16x32_f16 a[164:167], v[40:43], v[84:87], a[164:167]\n" \
  "v_mfma_f32_16x16x32_f16 a[168:171], v[40:43], v[88:91], a[168:171]\n" \
  "s_mov_b32 s40, 0x2000\n" \
  "v_mfma_f32_16x16x32_f16 a[172:175], v[40:43], v[92:95], a[172:175]\n" \
  "ds_read_b128 v[40:43], v26 offset:10240\n" \
  "v_mfma_f32_16x16x32_f16 a[188:191], v[44:47], v[92:95], a[188:191]\n" \
  "buffer_store_dwordx4 a[32:35], v12, s[16:19], s40 offen offset:0 sc1\n" \
  "v_mfma_f32_16x16x32_f16 a[184:187], v[44:47], v[88:91], a[184:187]\n" \
  "v_mfma_f32_16x16x32_f16 a[180:183], v[44:47], v[84:87], a[180:183]\n" \
  "s_waitcnt lgkmcnt(5)\n" \
  "v_mfma_f32_16x16x32_f16 a[176:179], v[44:47], v[80:83], a[176:179]\n" \
  "ds_read_b128 v[44:47], v26 offset:11264\n" \
  "v_mfma_f32_16x16x32_f16 a[192:195], v[48:51], v[80:83], a[192:195]\n" \
  "v_mfma_f32_16x16x32_f16 a[196:199], v[48:51], v[84:87], a[196:199]\n" \
  "v_mfma_f32_16x16x32_f16 a[200:203], v[48:51], v[88:91], a[200:203]\n" \
  "v_mfma_f32_16x16x32_f16 a[204:207], v[48:51], v[92:95], a[204:207]\n" \
  "ds_read_b128 v[48:51], v26 offset:12288\n" \
  "v_mfma_f32_16x16x32_f16 a[220:223], v[52:55], v[92:95], a[220:223]\n" \
  "buffer_store_dwordx4 a[36:39], v12, s[16:19], s40 offen offset:1024 sc1\n" \
  "v_mfma_f32_16x16x32_f16 a[216:219], v[52:55], v[88:91], a[216:219]\n" \
  "v_mfma_f32_16x16x32_f16 a[212:215], v[52:55], v[84:87], a[212:215]\n" \
  "s_waitcnt lgkmcnt(5)\n" \
  "v_mfma_f32_16x16x32_f16 a[208:211], v[52:55], v[80:83], a[208:211]\n" \
  "ds_read_b128 v[52:55], v26 offset:13312\n" \
  "v_mfma_f32_16x16x32_f16 a[224:227], v[56:59], v[80:83], a[224:227]\n" \
  "v_mfma_f32_16x16x32_f16 a[228:231], v[56:59], v[84:87], a[228:231]\n" \
  "v_mfma_f32_16x16x32_f16 a[232:235], v[56:59], v[88:91], a[232:235]\n" \
  "v_mfma_f32_16x16x32_f16 a[236:239], v[56:59], v[92:95], a[236:239]\n" \
  "ds_read_b128 v[56:59], v26 offset:14336\n" \
  "v_mfma_f32_16x16x32_f16 a[252:255], v[60:63], v[92:95], a[252:255]\n" \
  "buffer_store_dwordx4 a[40:43], v12, s[16:19], s40 offen offset:2048 sc1\n" \
  "v_mfma_f32_16x16x32_f16 a[248:251], v[60:63], v[88:91], a[248:251]\n" \
  "v_mfma_f32_16x16x32_f16 a[244:247], v[60:63], v[84:87], a[244:247]\n" \
  "s_waitcnt lgkmcnt(5)\n" \
  "v_mfma_f32_16x16x32_f16 a[240:243], v[60:63], v[80:83], a[240:243]\n" \
  "ds_read_b128 v[60:63], v26 offset:15360\n" \
  "v_mfma_f32_16x16x32_f16 a[128:131], v[32:35], v[224:227], a[128:131]\n" \
  "v_mfma_f32_16x16x32_f16 a[132:135], v[32:35], v[228:231], a[132:135]\n" \
  "v_mfma_f32_16x16x32_f16 a[136:139], v[32:35], v[232:235], a[136:139]\n" \
  "v_mfma_f32_16x16x32_f16 a[140:143], v[32:35], v[236:239], a[140:143]\n" \
  "ds_read_b128 v[32:35], v26 offset:24576\n" \
  "v_mfma_f32_16x16x32_f16 a[156:159], v[36:39], v[236:239], a[156:159]\n" \
  "buffer_store_dwordx4 a[44:47], v12, s[16:19], s40 offen offset:3072 sc1\n" \
  "v_mfma_f32_16x16x32_f16 a[152:155], v[36:39], v[232:235], a[152:155]\n" \
  "v_mfma_f32_16x16x32_f16 a[148:151], v[36:39], v[228:231], a[148:151]\n" \
  "s_waitcnt lgkmcnt(5)\n" \
  "v_mfma_f32_16x16x32_f16 a[144:147], v[36:39], v[224:227], a[144:147]\n" \
  "ds_read_b128 v[36:39], v26 offset:25600\n" \
  "v_mfma_f32_16x16x32_f16 a[160:163], v[40:43], v[224:227], a[160:163]\n" \
  "v_mfma_f32_16x16x32_f16 a[164:167], v[40:43], v[228:231], a[164:167]\n" \
  "v_mfma_f32_16x16x32_f16 a[168:171], v[40:43], v[232:235], a[168:171]\n" \
  "s_mov_b32 s40, 0x3000\n" \
  "v_mfma_f32_16x16x32_f16 a[172:175], v[40:43], v[236:239], a[172:175]\n" \
  "ds_read_b128 v[40:43], v26 offset:26624\n" \
  "v_mfma_f32_16x16x32_f16 a[188:191], v[44:47], v[236:239], a[188:191]\n" \
  "buffer_store_dwordx4 a[48:51], v12, s[16:19], s40 offen offset:0 sc1\n" \
  "v_mfma_f32_16x16x32_f16 a[184:187], v[44:47], v[232:235], a[184:187]\n" \
  "v_mfma_f32_16x16x32_f16 a[180:183], v[44:47], v[228:231], a[180:183]\n" \
  "s_waitcnt lgkmcnt(5)\n" \
  "v_mfma_f32_16x16x32_f16 a[176:179], v[44:47], v[224:227], a[176:179]\n" \
  "ds_read_b128 v[44:47], v26 offset:27648\n" \
  "v_mfma_f32_16x16x32_f16 a[192:195], v[48:51], v[224:227], a[192:195]\n" \
  "v_mfma_f32_16x16x32_f16 a[196:199], v[48:51], v[228:231], a[196:199]\n" \
  "v_mfma_f32_16x16x32_f16 a[200:203], v[48:51], v[232:235], a[200:203]\n" \
  "v_mfma_f32_16x16x32_f16 a[204:207], v[48:51], v[236:239], a[204:207]\n" \
  "ds_read_b128 v[48:51], v26 offset:28672\n" \
  "v_mfma_f32_16x16x32_f16 a[220:223], v[52:55], v[236:239], a[220:223]\n" \
  "buffer_store_dwordx4 a[52:55], v12, s[16:19], s40 offen offset:1024 sc1\n" \
  "v_mfma_f32_16x16x32_f16 a[216:219], v[52:55], v[232:235], a[216:219]\n" \
  "v_mfma_f32_16x16x32_f16 a[212:215], v[52:55], v[228:231], a[212:215]\n" \
  "s_waitcnt lgkmcnt(5)\n" \
  "v_mfma_f32_16x16x32_f16 a[208:211], v[52:55], v[224:227], a[208:211]\n" \
  "ds_read_b128 v[52:55], v26 offset:29696\n" \
  "v_mfma_f32_16x16x32_f16 a[224:227], v[56:59], v[224:227], a[224:227]\n" \
  "v_mfma_f32_16x16x32_f16 a[228:231], v[56:59], v[228:231], a[228:231]\n" \
  "v_mfma_f32_16x16x32_f16 a[232:235], v[56:59], v[232:235], a[232:235]\n" \
  "v_mfma_f32_16x16x32_f16 a[236:239], v[56:59], v[236:239], a[236:239]\n" \
  "ds_read_b128 v[56:59], v26 offset:30720\n" \
  "v_mfma_f32_16x16x32_f16 a[252:255], v[60:63], v[236:239], a[252:255]\n" \
  "buffer_store_dwordx4 a[56:59], v12, s[16:19], s40 offen offset:2048 sc1\n" \
  "v_mfma_f32_16x16x32_f16 a[248:251], v[60:63], v[232:235], a[248:251]\n" \
  "v_mfma_f32_16x16x32_f16 a[244:247], v[60:63], v[228:231], a[244:247]\n" \
  "s_waitcnt lgkmcnt(5)\n" \
  "v_mfma_f32_16x16x32_f16 a[240:243], v[60:63], v[224:227], a[240:243]\n" \
  "ds_read_b128 v[60:63], v26 offset:31744\n" \
  "v_mfma_f32_16x16x32_f16 a[128:131], v[32:35], v[240:243], a[128:131]\n" \
  "v_mfma_f32_16x16x32_f16 a[132:135], v[32:35], v[244:247], a[132:135]\n" \
  "v_mfma_f32_16x16x32_f16 a[136:139], v[32:35], v[248:251], a[136:139]\n" \
  "v_mfma_f32_16x16x32_f16 a[140:143], v[32:35], v[252:255], a[140:143]\n" \
  "ds_read_b128 v[32:35], v26 offset:40960\n" \
  "v_mfma_f32_16x16x32_f16 a[156:159], v[36:39], v[252:255], a[156:159]\n" \
  "buffer_store_dwordx4 a[60:63], v12, s[16:19], s40 offen offset:3072 sc1\n" \
  "v_mfma_f32_16x16x32_f16 a[152:155], v[36:39], v[248:251], a[152:155]\n" \
  "v_mfma_f32_16x16x32_f16 a[148:151], v[36:39], v[244:247], a[148:151]\n" \
  "s_waitcnt lgkmcnt(5)\n" \
  "v_mfma_f32_16x16x32_f16 a[144:147], v[36:39], v[240:243], a[144:147]\n" \
  "ds_read_b128 v[36:39], v26 offset:41984\n" \
  "v_mfma_f32_16x16x32_f16 a[160:163], v[40:43], v[240:243], a[160:163]\n" \
  "v_mfma_f32_16x16x32_f16 a[164:167], v[40:43], v[244:247], a[164:167]\n" \
  "v_mfma_f32_16x16x32_f16 a[168:171], v[40:43], v[248:251], a[168:171]\n" \
  "s_mov_b32 s40, 0x4000\n" \
  "v_mfma_f32_16x16x32_f16 a[172:175], v[40:43], v[252:255], a[172:175]\n" \
  "ds_read_b128 v[40:43], v26 offset:43008\n" \
  "v_mfma_f32_16x16x32_f16 a[188:191], v[44:47], v[252:255], a[188:191]\n" \
  "buffer_store_dwordx4 a[64:67], v12, s[16:19], s40 offen offset:0 sc1\n" \
  "v_mfma_f32_16x16x32_f16 a[184:187], v[44:47], v[248:251], a[184:187]\n" \
  "v_mfma_f32_16x16x32_f16 a[180:183], v[44:47], v[244:247], a[180:183]\n" \
  "s_waitcnt lgkmcnt(5)\n" \
  "v_mfma_f32_16x16x32_f16 a[176:179], v[44:47], v[240:243], a[176:179]\n" \
  "ds_read_b128 v[44:47], v26 offset:44032\n" \
  "v_mfma_f32_16x16x32_f16 a[192:195], v[48:51], v[240:243], a[192:195]\n" \
  "v_mfma_f32_16x16x32_f16 a[196:199], v[48:51], v[244:247], a[196:199]\n" \
  "v_mfma_f32_16x16x32_f16 a[200:203], v[48:51], v[248:251], a[200:203]\n" \
  "v_mfma_f32_16x16x32_f16 a[204:207], v[48:51], v[252:255], a[204:207]\n" \
  "ds_read_b128 v[48:51], v26 offset:45056\n" \
  "v_mfma_f32_16x16x32_f16 a[220:223], v[52:55], v[252:255], a[220:223]\n" \
  "buffer_store_dwordx4 a[68:71], v12, s[16:19], s40 offen offset:1024 sc1\n" \
  "v_mfma_f32_16x16x32_f16 a[216:219], v[52:55], v[248:251], a[216:219]\n" \
  "v_mfma_f32_16x16x32_f16 a[212:215], v[52:55], v[244:247], a[212:215]\n" \
  "s_waitcnt lgkmcnt(5)\n" \
  "v_mfma_f32_16x16x32_f16 a[208:211], v[52:55], v[240:243], a[208:211]\n" \
  "ds_read_b128 v[52:55], v26 offset:46080\n" \
  "v_mfma_f32_16x16x32_f16 a[224:227], v[56:59], v[240:243], a[224:227]\n" \
  "v_mfma_f32_16x16x32_f16 a[228:231], v[56:59], v[244:247], a[228:231]\n" \
  "v_mfma_f32_16x16x32_f16 a[232:235], v[56:59], v[248:251], a[232:235]\n" \
  "v_mfma_f32_16x16x32_f16 a[236:239], v[56:59], v[252:255], a[236:239]\n" \
  "ds_read_b128 v[56:59], v26 offset:47104\n" \
  "v_mfma_f32_16x16x32_f16 a[252:255], v[60:63], v[252:255], a[252:255]\n" \
  "buffer_store_dwordx4 a[72:75], v12, s[16:19], s40 offen offset:2048 sc1\n" \
  "v_mfma_f32_16x16x32_f16 a[248:251], v[60:63], v[248:251], a[248:251]\n" \
  "v_mfma_f32_16x16x32_f16 a[244:247], v[60:63], v[244:247], a[244:247]\n" \
  "s_waitcnt lgkmcnt(5)\n" \
  "v_mfma_f32_16x16x32_f16 a[240:243], v[60:63], v[240:243], a[240:243]\n" \
  "ds_read_b128 v[60:63], v26 offset:48128\n" \
  "v_mfma_f32_16x16x32_f16 a[128:131], v[32:35], v[0:3], a[128:131]\n" \
  "v_mfma_f32_16x16x32_f16 a[132:135], v[32:35], v[4:7], a[132:135]\n" \
  "v_mfma_f32_16x16x32_f16 a[136:139], v[32:35], v[8:11], a[136:139]\n" \
  "v_mfma_f32_16x16x32_f16 a[140:143], v[32:35], v[20:23], a[140:143]\n" \
  "ds_read_b128 v[32:35], v26 offset:57344\n" \
  "v_mfma_f32_16x16x32_f16 a[156:159], v[36:39], v[20:23], a[156:159]\n" \
  "buffer_store_dwordx4 a[76:79], v12, s[16:19], s40 offen offset:3072 sc1\n" \
  "v_mfma_f32_16x16x32_f16 a[152:155], v[36:39], v[8:11], a[152:155]\n" \
  "v_mfma_f32_16x16x32_f16 a[148:151], v[36:39], v[4:7], a[148:151]\n" \
  "s_waitcnt lgkmcnt(5)\n" \
  "v_mfma_f32_16x16x32_f16 a[144:147], v[36:39], v[0:3], a[144:147]\n" \
  "ds_read_b128 v[36:39], v26 offset:58368\n" \
  "v_mfma_f32_16x16x32_f16 a[160:163], v[40:43], v[0:3], a[160:163]\n" \
  "v_mfma_f32_16x16x32_f16 a[164:167], v[40:43], v[4:7], a[164:167]\n" \
  "v_mfma_f32_16x16x32_f16 a[168:171], v[40:43], v[8:11], a[168:171]\n" \
  "s_mov_b32 s40, 0x5000\n" \
  "v_mfma_f32_16x16x32_f16 a[172:175], v[40:43], v[20:23], a[172:175]\n" \
  "ds_read_b128 v[40:43], v26 offset:59392\n" \
  "v_mfma_f32_16x16x32_f16 a[188:191], v[44:47], v[20:23], a[188:191]\n" \
  "buffer_store_dwordx4 a[80:83], v12, s[16:19], s40 offen offset:0 sc1\n" \
  "v_mfma_f32_16x16x32_f16 a[184:187], v[44:47], v[8:11], a[184:187]\n" \
  "v_mfma_f32_16x16x32_f16 a[180:183], v[44:47], v[4:7], a[180:183]\n" \
  "s_waitcnt lgkmcnt(5)\n" \
  "v_mfma_f32_16x16x32_f16 a[176:179], v[44:47], v[0:3], a[176:179]\n" \
  "ds_read_b128 v[44:47], v26 offset:60416\n" \
  "v_mfma_f32_16x16x32_f16 a[192:195], v[48:51], v[0:3], a[192:195]\n" \
  "v_mfma_f32_16x16x32_f16 a[196:199], v[48:51], v[4:7], a[196:199]\n" \
  "v_mfma_f32_16x16x32_f16 a[200:203], v[48:51], v[8:11], a[200:203]\n" \
  "v_mfma_f32_16x16x32_f16 a[204:207], v[48:51], v[20:23], a[204:207]\n" \
  "ds_read_b128 v[48:51], v26 offset:61440\n" \
  "v_mfma_f32_16x16x32_f16 a[220:223], v[52:55], v[20:23], a[220:223]\n" \
  "buffer_store_dwordx4 a[84:87], v12, s[16:19], s40 offen offset:1024 sc1\n" \
  "v_mfma_f32_16x16x32_f16 a[216:219], v[52:55], v[8:11], a[216:219]\n" \
  "v_mfma_f32_16x16x32_f16 a[212:215], v[52:55], v[4:7], a[212:215]\n" \
  "s_waitcnt lgkmcnt(5)\n" \
  "v_mfma_f32_16x16x32_f16 a[208:211], v[52:55], v[0:3], a[208:211]\n" \
  "ds_read_b128 v[52:55], v26 offset:62464\n" \
  "v_mfma_f32_16x16x32_f16 a[224:227], v[56:59], v[0:3], a[224:227]\n" \
  "v_mfma_f32_16x16x32_f16 a[228:231], v[56:59], v[4:7], a[228:231]\n" \
  "v_mfma_f32_16x16x32_f16 a[232:235], v[56:59], v[8:11], a[232:235]\n" \
  "v_mfma_f32_16x16x32_f16 a[236:239], v[56:59], v[20:23], a[236:239]\n" \
  "ds_read_b128 v[56:59], v26 offset:63488\n" \
  "v_mfma_f32_16x16x32_f16 a[252:255], v[60:63], v[20:23], a[252:255]\n" \
  "buffer_store_dwordx4 a[88:91], v12, s[16:19], s40 offen offset:2048 sc1\n" \
  "v_mfma_f32_16x16x32_f16 a[248:251], v[60:63], v[8:11], a[248:251]\n" \
  "v_mfma_f32_16x16x32_f16 a[244:247], v[60:63], v[4:7], a[244:247]\n" \
  "s_waitcnt lgkmcnt(5)\n" \
  "v_mfma_f32_16x16x32_f16 a[240:243], v[60:63], v[0:3], a[240:243]\n" \
  "ds_read_b128 v[60:63], v26 offset:64512\n" \
  "v_mfma_f32_16x16x32_f16 a[128:131], v[32:35], v[160:163], a[128:131]\n" \
  "v_mfma_f32_16x16x32_f16 a[132:135], v[32:35], v[164:167], a[132:135]\n" \
  "v_mfma_f32_16x16x32_f16 a[136:139], v[32:35], v[168:171], a[136:139]\n" \
  "v_mfma_f32_16x16x32_f16 a[140:143], v[32:35], v[172:175], a[140:143]\n" \
  "ds_read_b128 v[32:35], v25 offset:8192\n" \
  "v_mfma_f32_16x16x32_f16 a[156:159], v[36:39], v[172:175], a[156:159]\n" \
  "buffer_store_dwordx4 a[92:95], v12, s[16:19], s40 offen offset:3072 sc1\n" \
  "v_mfma_f32_16x16x32_f16 a[152:155], v[36:39], v[168:171], a[152:155]\n" \
  "v_mfma_f32_16x16x32_f16 a[148:151], v[36:39], v[164:167], a[148:151]\n" \
  "s_waitcnt lgkmcnt(5)\n" \
  "v_mfma_f32_16x16x32_f16 a[144:147], v[36:39], v[160:163], a[144:147]\n" \
  "ds_read_b128 v[36:39], v25 offset:9216\n" \
  "v_mfma_f32_16x16x32_f16 a[160:163], v[40:43], v[160:163], a[160:163]\n" \
  "v_mfma_f32_16x16x32_f16 a[164:167], v[40:43], v[164:167], a[164:167]\n" \
  "v_mfma_f32_16x16x32_f16 a[168:171], v[40:43], v[168:171], a[168:171]\n" \
  "s_mov_b32 s40, 0x6000\n" \
  "v_mfma_f32_16x16x32_f16 a[172:175], v[40:43], v[172:175], a[172:175]\n" \
  "ds_read_b128 v[40:43], v25 offset:10240\n" \
  "v_mfma_f32_16x16x32_f16 a[188:191], v[44:47], v[172:175], a[188:191]\n" \
  "buffer_store_dwordx4 a[96:99], v12, s[16:19], s40 offen offset:0 sc1\n" \
  "v_mfma_f32_16x16x32_f16 a[184:187], v[44:47], v[168:171], a[184:187]\n" \
  "v_mfma_f32_16x16x32_f16 a[180:183], v[44:47], v[164:167], a[180:183]\n" \
  "s_waitcnt lgkmcnt(5)\n" \
  "v_mfma_f32_16x16x32_f16 a[176:179], v[44:47], v[160:163], a[176:179]\n" \
  "ds_read_b128 v[44:47], v25 offset:11264\n" \
  "v_mfma_f32_16x16x32_f16 a[192:195], v[48:51], v[160:163], a[192:195]\n" \
  "v_mfma_f32_16x16x32_f16 a[196:199], v[48:51], v[164:167], a[196:199]\n" \
  "v_mfma_f32_16x16x32_f16 a[200:203], v[48:51], v[168:171], a[200:203]\n" \
  "v_mfma_f32_16x16x32_f16 a[204:207], v[48:51], v[172:175], a[204:207]\n" \
  "ds_read_b128 v[48:51], v25 offset:12288\n" \
  "v_mfma_f32_16x16x32_f16 a[220:223], v[52:55], v[172:175], a[220:223]\n" \
  "buffer_store_dwordx4 a[100:103], v12, s[16:19], s40 offen offset:1024 sc1\n" \
  "v_mfma_f32_16x16x32_f16 a[216:219], v[52:55], v[168:171], a[216:219]\n" \
  "v_mfma_f32_16x16x32_f16 a[212:215], v[52:55], v[164:167], a[212:215]\n" \
  "s_waitcnt lgkmcnt(5)\n" \
  "v_mfma_f32_16x16x32_f16 a[208:211], v[52:55], v[160:163], a[208:211]\n" \
  "ds_read_b128 v[52:55], v25 offset:13312\n" \
  "v_mfma_f32_16x16x32_f16 a[224:227], v[56:59], v[160:163], a[224:227]\n" \
  "v_mfma_f32_16x16x32_f16 a[228:231], v[56:59], v[164:167], a[228:231]\n" \
  "v_mfma_f32_16x16x32_f16 a[232:235], v[56:59], v[168:171], a[232:235]\n" \
  "v_mfma_f32_16x16x32_f16 a[236:239], v[56:59], v[172:175], a[236:239]\n" \
  "ds_read_b128 v[56:59], v25 offset:14336\n" \
  "v_mfma_f32_16x16x32_f16 a[252:255], v[60:63], v[172:175], a[252:255]\n" \
  "buffer_store_dwordx4 a[104:107], v12, s[16:19], s40 offen offset:2048 sc1\n" \
  "v_mfma_f32_16x16x32_f16 a[248:251], v[60:63], v[168:171], a[248:251]\n" \
  "v_mfma_f32_16x16x32_f16 a[244:247], v[60:63], v[164:167], a[244:247]\n" \
  "s_waitcnt lgkmcnt(5)\n" \
  "v_mfma_f32_16x16x32_f16 a[240:243], v[60:63], v[160:163], a[240:243]\n" \
  "ds_read_b128 v[60:63], v25 offset:15360\n" \
  "v_mfma_f32_16x16x32_f16 a[128:131], v[32:35], v[176:179], a[128:131]\n" \
  "v_mfma_f32_16x16x32_f16 a[132:135], v[32:35], v[180:183], a[132:135]\n" \
  "v_mfma_f32_16x16x32_f16 a[136:139], v[32:35], v[184:187], a[136:139]\n" \
  "v_mfma_f32_16x16x32_f16 a[140:143], v[32:35], v[188:191], a[140:143]\n" \
  "ds_read_b128 v[32:35], v25 offset:24576\n" \
  "v_mfma_f32_16x16x32_f16 a[156:159], v[36:39], v[188:191], a[156:159]\n" \
  "buffer_store_dwordx4 a[108:111], v12, s[16:19], s40 offen offset:3072 sc1\n" \
  "v_mfma_f32_16x16x32_f16 a[152:155], v[36:39], v[184:187], a[152:155]\n" \
  "v_mfma_f32_16x16x32_f16 a[148:151], v[36:39], v[180:183], a[148:151]\n" \
  "s_waitcnt lgkmcnt(5)\n" \
  "v_mfma_f32_16x16x32_f16 a[144:147], v[36:39], v[176:179], a[144:147]\n" \
  "ds_read_b128 v[36:39], v25 offset:25600\n" \
  "v_mfma_f32_16x16x32_f16 a[160:163], v[40:43], v[176:179], a[160:163]\n" \
  "v_mfma_f32_16x16x32_f16 a[164:167], v[40:43], v[180:183], a[164:167]\n" \
  "v_mfma_f32_16x16x32_f16 a[168:171], v[40:43], v[184:187], a[168:171]\n" \
  "s_mov_b32 s40, 0x7000\n" \
  "v_mfma_f32_16x16x32_f16 a[172:175], v[40:43], v[188:191], a[172:175]\n" \
  "ds_read_b128 v[40:43], v25 offset:26624\n" \
  "v_mfma_f32_16x16x32_f16 a[188:191], v[44:47], v[188:191], a[188:191]\n" \
  "buffer_store_dwordx4 a[112:115], v12, s[16:19], s40 offen offset:0 sc1\n" \
  "v_mfma_f32_16x16x32_f16 a[184:187], v[44:47], v[184:187], a[184:187]\n" \
  "v_mfma_f32_16x16x32_f16 a[180:183], v[44:47], v[180:183], a[180:183]\n" \
  "s_waitcnt lgkmcnt(5)\n" \
  "v_mfma_f32_16x16x32_f16 a[176:179], v[44:47], v[176:179], a[176:179]\n" \
  "ds_read_b128 v[44:47], v25 offset:27648\n" \
  "v_mfma_f32_16x16x32_f16 a[192:195], v[48:51], v[176:179], a[192:195]\n" \
  "v_mfma_f32_16x16x32_f16 a[196:199], v[48:51], v[180:183], a[196:199]\n" \
  "v_mfma_f32_16x16x32_f16 a[200:203], v[48:51], v[184:187], a[200:203]\n" \
  "v_mfma_f32_16x16x32_f16 a[204:207], v[48:51], v[188:191], a[204:207]\n" \
  "ds_read_b128 v[48:51], v25 offset:28672\n" \
  "v_mfma_f32_16x16x32_f16 a[220:223], v[52:55], v[188:191], a[220:223]\n" \
  "buffer_store_dwordx4 a[116:119], v12, s[16:19], s40 offen offset:1024 sc1\n" \
  "v_mfma_f32_16x16x32_f16 a[216:219], v[52:55], v[184:187], a[216:219]\n" \
  "v_mfma_f32_16x16x32_f16 a[212:215], v[52:55], v[180:183], a[212:215]\n" \
  "s_waitcnt lgkmcnt(5)\n" \
  "v_mfma_f32_16x16x32_f16 a[208:211], v[52:55], v[176:179], a[208:211]\n" \
  "ds_read_b128 v[52:55], v25 offset:29696\n" \
  "v_mfma_f32_16x16x32_f16 a[224:227], v[56:59], v[176:179], a[224:227]\n" \
  "v_mfma_f32_16x16x32_f16 a[228:231], v[56:59], v[180:183], a[228:231]\n" \
  "v_mfma_f32_16x16x32_f16 a[232:235], v[56:59], v[184:187], a[232:235]\n" \
  "v_mfma_f32_16x16x32_f16 a[236:239], v[56:59], v[188:191], a[236:239]\n" \
  "ds_read_b128 v[56:59], v25 offset:30720\n" \
  "v_mfma_f32_16x16x32_f16 a[252:255], v[60:63], v[188:191], a[252:255]\n" \
  "buffer_store_dwordx4 a[120:123], v12, s[16:19], s40 offen offset:2048 sc1\n" \
  "v_mfma_f32_16x16x32_f16 a[248:251], v[60:63], v[184:187], a[248:251]\n" \
  "v_mfma_f32_16x16x32_f16 a[244:247], v[60:63], v[180:183], a[244:247]\n" \
  "s_waitcnt lgkmcnt(5)\n" \
  "v_mfma_f32_16x16x32_f16 a[240:243], v[60:63], v[176:179], a[240:243]\n" \
  "ds_read_b128 v[60:63], v25 offset:31744\n" \
  "v_mfma_f32_16x16x32_f16 a[128:131], v[32:35], v[192:195], a[128:131]\n" \
  "v_mfma_f32_16x16x32_f16 a[132:135], v[32:35], v[196:199], a[132:135]\n" \
  "v_mfma_f32_16x16x32_f16 a[136:139], v[32:35], v[200:203], a[136:139]\n" \
  "v_mfma_f32_16x16x32_f16 a[140:143], v[32:35], v[204:207], a[140:143]\n" \
  "v_mfma_f32_16x16x32_f16 a[156:159], v[36:39], v[204:207], a[156:159]\n" \
  "buffer_store_dwordx4 a[124:127], v12, s[16:19], s40 offen offset:3072 sc1\n" \
  "v_mfma_f32_16x16x32_f16 a[152:155], v[36:39], v[200:203], a[152:155]\n" \
  "v_mfma_f32_16x16x32_f16 a[148:151], v[36:39], v[196:199], a[148:151]\n" \
  "s_waitcnt lgkmcnt(4)\n" \
  "v_mfma_f32_16x16x32_f16 a[144:147], v[36:39], v[192:195], a[144:147]\n" \
  "v_mfma_f32_16x16x32_f16 a[160:163], v[40:43], v[192:195], a[160:163]\n" \
  "v_mfma_f32_16x16x32_f16 a[164:167], v[40:43], v[196:199], a[164:167]\n" \
  "v_mfma_f32_16x16x32_f16 a[168:171], v[40:43], v[200:203], a[168:171]\n" \
  "v_mfma_f32_16x16x32_f16 a[172:175], v[40:43], v[204:207], a[172:175]\n" \
  "v_mfma_f32_16x16x32_f16 a[188:191], v[44:47], v[204:207], a[188:191]\n" \
  "v_mfma_f32_16x16x32_f16 a[184:187], v[44:47], v[200:203], a[184:187]\n" \
  "v_mfma_f32_16x16x32_f16 a[180:183], v[44:47], v[196:199], a[180:183]\n" \
  "s_waitcnt lgkmcnt(2)\n" \
  "v_mfma_f32_16x16x32_f16 a[176:179], v[44:47], v[192:195], a[176:179]\n" \
  "v_mfma_f32_16x16x32_f16 a[192:195], v[48:51], v[192:195], a[192:195]\n" \
  "s_waitcnt vmcnt(16)\n" \
  "buffer_store_dword v15, v14, s[16:19], s59 offen sc1\n" \
  "v_mfma_f32_16x16x32_f16 a[196:199], v[48:51], v[196:199], a[196:199]\n" \
  "v_mfma_f32_16x16x32_f16 a[200:203], v[48:51], v[200:203], a[200:203]\n" \
  "v_mfma_f32_16x16x32_f16 a[204:207], v[48:51], v[204:207], a[204:207]\n" \
  "v_mfma_f32_16x16x32_f16 a[220:223], v[52:55], v[204:207], a[220:223]\n" \
  "v_mfma_f32_16x16x32_f16 a[216:219], v[52:55], v[200:203], a[216:219]\n" \
  "v_mfma_f32_16x16x32_f16 a[212:215], v[52:55], v[196:199], a[212:215]\n" \
  "s_waitcnt lgkmcnt(0)\n" \
  "v_mfma_f32_16x16x32_f16 a[208:211], v[52:55], v[192:195], a[208:211]\n" \
  "v_mfma_f32_16x16x32_f16 a[224:227], v[56:59], v[192:195], a[224:227]\n" \
  "v_mfma_f32_16x16x32_f16 a[228:231], v[56:59], v[196:199], a[228:231]\n" \
  "v_mfma_f32_16x16x32_f16 a[232:235], v[56:59], v[200:203], a[232:235]\n" \
  "v_mfma_f32_16x16x32_f16 a[236:239], v[56:59], v[204:207], a[236:239]\n" \
  "v_mfma_f32_16x16x32_f16 a[252:255], v[60:63], v[204:207], a[252:255]\n" \
  "v_mfma_f32_16x16x32_f16 a[248:251], v[60:63], v[200:203], a[248:251]\n" \
  "v_mfma_f32_16x16x32_f16 a[244:247], v[60:63], v[196:199], a[244:247]\n" \
  "v_mfma_f32_16x16x32_f16 a[240:243], v[60:63], v[192:195], a[240:243]\n" \
  "s_branch Lepi_%=\n" \
  ".p2align 6\n" \
  "Ltail3_%=:\n" \
  "v_mfma_f32_16x16x32_f16 a[240:243], v[60:63], v[64:67], a[240:243]\n" \
  "ds_read_b128 v[60:63], v25 offset:56320\n" \
  "v_mfma_f32_16x16x32_f16 a[0:3], v[32:35], v[80:83], a[0:3]\n" \
  "v_xor_b32 v26, 0x10000, v25\n" \
  "s_waitcnt vmcnt(47)\n" \
  "s_waitcnt vmcnt(37)\n" \
  "v_mfma_f32_16x16x32_f16 a[4:7], v[32:35], v[84:87], a[4:7]\n" \
  "v_cvt_pk_f16_f32 v64, v96, v100\n" \
  "v_cvt_pk_f16_f32 v68, v97, v101\n" \
  "s_barrier\n" \
  "v_mfma_f32_16x16x32_f16 a[8:11], v[32:35], v[88:91], a[8:11]\n" \
  "v_cvt_pk_f16_f32 v72, v98, v102\n" \
  "v_cvt_pk_f16_f32 v76, v99, v103\n" \
  "v_mfma_f32_16x16x32_f16 a[12:15], v[32:35], v[92:95], a[12:15]\n" \
  "ds_read_b128 v[32:35], v26 offset:0\n" \
  "v_mfma_f32_16x16x32_f16 a[28:31], v[36:39], v[92:95], a[28:31]\n" \
  "s_waitcnt vmcnt(17)\n" \
  "v_cmp_le_u32 vcc, s43, v28\n" \
  "v_mfma_f32_16x16x32_f16 a[24:27], v[36:39], v[88:91], a[24:27]\n" \
  "s_cmp_eq_u64 vcc, -1\n" \
  "s_cbranch_scc0 Lstub4_%=\n" \
  "Lback4_%=:\n" \
  "s_mov_b32 s39, s64\n" \
  "s_xor_b32 s36, s36, 0x10000\n" \
  "v_mfma_f32_16x16x32_f16 a[20:23], v[36:39], v[84:87], a[20:23]\n" \
  "s_waitcnt lgkmcnt(5)\n" \
  "s_bfe_u32 s40, s39, 0x80000\n" \
  "s_lshl_b32 s42, s40, 16\n" \
  "v_mfma_f32_16x16x32_f16 a[16:19], v[36:39], v[80:83], a[16:19]\n" \
  "ds_read_b128 v[36:39], v26 offset:1024\n" \
  "s_add_u32 m0, s36, 0x0\n" \
  "v_mfma_f32_16x16x32_f16 a[32:35], v[40:43], v[80:83], a[32:35]\n" \
  "buffer_load_dwordx4 v24, s[12:15], s42 offen offset:0 sc0 sc1 lds\n" \
  "s_waitcnt vmcnt(46)\n" \
  "v_mfma_f32_16x16x32_f16 a[36:39], v[40:43], v[84:87], a[36:39]\n" \
  "v_cvt_pk_f16_f32 v65, v104, v108\n" \
  "v_cvt_pk_f16_f32 v69, v105, v109\n" \
  "v_mfma_f32_16x16x32_f16 a[40:43], v[40:43], v[88:91], a[40:43]\n" \
  "v_cvt_pk_f16_f32 v73, v106, v110\n" \
  "v_cvt_pk_f16_f32 v77, v107, v111\n" \
  "v_mfma_f32_16x16x32_f16 a[44:47], v[40:43], v[92:95], a[44:47]\n" \
  "ds_read_b128 v[40:43], v26 offset:2048\n" \
  "v_mfma_f32_16x16x32_f16 a[60:63], v[44:47], v[92:95], a[60:63]\n" \
  "buffer_load_dwordx4 v24, s[12:15], s42 offen offset:1024 sc0 sc1 lds\n" \
  "s_waitcnt vmcnt(45)\n" \
  "v_mfma_f32_16x16x32_f16 a[56:59], v[44:47], v[88:91], a[56:59]\n" \
  "v_cvt_pk_f16_f32 v66, v112, v116\n" \
  "v_cvt_pk_f16_f32 v70, v113, v117\n" \
  "v_mfma_f32_16x16x32_f16 a[52:55], v[44:47], v[84:87], a[52:55]\n" \
  "s_waitcnt lgkmcnt(5)\n" \
  "v_mfma_f32_16x16x32_f16 a[48:51], v[44:47], v[80:83], a[48:51]\n" \
  "ds_read_b128 v[44:47], v26 offset:3072\n" \
  "v_mfma_f32_16x16x32_f16 a[64:67], v[48:51], v[80:83], a[64:67]\n" \
  "buffer_load_dwordx4 v24, s[12:15], s42 offen offset:2048 sc0 sc1 lds\n" \
  "v_mfma_f32_16x16x32_f16 a[68:71], v[48:51], v[84:87], a[68:71]\n" \
  "v_cvt_pk_f16_f32 v74, v114, v118\n" \
  "v_cvt_pk_f16_f32 v78, v115, v119\n" \
  "s_waitcnt vmcnt(44)\n" \
  "v_mfma_f32_16x16x32_f16 a[72:75], v[48:51], v[88:91], a[72:75]\n" \
  "v_cvt_pk_f16_f32 v67, v120, v124\n" \
  "v_cvt_pk_f16_f32 v71, v121, v125\n" \
  "v_mfma_f32_16x16x32_f16 a[76:79], v[48:51], v[92:95], a[76:79]\n" \
  "ds_read_b128 v[48:51], v26 offset:4096\n" \
  "v_mfma_f32_16x16x32_f16 a[92:95], v[52:55], v[92:95], a[92:95]\n" \
  "buffer_load_dwordx4 v24, s[12:15], s42 offen offset:3072 sc0 sc1 lds\n" \
  "v_mfma_f32_16x16x32_f16 a[88:91], v[52:55], v[88:91], a[88:91]\n" \
  "v_cvt_pk_f16_f32 v75, v122, v126\n" \
  "v_cvt_pk_f16_f32 v79, v123, v127\n" \
  "v_mfma_f32_16x16x32_f16 a[84:87], v[52:55], v[84:87], a[84:87]\n" \
  "s_waitcnt lgkmcnt(5)\n" \
  "s_lshl_b32 s41, s40, 20\n" \
  "v_mfma_f32_16x16x32_f16 a[80:83], v[52:55], v[80:83], a[80:83]\n" \
  "ds_read_b128 v[52:55], v26 offset:5120\n" \
  "v_mfma_f32_16x16x32_f16 a[96:99], v[56:59], v[80:83], a[96:99]\n" \
  "buffer_load_dwordx4 v[96:99], v16, s[8:11], s41 offen\n" \
  "v_mfma_f32_16x16x32_f16 a[100:103], v[56:59], v[84:87], a[100:103]\n" \
  "v_mfma_f32_16x16x32_f16 a[104:107], v[56:59], v[88:91], a[104:107]\n" \
  "v_mfma_f32_16x16x32_f16 a[108:111], v[56:59], v[92:95], a[108:111]\n" \
  "ds_read_b128 v[56:59], v26 offset:6144\n" \
  "v_mfma_f32_16x16x32_f16 a[124:127], v[60:63], v[92:95], a[124:127]\n" \
  "buffer_load_dwordx4 v[100:103], v17, s[8:11], s41 offen\n" \
  "v_mfma_f32_16x16x32_f16 a[120:123], v[60:63], v[88:91], a[120:123]\n" \
  "s_waitcnt vmcnt(31)\n" \
  "v_mfma_f32_16x16x32_f16 a[116:119], v[60:63], v[84:87], a[116:119]\n" \
  "s_waitcnt lgkmcnt(5)\n" \
  "s_barrier\n" \
  "v_mfma_f32_16x16x32_f16 a[112:115], v[60:63], v[80:83], a[112:115]\n" \
  "ds_read_b128 v[60:63], v26 offset:7168\n" \
  "v_mfma_f32_16x16x32_f16 a[0:3], v[32:35], v[64:67], a[0:3]\n" \
  "buffer_load_dwordx4 v[104:107], v18, s[8:11], s41 offen\n" \
  "s_waitcnt vmcnt(42)\n" \
  "v_mfma_f32_16x16x32_f16 a[4:7], v[32:35], v[68:71], a[4:7]\n" \
  "v_cvt_pk_f16_f32 v224, v128, v132\n" \
  "v_cvt_pk_f16_f32 v228, v129, v133\n" \
  "v_mfma_f32_16x16x32_f16 a[8:11], v[32:35], v[72:75], a[8:11]\n" \
  "v_cvt_pk_f16_f32 v232, v130, v134\n" \
  "v_cvt_pk_f16_f32 v236, v131, v135\n" \
  "v_mfma_f32_16x16x32_f16 a[12:15], v[32:35], v[76:79], a[12:15]\n" \
  "ds_read_b128 v[32:35], v26 offset:16384\n" \
  "s_add_u32 s46, s41, 0x80000\n" \
  "v_mfma_f32_16x16x32_f16 a[28:31], v[36:39], v[76:79], a[28:31]\n" \
  "buffer_load_dwordx4 v[108:111], v19, s[8:11], s41 offen\n" \
  "s_waitcnt vmcnt(41)\n" \
  "v_mfma_f32_16x16x32_f16 a[24:27], v[36:39], v[72:75], a[24:27]\n" \
  "v_cvt_pk_f16_f32 v225, v136, v140\n" \
  "v_cvt_pk_f16_f32 v229, v137, v141\n" \
  "v_mfma_f32_16x16x32_f16 a[20:23], v[36:39], v[68:71], a[20:23]\n" \
  "s_waitcnt lgkmcnt(5)\n" \
  "v_mfma_f32_16x16x32_f16 a[16:19], v[36:39], v[64:67], a[16:19]\n" \
  "ds_read_b128 v[36:39], v26 offset:17408\n" \
  "v_mfma_f32_16x16x32_f16 a[32:35], v[40:43], v[64:67], a[32:35]\n" \
  "buffer_load_dwordx4 v[112:115], v16, s[8:11], s46 offen\n" \
  "v_mfma_f32_16x16x32_f16 a[36:39], v[40:43], v[68:71], a[36:39]\n" \
  "v_cvt_pk_f16_f32 v233, v138, v142\n" \
  "v_cvt_pk_f16_f32 v237, v139, v143\n" \
  "s_waitcnt vmcnt(40)\n" \
  "v_mfma_f32_16x16x32_f16 a[40:43], v[40:43], v[72:75], a[40:43]\n" \
  "v_cvt_pk_f16_f32 v226, v144, v148\n" \
  "v_cvt_pk_f16_f32 v230, v145, v149\n" \
  "v_mfma_f32_16x16x32_f16 a[44:47], v[40:43], v[76:79], a[44:47]\n" \
  "ds_read_b128 v[40:43], v26 offset:18432\n" \
  "v_mfma_f32_16x16x32_f16 a[60:63], v[44:47], v[76:79], a[60:63]\n" \
  "buffer_load_dwordx4 v[116:119], v17, s[8:11], s46 offen\n" \
  "v_mfma_f32_16x16x32_f16 a[56:59], v[44:47], v[72:75], a[56:59]\n" \
  "v_cvt_pk_f16_f32 v234, v146, v150\n" \
  "v_cvt_pk_f16_f32 v238, v147, v151\n" \
  "v_mfma_f32_16x16x32_f16 a[52:55], v[44:47], v[68:71], a[52:55]\n" \
  "s_waitcnt lgkmcnt(5)\n" \
  "v_mfma_f32_16x16x32_f16 a[48:51], v[44:47], v[64:67], a[48:51]\n" \
  "ds_read_b128 v[44:47], v26 offset:19456\n" \
  "v_mfma_f32_16x16x32_f16 a[64:67], v[48:51], v[64:67], a[64:67]\n" \
  "buffer_load_dwordx4 v[120:123], v18, s[8:11], s46 offen\n" \
  "s_waitcnt vmcnt(40)\n" \
  "v_mfma_f32_16x16x32_f16 a[68:71], v[48:51], v[68:71], a[68:71]\n" \
  "v_cvt_pk_f16_f32 v227, v152, v156\n" \
  "v_cvt_pk_f16_f32 v231, v153, v157\n" \
  "v_mfma_f32_16x16x32_f16 a[72:75], v[48:51], v[72:75], a[72:75]\n" \
  "v_cvt_pk_f16_f32 v235, v154, v158\n" \
  "v_cvt_pk_f16_f32 v239, v155, v159\n" \
  "v_mfma_f32_16x16x32_f16 a[76:79], v[48:51], v[76:79], a[76:79]\n" \
  "ds_read_b128 v[48:51], v26 offset:20480\n" \
  "v_mfma_f32_16x16x32_f16 a[92:95], v[52:55], v[76:79], a[92:95]\n" \
  "buffer_load_dwordx4 v[124:127], v19, s[8:11], s46 offen\n" \
  "v_mfma_f32_16x16x32_f16 a[88:91], v[52:55], v[72:75], a[88:91]\n" \
  "v_mfma_f32_16x16x32_f16 a[84:87], v[52:55], v[68:71], a[84:87]\n" \
  "s_waitcnt lgkmcnt(5)\n" \
  "s_bfe_u32 s40, s39, 0x80008\n" \
  "s_lshl_b32 s42, s40, 16\n" \
  "v_mfma_f32_16x16x32_f16 a[80:83], v[52:55], v[64:67], a[80:83]\n" \
  "ds_read_b128 v[52:55], v26 offset:21504\n" \
  "s_add_u32 m0, s36, 0x4000\n" \
  "v_mfma_f32_16x16x32_f16 a[96:99], v[56:59], v[64:67], a[96:99]\n" \
  "buffer_load_dwordx4 v24, s[12:15], s42 offen offset:0 sc0 sc1 lds\n" \
  "v_mfma_f32_16x16x32_f16 a[100:103], v[56:59], v[68:71], a[100:103]\n" \
  "v_mfma_f32_16x16x32_f16 a[104:107], v[56:59], v[72:75], a[104:107]\n" \
  "v_mfma_f32_16x16x32_f16 a[108:111], v[56:59], v[76:79], a[108:111]\n" \
  "ds_read_b128 v[56:59], v26 offset:22528\n" \
  "v_mfma_f32_16x16x32_f16 a[124:127], v[60:63], v[76:79], a[124:127]\n" \
  "buffer_load_dwordx4 v24, s[12:15], s42 offen offset:1024 sc0 sc1 lds\n" \
  "v_mfma_f32_16x16x32_f16 a[120:123], v[60:63], v[72:75], a[120:123]\n" \
  "s_waitcnt vmcnt(26)\n" \
  "v_mfma_f32_16x16x32_f16 a[116:119], v[60:63], v[68:71], a[116:119]\n" \
  "s_waitcnt lgkmcnt(5)\n" \
  "s_barrier\n" \
  "v_mfma_f32_16x16x32_f16 a[112:115], v[60:63], v[64:67], a[112:115]\n" \
  "ds_read_b128 v[60:63], v26 offset:23552\n" \
  "v_mfma_f32_16x16x32_f16 a[0:3], v[32:35], v[224:227], a[0:3]\n" \
  "buffer_load_dwordx4 v24, s[12:15], s42 offen offset:2048 sc0 sc1 lds\n" \
  "s_waitcnt vmcnt(38)\n" \
  "v_mfma_f32_16x16x32_f16 a[4:7], v[32:35], v[228:231], a[4:7]\n" \
  "v_cvt_pk_f16_f32 v240, v160, v164\n" \
  "v_cvt_pk_f16_f32 v244, v161, v165\n" \
  "v_mfma_f32_16x16x32_f16 a[8:11], v[32:35], v[232:235], a[8:11]\n" \
  "v_cvt_pk_f16_f32 v248, v162, v166\n" \
  "v_cvt_pk_f16_f32 v252, v163, v167\n" \
  "v_mfma_f32_16x16x32_f16 a[12:15], v[32:35], v[236:239], a[12:15]\n" \
  "ds_read_b128 v[32:35], v26 offset:32768\n" \
  "v_mfma_f32_16x16x32_f16 a[28:31], v[36:39], v[236:239], a[28:31]\n" \
  "buffer_load_dwordx4 v24, s[12:15], s42 offen offset:3072 sc0 sc1 lds\n" \
  "s_waitcnt vmcnt(37)\n" \
  "v_mfma_f32_16x16x32_f16 a[24:27], v[36:39], v[232:235], a[24:27]\n" \
  "v_cvt_pk_f16_f32 v241, v168, v172\n" \
  "v_cvt_pk_f16_f32 v245, v169, v173\n" \
  "v_mfma_f32_16x16x32_f16 a[20:23], v[36:39], v[228:231], a[20:23]\n" \
  "s_waitcnt lgkmcnt(5)\n" \
  "s_lshl_b32 s41, s40, 20\n" \
  "v_mfma_f32_16x16x32_f16 a[16:19], v[36:39], v[224:227], a[16:19]\n" \
  "ds_read_b128 v[36:39], v26 offset:33792\n" \
  "v_mfma_f32_16x16x32_f16 a[32:35], v[40:43], v[224:227], a[32:35]\n" \
  "buffer_load_dwordx4 v[128:131], v16, s[8:11], s41 offen\n" \
  "v_mfma_f32_16x16x32_f16 a[36:39], v[40:43], v[228:231], a[36:39]\n" \
  "v_cvt_pk_f16_f32 v249, v170, v174\n" \
  "v_cvt_pk_f16_f32 v253, v171, v175\n" \
  "s_waitcnt vmcnt(36)\n" \
  "v_mfma_f32_16x16x32_f16 a[40:43], v[40:43], v[232:235], a[40:43]\n" \
  "v_cvt_pk_f16_f32 v242, v176, v180\n" \
  "v_cvt_pk_f16_f32 v246, v177, v181\n" \
  "v_mfma_f32_16x16x32_f16 a[44:47], v[40:43], v[236:239], a[44:47]\n" \
  "ds_read_b128 v[40:43], v26 offset:34816\n" \
  "v_mfma_f32_16x16x32_f16 a[60:63], v[44:47], v[236:239], a[60:63]\n" \
  "buffer_load_dwordx4 v[132:135], v17, s[8:11], s41 offen\n" \
  "v_mfma_f32_16x16x32_f16 a[56:59], v[44:47], v[232:235], a[56:59]\n" \
  "v_cvt_pk_f16_f32 v250, v178, v182\n" \
  "v_cvt_pk_f16_f32 v254, v179, v183\n" \
  "v_mfma_f32_16x16x32_f16 a[52:55], v[44:47], v[228:231], a[52:55]\n" \
  "s_waitcnt lgkmcnt(5)\n" \
  "v_mfma_f32_16x16x32_f16 a[48:51], v[44:47], v[224:227], a[48:51]\n" \
  "ds_read_b128 v[44:47], v26 offset:35840\n" \
  "v_mfma_f32_16x16x32_f16 a[64:67], v[48:51], v[224:227], a[64:67]\n" \
  "buffer_load_dwordx4 v[136:139], v18, s[8:11], s41 offen\n" \
  "s_waitcnt vmcnt(35)\n" \
  "v_mfma_f32_16x16x32_f16 a[68:71], v[48:51], v[228:231], a[68:71]\n" \
  "v_cvt_pk_f16_f32 v243, v184, v188\n" \
  "v_cvt_pk_f16_f32 v247, v185, v189\n" \
  "v_mfma_f32_16x16x32_f16 a[72:75], v[48:51], v[232:235], a[72:75]\n" \
  "v_cvt_pk_f16_f32 v251, v186, v190\n" \
  "v_cvt_pk_f16_f32 v255, v187, v191\n" \
  "v_mfma_f32_16x16x32_f16 a[76:79], v[48:51], v[236:239], a[76:79]\n" \
  "ds_read_b128 v[48:51], v26 offset:36864\n" \
  "s_add_u32 s46, s41, 0x80000\n" \
  "v_mfma_f32_16x16x32_f16 a[92:95], v[52:55], v[236:239], a[92:95]\n" \
  "buffer_load_dwordx4 v[140:143], v19, s[8:11], s41 offen\n" \
  "v_mfma_f32_16x16x32_f16 a[88:91], v[52:55], v[232:235], a[88:91]\n" \
  "v_mfma_f32_16x16x32_f16 a[84:87], v[52:55], v[228:231], a[84:87]\n" \
  "s_waitcnt lgkmcnt(5)\n" \
  "v_mfma_f32_16x16x32_f16 a[80:83], v[52:55], v[224:227], a[80:83]\n" \
  "ds_read_b128 v[52:55], v26 offset:37888\n" \
  "v_mfma_f32_16x16x32_f16 a[96:99], v[56:59], v[224:227], a[96:99]\n" \
  "buffer_load_dwordx4 v[144:147], v16, s[8:11], s46 offen\n" \
  "v_mfma_f32_16x16x32_f16 a[100:103], v[56:59], v[228:231], a[100:103]\n" \
  "v_mfma_f32_16x16x32_f16 a[104:107], v[56:59], v[232:235], a[104:107]\n" \
  "v_mfma_f32_16x16x32_f16 a[108:111], v[56:59], v[236:239], a[108:111]\n" \
  "ds_read_b128 v[56:59], v26 offset:38912\n" \
  "v_mfma_f32_16x16x32_f16 a[124:127], v[60:63], v[236:239], a[124:127]\n" \
  "buffer_load_dwordx4 v[148:151], v17, s[8:11], s46 offen\n" \
  "v_mfma_f32_16x16x32_f16 a[120:123], v[60:63], v[232:235], a[120:123]\n" \
  "s_waitcnt vmcnt(22)\n" \
  "v_mfma_f32_16x16x32_f16 a[116:119], v[60:63], v[228:231], a[116:119]\n" \
  "s_waitcnt lgkmcnt(5)\n" \
  "s_barrier\n" \
  "v_mfma_f32_16x16x32_f16 a[112:115], v[60:63], v[224:227], a[112:115]\n" \
  "ds_read_b128 v[60:63], v26 offset:39936\n" \
  "v_mfma_f32_16x16x32_f16 a[0:3], v[32:35], v[240:243], a[0:3]\n" \
  "buffer_load_dwordx4 v[152:155], v18, s[8:11], s46 offen\n" \
  "s_waitcnt vmcnt(33)\n" \
  "v_mfma_f32_16x16x32_f16 a[4:7], v[32:35], v[244:247], a[4:7]\n" \
  "v_cvt_pk_f16_f32 v0, v192, v196\n" \
  "v_cvt_pk_f16_f32 v4, v193, v197\n" \
  "v_mfma_f32_16x16x32_f16 a[8:11], v[32:35], v[248:251], a[8:11]\n" \
  "v_cvt_pk_f16_f32 v8, v194, v198\n" \
  "v_cvt_pk_f16_f32 v20, v195, v199\n" \
  "v_mfma_f32_16x16x32_f16 a[12:15], v[32:35], v[252:255], a[12:15]\n" \
  "ds_read_b128 v[32:35], v26 offset:49152\n" \
  "v_mfma_f32_16x16x32_f16 a[28:31], v[36:39], v[252:255], a[28:31]\n" \
  "buffer_load_dwordx4 v[156:159], v19, s[8:11], s46 offen\n" \
  "s_waitcnt vmcnt(32)\n" \
  "v_mfma_f32_16x16x32_f16 a[24:27], v[36:39], v[248:251], a[24:27]\n" \
  "v_cvt_pk_f16_f32 v1, v200, v204\n" \
  "v_cvt_pk_f16_f32 v5, v201, v205\n" \
  "v_mfma_f32_16x16x32_f16 a[20:23], v[36:39], v[244:247], a[20:23]\n" \
  "s_waitcnt lgkmcnt(5)\n" \
  "s_bfe_u32 s40, s39, 0x80010\n" \
  "s_lshl_b32 s42, s40, 16\n" \
  "v_mfma_f32_16x16x32_f16 a[16:19], v[36:39], v[240:243], a[16:19]\n" \
  "ds_read_b128 v[36:39], v26 offset:50176\n" \
  "s_add_u32 m0, s36, 0x8000\n" \
  "v_mfma_f32_16x16x32_f16 a[32:35], v[40:43], v[240:243], a[32:35]\n" \
  "buffer_load_dwordx4 v24, s[12:15], s42 offen offset:0 sc0 sc1 lds\n" \
  "v_mfma_f32_16x16x32_f16 a[36:39], v[40:43], v[244:247], a[36:39]\n" \
  "v_cvt_pk_f16_f32 v9, v202, v206\n" \
  "v_cvt_pk_f16_f32 v21, v203, v207\n" \
  "s_waitcnt vmcnt(31)\n" \
  "v_mfma_f32_16x16x32_f16 a[40:43], v[40:43], v[248:251], a[40:43]\n" \
  "v_cvt_pk_f16_f32 v2, v208, v212\n" \
  "v_cvt_pk_f16_f32 v6, v209, v213\n" \
  "v_mfma_f32_16x16x32_f16 a[44:47], v[40:43], v[252:255], a[44:47]\n" \
  "ds_read_b128 v[40:43], v26 offset:51200\n" \
  "v_mfma_f32_16x16x32_f16 a[60:63], v[44:47], v[252:255], a[60:63]\n" \
  "buffer_load_dwordx4 v24, s[12:15], s42 offen offset:1024 sc0 sc1 lds\n" \
  "v_mfma_f32_16x16x32_f16 a[56:59], v[44:47], v[248:251], a[56:59]\n" \
  "v_cvt_pk_f16_f32 v10, v210, v214\n" \
  "v_cvt_pk_f16_f32 v22, v211, v215\n" \
  "v_mfma_f32_16x16x32_f16 a[52:55], v[44:47], v[244:247], a[52:55]\n" \
  "s_waitcnt lgkmcnt(5)\n" \
  "v_mfma_f32_16x16x32_f16 a[48:51], v[44:47], v[240:243], a[48:51]\n" \
  "ds_read_b128 v[44:47], v26 offset:52224\n" \
  "v_mfma_f32_16x16x32_f16 a[64:67], v[48:51], v[240:243], a[64:67]\n" \
  "buffer_load_dwordx4 v24, s[12:15], s42 offen offset:2048 sc0 sc1 lds\n" \
  "s_waitcnt vmcnt(31)\n" \
  "v_mfma_f32_16x16x32_f16 a[68:71], v[48:51], v[244:247], a[68:71]\n" \
  "v_cvt_pk_f16_f32 v3, v216, v220\n" \
  "v_cvt_pk_f16_f32 v7, v217, v221\n" \
  "v_mfma_f32_16x16x32_f16 a[72:75], v[48:51], v[248:251], a[72:75]\n" \
  "v_cvt_pk_f16_f32 v11, v218, v222\n" \
  "v_cvt_pk_f16_f32 v23, v219, v223\n" \
  "v_mfma_f32_16x16x32_f16 a[76:79], v[48:51], v[252:255], a[76:79]\n" \
  "ds_read_b128 v[48:51], v26 offset:53248\n" \
  "v_mfma_f32_16x16x32_f16 a[92:95], v[52:55], v[252:255], a[92:95]\n" \
  "buffer_load_dwordx4 v24, s[12:15], s42 offen offset:3072 sc0 sc1 lds\n" \
  "v_mfma_f32_16x16x32_f16 a[88:91], v[52:55], v[248:251], a[88:91]\n" \
  "v_mfma_f32_16x16x32_f16 a[84:87], v[52:55], v[244:247], a[84:87]\n" \
  "s_waitcnt lgkmcnt(5)\n" \
  "s_lshl_b32 s41, s40, 20\n" \
  "v_mfma_f32_16x16x32_f16 a[80:83], v[52:55], v[240:243], a[80:83]\n" \
  "ds_read_b128 v[52:55], v26 offset:54272\n" \
  "v_mfma_f32_16x16x32_f16 a[96:99], v[56:59], v[240:243], a[96:99]\n" \
  "buffer_load_dwordx4 v[160:163], v16, s[8:11], s41 offen\n" \
  "v_mfma_f32_16x16x32_f16 a[100:103], v[56:59], v[244:247], a[100:103]\n" \
  "v_mfma_f32_16x16x32_f16 a[104:107], v[56:59], v[248:251], a[104:107]\n" \
  "v_mfma_f32_16x16x32_f16 a[108:111], v[56:59], v[252:255], a[108:111]\n" \
  "ds_read_b128 v[56:59], v26 offset:55296\n" \
  "v_mfma_f32_16x16x32_f16 a[124:127], v[60:63], v[252:255], a[124:127]\n" \
  "buffer_load_dwordx4 v[164:167], v17, s[8:11], s41 offen\n" \
  "v_mfma_f32_16x16x32_f16 a[120:123], v[60:63], v[248:251], a[120:123]\n" \
  "s_waitcnt vmcnt(26)\n" \
  "v_mfma_f32_16x16x32_f16 a[116:119], v[60:63], v[244:247], a[116:119]\n" \
  "s_waitcnt lgkmcnt(5)\n" \
  "s_barrier\n" \
  "v_mfma_f32_16x16x32_f16 a[112:115], v[60:63], v[240:243], a[112:115]\n" \
  "ds_read_b128 v[60:63], v26 offset:56320\n" \
  "v_mfma_f32_16x16x32_f16 a[0:3], v[32:35], v[0:3], a[0:3]\n" \
  "buffer_load_dwordx4 v[168:171], v18, s[8:11], s41 offen\n" \
  "s_waitcnt vmcnt(25)\n" \
  "v_mfma_f32_16x16x32_f16 a[4:7], v[32:35], v[4:7], a[4:7]\n" \
  "v_cvt_pk_f16_f32 v192, v96, v100\n" \
  "v_cvt_pk_f16_f32 v196, v97, v101\n" \
  "v_mfma_f32_16x16x32_f16 a[8:11], v[32:35], v[8:11], a[8:11]\n" \
  "v_cvt_pk_f16_f32 v200, v98, v102\n" \
  "v_cvt_pk_f16_f32 v204, v99, v103\n" \
  "v_mfma_f32_16x16x32_f16 a[12:15], v[32:35], v[20:23], a[12:15]\n" \
  "ds_read_b128 v[32:35], v25 offset:0\n" \
  "s_add_u32 s46, s41, 0x80000\n" \
  "v_mfma_f32_16x16x32_f16 a[28:31], v[36:39], v[20:23], a[28:31]\n" \
  "buffer_load_dwordx4 v[172:175], v19, s[8:11], s41 offen\n" \
  "s_waitcnt vmcnt(24)\n" \
  "v_mfma_f32_16x16x32_f16 a[24:27], v[36:39], v[8:11], a[24:27]\n" \
  "v_cvt_pk_f16_f32 v193, v104, v108\n" \
  "v_cvt_pk_f16_f32 v197, v105, v109\n" \
  "v_mfma_f32_16x16x32_f16 a[20:23], v[36:39], v[4:7], a[20:23]\n" \
  "s_waitcnt lgkmcnt(5)\n" \
  "v_mfma_f32_16x16x32_f16 a[16:19], v[36:39], v[0:3], a[16:19]\n" \
  "ds_read_b128 v[36:39], v25 offset:1024\n" \
  "v_mfma_f32_16x16x32_f16 a[32:35], v[40:43], v[0:3], a[32:35]\n" \
  "buffer_load_dwordx4 v[176:179], v16, s[8:11], s46 offen\n" \
  "v_mfma_f32_16x16x32_f16 a[36:39], v[40:43], v[4:7], a[36:39]\n" \
  "v_cvt_pk_f16_f32 v201, v106, v110\n" \
  "v_cvt_pk_f16_f32 v205, v107, v111\n" \
  "s_waitcnt vmcnt(23)\n" \
  "v_mfma_f32_16x16x32_f16 a[40:43], v[40:43], v[8:11], a[40:43]\n" \
  "v_cvt_pk_f16_f32 v194, v112, v116\n" \
  "v_cvt_pk_f16_f32 v198, v113, v117\n" \
  "v_mfma_f32_16x16x32_f16 a[44:47], v[40:43], v[20:23], a[44:47]\n" \
  "ds_read_b128 v[40:43], v25 offset:2048\n" \
  "v_mfma_f32_16x16x32_f16 a[60:63], v[44:47], v[20:23], a[60:63]\n" \
  "buffer_load_dwordx4 v[180:183], v17, s[8:11], s46 offen\n" \
  "v_mfma_f32_16x16x32_f16 a[56:59], v[44:47], v[8:11], a[56:59]\n" \
  "v_cvt_pk_f16_f32 v202, v114, v118\n" \
  "v_cvt_pk_f16_f32 v206, v115, v119\n" \
  "v_mfma_f32_16x16x32_f16 a[52:55], v[44:47], v[4:7], a[52:55]\n" \
  "s_waitcnt lgkmcnt(5)\n" \
  "v_mfma_f32_16x16x32_f16 a[48:51], v[44:47], v[0:3], a[48:51]\n" \
  "ds_read_b128 v[44:47], v25 offset:3072\n" \
  "v_mfma_f32_16x16x32_f16 a[64:67], v[48:51], v[0:3], a[64:67]\n" \
  "buffer_load_dwordx4 v[184:187], v18, s[8:11], s46 offen\n" \
  "s_waitcnt vmcnt(23)\n" \
  "v_mfma_f32_16x16x32_f16 a[68:71], v[48:51], v[4:7], a[68:71]\n" \
  "v_cvt_pk_f16_f32 v195, v120, v124\n" \
  "v_cvt_pk_f16_f32 v199, v121, v125\n" \
  "v_mfma_f32_16x16x32_f16 a[72:75], v[48:51], v[8:11], a[72:75]\n" \
  "v_cvt_pk_f16_f32 v203, v122, v126\n" \
  "v_cvt_pk_f16_f32 v207, v123, v127\n" \
  "v_mfma_f32_16x16x32_f16 a[76:79], v[48:51], v[20:23], a[76:79]\n" \
  "ds_read_b128 v[48:51], v25 offset:4096\n" \
  "v_mfma_f32_16x16x32_f16 a[92:95], v[52:55], v[20:23], a[92:95]\n" \
  "buffer_load_dwordx4 v[188:191], v19, s[8:11], s46 offen\n" \
  "v_mfma_f32_16x16x32_f16 a[88:91], v[52:55], v[8:11], a[88:91]\n" \
  "v_mfma_f32_16x16x32_f16 a[84:87], v[52:55], v[4:7], a[84:87]\n" \
  "s_waitcnt lgkmcnt(5)\n" \
  "v_mfma_f32_16x16x32_f16 a[80:83], v[52:55], v[0:3], a[80:83]\n" \
  "ds_read_b128 v[52:55], v25 offset:5120\n" \
  "v_mfma_f32_16x16x32_f16 a[96:99], v[56:59], v[0:3], a[96:99]\n" \
  "v_mfma_f32_16x16x32_f16 a[100:103], v[56:59], v[4:7], a[100:103]\n" \
  "v_mfma_f32_16x16x32_f16 a[104:107], v[56:59], v[8:11], a[104:107]\n" \
  "v_mfma_f32_16x16x32_f16 a[108:111], v[56:59], v[20:23], a[108:111]\n" \
  "ds_read_b128 v[56:59], v25 offset:6144\n" \
  "v_mfma_f32_16x16x32_f16 a[124:127], v[60:63], v[20:23], a[124:127]\n" \
  "v_mfma_f32_16x16x32_f16 a[120:123], v[60:63], v[8:11], a[120:123]\n" \
  "s_waitcnt vmcnt(20)\n" \
  "v_mfma_f32_16x16x32_f16 a[116:119], v[60:63], v[4:7], a[116:119]\n" \
  "s_waitcnt lgkmcnt(5)\n" \
  "s_barrier\n" \
  "v_mfma_f32_16x16x32_f16 a[112:115], v[60:63], v[0:3], a[112:115]\n" \
  "ds_read_b128 v[60:63], v25 offset:7168\n" \
  "v_mfma_f32_16x16x32_f16 a[0:3], v[32:35], v[192:195], a[0:3]\n" \
  "s_waitcnt vmcnt(18)\n" \
  "v_mfma_f32_16x16x32_f16 a[4:7], v[32:35], v[196:199], a[4:7]\n" \
  "v_cvt_pk_f16_f32 v208, v128, v132\n" \
  "v_cvt_pk_f16_f32 v212, v129, v133\n" \
  "v_mfma_f32_16x16x32_f16 a[8:11], v[32:35], v[200:203], a[8:11]\n" \
  "v_cvt_pk_f16_f32 v216, v130, v134\n" \
  "v_cvt_pk_f16_f32 v220, v131, v135\n" \
  "v_mfma_f32_16x16x32_f16 a[12:15], v[32:35], v[204:207], a[12:15]\n" \
  "ds_read_b128 v[32:35], v25 offset:16384\n" \
  "s_waitcnt vmcnt(16)\n" \
  "v_mfma_f32_16x16x32_f16 a[28:31], v[36:39], v[204:207], a[28:31]\n" \
  "v_cvt_pk_f16_f32 v209, v136, v140\n" \
  "v_cvt_pk_f16_f32 v213, v137, v141\n" \
  "v_mfma_f32_16x16x32_f16 a[24:27], v[36:39], v[200:203], a[24:27]\n" \
  "v_cvt_pk_f16_f32 v217, v138, v142\n" \
  "v_cvt_pk_f16_f32 v221, v139, v143\n" \
  "v_mfma_f32_16x16x32_f16 a[20:23], v[36:39], v[196:199], a[20:23]\n" \
  "s_waitcnt lgkmcnt(5)\n" \
  "v_mfma_f32_16x16x32_f16 a[16:19], v[36:39], v[192:195], a[16:19]\n" \
  "ds_read_b128 v[36:39], v25 offset:17408\n" \
  "s_waitcnt vmcnt(14)\n" \
  "v_mfma_f32_16x16x32_f16 a[32:35], v[40:43], v[192:195], a[32:35]\n" \
  "v_cvt_pk_f16_f32 v210, v144, v148\n" \
  "v_cvt_pk_f16_f32 v214, v145, v149\n" \
  "v_mfma_f32_16x16x32_f16 a[36:39], v[40:43], v[196:199], a[36:39]\n" \
  "v_cvt_pk_f16_f32 v218, v146, v150\n" \
  "v_cvt_pk_f16_f32 v222, v147, v151\n" \
  "s_waitcnt vmcnt(12)\n" \
  "v_mfma_f32_16x16x32_f16 a[40:43], v[40:43], v[200:203], a[40:43]\n" \
  "v_cvt_pk_f16_f32 v211, v152, v156\n" \
  "v_cvt_pk_f16_f32 v215, v153, v157\n" \
  "v_mfma_f32_16x16x32_f16 a[44:47], v[40:43], v[204:207], a[44:47]\n" \
  "ds_read_b128 v[40:43], v25 offset:18432\n" \
  "v_mfma_f32_16x16x32_f16 a[60:63], v[44:47], v[204:207], a[60:63]\n" \
  "v_cvt_pk_f16_f32 v219, v154, v158\n" \
  "v_cvt_pk_f16_f32 v223, v155, v159\n" \
  "v_mfma_f32_16x16x32_f16 a[56:59], v[44:47], v[200:203], a[56:59]\n" \
  "v_mfma_f32_16x16x32_f16 a[52:55], v[44:47], v[196:199], a[52:55]\n" \
  "s_waitcnt lgkmcnt(5)\n" \
  "v_mfma_f32_16x16x32_f16 a[48:51], v[44:47], v[192:195], a[48:51]\n" \
  "ds_read_b128 v[44:47], v25 offset:19456\n" \
  "v_mfma_f32_16x16x32_f16 a[64:67], v[48:51], v[192:195], a[64:67]\n" \
  "v_mfma_f32_16x16x32_f16 a[68:71], v[48:51], v[196:199], a[68:71]\n" \
  "v_mfma_f32_16x16x32_f16 a[72:75], v[48:51], v[200:203], a[72:75]\n" \
  "v_mfma_f32_16x16x32_f16 a[76:79], v[48:51], v[204:207], a[76:79]\n" \
  "ds_read_b128 v[48:51], v25 offset:20480\n" \
  "v_mfma_f32_16x16x32_f16 a[92:95], v[52:55], v[204:207], a[92:95]\n" \
  "v_mfma_f32_16x16x32_f16 a[88:91], v[52:55], v[200:203], a[88:91]\n" \
  "v_mfma_f32_16x16x32_f16 a[84:87], v[52:55], v[196:199], a[84:87]\n" \
  "s_waitcnt lgkmcnt(5)\n" \
  "v_mfma_f32_16x16x32_f16 a[80:83], v[52:55], v[192:195], a[80:83]\n" \
  "ds_read_b128 v[52:55], v25 offset:21504\n" \
  "v_mfma_f32_16x16x32_f16 a[96:99], v[56:59], v[192:195], a[96:99]\n" \
  "v_mfma_f32_16x16x32_f16 a[100:103], v[56:59], v[196:199], a[100:103]\n" \
  "v_mfma_f32_16x16x32_f16 a[104:107], v[56:59], v[200:203], a[104:107]\n" \
  "v_mfma_f32_16x16x32_f16 a[108:111], v[56:59], v[204:207], a[108:111]\n" \
  "ds_read_b128 v[56:59], v25 offset:22528\n" \
  "v_mfma_f32_16x16x32_f16 a[124:127], v[60:63], v[204:207], a[124:127]\n" \
  "v_mfma_f32_16x16x32_f16 a[120:123], v[60:63], v[200:203], a[120:123]\n" \
  "s_waitcnt vmcnt(8)\n" \
  "v_mfma_f32_16x16x32_f16 a[116:119], v[60:63], v[196:199], a[116:119]\n" \
  "s_waitcnt lgkmcnt(5)\n" \
  "s_barrier\n" \
  "v_mfma_f32_16x16x32_f16 a[112:115], v[60:63], v[192:195], a[112:115]\n" \
  "ds_read_b128 v[60:63], v25 offset:23552\n" \
  "v_mfma_f32_16x16x32_f16 a[0:3], v[32:35], v[208:211], a[0:3]\n" \
  "s_waitcnt vmcnt(6)\n" \
  "v_mfma_f32_16x16x32_f16 a[4:7], v[32:35], v[212:215], a[4:7]\n" \
  "v_cvt_pk_f16_f32 v96, v160, v164\n" \
  "v_cvt_pk_f16_f32 v100, v161, v165\n" \
  "v_mfma_f32_16x16x32_f16 a[8:11], v[32:35], v[216:219], a[8:11]\n" \
  "v_cvt_pk_f16_f32 v104, v162, v166\n" \
  "v_cvt_pk_f16_f32 v108, v163, v167\n" \
  "v_mfma_f32_16x16x32_f16 a[12:15], v[32:35], v[220:223], a[12:15]\n" \
  "ds_read_b128 v[32:35], v25 offset:32768\n" \
  "s_waitcnt vmcnt(4)\n" \
  "v_mfma_f32_16x16x32_f16 a[28:31], v[36:39], v[220:223], a[28:31]\n" \
  "v_cvt_pk_f16_f32 v97, v168, v172\n" \
  "v_cvt_pk_f16_f32 v101, v169, v173\n" \
  "v_mfma_f32_16x16x32_f16 a[24:27], v[36:39], v[216:219], a[24:27]\n" \
  "v_cvt_pk_f16_f32 v105, v170, v174\n" \
  "v_cvt_pk_f16_f32 v109, v171, v175\n" \
  "v_mfma_f32_16x16x32_f16 a[20:23], v[36:39], v[212:215], a[20:23]\n" \
  "s_waitcnt lgkmcnt(5)\n" \
  "v_mfma_f32_16x16x32_f16 a[16:19], v[36:39], v[208:211], a[16:19]\n" \
  "ds_read_b128 v[36:39], v25 offset:33792\n" \
  "s_waitcnt vmcnt(2)\n" \
  "v_mfma_f32_16x16x32_f16 a[32:35], v[40:43], v[208:211], a[32:35]\n" \
  "v_cvt_pk_f16_f32 v98, v176, v180\n" \
  "v_cvt_pk_f16_f32 v102, v177, v181\n" \
  "v_mfma_f32_16x16x32_f16 a[36:39], v[40:43], v[212:215], a[36:39]\n" \
  "v_cvt_pk_f16_f32 v106, v178, v182\n" \
  "v_cvt_pk_f16_f32 v110, v179, v183\n" \
  "s_waitcnt vmcnt(0)\n" \
  "v_mfma_f32_16x16x32_f16 a[40:43], v[40:43], v[216:219], a[40:43]\n" \
  "v_cvt_pk_f16_f32 v99, v184, v188\n" \
  "v_cvt_pk_f16_f32 v103, v185, v189\n" \
  "v_mfma_f32_16x16x32_f16 a[44:47], v[40:43], v[220:223], a[44:47]\n" \
  "ds_read_b128 v[40:43], v25 offset:34816\n" \
  "v_mfma_f32_16x16x32_f16 a[60:63], v[44:47], v[220:223], a[60:63]\n" \
  "v_cvt_pk_f16_f32 v107, v186, v190\n" \
  "v_cvt_pk_f16_f32 v111, v187, v191\n" \
  "v_mfma_f32_16x16x32_f16 a[56:59], v[44:47], v[216:219], a[56:59]\n" \
  "v_mfma_f32_16x16x32_f16 a[52:55], v[44:47], v[212:215], a[52:55]\n" \
  "s_waitcnt lgkmcnt(5)\n" \
  "v_mfma_f32_16x16x32_f16 a[48:51], v[44:47], v[208:211], a[48:51]\n" \
  "ds_read_b128 v[44:47], v25 offset:35840\n" \
  "v_mfma_f32_16x16x32_f16 a[64:67], v[48:51], v[208:211], a[64:67]\n" \
  "v_mfma_f32_16x16x32_f16 a[68:71], v[48:51], v[212:215], a[68:71]\n" \
  "v_mfma_f32_16x16x32_f16 a[72:75], v[48:51], v[216:219], a[72:75]\n" \
  "v_mfma_f32_16x16x32_f16 a[76:79], v[48:51], v[220:223], a[76:79]\n" \
  "ds_read_b128 v[48:51], v25 offset:36864\n" \
  "v_mfma_f32_16x16x32_f16 a[92:95], v[52:55], v[220:223], a[92:95]\n" \
  "v_mfma_f32_16x16x32_f16 a[88:91], v[52:55], v[216:219], a[88:91]\n" \
  "v_mfma_f32_16x16x32_f16 a[84:87], v[52:55], v[212:215], a[84:87]\n" \
  "s_waitcnt lgkmcnt(5)\n" \
  "v_mfma_f32_16x16x32_f16 a[80:83], v[52:55], v[208:211], a[80:83]\n" \
  "ds_read_b128 v[52:55], v25 offset:37888\n" \
  "v_mfma_f32_16x16x32_f16 a[96:99], v[56:59], v[208:211], a[96:99]\n" \
  "v_mfma_f32_16x16x32_f16 a[100:103], v[56:59], v[212:215], a[100:103]\n" \
  "v_mfma_f32_16x16x32_f16 a[104:107], v[56:59], v[216:219], a[104:107]\n" \
  "v_mfma_f32_16x16x32_f16 a[108:111], v[56:59], v[220:223], a[108:111]\n" \
  "ds_read_b128 v[56:59], v25 offset:38912\n" \
  "v_mfma_f32_16x16x32_f16 a[124:127], v[60:63], v[220:223], a[124:127]\n" \
  "v_mfma_f32_16x16x32_f16 a[120:123], v[60:63], v[216:219], a[120:123]\n" \
  "v_mfma_f32_16x16x32_f16 a[116:119], v[60:63], v[212:215], a[116:119]\n" \
  "s_waitcnt lgkmcnt(5)\n" \
  "v_mfma_f32_16x16x32_f16 a[112:115], v[60:63], v[208:211], a[112:115]\n" \
  "ds_read_b128 v[60:63], v25 offset:39936\n" \
  "v_mfma_f32_16x16x32_f16 a[0:3], v[32:35], v[96:99], a[0:3]\n" \
  "v_mfma_f32_16x16x32_f16 a[4:7], v[32:35], v[100:103], a[4:7]\n" \
  "v_mfma_f32_16x16x32_f16 a[8:11], v[32:35], v[104:107], a[8:11]\n" \
  "v_mfma_f32_16x16x32_f16 a[12:15], v[32:35], v[108:111], a[12:15]\n" \
  "ds_read_b128 v[32:35], v25 offset:57344\n" \
  "v_mfma_f32_16x16x32_f16 a[28:31], v[36:39], v[108:111], a[28:31]\n" \
  "v_mfma_f32_16x16x32_f16 a[24:27], v[36:39], v[104:107], a[24:27]\n" \
  "v_mfma_f32_16x16x32_f16 a[20:23], v[36:39], v[100:103], a[20:23]\n" \
  "s_waitcnt lgkmcnt(5)\n" \
  "v_mfma_f32_16x16x32_f16 a[16:19], v[36:39], v[96:99], a[16:19]\n" \
  "ds_read_b128 v[36:39], v25 offset:58368\n" \
  "v_mfma_f32_16x16x32_f16 a[32:35], v[40:43], v[96:99], a[32:35]\n" \
  "v_mfma_f32_16x16x32_f16 a[36:39], v[40:43], v[100:103], a[36:39]\n" \
  "v_mfma_f32_16x16x32_f16 a[40:43], v[40:43], v[104:107], a[40:43]\n" \
  "s_mov_b32 s40, 0x0\n" \
  "v_mfma_f32_16x16x32_f16 a[44:47], v[40:43], v[108:111], a[44:47]\n" \
  "ds_read_b128 v[40:43], v25 offset:59392\n" \
  "v_mfma_f32_16x16x32_f16 a[60:63], v[44:47], v[108:111], a[60:63]\n" \
  "buffer_store_dwordx4 a[0:3], v12, s[16:19], s40 offen offset:0 sc1\n" \
  "v_mfma_f32_16x16x32_f16 a[56:59], v[44:47], v[104:107], a[56:59]\n" \
  "v_mfma_f32_16x16x32_f16 a[52:55], v[44:47], v[100:103], a[52:55]\n" \
  "s_waitcnt lgkmcnt(5)\n" \
  "v_mfma_f32_16x16x32_f16 a[48:51], v[44:47], v[96:99], a[48:51]\n" \
  "ds_read_b128 v[44:47], v25 offset:60416\n" \
  "v_mfma_f32_16x16x32_f16 a[64:67], v[48:51], v[96:99], a[64:67]\n" \
  "v_mfma_f32_16x16x32_f16 a[68:71], v[48:51], v[100:103], a[68:71]\n" \
  "v_mfma_f32_16x16x32_f16 a[72:75], v[48:51], v[104:107], a[72:75]\n" \
  "v_mfma_f32_16x16x32_f16 a[76:79], v[48:51], v[108:111], a[76:79]\n" \
  "ds_read_b128 v[48:51], v25 offset:61440\n" \
  "v_mfma_f32_16x16x32_f16 a[92:95], v[52:55], v[108:111], a[92:95]\n" \
  "buffer_store_dwordx4 a[4:7], v12, s[16:19], s40 offen offset:1024 sc1\n" \
  "v_mfma_f32_16x16x32_f16 a[88:91], v[52:55], v[104:107], a[88:91]\n" \
  "v_mfma_f32_16x16x32_f16 a[84:87], v[52:55], v[100:103], a[84:87]\n" \
  "s_waitcnt lgkmcnt(5)\n" \
  "v_mfma_f32_16x16x32_f16 a[80:83], v[52:55], v[96:99], a[80:83]\n" \
  "ds_read_b128 v[52:55], v25 offset:62464\n" \
  "v_mfma_f32_16x16x32_f16 a[96:99], v[56:59], v[96:99], a[96:99]\n" \
  "v_mfma_f32_16x16x32_f16 a[100:103], v[56:59], v[100:103], a[100:103]\n" \
  "v_mfma_f32_16x16x32_f16 a[104:107], v[56:59], v[104:107], a[104:107]\n" \
  "v_mfma_f32_16x16x32_f16 a[108:111], v[56:59], v[108:111], a[108:111]\n" \
  "ds_read_b128 v[56:59], v25 offset:63488\n" \
  "v_mfma_f32_16x16x32_f16 a[124:127], v[60:63], v[108:111], a[124:127]\n" \
  "buffer_store_dwordx4 a[8:11], v12, s[16:19], s40 offen offset:2048 sc1\n" \
  "v_mfma_f32_16x16x32_f16 a[120:123], v[60:63], v[104:107], a[120:123]\n" \
  "v_mfma_f32_16x16x32_f16 a[116:119], v[60:63], v[100:103], a[116:119]\n" \
  "s_waitcnt lgkmcnt(5)\n" \
  "v_mfma_f32_16x16x32_f16 a[112:115], v[60:63], v[96:99], a[112:115]\n" \
  "ds_read_b128 v[60:63], v25 offset:64512\n" \
  "v_mfma_f32_16x16x32_f16 a[128:131], v[32:35], v[80:83], a[128:131]\n" \
  "v_mfma_f32_16x16x32_f16 a[132:135], v[32:35], v[84:87], a[132:135]\n" \
  "v_mfma_f32_16x16x32_f16 a[136:139], v[32:35], v[88:91], a[136:139]\n" \
  "v_mfma_f32_16x16x32_f16 a[140:143], v[32:35], v[92:95], a[140:143]\n" \
  "ds_read_b128 v[32:35], v26 offset:8192\n" \
  "v_mfma_f32_16x16x32_f16 a[156:159], v[36:39], v[92:95], a[156:159]\n" \
  "buffer_store_dwordx4 a[12:15], v12, s[16:19], s40 offen offset:3072 sc1\n" \
  "v_mfma_f32_16x16x32_f16 a[152:155], v[36:39], v[88:91], a[152:155]\n" \
  "v_mfma_f32_16x16x32_f16 a[148:151], v[36:39], v[84:87], a[148:151]\n" \
  "s_waitcnt lgkmcnt(5)\n" \
  "v_mfma_f32_16x16x32_f16 a[144:147], v[36:39], v[80:83], a[144:147]\n" \
  "ds_read_b128 v[36:39], v26 offset:9216\n" \
  "v_mfma_f32_16x16x32_f16 a[160:163], v[40:43], v[80:83], a[160:163]\n" \
  "v_mfma_f32_16x16x32_f16 a[164:167], v[40:43], v[84:87], a[164:167]\n" \
  "v_mfma_f32_16x16x32_f16 a[168:171], v[40:43], v[88:91], a[168:171]\n" \
  "s_mov_b32 s40, 0x1000\n" \
  "v_mfma_f32_16x16x32_f16 a[172:175], v[40:43], v[92:95], a[172:175]\n" \
  "ds_read_b128 v[40:43], v26 offset:10240\n" \
  "v_mfma_f32_16x16x32_f16 a[188:191], v[44:47], v[92:95], a[188:191]\n" \
  "buffer_store_dwordx4 a[16:19], v12, s[16:19], s40 offen offset:0 sc1\n" \
  "v_mfma_f32_16x16x32_f16 a[184:187], v[44:47], v[88:91], a[184:187]\n" \
  "v_mfma_f32_16x16x32_f16 a[180:183], v[44:47], v[84:87], a[180:183]\n" \
  "s_waitcnt lgkmcnt(5)\n" \
  "v_mfma_f32_16x16x32_f16 a[176:179], v[44:47], v[80:83], a[176:179]\n" \
  "ds_read_b128 v[44:47], v26 offset:11264\n" \
  "v_mfma_f32_16x16x32_f16 a[192:195], v[48:51], v[80:83], a[192:195]\n" \
  "v_mfma_f32_16x16x32_f16 a[196:199], v[48:51], v[84:87], a[196:199]\n" \
  "v_mfma_f32_16x16x32_f16 a[200:203], v[48:51], v[88:91], a[200:203]\n" \
  "v_mfma_f32_16x16x32_f16 a[204:207], v[48:51], v[92:95], a[204:207]\n" \
  "ds_read_b128 v[48:51], v26 offset:12288\n" \
  "v_mfma_f32_16x16x32_f16 a[220:223], v[52:55], v[92:95], a[220:223]\n" \
  "buffer_store_dwordx4 a[20:23], v12, s[16:19], s40 offen offset:1024 sc1\n" \
  "v_mfma_f32_16x16x32_f16 a[216:219], v[52:55], v[88:91], a[216:219]\n" \
  "v_mfma_f32_16x16x32_f16 a[212:215], v[52:55], v[84:87], a[212:215]\n" \
  "s_waitcnt lgkmcnt(5)\n" \
  "v_mfma_f32_16x16x32_f16 a[208:211], v[52:55], v[80:83], a[208:211]\n" \
  "ds_read_b128 v[52:55], v26 offset:13312\n" \
  "v_mfma_f32_16x16x32_f16 a[224:227], v[56:59], v[80:83], a[224:227]\n" \
  "v_mfma_f32_16x16x32_f16 a[228:231], v[56:59], v[84:87], a[228:231]\n" \
  "v_mfma_f32_16x16x32_f16 a[232:235], v[56:59], v[88:91], a[232:235]\n" \
  "v_mfma_f32_16x16x32_f16 a[236:239], v[56:59], v[92:95], a[236:239]\n" \
  "ds_read_b128 v[56:59], v26 offset:14336\n" \
  "v_mfma_f32_16x16x32_f16 a[252:255], v[60:63], v[92:95], a[252:255]\n" \
  "buffer_store_dwordx4 a[24:27], v12, s[16:19], s40 offen offset:2048 sc1\n" \
  "v_mfma_f32_16x16x32_f16 a[248:251], v[60:63], v[88:91], a[248:251]\n" \
  "v_mfma_f32_16x16x32_f16 a[244:247], v[60:63], v[84:87], a[244:247]\n" \
  "s_waitcnt lgkmcnt(5)\n" \
  "v_mfma_f32_16x16x32_f16 a[240:243], v[60:63], v[80:83], a[240:243]\n" \
  "ds_read_b128 v[60:63], v26 offset:15360\n" \
  "v_mfma_f32_16x16x32_f16 a[128:131], v[32:35], v[64:67], a[128:131]\n" \
  "v_mfma_f32_16x16x32_f16 a[132:135], v[32:35], v[68:71], a[132:135]\n" \
  "v_mfma_f32_16x16x32_f16 a[136:139], v[32:35], v[72:75], a[136:139]\n" \
  "v_mfma_f32_16x16x32_f16 a[140:143], v[32:35], v[76:79], a[140:143]\n" \
  "ds_read_b128 v[32:35], v26 offset:24576\n" \
  "v_mfma_f32_16x16x32_f16 a[156:159], v[36:39], v[76:79], a[156:159]\n" \
  "buffer_store_dwordx4 a[28:31], v12, s[16:19], s40 offen offset:3072 sc1\n" \
  "v_mfma_f32_16x16x32_f16 a[152:155], v[36:39], v[72:75], a[152:155]\n" \
  "v_mfma_f32_16x16x32_f16 a[148:151], v[36:39], v[68:71], a[148:151]\n" \
  "s_waitcnt lgkmcnt(5)\n" \
  "v_mfma_f32_16x16x32_f16 a[144:147], v[36:39], v[64:67], a[144:147]\n" \
  "ds_read_b128 v[36:39], v26 offset:25600\n" \
  "v_mfma_f32_16x16x32_f16 a[160:163], v[40:43], v[64:67], a[160:163]\n" \
  "v_mfma_f32_16x16x32_f16 a[164:167], v[40:43], v[68:71], a[164:167]\n" \
  "v_mfma_f32_16x16x32_f16 a[168:171], v[40:43], v[72:75], a[168:171]\n" \
  "s_mov_b32 s40, 0x2000\n" \
  "v_mfma_f32_16x16x32_f16 a[172:175], v[40:43], v[76:79], a[172:175]\n" \
  "ds_read_b128 v[40:43], v26 offset:26624\n" \
  "v_mfma_f32_16x16x32_f16 a[188:191], v[44:47], v[76:79], a[188:191]\n" \
  "buffer_store_dwordx4 a[32:35], v12, s[16:19], s40 offen offset:0 sc1\n" \
  "v_mfma_f32_16x16x32_f16 a[184:187], v[44:47], v[72:75], a[184:187]\n" \
  "v_mfma_f32_16x16x32_f16 a[180:183], v[44:47], v[68:71], a[180:183]\n" \
  "s_waitcnt lgkmcnt(5)\n" \
  "v_mfma_f32_16x16x32_f16 a[176:179], v[44:47], v[64:67], a[176:179]\n" \
  "ds_read_b128 v[44:47], v26 offset:27648\n" \
  "v_mfma_f32_16x16x32_f16 a[192:195], v[48:51], v[64:67], a[192:195]\n" \
  "v_mfma_f32_16x16x32_f16 a[196:199], v[48:51], v[68:71], a[196:199]\n" \
  "v_mfma_f32_16x16x32_f16 a[200:203], v[48:51], v[72:75], a[200:203]\n" \
  "v_mfma_f32_16x16x32_f16 a[204:207], v[48:51], v[76:79], a[204:207]\n" \
  "ds_read_b128 v[48:51], v26 offset:28672\n" \
  "v_mfma_f32_16x16x32_f16 a[220:223], v[52:55], v[76:79], a[220:223]\n" \
  "buffer_store_dwordx4 a[36:39], v12, s[16:19], s40 offen offset:1024 sc1\n" \
  "v_mfma_f32_16x16x32_f16 a[216:219], v[52:55], v[72:75], a[216:219]\n" \
  "v_mfma_f32_16x16x32_f16 a[212:215], v[52:55], v[68:71], a[212:215]\n" \
  "s_waitcnt lgkmcnt(5)\n" \
  "v_mfma_f32_16x16x32_f16 a[208:211], v[52:55], v[64:67], a[208:211]\n" \
  "ds_read_b128 v[52:55], v26 offset:29696\n" \
  "v_mfma_f32_16x16x32_f16 a[224:227], v[56:59], v[64:67], a[224:227]\n" \
  "v_mfma_f32_16x16x32_f16 a[228:231], v[56:59], v[68:71], a[228:231]\n" \
  "v_mfma_f32_16x16x32_f16 a[232:235], v[56:59], v[72:75], a[232:235]\n" \
  "v_mfma_f32_16x16x32_f16 a[236:239], v[56:59], v[76:79], a[236:239]\n" \
  "ds_read_b128 v[56:59], v26 offset:30720\n" \
  "v_mfma_f32_16x16x32_f16 a[252:255], v[60:63], v[76:79], a[252:255]\n" \
  "buffer_store_dwordx4 a[40:43], v12, s[16:19], s40 offen offset:2048 sc1\n" \
  "v_mfma_f32_16x16x32_f16 a[248:251], v[60:63], v[72:75], a[248:251]\n" \
  "v_mfma_f32_16x16x32_f16 a[244:247], v[60:63], v[68:71], a[244:247]\n" \
  "s_waitcnt lgkmcnt(5)\n" \
  "v_mfma_f32_16x16x32_f16 a[240:243], v[60:63], v[64:67], a[240:243]\n" \
  "ds_read_b128 v[60:63], v26 offset:31744\n" \
  "v_mfma_f32_16x16x32_f16 a[128:131], v[32:35], v[224:227], a[128:131]\n" \
  "v_mfma_f32_16x16x32_f16 a[132:135], v[32:35], v[228:231], a[132:135]\n" \
  "v_mfma_f32_16x16x32_f16 a[136:139], v[32:35], v[232:235], a[136:139]\n" \
  "v_mfma_f32_16x16x32_f16 a[140:143], v[32:35], v[236:239], a[140:143]\n" \
  "ds_read_b128 v[32:35], v26 offset:40960\n" \
  "v_mfma_f32_16x16x32_f16 a[156:159], v[36:39], v[236:239], a[156:159]\n" \
  "buffer_store_dwordx4 a[44:47], v12, s[16:19], s40 offen offset:3072 sc1\n" \
  "v_mfma_f32_16x16x32_f16 a[152:155], v[36:39], v[232:235], a[152:155]\n" \
  "v_mfma_f32_16x16x32_f16 a[148:151], v[36:39], v[228:231], a[148:151]\n" \
  "s_waitcnt lgkmcnt(5)\n" \
  "v_mfma_f32_16x16x32_f16 a[144:147], v[36:39], v[224:227], a[144:147]\n" \
  "ds_read_b128 v[36:39], v26 offset:41984\n" \
  "v_mfma_f32_16x16x32_f16 a[160:163], v[40:43], v[224:227], a[160:163]\n" \
  "v_mfma_f32_16x16x32_f16 a[164:167], v[40:43], v[228:231], a[164:167]\n" \
  "v_mfma_f32_16x16x32_f16 a[168:171], v[40:43], v[232:235], a[168:171]\n" \
  "s_mov_b32 s40, 0x3000\n" \
  "v_mfma_f32_16x16x32_f16 a[172:175], v[40:43], v[236:239], a[172:175]\n" \
  "ds_read_b128 v[40:43], v26 offset:43008\n" \
  "v_mfma_f32_16x16x32_f16 a[188:191], v[44:47], v[236:239], a[188:191]\n" \
  "buffer_store_dwordx4 a[48:51], v12, s[16:19], s40 offen offset:0 sc1\n" \
  "v_mfma_f32_16x16x32_f16 a[184:187], v[44:47], v[232:235], a[184:187]\n" \
  "v_mfma_f32_16x16x32_f16 a[180:183], v[44:47], v[228:231], a[180:183]\n" \
  "s_waitcnt lgkmcnt(5)\n" \
  "v_mfma_f32_16x16x32_f16 a[176:179], v[44:47], v[224:227], a[176:179]\n" \
  "ds_read_b128 v[44:47], v26 offset:44032\n" \
  "v_mfma_f32_16x16x32_f16 a[192:195], v[48:51], v[224:227], a[192:195]\n" \
  "v_mfma_f32_16x16x32_f16 a[196:199], v[48:51], v[228:231], a[196:199]\n" \
  "v_mfma_f32_16x16x32_f16 a[200:203], v[48:51], v[232:235], a[200:203]\n" \
  "v_mfma_f32_16x16x32_f16 a[204:207], v[48:51], v[236:239], a[204:207]\n" \
  "ds_read_b128 v[48:51], v26 offset:45056\n" \
  "v_mfma_f32_16x16x32_f16 a[220:223], v[52:55], v[236:239], a[220:223]\n" \
  "buffer_store_dwordx4 a[52:55], v12, s[16:19], s40 offen offset:1024 sc1\n" \
  "v_mfma_f32_16x16x32_f16 a[216:219], v[52:55], v[232:235], a[216:219]\n" \
  "v_mfma_f32_16x16x32_f16 a[212:215], v[52:55], v[228:231], a[212:215]\n" \
  "s_waitcnt lgkmcnt(5)\n" \
  "v_mfma_f32_16x16x32_f16 a[208:211], v[52:55], v[224:227], a[208:211]\n" \
  "ds_read_b128 v[52:55], v26 offset:46080\n" \
  "v_mfma_f32_16x16x32_f16 a[224:227], v[56:59], v[224:227], a[224:227]\n" \
  "v_mfma_f32_16x16x32_f16 a[228:231], v[56:59], v[228:231], a[228:231]\n" \
  "v_mfma_f32_16x16x32_f16 a[232:235], v[56:59], v[232:235], a[232:235]\n" \
  "v_mfma_f32_16x16x32_f16 a[236:239], v[56:59], v[236:239], a[236:239]\n" \
  "ds_read_b128 v[56:59], v26 offset:47104\n" \
  "v_mfma_f32_16x16x32_f16 a[252:255], v[60:63], v[236:239], a[252:255]\n" \
  "buffer_store_dwordx4 a[56:59], v12, s[16:19], s40 offen offset:2048 sc1\n" \
  "v_mfma_f32_16x16x32_f16 a[248:251], v[60:63], v[232:235], a[248:251]\n" \
  "v_mfma_f32_16x16x32_f16 a[244:247], v[60:63], v[228:231], a[244:247]\n" \
  "s_waitcnt lgkmcnt(5)\n" \
  "v_mfma_f32_16x16x32_f16 a[240:243], v[60:63], v[224:227], a[240:243]\n" \
  "ds_read_b128 v[60:63], v26 offset:48128\n" \
  "v_mfma_f32_16x16x32_f16 a[128:131], v[32:35], v[240:243], a[128:131]\n" \
  "v_mfma_f32_16x16x32_f16 a[132:135], v[32:35], v[244:247], a[132:135]\n" \
  "v_mfma_f32_16x16x32_f16 a[136:139], v[32:35], v[248:251], a[136:139]\n" \
  "v_mfma_f32_16x16x32_f16 a[140:143], v[32:35], v[252:255], a[140:143]\n" \
  "ds_read_b128 v[32:35], v26 offset:57344\n" \
  "v_mfma_f32_16x16x32_f16 a[156:159], v[36:39], v[252:255], a[156:159]\n" \
  "buffer_store_dwordx4 a[60:63], v12, s[16:19], s40 offen offset:3072 sc1\n" \
  "v_mfma_f32_16x16x32_f16 a[152:155], v[36:39], v[248:251], a[152:155]\n" \
  "v_mfma_f32_16x16x32_f16 a[148:151], v[36:39], v[244:247], a[148:151]\n" \
  "s_waitcnt lgkmcnt(5)\n" \
  "v_mfma_f32_16x16x32_f16 a[144:147], v[36:39], v[240:243], a[144:147]\n" \
  "ds_read_b128 v[36:39], v26 offset:58368\n" \
  "v_mfma_f32_16x16x32_f16 a[160:163], v[40:43], v[240:243], a[160:163]\n" \
  "v_mfma_f32_16x16x32_f16 a[164:167], v[40:43], v[244:247], a[164:167]\n" \
  "v_mfma_f32_16x16x32_f16 a[168:171], v[40:43], v[248:251], a[168:171]\n" \
  "s_mov_b32 s40, 0x4000\n" \
  "v_mfma_f32_16x16x32_f16 a[172:175], v[40:43], v[252:255], a[172:175]\n" \
  "ds_read_b128 v[40:43], v26 offset:59392\n" \
  "v_mfma_f32_16x16x32_f16 a[188:191], v[44:47], v[252:255], a[188:191]\n" \
  "buffer_store_dwordx4 a[64:67], v12, s[16:19], s40 offen offset:0 sc1\n" \
  "v_mfma_f32_16x16x32_f16 a[184:187], v[44:47], v[248:251], a[184:187]\n" \
  "v_mfma_f32_16x16x32_f16 a[180:183], v[44:47], v[244:247], a[180:183]\n" \
  "s_waitcnt lgkmcnt(5)\n" \
  "v_mfma_f32_16x16x32_f16 a[176:179], v[44:47], v[240:243], a[176:179]\n" \
  "ds_read_b128 v[44:47], v26 offset:60416\n" \
  "v_mfma_f32_16x16x32_f16 a[192:195], v[48:51], v[240:243], a[192:195]\n" \
  "v_mfma_f32_16x16x32_f16 a[196:199], v[48:51], v[244:247], a[196:199]\n" \
  "v_mfma_f32_16x16x32_f16 a[200:203], v[48:51], v[248:251], a[200:203]\n" \
  "v_mfma_f32_16x16x32_f16 a[204:207], v[48:51], v[252:255], a[204:207]\n" \
  "ds_read_b128 v[48:51], v26 offset:61440\n" \
  "v_mfma_f32_16x16x32_f16 a[220:223], v[52:55], v[252:255], a[220:223]\n" \
  "buffer_store_dwordx4 a[68:71], v12, s[16:19], s40 offen offset:1024 sc1\n" \
  "v_mfma_f32_16x16x32_f16 a[216:219], v[52:55], v[248:251], a[216:219]\n" \
  "v_mfma_f32_16x16x32_f16 a[212:215], v[52:55], v[244:247], a[212:215]\n" \
  "s_waitcnt lgkmcnt(5)\n" \
  "v_mfma_f32_16x16x32_f16 a[208:211], v[52:55], v[240:243], a[208:211]\n" \
  "ds_read_b128 v[52:55], v26 offset:62464\n" \
  "v_mfma_f32_16x16x32_f16 a[224:227], v[56:59], v[240:243], a[224:227]\n" \
  "v_mfma_f32_16x16x32_f16 a[228:231], v[56:59], v[244:247], a[228:231]\n" \
  "v_mfma_f32_16x16x32_f16 a[232:235], v[56:59], v[248:251], a[232:235]\n" \
  "v_mfma_f32_16x16x32_f16 a[236:239], v[56:59], v[252:255], a[236:239]\n" \
  "ds_read_b128 v[56:59], v26 offset:63488\n" \
  "v_mfma_f32_16x16x32_f16 a[252:255], v[60:63], v[252:255], a[252:255]\n" \
  "buffer_store_dwordx4 a[72:75], v12, s[16:19], s40 offen offset:2048 sc1\n" \
  "v_mfma_f32_16x16x32_f16 a[248:251], v[60:63], v[248:251], a[248:251]\n" \
  "v_mfma_f32_16x16x32_f16 a[244:247], v[60:63], v[244:247], a[244:247]\n" \
  "s_waitcnt lgkmcnt(5)\n" \
  "v_mfma_f32_16x16x32_f16 a[240:243], v[60:63], v[240:243], a[240:243]\n" \
  "ds_read_b128 v[60:63], v26 offset:64512\n" \
  "v_mfma_f32_16x16x32_f16 a[128:131], v[32:35], v[0:3], a[128:131]\n" \
  "v_mfma_f32_16x16x32_f16 a[132:135], v[32:35], v[4:7], a[132:135]\n" \
  "v_mfma_f32_16x16x32_f16 a[136:139], v[32:35], v[8:11], a[136:139]\n" \
  "v_mfma_f32_16x16x32_f16 a[140:143], v[32:35], v[20:23], a[140:143]\n" \
  "ds_read_b128 v[32:35], v25 offset:8192\n" \
  "v_mfma_f32_16x16x32_f16 a[156:159], v[36:39], v[20:23], a[156:159]\n" \
  "buffer_store_dwordx4 a[76:79], v12, s[16:19], s40 offen offset:3072 sc1\n" \
  "v_mfma_f32_16x16x32_f16 a[152:155], v[36:39], v[8:11], a[152:155]\n" \
  "v_mfma_f32_16x16x32_f16 a[148:151], v[36:39], v[4:7], a[148:151]\n" \
  "s_waitcnt lgkmcnt(5)\n" \
  "v_mfma_f32_16x16x32_f16 a[144:147], v[36:39], v[0:3], a[144:147]\n" \
  "ds_read_b128 v[36:39], v25 offset:9216\n" \
  "v_mfma_f32_16x16x32_f16 a[160:163], v[40:43], v[0:3], a[160:163]\n" \
  "v_mfma_f32_16x16x32_f16 a[164:167], v[40:43], v[4:7], a[164:167]\n" \
  "v_mfma_f32_16x16x32_f16 a[168:171], v[40:43], v[8:11], a[168:171]\n" \
  "s_mov_b32 s40, 0x5000\n" \
  "v_mfma_f32_16x16x32_f16 a[172:175], v[40:43], v[20:23], a[172:175]\n" \
  "ds_read_b128 v[40:43], v25 offset:10240\n" \
  "v_mfma_f32_16x16x32_f16 a[188:191], v[44:47], v[20:23], a[188:191]\n" \
  "buffer_store_dwordx4 a[80:83], v12, s[16:19], s40 offen offset:0 sc1\n" \
  "v_mfma_f32_16x16x32_f16 a[184:187], v[44:47], v[8:11], a[184:187]\n" \
  "v_mfma_f32_16x16x32_f16 a[180:183], v[44:47], v[4:7], a[180:183]\n" \
  "s_waitcnt lgkmcnt(5)\n" \
  "v_mfma_f32_16x16x32_f16 a[176:179], v[44:47], v[0:3], a[176:179]\n" \
  "ds_read_b128 v[44:47], v25 offset:11264\n" \
  "v_mfma_f32_16x16x32_f16 a[192:195], v[48:51], v[0:3], a[192:195]\n" \
  "v_mfma_f32_16x16x32_f16 a[196:199], v[48:51], v[4:7], a[196:199]\n" \
  "v_mfma_f32_16x16x32_f16 a[200:203], v[48:51], v[8:11], a[200:203]\n" \
  "v_mfma_f32_16x16x32_f16 a[204:207], v[48:51], v[20:23], a[204:207]\n" \
  "ds_read_b128 v[48:51], v25 offset:12288\n" \
  "v_mfma_f32_16x16x32_f16 a[220:223], v[52:55], v[20:23], a[220:223]\n" \
  "buffer_store_dwordx4 a[84:87], v12, s[16:19], s40 offen offset:1024 sc1\n" \
  "v_mfma_f32_16x16x32_f16 a[216:219], v[52:55], v[8:11], a[216:219]\n" \
  "v_mfma_f32_16x16x32_f16 a[212:215], v[52:55], v[4:7], a[212:215]\n" \
  "s_waitcnt lgkmcnt(5)\n" \
  "v_mfma_f32_16x16x32_f16 a[208:211], v[52:55], v[0:3], a[208:211]\n" \
  "ds_read_b128 v[52:55], v25 offset:13312\n" \
  "v_mfma_f32_16x16x32_f16 a[224:227], v[56:59], v[0:3], a[224:227]\n" \
  "v_mfma_f32_16x16x32_f16 a[228:231], v[56:59], v[4:7], a[228:231]\n" \
  "v_mfma_f32_16x16x32_f16 a[232:235], v[56:59], v[8:11], a[232:235]\n" \
  "v_mfma_f32_16x16x32_f16 a[236:239], v[56:59], v[20:23], a[236:239]\n" \
  "ds_read_b128 v[56:59], v25 offset:14336\n" \
  "v_mfma_f32_16x16x32_f16 a[252:255], v[60:63], v[20:23], a[252:255]\n" \
  "buffer_store_dwordx4 a[88:91], v12, s[16:19], s40 offen offset:2048 sc1\n" \
  "v_mfma_f32_16x16x32_f16 a[248:251], v[60:63], v[8:11], a[248:251]\n" \
  "v_mfma_f32_16x16x32_f16 a[244:247], v[60:63], v[4:7], a[244:247]\n" \
  "s_waitcnt lgkmcnt(5)\n" \
  "v_mfma_f32_16x16x32_f16 a[240:243], v[60:63], v[0:3], a[240:243]\n" \
  "ds_read_b128 v[60:63], v25 offset:15360\n" \
  "v_mfma_f32_16x16x32_f16 a[128:131], v[32:35], v[192:195], a[128:131]\n" \
  "v_mfma_f32_16x16x32_f16 a[132:135], v[32:35], v[196:199], a[132:135]\n" \
  "v_mfma_f32_16x16x32_f16 a[136:139], v[32:35], v[200:203], a[136:139]\n" \
  "v_mfma_f32_16x16x32_f16 a[140:143], v[32:35], v[204:207], a[140:143]\n" \
  "ds_read_b128 v[32:35], v25 offset:24576\n" \
  "v_mfma_f32_16x16x32_f16 a[156:159], v[36:39], v[204:207], a[156:159]\n" \
  "buffer_store_dwordx4 a[92:95], v12, s[16:19], s40 offen offset:3072 sc1\n" \
  "v_mfma_f32_16x16x32_f16 a[152:155], v[36:39], v[200:203], a[152:155]\n" \
  "v_mfma_f32_16x16x32_f16 a[148:151], v[36:39], v[196:199], a[148:151]\n" \
  "s_waitcnt lgkmcnt(5)\n" \
  "v_mfma_f32_16x16x32_f16 a[144:147], v[36:39], v[192:195], a[144:147]\n" \
  "ds_read_b128 v[36:39], v25 offset:25600\n" \
  "v_mfma_f32_16x16x32_f16 a[160:163], v[40:43], v[192:195], a[160:163]\n" \
  "v_mfma_f32_16x16x32_f16 a[164:167], v[40:43], v[196:199], a[164:167]\n" \
  "v_mfma_f32_16x16x32_f16 a[168:171], v[40:43], v[200:203], a[168:171]\n" \
  "s_mov_b32 s40, 0x6000\n" \
  "v_mfma_f32_16x16x32_f16 a[172:175], v[40:43], v[204:207], a[172:175]\n" \
  "ds_read_b128 v[40:43], v25 offset:26624\n" \
  "v_mfma_f32_16x16x32_f16 a[188:191], v[44:47], v[204:207], a[188:191]\n" \
  "buffer_store_dwordx4 a[96:99], v12, s[16:19], s40 offen offset:0 sc1\n" \
  "v_mfma_f32_16x16x32_f16 a[184:187], v[44:47], v[200:203], a[184:187]\n" \
  "v_mfma_f32_16x16x32_f16 a[180:183], v[44:47], v[196:199], a[180:183]\n" \
  "s_waitcnt lgkmcnt(5)\n" \
  "v_mfma_f32_16x16x32_f16 a[176:179], v[44:47], v[192:195], a[176:179]\n" \
  "ds_read_b128 v[44:47], v25 offset:27648\n" \
  "v_mfma_f32_16x16x32_f16 a[192:195], v[48:51], v[192:195], a[192:195]\n" \
  "v_mfma_f32_16x16x32_f16 a[196:199], v[48:51], v[196:199], a[196:199]\n" \
  "v_mfma_f32_16x16x32_f16 a[200:203], v[48:51], v[200:203], a[200:203]\n" \
  "v_mfma_f32_16x16x32_f16 a[204:207], v[48:51], v[204:207], a[204:207]\n" \
  "ds_read_b128 v[48:51], v25 offset:28672\n" \
  "v_mfma_f32_16x16x32_f16 a[220:223], v[52:55], v[204:207], a[220:223]\n" \
  "buffer_store_dwordx4 a[100:103], v12, s[16:19], s40 offen offset:1024 sc1\n" \
  "v_mfma_f32_16x16x32_f16 a[216:219], v[52:55], v[200:203], a[216:219]\n" \
  "v_mfma_f32_16x16x32_f16 a[212:215], v[52:55], v[196:199], a[212:215]\n" \
  "s_waitcnt lgkmcnt(5)\n" \
  "v_mfma_f32_16x16x32_f16 a[208:211], v[52:55], v[192:195], a[208:211]\n" \
  "ds_read_b128 v[52:55], v25 offset:29696\n" \
  "v_mfma_f32_16x16x32_f16 a[224:227], v[56:59], v[192:195], a[224:227]\n" \
  "v_mfma_f32_16x16x32_f16 a[228:231], v[56:59], v[196:199], a[228:231]\n" \
  "v_mfma_f32_16x16x32_f16 a[232:235], v[56:59], v[200:203], a[232:235]\n" \
  "v_mfma_f32_16x16x32_f16 a[236:239], v[56:59], v[204:207], a[236:239]\n" \
  "ds_read_b128 v[56:59], v25 offset:30720\n" \
  "v_mfma_f32_16x16x32_f16 a[252:255], v[60:63], v[204:207], a[252:255]\n" \
  "buffer_store_dwordx4 a[104:107], v12, s[16:19], s40 offen offset:2048 sc1\n" \
  "v_mfma_f32_16x16x32_f16 a[248:251], v[60:63], v[200:203], a[248:251]\n" \
  "v_mfma_f32_16x16x32_f16 a[244:247], v[60:63], v[196:199], a[244:247]\n" \
  "s_waitcnt lgkmcnt(5)\n" \
  "v_mfma_f32_16x16x32_f16 a[240:243], v[60:63], v[192:195], a[240:243]\n" \
  "ds_read_b128 v[60:63], v25 offset:31744\n" \
  "v_mfma_f32_16x16x32_f16 a[128:131], v[32:35], v[208:211], a[128:131]\n" \
  "v_mfma_f32_16x16x32_f16 a[132:135], v[32:35], v[212:215], a[132:135]\n" \
  "v_mfma_f32_16x16x32_f16 a[136:139], v[32:35], v[216:219], a[136:139]\n" \
  "v_mfma_f32_16x16x32_f16 a[140:143], v[32:35], v[220:223], a[140:143]\n" \
  "ds_read_b128 v[32:35], v25 offset:40960\n" \
  "v_mfma_f32_16x16x32_f16 a[156:159], v[36:39], v[220:223], a[156:159]\n" \
  "buffer_store_dwordx4 a[108:111], v12, s[16:19], s40 offen offset:3072 sc1\n" \
  "v_mfma_f32_16x16x32_f16 a[152:155], v[36:39], v[216:219], a[152:155]\n" \
  "v_mfma_f32_16x16x32_f16 a[148:151], v[36:39], v[212:215], a[148:151]\n" \
  "s_waitcnt lgkmcnt(5)\n" \
  "v_mfma_f32_16x16x32_f16 a[144:147], v[36:39], v[208:211], a[144:147]\n" \
  "ds_read_b128 v[36:39], v25 offset:41984\n" \
  "v_mfma_f32_16x16x32_f16 a[160:163], v[40:43], v[208:211], a[160:163]\n" \
  "v_mfma_f32_16x16x32_f16 a[164:167], v[40:43], v[212:215], a[164:167]\n" \
  "v_mfma_f32_16x16x32_f16 a[168:171], v[40:43], v[216:219], a[168:171]\n" \
  "s_mov_b32 s40, 0x7000\n" \
  "v_mfma_f32_16x16x32_f16 a[172:175], v[40:43], v[220:223], a[172:175]\n" \
  "ds_read_b128 v[40:43], v25 offset:43008\n" \
  "v_mfma_f32_16x16x32_f16 a[188:191], v[44:47], v[220:223], a[188:191]\n" \
  "buffer_store_dwordx4 a[112:115], v12, s[16:19], s40 offen offset:0 sc1\n" \
  "v_mfma_f32_16x16x32_f16 a[184:187], v[44:47], v[216:219], a[184:187]\n" \
  "v_mfma_f32_16x16x32_f16 a[180:183], v[44:47], v[212:215], a[180:183]\n" \
  "s_waitcnt lgkmcnt(5)\n" \
  "v_mfma_f32_16x16x32_f16 a[176:179], v[44:47], v[208:211], a[176:179]\n" \
  "ds_read_b128 v[44:47], v25 offset:44032\n" \
  "v_mfma_f32_16x16x32_f16 a[192:195], v[48:51], v[208:211], a[192:195]\n" \
  "v_mfma_f32_16x16x32_f16 a[196:199], v[48:51], v[212:215], a[196:199]\n" \
  "v_mfma_f32_16x16x32_f16 a[200:203], v[48:51], v[216:219], a[200:203]\n" \
  "v_mfma_f32_16x16x32_f16 a[204:207], v[48:51], v[220:223], a[204:207]\n" \
  "ds_read_b128 v[48:51], v25 offset:45056\n" \
  "v_mfma_f32_16x16x32_f16 a[220:223], v[52:55], v[220:223], a[220:223]\n" \
  "buffer_store_dwordx4 a[116:119], v12, s[16:19], s40 offen offset:1024 sc1\n" \
  "v_mfma_f32_16x16x32_f16 a[216:219], v[52:55], v[216:219], a[216:219]\n" \
  "v_mfma_f32_16x16x32_f16 a[212:215], v[52:55], v[212:215], a[212:215]\n" \
  "s_waitcnt lgkmcnt(5)\n" \
  "v_mfma_f32_16x16x32_f16 a[208:211], v[52:55], v[208:211], a[208:211]\n" \
  "ds_read_b128 v[52:55], v25 offset:46080\n" \
  "v_mfma_f32_16x16x32_f16 a[224:227], v[56:59], v[208:211], a[224:227]\n" \
  "v_mfma_f32_16x16x32_f16 a[228:231], v[56:59], v[212:215], a[228:231]\n" \
  "v_mfma_f32_16x16x32_f16 a[232:235], v[56:59], v[216:219], a[232:235]\n" \
  "v_mfma_f32_16x16x32_f16 a[236:239], v[56:59], v[220:223], a[236:239]\n" \
  "ds_read_b128 v[56:59], v25 offset:47104\n" \
  "v_mfma_f32_16x16x32_f16 a[252:255], v[60:63], v[220:223], a[252:255]\n" \
  "buffer_store_dwordx4 a[120:123], v12, s[16:19], s40 offen offset:2048 sc1\n" \
  "v_mfma_f32_16x16x32_f16 a[248:251], v[60:63], v[216:219], a[248:251]\n" \
  "v_mfma_f32_16x16x32_f16 a[244:247], v[60:63], v[212:215], a[244:247]\n" \
  "s_waitcnt lgkmcnt(5)\n" \
  "v_mfma_f32_16x16x32_f16 a[240:243], v[60:63], v[208:211], a[240:243]\n" \
  "ds_read_b128 v[60:63], v25 offset:48128\n" \
  "v_mfma_f32_16x16x32_f16 a[128:131], v[32:35], v[96:99], a[128:131]\n" \
  "v_mfma_f32_16x16x32_f16 a[132:135], v[32:35], v[100:103], a[132:135]\n" \
  "v_mfma_f32_16x16x32_f16 a[136:139], v[32:35], v[104:107], a[136:139]\n" \
  "v_mfma_f32_16x16x32_f16 a[140:143], v[32:35], v[108:111], a[140:143]\n" \
  "v_mfma_f32_16x16x32_f16 a[156:159], v[36:39], v[108:111], a[156:159]\n" \
  "buffer_store_dwordx4 a[124:127], v12, s[16:19], s40 offen offset:3072 sc1\n" \
  "v_mfma_f32_16x16x32_f16 a[152:155], v[36:39], v[104:107], a[152:155]\n" \
  "v_mfma_f32_16x16x32_f16 a[148:151], v[36:39], v[100:103], a[148:151]\n" \
  "s_waitcnt lgkmcnt(4)\n" \
  "v_mfma_f32_16x16x32_f16 a[144:147], v[36:39], v[96:99], a[144:147]\n" \
  "v_mfma_f32_16x16x32_f16 a[160:163], v[40:43], v[96:99], a[160:163]\n" \
  "v_mfma_f32_16x16x32_f16 a[164:167], v[40:43], v[100:103], a[164:167]\n" \
  "v_mfma_f32_16x16x32_f16 a[168:171], v[40:43], v[104:107], a[168:171]\n" \
  "v_mfma_f32_16x16x32_f16 a[172:175], v[40:43], v[108:111], a[172:175]\n" \
  "v_mfma_f32_16x16x32_f16 a[188:191], v[44:47], v[108:111], a[188:191]\n" \
  "v_mfma_f32_16x16x32_f16 a[184:187], v[44:47], v[104:107], a[184:187]\n" \
  "v_mfma_f32_16x16x32_f16 a[180:183], v[44:47], v[100:103], a[180:183]\n" \
  "s_waitcnt lgkmcnt(2)\n" \
  "v_mfma_f32_16x16x32_f16 a[176:179], v[44:47], v[96:99], a[176:179]\n" \
  "v_mfma_f32_16x16x32_f16 a[192:195], v[48:51], v[96:99], a[192:195]\n" \
  "s_waitcnt vmcnt(16)\n" \
  "buffer_store_dword v15, v14, s[16:19], s59 offen sc1\n" \
  "v_mfma_f32_16x16x32_f16 a[196:199], v[48:51], v[100:103], a[196:199]\n" \
  "v_mfma_f32_16x16x32_f16 a[200:203], v[48:51], v[104:107], a[200:203]\n" \
  "v_mfma_f32_16x16x32_f16 a[204:207], v[48:51], v[108:111], a[204:207]\n" \
  "v_mfma_f32_16x16x32_f16 a[220:223], v[52:55], v[108:111], a[220:223]\n" \
  "v_mfma_f32_16x16x32_f16 a[216:219], v[52:55], v[104:107], a[216:219]\n" \
  "v_mfma_f32_16x16x32_f16 a[212:215], v[52:55], v[100:103], a[212:215]\n" \
  "s_waitcnt lgkmcnt(0)\n" \
  "v_mfma_f32_16x16x32_f16 a[208:211], v[52:55], v[96:99], a[208:211]\n" \
  "v_mfma_f32_16x16x32_f16 a[224:227], v[56:59], v[96:99], a[224:227]\n" \
  "v_mfma_f32_16x16x32_f16 a[228:231], v[56:59], v[100:103], a[228:231]\n" \
  "v_mfma_f32_16x16x32_f16 a[232:235], v[56:59], v[104:107], a[232:235]\n" \
  "v_mfma_f32_16x16x32_f16 a[236:239], v[56:59], v[108:111], a[236:239]\n" \
  "v_mfma_f32_16x16x32_f16 a[252:255], v[60:63], v[108:111], a[252:255]\n" \
  "v_mfma_f32_16x16x32_f16 a[248:251], v[60:63], v[104:107], a[248:251]\n" \
  "v_mfma_f32_16x16x32_f16 a[244:247], v[60:63], v[100:103], a[244:247]\n" \
  "v_mfma_f32_16x16x32_f16 a[240:243], v[60:63], v[96:99], a[240:243]\n" \
  "s_branch Lepi_%=\n" \
  ".p2align 6\n" \
  "Ltail0_%=:\n" \
  "v_mfma_f32_16x16x32_f16 a[240:243], v[60:63], v[80:83], a[240:243]\n" \
  "ds_read_b128 v[60:63], v25 offset:7168\n" \
  "v_mfma_f32_16x16x32_f16 a[0:3], v[32:35], v[64:67], a[0:3]\n" \
  "v_xor_b32 v26, 0x10000, v25\n" \
  "s_waitcnt vmcnt(47)\n" \
  "s_waitcnt vmcnt(37)\n" \
  "v_mfma_f32_16x16x32_f16 a[4:7], v[32:35], v[68:71], a[4:7]\n" \
  "v_cvt_pk_f16_f32 v80, v128, v132\n" \
  "v_cvt_pk_f16_f32 v84, v129, v133\n" \
  "s_barrier\n" \
  "v_mfma_f32_16x16x32_f16 a[8:11], v[32:35], v[72:75], a[8:11]\n" \
  "v_cvt_pk_f16_f32 v88, v130, v134\n" \
  "v_cvt_pk_f16_f32 v92, v131, v135\n" \
  "v_mfma_f32_16x16x32_f16 a[12:15], v[32:35], v[76:79], a[12:15]\n" \
  "ds_read_b128 v[32:35], v25 offset:16384\n" \
  "s_waitcnt vmcnt(45)\n" \
  "v_mfma_f32_16x16x32_f16 a[28:31], v[36:39], v[76:79], a[28:31]\n" \
  "v_cvt_pk_f16_f32 v81, v136, v140\n" \
  "v_cvt_pk_f16_f32 v85, v137, v141\n" \
  "v_mfma_f32_16x16x32_f16 a[24:27], v[36:39], v[72:75], a[24:27]\n" \
  "v_cvt_pk_f16_f32 v89, v138, v142\n" \
  "v_cvt_pk_f16_f32 v93, v139, v143\n" \
  "v_mfma_f32_16x16x32_f16 a[20:23], v[36:39], v[68:71], a[20:23]\n" \
  "s_waitcnt lgkmcnt(5)\n" \
  "s_bfe_u32 s40, s39, 0x80008\n" \
  "s_lshl_b32 s42, s40, 16\n" \
  "v_mfma_f32_16x16x32_f16 a[16:19], v[36:39], v[64:67], a[16:19]\n" \
  "ds_read_b128 v[36:39], v25 offset:17408\n" \
  "s_add_u32 m0, s36, 0x4000\n" \
  "v_mfma_f32_16x16x32_f16 a[32:35], v[40:43], v[64:67], a[32:35]\n" \
  "buffer_load_dwordx4 v24, s[12:15], s42 offen offset:0 sc0 sc1 lds\n" \
  "s_waitcnt vmcnt(44)\n" \
  "v_mfma_f32_16x16x32_f16 a[36:39], v[40:43], v[68:71], a[36:39]\n" \
  "v_cvt_pk_f16_f32 v82, v144, v148\n" \
  "v_cvt_pk_f16_f32 v86, v145, v149\n" \
  "v_mfma_f32_16x16x32_f16 a[40:43], v[40:43], v[72:75], a[40:43]\n" \
  "v_cvt_pk_f16_f32 v90, v146, v150\n" \
  "v_cvt_pk_f16_f32 v94, v147, v151\n" \
  "v_mfma_f32_16x16x32_f16 a[44:47], v[40:43], v[76:79], a[44:47]\n" \
  "ds_read_b128 v[40:43], v25 offset:18432\n" \
  "v_mfma_f32_16x16x32_f16 a[60:63], v[44:47], v[76:79], a[60:63]\n" \
  "buffer_load_dwordx4 v24, s[12:15], s42 offen offset:1024 sc0 sc1 lds\n" \
  "s_waitcnt vmcnt(43)\n" \
  "v_mfma_f32_16x16x32_f16 a[56:59], v[44:47], v[72:75], a[56:59]\n" \
  "v_cvt_pk_f16_f32 v83, v152, v156\n" \
  "v_cvt_pk_f16_f32 v87, v153, v157\n" \
  "v_mfma_f32_16x16x32_f16 a[52:55], v[44:47], v[68:71], a[52:55]\n" \
  "s_waitcnt lgkmcnt(5)\n" \
  "v_mfma_f32_16x16x32_f16 a[48:51], v[44:47], v[64:67], a[48:51]\n" \
  "ds_read_b128 v[44:47], v25 offset:19456\n" \
  "v_mfma_f32_16x16x32_f16 a[64:67], v[48:51], v[64:67], a[64:67]\n" \
  "buffer_load_dwordx4 v24, s[12:15], s42 offen offset:2048 sc0 sc1 lds\n" \
  "v_mfma_f32_16x16x32_f16 a[68:71], v[48:51], v[68:71], a[68:71]\n" \
  "v_cvt_pk_f16_f32 v91, v154, v158\n" \
  "v_cvt_pk_f16_f32 v95, v155, v159\n" \
  "v_mfma_f32_16x16x32_f16 a[72:75], v[48:51], v[72:75], a[72:75]\n" \
  "v_mfma_f32_16x16x32_f16 a[76:79], v[48:51], v[76:79], a[76:79]\n" \
  "ds_read_b128 v[48:51], v25 offset:20480\n" \
  "v_mfma_f32_16x16x32_f16 a[92:95], v[52:55], v[76:79], a[92:95]\n" \
  "buffer_load_dwordx4 v24, s[12:15], s42 offen offset:3072 sc0 sc1 lds\n" \
  "v_mfma_f32_16x16x32_f16 a[88:91], v[52:55], v[72:75], a[88:91]\n" \
  "v_mfma_f32_16x16x32_f16 a[84:87], v[52:55], v[68:71], a[84:87]\n" \
  "s_waitcnt lgkmcnt(5)\n" \
  "s_lshl_b32 s41, s40, 20\n" \
  "v_mfma_f32_16x16x32_f16 a[80:83], v[52:55], v[64:67], a[80:83]\n" \
  "ds_read_b128 v[52:55], v25 offset:21504\n" \
  "v_mfma_f32_16x16x32_f16 a[96:99], v[56:59], v[64:67], a[96:99]\n" \
  "buffer_load_dwordx4 v[128:131], v16, s[8:11], s41 offen\n" \
  "v_mfma_f32_16x16x32_f16 a[100:103], v[56:59], v[68:71], a[100:103]\n" \
  "v_mfma_f32_16x16x32_f16 a[104:107], v[56:59], v[72:75], a[104:107]\n" \
  "v_mfma_f32_16x16x32_f16 a[108:111], v[56:59], v[76:79], a[108:111]\n" \
  "ds_read_b128 v[56:59], v25 offset:22528\n" \
  "v_mfma_f32_16x16x32_f16 a[124:127], v[60:63], v[76:79], a[124:127]\n" \
  "buffer_load_dwordx4 v[132:135], v17, s[8:11], s41 offen\n" \
  "v_mfma_f32_16x16x32_f16 a[120:123], v[60:63], v[72:75], a[120:123]\n" \
  "s_waitcnt vmcnt(30)\n" \
  "v_mfma_f32_16x16x32_f16 a[116:119], v[60:63], v[68:71], a[116:119]\n" \
  "s_waitcnt lgkmcnt(5)\n" \
  "s_barrier\n" \
  "v_mfma_f32_16x16x32_f16 a[112:115], v[60:63], v[64:67], a[112:115]\n" \
  "ds_read_b128 v[60:63], v25 offset:23552\n" \
  "v_mfma_f32_16x16x32_f16 a[0:3], v[32:35], v[80:83], a[0:3]\n" \
  "buffer_load_dwordx4 v[136:139], v18, s[8:11], s41 offen\n" \
  "s_waitcnt vmcnt(42)\n" \
  "v_mfma_f32_16x16x32_f16 a[4:7], v[32:35], v[84:87], a[4:7]\n" \
  "v_cvt_pk_f16_f32 v224, v160, v164\n" \
  "v_cvt_pk_f16_f32 v228, v161, v165\n" \
  "v_mfma_f32_16x16x32_f16 a[8:11], v[32:35], v[88:91], a[8:11]\n" \
  "v_cvt_pk_f16_f32 v232, v162, v166\n" \
  "v_cvt_pk_f16_f32 v236, v163, v167\n" \
  "v_mfma_f32_16x16x32_f16 a[12:15], v[32:35], v[92:95], a[12:15]\n" \
  "ds_read_b128 v[32:35], v25 offset:32768\n" \
  "s_add_u32 s46, s41, 0x80000\n" \
  "v_mfma_f32_16x16x32_f16 a[28:31], v[36:39], v[92:95], a[28:31]\n" \
  "buffer_load_dwordx4 v[140:143], v19, s[8:11], s41 offen\n" \
  "s_waitcnt vmcnt(41)\n" \
  "v_mfma_f32_16x16x32_f16 a[24:27], v[36:39], v[88:91], a[24:27]\n" \
  "v_cvt_pk_f16_f32 v225, v168, v172\n" \
  "v_cvt_pk_f16_f32 v229, v169, v173\n" \
  "v_mfma_f32_16x16x32_f16 a[20:23], v[36:39], v[84:87], a[20:23]\n" \
  "s_waitcnt lgkmcnt(5)\n" \
  "v_mfma_f32_16x16x32_f16 a[16:19], v[36:39], v[80:83], a[16:19]\n" \
  "ds_read_b128 v[36:39], v25 offset:33792\n" \
  "v_mfma_f32_16x16x32_f16 a[32:35], v[40:43], v[80:83], a[32:35]\n" \
  "buffer_load_dwordx4 v[144:147], v16, s[8:11], s46 offen\n" \
  "v_mfma_f32_16x16x32_f16 a[36:39], v[40:43], v[84:87], a[36:39]\n" \
  "v_cvt_pk_f16_f32 v233, v170, v174\n" \
  "v_cvt_pk_f16_f32 v237, v171, v175\n" \
  "s_waitcnt vmcnt(40)\n" \
  "v_mfma_f32_16x16x32_f16 a[40:43], v[40:43], v[88:91], a[40:43]\n" \
  "v_cvt_pk_f16_f32 v226, v176, v180\n" \
  "v_cvt_pk_f16_f32 v230, v177, v181\n" \
  "v_mfma_f32_16x16x32_f16 a[44:47], v[40:43], v[92:95], a[44:47]\n" \
  "ds_read_b128 v[40:43], v25 offset:34816\n" \
  "v_mfma_f32_16x16x32_f16 a[60:63], v[44:47], v[92:95], a[60:63]\n" \
  "buffer_load_dwordx4 v[148:151], v17, s[8:11], s46 offen\n" \
  "v_mfma_f32_16x16x32_f16 a[56:59], v[44:47], v[88:91], a[56:59]\n" \
  "v_cvt_pk_f16_f32 v234, v178, v182\n" \
  "v_cvt_pk_f16_f32 v238, v179, v183\n" \
  "v_mfma_f32_16x16x32_f16 a[52:55], v[44:47], v[84:87], a[52:55]\n" \
  "s_waitcnt lgkmcnt(5)\n" \
  "v_mfma_f32_16x16x32_f16 a[48:51], v[44:47], v[80:83], a[48:51]\n" \
  "ds_read_b128 v[44:47], v25 offset:35840\n" \
  "v_mfma_f32_16x16x32_f16 a[64:67], v[48:51], v[80:83], a[64:67]\n" \
  "buffer_load_dwordx4 v[152:155], v18, s[8:11], s46 offen\n" \
  "s_waitcnt vmcnt(39)\n" \
  "v_mfma_f32_16x16x32_f16 a[68:71], v[48:51], v[84:87], a[68:71]\n" \
  "v_cvt_pk_f16_f32 v227, v184, v188\n" \
  "v_cvt_pk_f16_f32 v231, v185, v189\n" \
  "v_mfma_f32_16x16x32_f16 a[72:75], v[48:51], v[88:91], a[72:75]\n" \
  "v_cvt_pk_f16_f32 v235, v186, v190\n" \
  "v_cvt_pk_f16_f32 v239, v187, v191\n" \
  "v_mfma_f32_16x16x32_f16 a[76:79], v[48:51], v[92:95], a[76:79]\n" \
  "ds_read_b128 v[48:51], v25 offset:36864\n" \
  "v_mfma_f32_16x16x32_f16 a[92:95], v[52:55], v[92:95], a[92:95]\n" \
  "buffer_load_dwordx4 v[156:159], v19, s[8:11], s46 offen\n" \
  "v_mfma_f32_16x16x32_f16 a[88:91], v[52:55], v[88:91], a[88:91]\n" \
  "v_mfma_f32_16x16x32_f16 a[84:87], v[52:55], v[84:87], a[84:87]\n" \
  "s_waitcnt lgkmcnt(5)\n" \
  "s_bfe_u32 s40, s39, 0x80010\n" \
  "s_lshl_b32 s42, s40, 16\n" \
  "v_mfma_f32_16x16x32_f16 a[80:83], v[52:55], v[80:83], a[80:83]\n" \
  "ds_read_b128 v[52:55], v25 offset:37888\n" \
  "s_add_u32 m0, s36, 0x8000\n" \
  "v_mfma_f32_16x16x32_f16 a[96:99], v[56:59], v[80:83], a[96:99]\n" \
  "buffer_load_dwordx4 v24, s[12:15], s42 offen offset:0 sc0 sc1 lds\n" \
  "v_mfma_f32_16x16x32_f16 a[100:103], v[56:59], v[84:87], a[100:103]\n" \
  "v_mfma_f32_16x16x32_f16 a[104:107], v[56:59], v[88:91], a[104:107]\n" \
  "v_mfma_f32_16x16x32_f16 a[108:111], v[56:59], v[92:95], a[108:111]\n" \
  "ds_read_b128 v[56:59], v25 offset:38912\n" \
  "v_mfma_f32_16x16x32_f16 a[124:127], v[60:63], v[92:95], a[124:127]\n" \
  "buffer_load_dwordx4 v24, s[12:15], s42 offen offset:1024 sc0 sc1 lds\n" \
  "v_mfma_f32_16x16x32_f16 a[120:123], v[60:63], v[88:91], a[120:123]\n" \
  "s_waitcnt vmcnt(26)\n" \
  "v_mfma_f32_16x16x32_f16 a[116:119], v[60:63], v[84:87], a[116:119]\n" \
  "s_waitcnt lgkmcnt(5)\n" \
  "s_barrier\n" \
  "v_mfma_f32_16x16x32_f16 a[112:115], v[60:63], v[80:83], a[112:115]\n" \
  "ds_read_b128 v[60:63], v25 offset:39936\n" \
  "v_mfma_f32_16x16x32_f16 a[0:3], v[32:35], v[224:227], a[0:3]\n" \
  "buffer_load_dwordx4 v24, s[12:15], s42 offen offset:2048 sc0 sc1 lds\n" \
  "s_waitcnt vmcnt(37)\n" \
  "v_mfma_f32_16x16x32_f16 a[4:7], v[32:35], v[228:231], a[4:7]\n" \
  "v_cvt_pk_f16_f32 v240, v192, v196\n" \
  "v_cvt_pk_f16_f32 v244, v193, v197\n" \
  "v_mfma_f32_16x16x32_f16 a[8:11], v[32:35], v[232:235], a[8:11]\n" \
  "v_cvt_pk_f16_f32 v248, v194, v198\n" \
  "v_cvt_pk_f16_f32 v252, v195, v199\n" \
  "v_mfma_f32_16x16x32_f16 a[12:15], v[32:35], v[236:239], a[12:15]\n" \
  "ds_read_b128 v[32:35], v25 offset:49152\n" \
  "v_mfma_f32_16x16x32_f16 a[28:31], v[36:39], v[236:239], a[28:31]\n" \
  "buffer_load_dwordx4 v24, s[12:15], s42 offen offset:3072 sc0 sc1 lds\n" \
  "s_waitcnt vmcnt(36)\n" \
  "v_mfma_f32_16x16x32_f16 a[24:27], v[36:39], v[232:235], a[24:27]\n" \
  "v_cvt_pk_f16_f32 v241, v200, v204\n" \
  "v_cvt_pk_f16_f32 v245, v201, v205\n" \
  "v_mfma_f32_16x16x32_f16 a[20:23], v[36:39], v[228:231], a[20:23]\n" \
  "s_waitcnt lgkmcnt(5)\n" \
  "s_lshl_b32 s41, s40, 20\n" \
  "v_mfma_f32_16x16x32_f16 a[16:19], v[36:39], v[224:227], a[16:19]\n" \
  "ds_read_b128 v[36:39], v25 offset:50176\n" \
  "v_mfma_f32_16x16x32_f16 a[32:35], v[40:43], v[224:227], a[32:35]\n" \
  "buffer_load_dwordx4 v[160:163], v16, s[8:11], s41 offen\n" \
  "v_mfma_f32_16x16x32_f16 a[36:39], v[40:43], v[228:231], a[36:39]\n" \
  "v_cvt_pk_f16_f32 v249, v202, v206\n" \
  "v_cvt_pk_f16_f32 v253, v203, v207\n" \
  "s_waitcnt vmcnt(35)\n" \
  "v_mfma_f32_16x16x32_f16 a[40:43], v[40:43], v[232:235], a[40:43]\n" \
  "v_cvt_pk_f16_f32 v242, v208, v212\n" \
  "v_cvt_pk_f16_f32 v246, v209, v213\n" \
  "v_mfma_f32_16x16x32_f16 a[44:47], v[40:43], v[236:239], a[44:47]\n" \
  "ds_read_b128 v[40:43], v25 offset:51200\n" \
  "v_mfma_f32_16x16x32_f16 a[60:63], v[44:47], v[236:239], a[60:63]\n" \
  "buffer_load_dwordx4 v[164:167], v17, s[8:11], s41 offen\n" \
  "v_mfma_f32_16x16x32_f16 a[56:59], v[44:47], v[232:235], a[56:59]\n" \
  "v_cvt_pk_f16_f32 v250, v210, v214\n" \
  "v_cvt_pk_f16_f32 v254, v211, v215\n" \
  "v_mfma_f32_16x16x32_f16 a[52:55], v[44:47], v[228:231], a[52:55]\n" \
  "s_waitcnt lgkmcnt(5)\n" \
  "v_mfma_f32_16x16x32_f16 a[48:51], v[44:47], v[224:227], a[48:51]\n" \
  "ds_read_b128 v[44:47], v25 offset:52224\n" \
  "v_mfma_f32_16x16x32_f16 a[64:67], v[48:51], v[224:227], a[64:67]\n" \
  "buffer_load_dwordx4 v[168:171], v18, s[8:11], s41 offen\n" \
  "s_waitcnt vmcnt(35)\n" \
  "v_mfma_f32_16x16x32_f16 a[68:71], v[48:51], v[228:231], a[68:71]\n" \
  "v_cvt_pk_f16_f32 v243, v216, v220\n" \
  "v_cvt_pk_f16_f32 v247, v217, v221\n" \
  "v_mfma_f32_16x16x32_f16 a[72:75], v[48:51], v[232:235], a[72:75]\n" \
  "v_cvt_pk_f16_f32 v251, v218, v222\n" \
  "v_cvt_pk_f16_f32 v255, v219, v223\n" \
  "v_mfma_f32_16x16x32_f16 a[76:79], v[48:51], v[236:239], a[76:79]\n" \
  "ds_read_b128 v[48:51], v25 offset:53248\n" \
  "s_add_u32 s46, s41, 0x80000\n" \
  "v_mfma_f32_16x16x32_f16 a[92:95], v[52:55], v[236:239], a[92:95]\n" \
  "buffer_load_dwordx4 v[172:175], v19, s[8:11], s41 offen\n" \
  "v_mfma_f32_16x16x32_f16 a[88:91], v[52:55], v[232:235], a[88:91]\n" \
  "v_mfma_f32_16x16x32_f16 a[84:87], v[52:55], v[228:231], a[84:87]\n" \
  "s_waitcnt lgkmcnt(5)\n" \
  "v_mfma_f32_16x16x32_f16 a[80:83], v[52:55], v[224:227], a[80:83]\n" \
  "ds_read_b128 v[52:55], v25 offset:54272\n" \
  "v_mfma_f32_16x16x32_f16 a[96:99], v[56:59], v[224:227], a[96:99]\n" \
  "buffer_load_dwordx4 v[176:179], v16, s[8:11], s46 offen\n" \
  "v_mfma_f32_16x16x32_f16 a[100:103], v[56:59], v[228:231], a[100:103]\n" \
  "v_mfma_f32_16x16x32_f16 a[104:107], v[56:59], v[232:235], a[104:107]\n" \
  "v_mfma_f32_16x16x32_f16 a[108:111], v[56:59], v[236:239], a[108:111]\n" \
  "ds_read_b128 v[56:59], v25 offset:55296\n" \
  "v_mfma_f32_16x16x32_f16 a[124:127], v[60:63], v[236:239], a[124:127]\n" \
  "buffer_load_dwordx4 v[180:183], v17, s[8:11], s46 offen\n" \
  "v_mfma_f32_16x16x32_f16 a[120:123], v[60:63], v[232:235], a[120:123]\n" \
  "s_waitcnt vmcnt(22)\n" \
  "v_mfma_f32_16x16x32_f16 a[116:119], v[60:63], v[228:231], a[116:119]\n" \
  "s_waitcnt lgkmcnt(5)\n" \
  "s_barrier\n" \
  "v_mfma_f32_16x16x32_f16 a[112:115], v[60:63], v[224:227], a[112:115]\n" \
  "ds_read_b128 v[60:63], v25 offset:56320\n" \
  "v_mfma_f32_16x16x32_f16 a[0:3], v[32:35], v[240:243], a[0:3]\n" \
  "buffer_load_dwordx4 v[184:187], v18, s[8:11], s46 offen\n" \
  "s_waitcnt vmcnt(33)\n" \
  "v_mfma_f32_16x16x32_f16 a[4:7], v[32:35], v[244:247], a[4:7]\n" \
  "v_cvt_pk_f16_f32 v0, v96, v100\n" \
  "v_cvt_pk_f16_f32 v4, v97, v101\n" \
  "v_mfma_f32_16x16x32_f16 a[8:11], v[32:35], v[248:251], a[8:11]\n" \
  "v_cvt_pk_f16_f32 v8, v98, v102\n" \
  "v_cvt_pk_f16_f32 v20, v99, v103\n" \
  "v_mfma_f32_16x16x32_f16 a[12:15], v[32:35], v[252:255], a[12:15]\n" \
  "ds_read_b128 v[32:35], v26 offset:0\n" \
  "v_mfma_f32_16x16x32_f16 a[28:31], v[36:39], v[252:255], a[28:31]\n" \
  "buffer_load_dwordx4 v[188:191], v19, s[8:11], s46 offen\n" \
  "s_waitcnt vmcnt(32)\n" \
  "v_mfma_f32_16x16x32_f16 a[24:27], v[36:39], v[248:251], a[24:27]\n" \
  "v_cvt_pk_f16_f32 v1, v104, v108\n" \
  "v_cvt_pk_f16_f32 v5, v105, v109\n" \
  "v_mfma_f32_16x16x32_f16 a[20:23], v[36:39], v[244:247], a[20:23]\n" \
  "s_waitcnt lgkmcnt(5)\n" \
  "s_bfe_u32 s40, s39, 0x80018\n" \
  "s_lshl_b32 s42, s40, 16\n" \
  "v_mfma_f32_16x16x32_f16 a[16:19], v[36:39], v[240:243], a[16:19]\n" \
  "ds_read_b128 v[36:39], v26 offset:1024\n" \
  "s_add_u32 m0, s36, 0xc000\n" \
  "v_mfma_f32_16x16x32_f16 a[32:35], v[40:43], v[240:243], a[32:35]\n" \
  "buffer_load_dwordx4 v24, s[12:15], s42 offen offset:0 sc0 sc1 lds\n" \
  "v_mfma_f32_16x16x32_f16 a[36:39], v[40:43], v[244:247], a[36:39]\n" \
  "v_cvt_pk_f16_f32 v9, v106, v110\n" \
  "v_cvt_pk_f16_f32 v21, v107, v111\n" \
  "s_waitcnt vmcnt(31)\n" \
  "v_mfma_f32_16x16x32_f16 a[40:43], v[40:43], v[248:251], a[40:43]\n" \
  "v_cvt_pk_f16_f32 v2, v112, v116\n" \
  "v_cvt_pk_f16_f32 v6, v113, v117\n" \
  "v_mfma_f32_16x16x32_f16 a[44:47], v[40:43], v[252:255], a[44:47]\n" \
  "ds_read_b128 v[40:43], v26 offset:2048\n" \
  "v_mfma_f32_16x16x32_f16 a[60:63], v[44:47], v[252:255], a[60:63]\n" \
  "buffer_load_dwordx4 v24, s[12:15], s42 offen offset:1024 sc0 sc1 lds\n" \
  "v_mfma_f32_16x16x32_f16 a[56:59], v[44:47], v[248:251], a[56:59]\n" \
  "v_cvt_pk_f16_f32 v10, v114, v118\n" \
  "v_cvt_pk_f16_f32 v22, v115, v119\n" \
  "v_mfma_f32_16x16x32_f16 a[52:55], v[44:47], v[244:247], a[52:55]\n" \
  "s_waitcnt lgkmcnt(5)\n" \
  "v_mfma_f32_16x16x32_f16 a[48:51], v[44:47], v[240:243], a[48:51]\n" \
  "ds_read_b128 v[44:47], v26 offset:3072\n" \
  "v_mfma_f32_16x16x32_f16 a[64:67], v[48:51], v[240:243], a[64:67]\n" \
  "buffer_load_dwordx4 v24, s[12:15], s42 offen offset:2048 sc0 sc1 lds\n" \
  "s_waitcnt vmcnt(31)\n" \
  "v_mfma_f32_16x16x32_f16 a[68:71], v[48:51], v[244:247], a[68:71]\n" \
  "v_cvt_pk_f16_f32 v3, v120, v124\n" \
  "v_cvt_pk_f16_f32 v7, v121, v125\n" \
  "v_mfma_f32_16x16x32_f16 a[72:75], v[48:51], v[248:251], a[72:75]\n" \
  "v_cvt_pk_f16_f32 v11, v122, v126\n" \
  "v_cvt_pk_f16_f32 v23, v123, v127\n" \
  "v_mfma_f32_16x16x32_f16 a[76:79], v[48:51], v[252:255], a[76:79]\n" \
  "ds_read_b128 v[48:51], v26 offset:4096\n" \
  "v_mfma_f32_16x16x32_f16 a[92:95], v[52:55], v[252:255], a[92:95]\n" \
  "buffer_load_dwordx4 v24, s[12:15], s42 offen offset:3072 sc0 sc1 lds\n" \
  "v_mfma_f32_16x16x32_f16 a[88:91], v[52:55], v[248:251], a[88:91]\n" \
  "v_mfma_f32_16x16x32_f16 a[84:87], v[52:55], v[244:247], a[84:87]\n" \
  "s_waitcnt lgkmcnt(5)\n" \
  "s_lshl_b32 s41, s40, 20\n" \
  "v_mfma_f32_16x16x32_f16 a[80:83], v[52:55], v[240:243], a[80:83]\n" \
  "ds_read_b128 v[52:55], v26 offset:5120\n" \
  "v_mfma_f32_16x16x32_f16 a[96:99], v[56:59], v[240:243], a[96:99]\n" \
  "buffer_load_dwordx4 v[192:195], v16, s[8:11], s41 offen\n" \
  "v_mfma_f32_16x16x32_f16 a[100:103], v[56:59], v[244:247], a[100:103]\n" \
  "v_mfma_f32_16x16x32_f16 a[104:107], v[56:59], v[248:251], a[104:107]\n" \
  "v_mfma_f32_16x16x32_f16 a[108:111], v[56:59], v[252:255], a[108:111]\n" \
  "ds_read_b128 v[56:59], v26 offset:6144\n" \
  "v_mfma_f32_16x16x32_f16 a[124:127], v[60:63], v[252:255], a[124:127]\n" \
  "buffer_load_dwordx4 v[196:199], v17, s[8:11], s41 offen\n" \
  "v_mfma_f32_16x16x32_f16 a[120:123], v[60:63], v[248:251], a[120:123]\n" \
  "s_waitcnt vmcnt(26)\n" \
  "v_mfma_f32_16x16x32_f16 a[116:119], v[60:63], v[244:247], a[116:119]\n" \
  "s_waitcnt lgkmcnt(5)\n" \
  "s_barrier\n" \
  "v_mfma_f32_16x16x32_f16 a[112:115], v[60:63], v[240:243], a[112:115]\n" \
  "ds_read_b128 v[60:63], v26 offset:7168\n" \
  "v_mfma_f32_16x16x32_f16 a[0:3], v[32:35], v[0:3], a[0:3]\n" \
  "buffer_load_dwordx4 v[200:203], v18, s[8:11], s41 offen\n" \
  "s_waitcnt vmcnt(25)\n" \
  "v_mfma_f32_16x16x32_f16 a[4:7], v[32:35], v[4:7], a[4:7]\n" \
  "v_cvt_pk_f16_f32 v96, v128, v132\n" \
  "v_cvt_pk_f16_f32 v100, v129, v133\n" \
  "v_mfma_f32_16x16x32_f16 a[8:11], v[32:35], v[8:11], a[8:11]\n" \
  "v_cvt_pk_f16_f32 v104, v130, v134\n" \
  "v_cvt_pk_f16_f32 v108, v131, v135\n" \
  "v_mfma_f32_16x16x32_f16 a[12:15], v[32:35], v[20:23], a[12:15]\n" \
  "ds_read_b128 v[32:35], v26 offset:16384\n" \
  "s_add_u32 s46, s41, 0x80000\n" \
  "v_mfma_f32_16x16x32_f16 a[28:31], v[36:39], v[20:23], a[28:31]\n" \
  "buffer_load_dwordx4 v[204:207], v19, s[8:11], s41 offen\n" \
  "s_waitcnt vmcnt(24)\n" \
  "v_mfma_f32_16x16x32_f16 a[24:27], v[36:39], v[8:11], a[24:27]\n" \
  "v_cvt_pk_f16_f32 v97, v136, v140\n" \
  "v_cvt_pk_f16_f32 v101, v137, v141\n" \
  "v_mfma_f32_16x16x32_f16 a[20:23], v[36:39], v[4:7], a[20:23]\n" \
  "s_waitcnt lgkmcnt(5)\n" \
  "v_mfma_f32_16x16x32_f16 a[16:19], v[36:39], v[0:3], a[16:19]\n" \
  "ds_read_b128 v[36:39], v26 offset:17408\n" \
  "v_mfma_f32_16x16x32_f16 a[32:35], v[40:43], v[0:3], a[32:35]\n" \
  "buffer_load_dwordx4 v[208:211], v16, s[8:11], s46 offen\n" \
  "v_mfma_f32_16x16x32_f16 a[36:39], v[40:43], v[4:7], a[36:39]\n" \
  "v_cvt_pk_f16_f32 v105, v138, v142\n" \
  "v_cvt_pk_f16_f32 v109, v139, v143\n" \
  "s_waitcnt vmcnt(23)\n" \
  "v_mfma_f32_16x16x32_f16 a[40:43], v[40:43], v[8:11], a[40:43]\n" \
  "v_cvt_pk_f16_f32 v98, v144, v148\n" \
  "v_cvt_pk_f16_f32 v102, v145, v149\n" \
  "v_mfma_f32_16x16x32_f16 a[44:47], v[40:43], v[20:23], a[44:47]\n" \
  "ds_read_b128 v[40:43], v26 offset:18432\n" \
  "v_mfma_f32_16x16x32_f16 a[60:63], v[44:47], v[20:23], a[60:63]\n" \
  "buffer_load_dwordx4 v[212:215], v17, s[8:11], s46 offen\n" \
  "v_mfma_f32_16x16x32_f16 a[56:59], v[44:47], v[8:11], a[56:59]\n" \
  "v_cvt_pk_f16_f32 v106, v146, v150\n" \
  "v_cvt_pk_f16_f32 v110, v147, v151\n" \
  "v_mfma_f32_16x16x32_f16 a[52:55], v[44:47], v[4:7], a[52:55]\n" \
  "s_waitcnt lgkmcnt(5)\n" \
  "v_mfma_f32_16x16x32_f16 a[48:51], v[44:47], v[0:3], a[48:51]\n" \
  "ds_read_b128 v[44:47], v26 offset:19456\n" \
  "v_mfma_f32_16x16x32_f16 a[64:67], v[48:51], v[0:3], a[64:67]\n" \
  "buffer_load_dwordx4 v[216:219], v18, s[8:11], s46 offen\n" \
  "s_waitcnt vmcnt(23)\n" \
  "v_mfma_f32_16x16x32_f16 a[68:71], v[48:51], v[4:7], a[68:71]\n" \
  "v_cvt_pk_f16_f32 v99, v152, v156\n" \
  "v_cvt_pk_f16_f32 v103, v153, v157\n" \
  "v_mfma_f32_16x16x32_f16 a[72:75], v[48:51], v[8:11], a[72:75]\n" \
  "v_cvt_pk_f16_f32 v107, v154, v158\n" \
  "v_cvt_pk_f16_f32 v111, v155, v159\n" \
  "v_mfma_f32_16x16x32_f16 a[76:79], v[48:51], v[20:23], a[76:79]\n" \
  "ds_read_b128 v[48:51], v26 offset:20480\n" \
  "v_mfma_f32_16x16x32_f16 a[92:95], v[52:55], v[20:23], a[92:95]\n" \
  "buffer_load_dwordx4 v[220:223], v19, s[8:11], s46 offen\n" \
  "v_mfma_f32_16x16x32_f16 a[88:91], v[52:55], v[8:11], a[88:91]\n" \
  "v_mfma_f32_16x16x32_f16 a[84:87], v[52:55], v[4:7], a[84:87]\n" \
  "s_waitcnt lgkmcnt(5)\n" \
  "v_mfma_f32_16x16x32_f16 a[80:83], v[52:55], v[0:3], a[80:83]\n" \
  "ds_read_b128 v[52:55], v26 offset:21504\n" \
  "v_mfma_f32_16x16x32_f16 a[96:99], v[56:59], v[0:3], a[96:99]\n" \
  "v_mfma_f32_16x16x32_f16 a[100:103], v[56:59], v[4:7], a[100:103]\n" \
  "v_mfma_f32_16x16x32_f16 a[104:107], v[56:59], v[8:11], a[104:107]\n" \
  "v_mfma_f32_16x16x32_f16 a[108:111], v[56:59], v[20:23], a[108:111]\n" \
  "ds_read_b128 v[56:59], v26 offset:22528\n" \
  "v_mfma_f32_16x16x32_f16 a[124:127], v[60:63], v[20:23], a[124:127]\n" \
  "v_mfma_f32_16x16x32_f16 a[120:123], v[60:63], v[8:11], a[120:123]\n" \
  "s_waitcnt vmcnt(20)\n" \
  "v_mfma_f32_16x16x32_f16 a[116:119], v[60:63], v[4:7], a[116:119]\n" \
  "s_waitcnt lgkmcnt(5)\n" \
  "s_barrier\n" \
  "v_mfma_f32_16x16x32_f16 a[112:115], v[60:63], v[0:3], a[112:115]\n" \
  "ds_read_b128 v[60:63], v26 offset:23552\n" \
  "v_mfma_f32_16x16x32_f16 a[0:3], v[32:35], v[96:99], a[0:3]\n" \
  "s_waitcnt vmcnt(18)\n" \
  "v_mfma_f32_16x16x32_f16 a[4:7], v[32:35], v[100:103], a[4:7]\n" \
  "v_cvt_pk_f16_f32 v112, v160, v164\n" \
  "v_cvt_pk_f16_f32 v116, v161, v165\n" \
  "v_mfma_f32_16x16x32_f16 a[8:11], v[32:35], v[104:107], a[8:11]\n" \
  "v_cvt_pk_f16_f32 v120, v162, v166\n" \
  "v_cvt_pk_f16_f32 v124, v163, v167\n" \
  "v_mfma_f32_16x16x32_f16 a[12:15], v[32:35], v[108:111], a[12:15]\n" \
  "ds_read_b128 v[32:35], v26 offset:32768\n" \
  "s_waitcnt vmcnt(16)\n" \
  "v_mfma_f32_16x16x32_f16 a[28:31], v[36:39], v[108:111], a[28:31]\n" \
  "v_cvt_pk_f16_f32 v113, v168, v172\n" \
  "v_cvt_pk_f16_f32 v117, v169, v173\n" \
  "v_mfma_f32_16x16x32_f16 a[24:27], v[36:39], v[104:107], a[24:27]\n" \
  "v_cvt_pk_f16_f32 v121, v170, v174\n" \
  "v_cvt_pk_f16_f32 v125, v171, v175\n" \
  "v_mfma_f32_16x16x32_f16 a[20:23], v[36:39], v[100:103], a[20:23]\n" \
  "s_waitcnt lgkmcnt(5)\n" \
  "v_mfma_f32_16x16x32_f16 a[16:19], v[36:39], v[96:99], a[16:19]\n" \
  "ds_read_b128 v[36:39], v26 offset:33792\n" \
  "s_waitcnt vmcnt(14)\n" \
  "v_mfma_f32_16x16x32_f16 a[32:35], v[40:43], v[96:99], a[32:35]\n" \
  "v_cvt_pk_f16_f32 v114, v176, v180\n" \
  "v_cvt_pk_f16_f32 v118, v177, v181\n" \
  "v_mfma_f32_16x16x32_f16 a[36:39], v[40:43], v[100:103], a[36:39]\n" \
  "v_cvt_pk_f16_f32 v122, v178, v182\n" \
  "v_cvt_pk_f16_f32 v126, v179, v183\n" \
  "s_waitcnt vmcnt(12)\n" \
  "v_mfma_f32_16x16x32_f16 a[40:43], v[40:43], v[104:107], a[40:43]\n" \
  "v_cvt_pk_f16_f32 v115, v184, v188\n" \
  "v_cvt_pk_f16_f32 v119, v185, v189\n" \
  "v_mfma_f32_16x16x32_f16 a[44:47], v[40:43], v[108:111], a[44:47]\n" \
  "ds_read_b128 v[40:43], v26 offset:34816\n" \
  "v_mfma_f32_16x16x32_f16 a[60:63], v[44:47], v[108:111], a[60:63]\n" \
  "v_cvt_pk_f16_f32 v123, v186, v190\n" \
  "v_cvt_pk_f16_f32 v127, v187, v191\n" \
  "v_mfma_f32_16x16x32_f16 a[56:59], v[44:47], v[104:107], a[56:59]\n" \
  "v_mfma_f32_16x16x32_f16 a[52:55], v[44:47], v[100:103], a[52:55]\n" \
  "s_waitcnt lgkmcnt(5)\n" \
  "v_mfma_f32_16x16x32_f16 a[48:51], v[44:47], v[96:99], a[48:51]\n" \
  "ds_read_b128 v[44:47], v26 offset:35840\n" \
  "v_mfma_f32_16x16x32_f16 a[64:67], v[48:51], v[96:99], a[64:67]\n" \
  "v_mfma_f32_16x16x32_f16 a[68:71], v[48:51], v[100:103], a[68:71]\n" \
  "v_mfma_f32_16x16x32_f16 a[72:75], v[48:51], v[104:107], a[72:75]\n" \
  "v_mfma_f32_16x16x32_f16 a[76:79], v[48:51], v[108:111], a[76:79]\n" \
  "ds_read_b128 v[48:51], v26 offset:36864\n" \
  "v_mfma_f32_16x16x32_f16 a[92:95], v[52:55], v[108:111], a[92:95]\n" \
  "v_mfma_f32_16x16x32_f16 a[88:91], v[52:55], v[104:107], a[88:91]\n" \
  "v_mfma_f32_16x16x32_f16 a[84:87], v[52:55], v[100:103], a[84:87]\n" \
  "s_waitcnt lgkmcnt(5)\n" \
  "v_mfma_f32_16x16x32_f16 a[80:83], v[52:55], v[96:99], a[80:83]\n" \
  "ds_read_b128 v[52:55], v26 offset:37888\n" \
  "v_mfma_f32_16x16x32_f16 a[96:99], v[56:59], v[96:99], a[96:99]\n" \
  "v_mfma_f32_16x16x32_f16 a[100:103], v[56:59], v[100:103], a[100:103]\n" \
  "v_mfma_f32_16x16x32_f16 a[104:107], v[56:59], v[104:107], a[104:107]\n" \
  "v_mfma_f32_16x16x32_f16 a[108:111], v[56:59], v[108:111], a[108:111]\n" \
  "ds_read_b128 v[56:59], v26 offset:38912\n" \
  "v_mfma_f32_16x16x32_f16 a[124:127], v[60:63], v[108:111], a[124:127]\n" \
  "v_mfma_f32_16x16x32_f16 a[120:123], v[60:63], v[104:107], a[120:123]\n" \
  "s_waitcnt vmcnt(8)\n" \
  "v_mfma_f32_16x16x32_f16 a[116:119], v[60:63], v[100:103], a[116:119]\n" \
  "s_waitcnt lgkmcnt(5)\n" \
  "s_barrier\n" \
  "v_mfma_f32_16x16x32_f16 a[112:115], v[60:63], v[96:99], a[112:115]\n" \
  "ds_read_b128 v[60:63], v26 offset:39936\n" \
  "v_mfma_f32_16x16x32_f16 a[0:3], v[32:35], v[112:115], a[0:3]\n" \
  "s_waitcnt vmcnt(6)\n" \
  "v_mfma_f32_16x16x32_f16 a[4:7], v[32:35], v[116:119], a[4:7]\n" \
  "v_cvt_pk_f16_f32 v128, v192, v196\n" \
  "v_cvt_pk_f16_f32 v132, v193, v197\n" \
  "v_mfma_f32_16x16x32_f16 a[8:11], v[32:35], v[120:123], a[8:11]\n" \
  "v_cvt_pk_f16_f32 v136, v194, v198\n" \
  "v_cvt_pk_f16_f32 v140, v195, v199\n" \
  "v_mfma_f32_16x16x32_f16 a[12:15], v[32:35], v[124:127], a[12:15]\n" \
  "ds_read_b128 v[32:35], v26 offset:49152\n" \
  "s_waitcnt vmcnt(4)\n" \
  "v_mfma_f32_16x16x32_f16 a[28:31], v[36:39], v[124:127], a[28:31]\n" \
  "v_cvt_pk_f16_f32 v129, v200, v204\n" \
  "v_cvt_pk_f16_f32 v133, v201, v205\n" \
  "v_mfma_f32_16x16x32_f16 a[24:27], v[36:39], v[120:123], a[24:27]\n" \
  "v_cvt_pk_f16_f32 v137, v202, v206\n" \
  "v_cvt_pk_f16_f32 v141, v203, v207\n" \
  "v_mfma_f32_16x16x32_f16 a[20:23], v[36:39], v[116:119], a[20:23]\n" \
  "s_waitcnt lgkmcnt(5)\n" \
  "v_mfma_f32_16x16x32_f16 a[16:19], v[36:39], v[112:115], a[16:19]\n" \
  "ds_read_b128 v[36:39], v26 offset:50176\n" \
  "s_waitcnt vmcnt(2)\n" \
  "v_mfma_f32_16x16x32_f16 a[32:35], v[40:43], v[112:115], a[32:35]\n" \
  "v_cvt_pk_f16_f32 v130, v208, v212\n" \
  "v_cvt_pk_f16_f32 v134, v209, v213\n" \
  "v_mfma_f32_16x16x32_f16 a[36:39], v[40:43], v[116:119], a[36:39]\n" \
  "v_cvt_pk_f16_f32 v138, v210, v214\n" \
  "v_cvt_pk_f16_f32 v142, v211, v215\n" \
  "s_waitcnt vmcnt(0)\n" \
  "v_mfma_f32_16x16x32_f16 a[40:43], v[40:43], v[120:123], a[40:43]\n" \
  "v_cvt_pk_f16_f32 v131, v216, v220\n" \
  "v_cvt_pk_f16_f32 v135, v217, v221\n" \
  "v_mfma_f32_16x16x32_f16 a[44:47], v[40:43], v[124:127], a[44:47]\n" \
  "ds_read_b128 v[40:43], v26 offset:51200\n" \
  "v_mfma_f32_16x16x32_f16 a[60:63], v[44:47], v[124:127], a[60:63]\n" \
  "v_cvt_pk_f16_f32 v139, v218, v222\n" \
  "v_cvt_pk_f16_f32 v143, v219, v223\n" \
  "v_mfma_f32_16x16x32_f16 a[56:59], v[44:47], v[120:123], a[56:59]\n" \
  "v_mfma_f32_16x16x32_f16 a[52:55], v[44:47], v[116:119], a[52:55]\n" \
  "s_waitcnt lgkmcnt(5)\n" \
  "v_mfma_f32_16x16x32_f16 a[48:51], v[44:47], v[112:115], a[48:51]\n" \
  "ds_read_b128 v[44:47], v26 offset:52224\n" \
  "v_mfma_f32_16x16x32_f16 a[64:67], v[48:51], v[112:115], a[64:67]\n" \
  "v_mfma_f32_16x16x32_f16 a[68:71], v[48:51], v[116:119], a[68:71]\n" \
  "v_mfma_f32_16x16x32_f16 a[72:75], v[48:51], v[120:123], a[72:75]\n" \
  "v_mfma_f32_16x16x32_f16 a[76:79], v[48:51], v[124:127], a[76:79]\n" \
  "ds_read_b128 v[48:51], v26 offset:53248\n" \
  "v_mfma_f32_16x16x32_f16 a[92:95], v[52:55], v[124:127], a[92:95]\n" \
  "v_mfma_f32_16x16x32_f16 a[88:91], v[52:55], v[120:123], a[88:91]\n" \
  "v_mfma_f32_16x16x32_f16 a[84:87], v[52:55], v[116:119], a[84:87]\n" \
  "s_waitcnt lgkmcnt(5)\n" \
  "v_mfma_f32_16x16x32_f16 a[80:83], v[52:55], v[112:115], a[80:83]\n" \
  "ds_read_b128 v[52:55], v26 offset:54272\n" \
  "v_mfma_f32_16x16x32_f16 a[96:99], v[56:59], v[112:115], a[96:99]\n" \
  "v_mfma_f32_16x16x32_f16 a[100:103], v[56:59], v[116:119], a[100:103]\n" \
  "v_mfma_f32_16x16x32_f16 a[104:107], v[56:59], v[120:123], a[104:107]\n" \
  "v_mfma_f32_16x16x32_f16 a[108:111], v[56:59], v[124:127], a[108:111]\n" \
  "ds_read_b128 v[56:59], v26 offset:55296\n" \
  "v_mfma_f32_16x16x32_f16 a[124:127], v[60:63], v[124:127], a[124:127]\n" \
  "v_mfma_f32_16x16x32_f16 a[120:123], v[60:63], v[120:123], a[120:123]\n" \
  "v_mfma_f32_16x16x32_f16 a[116:119], v[60:63], v[116:119], a[116:119]\n" \
  "s_waitcnt lgkmcnt(5)\n" \
  "v_mfma_f32_16x16x32_f16 a[112:115], v[60:63], v[112:115], a[112:115]\n" \
  "ds_read_b128 v[60:63], v26 offset:56320\n" \
  "v_mfma_f32_16x16x32_f16 a[0:3], v[32:35], v[128:131], a[0:3]\n" \
  "v_mfma_f32_16x16x32_f16 a[4:7], v[32:35], v[132:135], a[4:7]\n" \
  "v_mfma_f32_16x16x32_f16 a[8:11], v[32:35], v[136:139], a[8:11]\n" \
  "v_mfma_f32_16x16x32_f16 a[12:15], v[32:35], v[140:143], a[12:15]\n" \
  "ds_read_b128 v[32:35], v25 offset:8192\n" \
  "v_mfma_f32_16x16x32_f16 a[28:31], v[36:39], v[140:143], a[28:31]\n" \
  "v_mfma_f32_16x16x32_f16 a[24:27], v[36:39], v[136:139], a[24:27]\n" \
  "v_mfma_f32_16x16x32_f16 a[20:23], v[36:39], v[132:135], a[20:23]\n" \
  "s_waitcnt lgkmcnt(5)\n" \
  "v_mfma_f32_16x16x32_f16 a[16:19], v[36:39], v[128:131], a[16:19]\n" \
  "ds_read_b128 v[36:39], v25 offset:9216\n" \
  "v_mfma_f32_16x16x32_f16 a[32:35], v[40:43], v[128:131], a[32:35]\n" \
  "v_mfma_f32_16x16x32_f16 a[36:39], v[40:43], v[132:135], a[36:39]\n" \
  "v_mfma_f32_16x16x32_f16 a[40:43], v[40:43], v[136:139], a[40:43]\n" \
  "s_mov_b32 s40, 0x0\n" \
  "v_mfma_f32_16x16x32_f16 a[44:47], v[40:43], v[140:143], a[44:47]\n" \
  "ds_read_b128 v[40:43], v25 offset:10240\n" \
  "v_mfma_f32_16x16x32_f16 a[60:63], v[44:47], v[140:143], a[60:63]\n" \
  "buffer_store_dwordx4 a[0:3], v12, s[16:19], s40 offen offset:0 sc1\n" \
  "v_mfma_f32_16x16x32_f16 a[56:59], v[44:47], v[136:139], a[56:59]\n" \
  "v_mfma_f32_16x16x32_f16 a[52:55], v[44:47], v[132:135], a[52:55]\n" \
  "s_waitcnt lgkmcnt(5)\n" \
  "v_mfma_f32_16x16x32_f16 a[48:51], v[44:47], v[128:131], a[48:51]\n" \
  "ds_read_b128 v[44:47], v25 offset:11264\n" \
  "v_mfma_f32_16x16x32_f16 a[64:67], v[48:51], v[128:131], a[64:67]\n" \
  "v_mfma_f32_16x16x32_f16 a[68:71], v[48:51], v[132:135], a[68:71]\n" \
  "v_mfma_f32_16x16x32_f16 a[72:75], v[48:51], v[136:139], a[72:75]\n" \
  "v_mfma_f32_16x16x32_f16 a[76:79], v[48:51], v[140:143], a[76:79]\n" \
  "ds_read_b128 v[48:51], v25 offset:12288\n" \
  "v_mfma_f32_16x16x32_f16 a[92:95], v[52:55], v[140:143], a[92:95]\n" \
  "buffer_store_dwordx4 a[4:7], v12, s[16:19], s40 offen offset:1024 sc1\n" \
  "v_mfma_f32_16x16x32_f16 a[88:91], v[52:55], v[136:139], a[88:91]\n" \
  "v_mfma_f32_16x16x32_f16 a[84:87], v[52:55], v[132:135], a[84:87]\n" \
  "s_waitcnt lgkmcnt(5)\n" \
  "v_mfma_f32_16x16x32_f16 a[80:83], v[52:55], v[128:131], a[80:83]\n" \
  "ds_read_b128 v[52:55], v25 offset:13312\n" \
  "v_mfma_f32_16x16x32_f16 a[96:99], v[56:59], v[128:131], a[96:99]\n" \
  "v_mfma_f32_16x16x32_f16 a[100:103], v[56:59], v[132:135], a[100:103]\n" \
  "v_mfma_f32_16x16x32_f16 a[104:107], v[56:59], v[136:139], a[104:107]\n" \
  "v_mfma_f32_16x16x32_f16 a[108:111], v[56:59], v[140:143], a[108:111]\n" \
  "ds_read_b128 v[56:59], v25 offset:14336\n" \
  "v_mfma_f32_16x16x32_f16 a[124:127], v[60:63], v[140:143], a[124:127]\n" \
  "buffer_store_dwordx4 a[8:11], v12, s[16:19], s40 offen offset:2048 sc1\n" \
  "v_mfma_f32_16x16x32_f16 a[120:123], v[60:63], v[136:139], a[120:123]\n" \
  "v_mfma_f32_16x16x32_f16 a[116:119], v[60:63], v[132:135], a[116:119]\n" \
  "s_waitcnt lgkmcnt(5)\n" \
  "v_mfma_f32_16x16x32_f16 a[112:115], v[60:63], v[128:131], a[112:115]\n" \
  "ds_read_b128 v[60:63], v25 offset:15360\n" \
  "v_mfma_f32_16x16x32_f16 a[128:131], v[32:35], v[64:67], a[128:131]\n" \
  "v_mfma_f32_16x16x32_f16 a[132:135], v[32:35], v[68:71], a[132:135]\n" \
  "v_mfma_f32_16x16x32_f16 a[136:139], v[32:35], v[72:75], a[136:139]\n" \
  "v_mfma_f32_16x16x32_f16 a[140:143], v[32:35], v[76:79], a[140:143]\n" \
  "ds_read_b128 v[32:35], v25 offset:24576\n" \
  "v_mfma_f32_16x16x32_f16 a[156:159], v[36:39], v[76:79], a[156:159]\n" \
  "buffer_store_dwordx4 a[12:15], v12, s[16:19], s40 offen offset:3072 sc1\n" \
  "v_mfma_f32_16x16x32_f16 a[152:155], v[36:39], v[72:75], a[152:155]\n" \
  "v_mfma_f32_16x16x32_f16 a[148:151], v[36:39], v[68:71], a[148:151]\n" \
  "s_waitcnt lgkmcnt(5)\n" \
  "v_mfma_f32_16x16x32_f16 a[144:147], v[36:39], v[64:67], a[144:147]\n" \
  "ds_read_b128 v[36:39], v25 offset:25600\n" \
  "v_mfma_f32_16x16x32_f16 a[160:163], v[40:43], v[64:67], a[160:163]\n" \
  "v_mfma_f32_16x16x32_f16 a[164:167], v[40:43], v[68:71], a[164:167]\n" \
  "v_mfma_f32_16x16x32_f16 a[168:171], v[40:43], v[72:75], a[168:171]\n" \
  "s_mov_b32 s40, 0x1000\n" \
  "v_mfma_f32_16x16x32_f16 a[172:175], v[40:43], v[76:79], a[172:175]\n" \
  "ds_read_b128 v[40:43], v25 offset:26624\n" \
  "v_mfma_f32_16x16x32_f16 a[188:191], v[44:47], v[76:79], a[188:191]\n" \
  "buffer_store_dwordx4 a[16:19], v12, s[16:19], s40 offen offset:0 sc1\n" \
  "v_mfma_f32_16x16x32_f16 a[184:187], v[44:47], v[72:75], a[184:187]\n" \
  "v_mfma_f32_16x16x32_f16 a[180:183], v[44:47], v[68:71], a[180:183]\n" \
  "s_waitcnt lgkmcnt(5)\n" \
  "v_mfma_f32_16x16x32_f16 a[176:179], v[44:47], v[64:67], a[176:179]\n" \
  "ds_read_b128 v[44:47], v25 offset:27648\n" \
  "v_mfma_f32_16x16x32_f16 a[192:195], v[48:51], v[64:67], a[192:195]\n" \
  "v_mfma_f32_16x16x32_f16 a[196:199], v[48:51], v[68:71], a[196:199]\n" \
  "v_mfma_f32_16x16x32_f16 a[200:203], v[48:51], v[72:75], a[200:203]\n" \
  "v_mfma_f32_16x16x32_f16 a[204:207], v[48:51], v[76:79], a[204:207]\n" \
  "ds_read_b128 v[48:51], v25 offset:28672\n" \
  "v_mfma_f32_16x16x32_f16 a[220:223], v[52:55], v[76:79], a[220:223]\n" \
  "buffer_store_dwordx4 a[20:23], v12, s[16:19], s40 offen offset:1024 sc1\n" \
  "v_mfma_f32_16x16x32_f16 a[216:219], v[52:55], v[72:75], a[216:219]\n" \
  "v_mfma_f32_16x16x32_f16 a[212:215], v[52:55], v[68:71], a[212:215]\n" \
  "s_waitcnt lgkmcnt(5)\n" \
  "v_mfma_f32_16x16x32_f16 a[208:211], v[52:55], v[64:67], a[208:211]\n" \
  "ds_read_b128 v[52:55], v25 offset:29696\n" \
  "v_mfma_f32_16x16x32_f16 a[224:227], v[56:59], v[64:67], a[224:227]\n" \
  "v_mfma_f32_16x16x32_f16 a[228:231], v[56:59], v[68:71], a[228:231]\n" \
  "v_mfma_f32_16x16x32_f16 a[232:235], v[56:59], v[72:75], a[232:235]\n" \
  "v_mfma_f32_16x16x32_f16 a[236:239], v[56:59], v[76:79], a[236:239]\n" \
  "ds_read_b128 v[56:59], v25 offset:30720\n" \
  "v_mfma_f32_16x16x32_f16 a[252:255], v[60:63], v[76:79], a[252:255]\n" \
  "buffer_store_dwordx4 a[24:27], v12, s[16:19], s40 offen offset:2048 sc1\n" \
  "v_mfma_f32_16x16x32_f16 a[248:251], v[60:63], v[72:75], a[248:251]\n" \
  "v_mfma_f32_16x16x32_f16 a[244:247], v[60:63], v[68:71], a[244:247]\n" \
  "s_waitcnt lgkmcnt(5)\n" \
  "v_mfma_f32_16x16x32_f16 a[240:243], v[60:63], v[64:67], a[240:243]\n" \
  "ds_read_b128 v[60:63], v25 offset:31744\n" \
  "v_mfma_f32_16x16x32_f16 a[128:131], v[32:35], v[80:83], a[128:131]\n" \
  "v_mfma_f32_16x16x32_f16 a[132:135], v[32:35], v[84:87], a[132:135]\n" \
  "v_mfma_f32_16x16x32_f16 a[136:139], v[32:35], v[88:91], a[136:139]\n" \
  "v_mfma_f32_16x16x32_f16 a[140:143], v[32:35], v[92:95], a[140:143]\n" \
  "ds_read_b128 v[32:35], v25 offset:40960\n" \
  "v_mfma_f32_16x16x32_f16 a[156:159], v[36:39], v[92:95], a[156:159]\n" \
  "buffer_store_dwordx4 a[28:31], v12, s[16:19], s40 offen offset:3072 sc1\n" \
  "v_mfma_f32_16x16x32_f16 a[152:155], v[36:39], v[88:91], a[152:155]\n" \
  "v_mfma_f32_16x16x32_f16 a[148:151], v[36:39], v[84:87], a[148:151]\n" \
  "s_waitcnt lgkmcnt(5)\n" \
  "v_mfma_f32_16x16x32_f16 a[144:147], v[36:39], v[80:83], a[144:147]\n" \
  "ds_read_b128 v[36:39], v25 offset:41984\n" \
  "v_mfma_f32_16x16x32_f16 a[160:163], v[40:43], v[80:83], a[160:163]\n" \
  "v_mfma_f32_16x16x32_f16 a[164:167], v[40:43], v[84:87], a[164:167]\n" \
  "v_mfma_f32_16x16x32_f16 a[168:171], v[40:43], v[88:91], a[168:171]\n" \
  "s_mov_b32 s40, 0x2000\n" \
  "v_mfma_f32_16x16x32_f16 a[172:175], v[40:43], v[92:95], a[172:175]\n" \
  "ds_read_b128 v[40:43], v25 offset:43008\n" \
  "v_mfma_f32_16x16x32_f16 a[188:191], v[44:47], v[92:95], a[188:191]\n" \
  "buffer_store_dwordx4 a[32:35], v12, s[16:19], s40 offen offset:0 sc1\n" \
  "v_mfma_f32_16x16x32_f16 a[184:187], v[44:47], v[88:91], a[184:187]\n" \
  "v_mfma_f32_16x16x32_f16 a[180:183], v[44:47], v[84:87], a[180:183]\n" \
  "s_waitcnt lgkmcnt(5)\n" \
  "v_mfma_f32_16x16x32_f16 a[176:179], v[44:47], v[80:83], a[176:179]\n" \
  "ds_read_b128 v[44:47], v25 offset:44032\n" \
  "v_mfma_f32_16x16x32_f16 a[192:195], v[48:51], v[80:83], a[192:195]\n" \
  "v_mfma_f32_16x16x32_f16 a[196:199], v[48:51], v[84:87], a[196:199]\n" \
  "v_mfma_f32_16x16x32_f16 a[200:203], v[48:51], v[88:91], a[200:203]\n" \
  "v_mfma_f32_16x16x32_f16 a[204:207], v[48:51], v[92:95], a[204:207]\n" \
  "ds_read_b128 v[48:51], v25 offset:45056\n" \
  "v_mfma_f32_16x16x32_f16 a[220:223], v[52:55], v[92:95], a[220:223]\n" \
  "buffer_store_dwordx4 a[36:39], v12, s[16:19], s40 offen offset:1024 sc1\n" \
  "v_mfma_f32_16x16x32_f16 a[216:219], v[52:55], v[88:91], a[216:219]\n" \
  "v_mfma_f32_16x16x32_f16 a[212:215], v[52:55], v[84:87], a[212:215]\n" \
  "s_waitcnt lgkmcnt(5)\n" \
  "v_mfma_f32_16x16x32_f16 a[208:211], v[52:55], v[80:83], a[208:211]\n" \
  "ds_read_b128 v[52:55], v25 offset:46080\n" \
  "v_mfma_f32_16x16x32_f16 a[224:227], v[56:59], v[80:83], a[224:227]\n" \
  "v_mfma_f32_16x16x32_f16 a[228:231], v[56:59], v[84:87], a[228:231]\n" \
  "v_mfma_f32_16x16x32_f16 a[232:235], v[56:59], v[88:91], a[232:235]\n" \
  "v_mfma_f32_16x16x32_f16 a[236:239], v[56:59], v[92:95], a[236:239]\n" \
  "ds_read_b128 v[56:59], v25 offset:47104\n" \
  "v_mfma_f32_16x16x32_f16 a[252:255], v[60:63], v[92:95], a[252:255]\n" \
  "buffer_store_dwordx4 a[40:43], v12, s[16:19], s40 offen offset:2048 sc1\n" \
  "v_mfma_f32_16x16x32_f16 a[248:251], v[60:63], v[88:91], a[248:251]\n" \
  "v_mfma_f32_16x16x32_f16 a[244:247], v[60:63], v[84:87], a[244:247]\n" \
  "s_waitcnt lgkmcnt(5)\n" \
  "v_mfma_f32_16x16x32_f16 a[240:243], v[60:63], v[80:83], a[240:243]\n" \
  "ds_read_b128 v[60:63], v25 offset:48128\n" \
  "v_mfma_f32_16x16x32_f16 a[128:131], v[32:35], v[224:227], a[128:131]\n" \
  "v_mfma_f32_16x16x32_f16 a[132:135], v[32:35], v[228:231], a[132:135]\n" \
  "v_mfma_f32_16x16x32_f16 a[136:139], v[32:35], v[232:235], a[136:139]\n" \
  "v_mfma_f32_16x16x32_f16 a[140:143], v[32:35], v[236:239], a[140:143]\n" \
  "ds_read_b128 v[32:35], v25 offset:57344\n" \
  "v_mfma_f32_16x16x32_f16 a[156:159], v[36:39], v[236:239], a[156:159]\n" \
  "buffer_store_dwordx4 a[44:47], v12, s[16:19], s40 offen offset:3072 sc1\n" \
  "v_mfma_f32_16x16x32_f16 a[152:155], v[36:39], v[232:235], a[152:155]\n" \
  "v_mfma_f32_16x16x32_f16 a[148:151], v[36:39], v[228:231], a[148:151]\n" \
  "s_waitcnt lgkmcnt(5)\n" \
  "v_mfma_f32_16x16x32_f16 a[144:147], v[36:39], v[224:227], a[144:147]\n" \
  "ds_read_b128 v[36:39], v25 offset:58368\n" \
  "v_mfma_f32_16x16x32_f16 a[160:163], v[40:43], v[224:227], a[160:163]\n" \
  "v_mfma_f32_16x16x32_f16 a[164:167], v[40:43], v[228:231], a[164:167]\n" \
  "v_mfma_f32_16x16x32_f16 a[168:171], v[40:43], v[232:235], a[168:171]\n" \
  "s_mov_b32 s40, 0x3000\n" \
  "v_mfma_f32_16x16x32_f16 a[172:175], v[40:43], v[236:239], a[172:175]\n" \
  "ds_read_b128 v[40:43], v25 offset:59392\n" \
  "v_mfma_f32_16x16x32_f16 a[188:191], v[44:47], v[236:239], a[188:191]\n" \
  "buffer_store_dwordx4 a[48:51], v12, s[16:19], s40 offen offset:0 sc1\n" \
  "v_mfma_f32_16x16x32_f16 a[184:187], v[44:47], v[232:235], a[184:187]\n" \
  "v_mfma_f32_16x16x32_f16 a[180:183], v[44:47], v[228:231], a[180:183]\n" \
  "s_waitcnt lgkmcnt(5)\n" \
  "v_mfma_f32_16x16x32_f16 a[176:179], v[44:47], v[224:227], a[176:179]\n" \
  "ds_read_b128 v[44:47], v25 offset:60416\n" \
  "v_mfma_f32_16x16x32_f16 a[192:195], v[48:51], v[224:227], a[192:195]\n" \
  "v_mfma_f32_16x16x32_f16 a[196:199], v[48:51], v[228:231], a[196:199]\n" \
  "v_mfma_f32_16x16x32_f16 a[200:203], v[48:51], v[232:235], a[200:203]\n" \
  "v_mfma_f32_16x16x32_f16 a[204:207], v[48:51], v[236:239], a[204:207]\n" \
  "ds_read_b128 v[48:51], v25 offset:61440\n" \
  "v_mfma_f32_16x16x32_f16 a[220:223], v[52:55], v[236:239], a[220:223]\n" \
  "buffer_store_dwordx4 a[52:55], v12, s[16:19], s40 offen offset:1024 sc1\n" \
  "v_mfma_f32_16x16x32_f16 a[216:219], v[52:55], v[232:235], a[216:219]\n" \
  "v_mfma_f32_16x16x32_f16 a[212:215], v[52:55], v[228:231], a[212:215]\n" \
  "s_waitcnt lgkmcnt(5)\n" \
  "v_mfma_f32_16x16x32_f16 a[208:211], v[52:55], v[224:227], a[208:211]\n" \
  "ds_read_b128 v[52:55], v25 offset:62464\n" \
  "v_mfma_f32_16x16x32_f16 a[224:227], v[56:59], v[224:227], a[224:227]\n" \
  "v_mfma_f32_16x16x32_f16 a[228:231], v[56:59], v[228:231], a[228:231]\n" \
  "v_mfma_f32_16x16x32_f16 a[232:235], v[56:59], v[232:235], a[232:235]\n" \
  "v_mfma_f32_16x16x32_f16 a[236:239], v[56:59], v[236:239], a[236:239]\n" \
  "ds_read_b128 v[56:59], v25 offset:63488\n" \
  "v_mfma_f32_16x16x32_f16 a[252:255], v[60:63], v[236:239], a[252:255]\n" \
  "buffer_store_dwordx4 a[56:59], v12, s[16:19], s40 offen offset:2048 sc1\n" \
  "v_mfma_f32_16x16x32_f16 a[248:251], v[60:63], v[232:235], a[248:251]\n" \
  "v_mfma_f32_16x16x32_f16 a[244:247], v[60:63], v[228:231], a[244:247]\n" \
  "s_waitcnt lgkmcnt(5)\n" \
  "v_mfma_f32_16x16x32_f16 a[240:243], v[60:63], v[224:227], a[240:243]\n" \
  "ds_read_b128 v[60:63], v25 offset:64512\n" \
  "v_mfma_f32_16x16x32_f16 a[128:131], v[32:35], v[240:243], a[128:131]\n" \
  "v_mfma_f32_16x16x32_f16 a[132:135], v[32:35], v[244:247], a[132:135]\n" \
  "v_mfma_f32_16x16x32_f16 a[136:139], v[32:35], v[248:251], a[136:139]\n" \
  "v_mfma_f32_16x16x32_f16 a[140:143], v[32:35], v[252:255], a[140:143]\n" \
  "ds_read_b128 v[32:35], v26 offset:8192\n" \
  "v_mfma_f32_16x16x32_f16 a[156:159], v[36:39], v[252:255], a[156:159]\n" \
  "buffer_store_dwordx4 a[60:63], v12, s[16:19], s40 offen offset:3072 sc1\n" \
  "v_mfma_f32_16x16x32_f16 a[152:155], v[36:39], v[248:251], a[152:155]\n" \
  "v_mfma_f32_16x16x32_f16 a[148:151], v[36:39], v[244:247], a[148:151]\n" \
  "s_waitcnt lgkmcnt(5)\n" \
  "v_mfma_f32_16x16x32_f16 a[144:147], v[36:39], v[240:243], a[144:147]\n" \
  "ds_read_b128 v[36:39], v26 offset:9216\n" \
  "v_mfma_f32_16x16x32_f16 a[160:163], v[40:43], v[240:243], a[160:163]\n" \
  "v_mfma_f32_16x16x32_f16 a[164:167], v[40:43], v[244:247], a[164:167]\n" \
  "v_mfma_f32_16x16x32_f16 a[168:171], v[40:43], v[248:251], a[168:171]\n" \
  "s_mov_b32 s40, 0x4000\n" \
  "v_mfma_f32_16x16x32_f16 a[172:175], v[40:43], v[252:255], a[172:175]\n" \
  "ds_read_b128 v[40:43], v26 offset:10240\n" \
  "v_mfma_f32_16x16x32_f16 a[188:191], v[44:47], v[252:255], a[188:191]\n" \
  "buffer_store_dwordx4 a[64:67], v12, s[16:19], s40 offen offset:0 sc1\n" \
  "v_mfma_f32_16x16x32_f16 a[184:187], v[44:47], v[248:251], a[184:187]\n" \
  "v_mfma_f32_16x16x32_f16 a[180:183], v[44:47], v[244:247], a[180:183]\n" \
  "s_waitcnt lgkmcnt(5)\n" \
  "v_mfma_f32_16x16x32_f16 a[176:179], v[44:47], v[240:243], a[176:179]\n" \
  "ds_read_b128 v[44:47], v26 offset:11264\n" \
  "v_mfma_f32_16x16x32_f16 a[192:195], v[48:51], v[240:243], a[192:195]\n" \
  "v_mfma_f32_16x16x32_f16 a[196:199], v[48:51], v[244:247], a[196:199]\n" \
  "v_mfma_f32_16x16x32_f16 a[200:203], v[48:51], v[248:251], a[200:203]\n" \
  "v_mfma_f32_16x16x32_f16 a[204:207], v[48:51], v[252:255], a[204:207]\n" \
  "ds_read_b128 v[48:51], v26 offset:12288\n" \
  "v_mfma_f32_16x16x32_f16 a[220:223], v[52:55], v[252:255], a[220:223]\n" \
  "buffer_store_dwordx4 a[68:71], v12, s[16:19], s40 offen offset:1024 sc1\n" \
  "v_mfma_f32_16x16x32_f16 a[216:219], v[52:55], v[248:251], a[216:219]\n" \
  "v_mfma_f32_16x16x32_f16 a[212:215], v[52:55], v[244:247], a[212:215]\n" \
  "s_waitcnt lgkmcnt(5)\n" \
  "v_mfma_f32_16x16x32_f16 a[208:211], v[52:55], v[240:243], a[208:211]\n" \
  "ds_read_b128 v[52:55], v26 offset:13312\n" \
  "v_mfma_f32_16x16x32_f16 a[224:227], v[56:59], v[240:243], a[224:227]\n" \
  "v_mfma_f32_16x16x32_f16 a[228:231], v[56:59], v[244:247], a[228:231]\n" \
  "v_mfma_f32_16x16x32_f16 a[232:235], v[56:59], v[248:251], a[232:235]\n" \
  "v_mfma_f32_16x16x32_f16 a[236:239], v[56:59], v[252:255], a[236:239]\n" \
  "ds_read_b128 v[56:59], v26 offset:14336\n" \
  "v_mfma_f32_16x16x32_f16 a[252:255], v[60:63], v[252:255], a[252:255]\n" \
  "buffer_store_dwordx4 a[72:75], v12, s[16:19], s40 offen offset:2048 sc1\n" \
  "v_mfma_f32_16x16x32_f16 a[248:251], v[60:63], v[248:251], a[248:251]\n" \
  "v_mfma_f32_16x16x32_f16 a[244:247], v[60:63], v[244:247], a[244:247]\n" \
  "s_waitcnt lgkmcnt(5)\n" \
  "v_mfma_f32_16x16x32_f16 a[240:243], v[60:63], v[240:243], a[240:243]\n" \
  "ds_read_b128 v[60:63], v26 offset:15360\n" \
  "v_mfma_f32_16x16x32_f16 a[128:131], v[32:35], v[0:3], a[128:131]\n" \
  "v_mfma_f32_16x16x32_f16 a[132:135], v[32:35], v[4:7], a[132:135]\n" \
  "v_mfma_f32_16x16x32_f16 a[136:139], v[32:35], v[8:11], a[136:139]\n" \
  "v_mfma_f32_16x16x32_f16 a[140:143], v[32:35], v[20:23], a[140:143]\n" \
  "ds_read_b128 v[32:35], v26 offset:24576\n" \
  "v_mfma_f32_16x16x32_f16 a[156:159], v[36:39], v[20:23], a[156:159]\n" \
  "buffer_store_dwordx4 a[76:79], v12, s[16:19], s40 offen offset:3072 sc1\n" \
  "v_mfma_f32_16x16x32_f16 a[152:155], v[36:39], v[8:11], a[152:155]\n" \
  "v_mfma_f32_16x16x32_f16 a[148:151], v[36:39], v[4:7], a[148:151]\n" \
  "s_waitcnt lgkmcnt(5)\n" \
  "v_mfma_f32_16x16x32_f16 a[144:147], v[36:39], v[0:3], a[144:147]\n" \
  "ds_read_b128 v[36:39], v26 offset:25600\n" \
  "v_mfma_f32_16x16x32_f16 a[160:163], v[40:43], v[0:3], a[160:163]\n" \
  "v_mfma_f32_16x16x32_f16 a[164:167], v[40:43], v[4:7], a[164:167]\n" \
  "v_mfma_f32_16x16x32_f16 a[168:171], v[40:43], v[8:11], a[168:171]\n" \
  "s_mov_b32 s40, 0x5000\n" \
  "v_mfma_f32_16x16x32_f16 a[172:175], v[40:43], v[20:23], a[172:175]\n" \
  "ds_read_b128 v[40:43], v26 offset:26624\n" \
  "v_mfma_f32_16x16x32_f16 a[188:191], v[44:47], v[20:23], a[188:191]\n" \
  "buffer_store_dwordx4 a[80:83], v12, s[16:19], s40 offen offset:0 sc1\n" \
  "v_mfma_f32_16x16x32_f16 a[184:187], v[44:47], v[8:11], a[184:187]\n" \
  "v_mfma_f32_16x16x32_f16 a[180:183], v[44:47], v[4:7], a[180:183]\n" \
  "s_waitcnt lgkmcnt(5)\n" \
  "v_mfma_f32_16x16x32_f16 a[176:179], v[44:47], v[0:3], a[176:179]\n" \
  "ds_read_b128 v[44:47], v26 offset:27648\n" \
  "v_mfma_f32_16x16x32_f16 a[192:195], v[48:51], v[0:3], a[192:195]\n" \
  "v_mfma_f32_16x16x32_f16 a[196:199], v[48:51], v[4:7], a[196:199]\n" \
  "v_mfma_f32_16x16x32_f16 a[200:203], v[48:51], v[8:11], a[200:203]\n" \
  "v_mfma_f32_16x16x32_f16 a[204:207], v[48:51], v[20:23], a[204:207]\n" \
  "ds_read_b128 v[48:51], v26 offset:28672\n" \
  "v_mfma_f32_16x16x32_f16 a[220:223], v[52:55], v[20:23], a[220:223]\n" \
  "buffer_store_dwordx4 a[84:87], v12, s[16:19], s40 offen offset:1024 sc1\n" \
  "v_mfma_f32_16x16x32_f16 a[216:219], v[52:55], v[8:11], a[216:219]\n" \
  "v_mfma_f32_16x16x32_f16 a[212:215], v[52:55], v[4:7], a[212:215]\n" \
  "s_waitcnt lgkmcnt(5)\n" \
  "v_mfma_f32_16x16x32_f16 a[208:211], v[52:55], v[0:3], a[208:211]\n" \
  "ds_read_b128 v[52:55], v26 offset:29696\n" \
  "v_mfma_f32_16x16x32_f16 a[224:227], v[56:59], v[0:3], a[224:227]\n" \
  "v_mfma_f32_16x16x32_f16 a[228:231], v[56:59], v[4:7], a[228:231]\n" \
  "v_mfma_f32_16x16x32_f16 a[232:235], v[56:59], v[8:11], a[232:235]\n" \
  "v_mfma_f32_16x16x32_f16 a[236:239], v[56:59], v[20:23], a[236:239]\n" \
  "ds_read_b128 v[56:59], v26 offset:30720\n" \
  "v_mfma_f32_16x16x32_f16 a[252:255], v[60:63], v[20:23], a[252:255]\n" \
  "buffer_store_dwordx4 a[88:91], v12, s[16:19], s40 offen offset:2048 sc1\n" \
  "v_mfma_f32_16x16x32_f16 a[248:251], v[60:63], v[8:11], a[248:251]\n" \
  "v_mfma_f32_16x16x32_f16 a[244:247], v[60:63], v[4:7], a[244:247]\n" \
  "s_waitcnt lgkmcnt(5)\n" \
  "v_mfma_f32_16x16x32_f16 a[240:243], v[60:63], v[0:3], a[240:243]\n" \
  "ds_read_b128 v[60:63], v26 offset:31744\n" \
  "v_mfma_f32_16x16x32_f16 a[128:131], v[32:35], v[96:99], a[128:131]\n" \
  "v_mfma_f32_16x16x32_f16 a[132:135], v[32:35], v[100:103], a[132:135]\n" \
  "v_mfma_f32_16x16x32_f16 a[136:139], v[32:35], v[104:107], a[136:139]\n" \
  "v_mfma_f32_16x16x32_f16 a[140:143], v[32:35], v[108:111], a[140:143]\n" \
  "ds_read_b128 v[32:35], v26 offset:40960\n" \
  "v_mfma_f32_16x16x32_f16 a[156:159], v[36:39], v[108:111], a[156:159]\n" \
  "buffer_store_dwordx4 a[92:95], v12, s[16:19], s40 offen offset:3072 sc1\n" \
  "v_mfma_f32_16x16x32_f16 a[152:155], v[36:39], v[104:107], a[152:155]\n" \
  "v_mfma_f32_16x16x32_f16 a[148:151], v[36:39], v[100:103], a[148:151]\n" \
  "s_waitcnt lgkmcnt(5)\n" \
  "v_mfma_f32_16x16x32_f16 a[144:147], v[36:39], v[96:99], a[144:147]\n" \
  "ds_read_b128 v[36:39], v26 offset:41984\n" \
  "v_mfma_f32_16x16x32_f16 a[160:163], v[40:43], v[96:99], a[160:163]\n" \
  "v_mfma_f32_16x16x32_f16 a[164:167], v[40:43], v[100:103], a[164:167]\n" \
  "v_mfma_f32_16x16x32_f16 a[168:171], v[40:43], v[104:107], a[168:171]\n" \
  "s_mov_b32 s40, 0x6000\n" \
  "v_mfma_f32_16x16x32_f16 a[172:175], v[40:43], v[108:111], a[172:175]\n" \
  "ds_read_b128 v[40:43], v26 offset:43008\n" \
  "v_mfma_f32_16x16x32_f16 a[188:191], v[44:47], v[108:111], a[188:191]\n" \
  "buffer_store_dwordx4 a[96:99], v12, s[16:19], s40 offen offset:0 sc1\n" \
  "v_mfma_f32_16x16x32_f16 a[184:187], v[44:47], v[104:107], a[184:187]\n" \
  "v_mfma_f32_16x16x32_f16 a[180:183], v[44:47], v[100:103], a[180:183]\n" \
  "s_waitcnt lgkmcnt(5)\n" \
  "v_mfma_f32_16x16x32_f16 a[176:179], v[44:47], v[96:99], a[176:179]\n" \
  "ds_read_b128 v[44:47], v26 offset:44032\n" \
  "v_mfma_f32_16x16x32_f16 a[192:195], v[48:51], v[96:99], a[192:195]\n" \
  "v_mfma_f32_16x16x32_f16 a[196:199], v[48:51], v[100:103], a[196:199]\n" \
  "v_mfma_f32_16x16x32_f16 a[200:203], v[48:51], v[104:107], a[200:203]\n" \
  "v_mfma_f32_16x16x32_f16 a[204:207], v[48:51], v[108:111], a[204:207]\n" \
  "ds_read_b128 v[48:51], v26 offset:45056\n" \
  "v_mfma_f32_16x16x32_f16 a[220:223], v[52:55], v[108:111], a[220:223]\n" \
  "buffer_store_dwordx4 a[100:103], v12, s[16:19], s40 offen offset:1024 sc1\n" \
  "v_mfma_f32_16x16x32_f16 a[216:219], v[52:55], v[104:107], a[216:219]\n" \
  "v_mfma_f32_16x16x32_f16 a[212:215], v[52:55], v[100:103], a[212:215]\n" \
  "s_waitcnt lgkmcnt(5)\n" \
  "v_mfma_f32_16x16x32_f16 a[208:211], v[52:55], v[96:99], a[208:211]\n" \
  "ds_read_b128 v[52:55], v26 offset:46080\n" \
  "v_mfma_f32_16x16x32_f16 a[224:227], v[56:59], v[96:99], a[224:227]\n" \
  "v_mfma_f32_16x16x32_f16 a[228:231], v[56:59], v[100:103], a[228:231]\n" \
  "v_mfma_f32_16x16x32_f16 a[232:235], v[56:59], v[104:107], a[232:235]\n" \
  "v_mfma_f32_16x16x32_f16 a[236:239], v[56:59], v[108:111], a[236:239]\n" \
  "ds_read_b128 v[56:59], v26 offset:47104\n" \
  "v_mfma_f32_16x16x32_f16 a[252:255], v[60:63], v[108:111], a[252:255]\n" \
  "buffer_store_dwordx4 a[104:107], v12, s[16:19], s40 offen offset:2048 sc1\n" \
  "v_mfma_f32_16x16x32_f16 a[248:251], v[60:63], v[104:107], a[248:251]\n" \
  "v_mfma_f32_16x16x32_f16 a[244:247], v[60:63], v[100:103], a[244:247]\n" \
  "s_waitcnt lgkmcnt(5)\n" \
  "v_mfma_f32_16x16x32_f16 a[240:243], v[60:63], v[96:99], a[240:243]\n" \
  "ds_read_b128 v[60:63], v26 offset:48128\n" \
  "v_mfma_f32_16x16x32_f16 a[128:131], v[32:35], v[112:115], a[128:131]\n" \
  "v_mfma_f32_16x16x32_f16 a[132:135], v[32:35], v[116:119], a[132:135]\n" \
  "v_mfma_f32_16x16x32_f16 a[136:139], v[32:35], v[120:123], a[136:139]\n" \
  "v_mfma_f32_16x16x32_f16 a[140:143], v[32:35], v[124:127], a[140:143]\n" \
  "ds_read_b128 v[32:35], v26 offset:57344\n" \
  "v_mfma_f32_16x16x32_f16 a[156:159], v[36:39], v[124:127], a[156:159]\n" \
  "buffer_store_dwordx4 a[108:111], v12, s[16:19], s40 offen offset:3072 sc1\n" \
  "v_mfma_f32_16x16x32_f16 a[152:155], v[36:39], v[120:123], a[152:155]\n" \
  "v_mfma_f32_16x16x32_f16 a[148:151], v[36:39], v[116:119], a[148:151]\n" \
  "s_waitcnt lgkmcnt(5)\n" \
  "v_mfma_f32_16x16x32_f16 a[144:147], v[36:39], v[112:115], a[144:147]\n" \
  "ds_read_b128 v[36:39], v26 offset:58368\n" \
  "v_mfma_f32_16x16x32_f16 a[160:163], v[40:43], v[112:115], a[160:163]\n" \
  "v_mfma_f32_16x16x32_f16 a[164:167], v[40:43], v[116:119], a[164:167]\n" \
  "v_mfma_f32_16x16x32_f16 a[168:171], v[40:43], v[120:123], a[168:171]\n" \
  "s_mov_b32 s40, 0x7000\n" \
  "v_mfma_f32_16x16x32_f16 a[172:175], v[40:43], v[124:127], a[172:175]\n" \
  "ds_read_b128 v[40:43], v26 offset:59392\n" \
  "v_mfma_f32_16x16x32_f16 a[188:191], v[44:47], v[124:127], a[188:191]\n" \
  "buffer_store_dwordx4 a[112:115], v12, s[16:19], s40 offen offset:0 sc1\n" \
  "v_mfma_f32_16x16x32_f16 a[184:187], v[44:47], v[120:123], a[184:187]\n" \
  "v_mfma_f32_16x16x32_f16 a[180:183], v[44:47], v[116:119], a[180:183]\n" \
  "s_waitcnt lgkmcnt(5)\n" \
  "v_mfma_f32_16x16x32_f16 a[176:179], v[44:47], v[112:115], a[176:179]\n" \
  "ds_read_b128 v[44:47], v26 offset:60416\n" \
  "v_mfma_f32_16x16x32_f16 a[192:195], v[48:51], v[112:115], a[192:195]\n" \
  "v_mfma_f32_16x16x32_f16 a[196:199], v[48:51], v[116:119], a[196:199]\n" \
  "v_mfma_f32_16x16x32_f16 a[200:203], v[48:51], v[120:123], a[200:203]\n" \
  "v_mfma_f32_16x16x32_f16 a[204:207], v[48:51], v[124:127], a[204:207]\n" \
  "ds_read_b128 v[48:51], v26 offset:61440\n" \
  "v_mfma_f32_16x16x32_f16 a[220:223], v[52:55], v[124:127], a[220:223]\n" \
  "buffer_store_dwordx4 a[116:119], v12, s[16:19], s40 offen offset:1024 sc1\n" \
  "v_mfma_f32_16x16x32_f16 a[216:219], v[52:55], v[120:123], a[216:219]\n" \
  "v_mfma_f32_16x16x32_f16 a[212:215], v[52:55], v[116:119], a[212:215]\n" \
  "s_waitcnt lgkmcnt(5)\n" \
  "v_mfma_f32_16x16x32_f16 a[208:211], v[52:55], v[112:115], a[208:211]\n" \
  "ds_read_b128 v[52:55], v26 offset:62464\n" \
  "v_mfma_f32_16x16x32_f16 a[224:227], v[56:59], v[112:115], a[224:227]\n" \
  "v_mfma_f32_16x16x32_f16 a[228:231], v[56:59], v[116:119], a[228:231]\n" \
  "v_mfma_f32_16x16x32_f16 a[232:235], v[56:59], v[120:123], a[232:235]\n" \
  "v_mfma_f32_16x16x32_f16 a[236:239], v[56:59], v[124:127], a[236:239]\n" \
  "ds_read_b128 v[56:59], v26 offset:63488\n" \
  "v_mfma_f32_16x16x32_f16 a[252:255], v[60:63], v[124:127], a[252:255]\n" \
  "buffer_store_dwordx4 a[120:123], v12, s[16:19], s40 offen offset:2048 sc1\n" \
  "v_mfma_f32_16x16x32_f16 a[248:251], v[60:63], v[120:123], a[248:251]\n" \
  "v_mfma_f32_16x16x32_f16 a[244:247], v[60:63], v[116:119], a[244:247]\n" \
  "s_waitcnt lgkmcnt(5)\n" \
  "v_mfma_f32_16x16x32_f16 a[240:243], v[60:63], v[112:115], a[240:243]\n" \
  "ds_read_b128 v[60:63], v26 offset:64512\n" \
  "v_mfma_f32_16x16x32_f16 a[128:131], v[32:35], v[128:131], a[128:131]\n" \
  "v_mfma_f32_16x16x32_f16 a[132:135], v[32:35], v[132:135], a[132:135]\n" \
  "v_mfma_f32_16x16x32_f16 a[136:139], v[32:35], v[136:139], a[136:139]\n" \
  "v_mfma_f32_16x16x32_f16 a[140:143], v[32:35], v[140:143], a[140:143]\n" \
  "v_mfma_f32_16x16x32_f16 a[156:159], v[36:39], v[140:143], a[156:159]\n" \
  "buffer_store_dwordx4 a[124:127], v12, s[16:19], s40 offen offset:3072 sc1\n" \
  "v_mfma_f32_16x16x32_f16 a[152:155], v[36:39], v[136:139], a[152:155]\n" \
  "v_mfma_f32_16x16x32_f16 a[148:151], v[36:39], v[132:135], a[148:151]\n" \
  "s_waitcnt lgkmcnt(4)\n" \
  "v_mfma_f32_16x16x32_f16 a[144:147], v[36:39], v[128:131], a[144:147]\n" \
  "v_mfma_f32_16x16x32_f16 a[160:163], v[40:43], v[128:131], a[160:163]\n" \
  "v_mfma_f32_16x16x32_f16 a[164:167], v[40:43], v[132:135], a[164:167]\n" \
  "v_mfma_f32_16x16x32_f16 a[168:171], v[40:43], v[136:139], a[168:171]\n" \
  "v_mfma_f32_16x16x32_f16 a[172:175], v[40:43], v[140:143], a[172:175]\n" \
  "v_mfma_f32_16x16x32_f16 a[188:191], v[44:47], v[140:143], a[188:191]\n" \
  "v_mfma_f32_16x16x32_f16 a[184:187], v[44:47], v[136:139], a[184:187]\n" \
  "v_mfma_f32_16x16x32_f16 a[180:183], v[44:47], v[132:135], a[180:183]\n" \
  "s_waitcnt lgkmcnt(2)\n" \
  "v_mfma_f32_16x16x32_f16 a[176:179], v[44:47], v[128:131], a[176:179]\n" \
  "v_mfma_f32_16x16x32_f16 a[192:195], v[48:51], v[128:131], a[192:195]\n" \
  "s_waitcnt vmcnt(16)\n" \
  "buffer_store_dword v15, v14, s[16:19], s59 offen sc1\n" \
  "v_mfma_f32_16x16x32_f16 a[196:199], v[48:51], v[132:135], a[196:199]\n" \
  "v_mfma_f32_16x16x32_f16 a[200:203], v[48:51], v[136:139], a[200:203]\n" \
  "v_mfma_f32_16x16x32_f16 a[204:207], v[48:51], v[140:143], a[204:207]\n" \
  "v_mfma_f32_16x16x32_f16 a[220:223], v[52:55], v[140:143], a[220:223]\n" \
  "v_mfma_f32_16x16x32_f16 a[216:219], v[52:55], v[136:139], a[216:219]\n" \
  "v_mfma_f32_16x16x32_f16 a[212:215], v[52:55], v[132:135], a[212:215]\n" \
  "s_waitcnt lgkmcnt(0)\n" \
  "v_mfma_f32_16x16x32_f16 a[208:211], v[52:55], v[128:131], a[208:211]\n" \
  "v_mfma_f32_16x16x32_f16 a[224:227], v[56:59], v[128:131], a[224:227]\n" \
  "v_mfma_f32_16x16x32_f16 a[228:231], v[56:59], v[132:135], a[228:231]\n" \
  "v_mfma_f32_16x16x32_f16 a[232:235], v[56:59], v[136:139], a[232:235]\n" \
  "v_mfma_f32_16x16x32_f16 a[236:239], v[56:59], v[140:143], a[236:239]\n" \
  "v_mfma_f32_16x16x32_f16 a[252:255], v[60:63], v[140:143], a[252:255]\n" \
  "v_mfma_f32_16x16x32_f16 a[248:251], v[60:63], v[136:139], a[248:251]\n" \
  "v_mfma_f32_16x16x32_f16 a[244:247], v[60:63], v[132:135], a[244:247]\n" \
  "v_mfma_f32_16x16x32_f16 a[240:243], v[60:63], v[128:131], a[240:243]\n" \
  "Lepi_%=:\n" \
  "s_nop 7\n" \
  "s_nop 7\n" \
  "v_mov_b32 v16, 0xc0135761\n" \
  "v_mov_b32 v17, 0xc0135761\n" \
  "v_mov_b32 v18, 0xbdd2d3e8\n" \
  "v_mov_b32 v19, 0xbdd2d3e8\n" \
  "v_mov_b32 v20, 0x3f800000\n" \
  "v_mov_b32 v21, 0x3f800000\n" \
  "s_xor_b32 s40, s59, 0x500\n" \
  "v_mov_b32 v27, s40\n" \
  "v_xor_b32 v12, 0xa0000, v12\n" \
  "s_waitcnt vmcnt(0)\n" \
  "buffer_store_dword v15, v14, s[16:19], s59 offen offset:4 sc1\n" \
  "s_call_b64 s[54:55], Lpoll_%=\n" \
  "s_mov_b32 s40, 0x0\n" \
  "buffer_load_dwordx4 v[96:99], v12, s[16:19], s40 offen offset:0 sc1 nt\n" \
  "buffer_load_dwordx4 v[100:103], v12, s[16:19], s40 offen offset:1024 sc1 nt\n" \
  "buffer_load_dwordx4 v[104:107], v12, s[16:19], s40 offen offset:2048 sc1 nt\n" \
  "buffer_load_dwordx4 v[108:111], v12, s[16:19], s40 offen offset:3072 sc1 nt\n" \
  "s_mov_b32 s40, 0x1000\n" \
  "buffer_load_dwordx4 v[112:115], v12, s[16:19], s40 offen offset:0 sc1 nt\n" \
  "buffer_load_dwordx4 v[116:119], v12, s[16:19], s40 offen offset:1024 sc1 nt\n" \
  "buffer_load_dwordx4 v[120:123], v12, s[16:19], s40 offen offset:2048 sc1 nt\n" \
  "buffer_load_dwordx4 v[124:127], v12, s[16:19], s40 offen offset:3072 sc1 nt\n" \
  "s_mov_b32 s40, 0x2000\n" \
  "buffer_load_dwordx4 v[128:131], v12, s[16:19], s40 offen offset:0 sc1 nt\n" \
  "buffer_load_dwordx4 v[132:135], v12, s[16:19], s40 offen offset:1024 sc1 nt\n" \
  "buffer_load_dwordx4 v[136:139], v12, s[16:19], s40 offen offset:2048 sc1 nt\n" \
  "buffer_load_dwordx4 v[140:143], v12, s[16:19], s40 offen offset:3072 sc1 nt\n" \
  "s_mov_b32 s40, 0x3000\n" \
  "buffer_load_dwordx4 v[144:147], v12, s[16:19], s40 offen offset:0 sc1 nt\n" \
  "buffer_load_dwordx4 v[148:151], v12, s[16:19], s40 offen offset:1024 sc1 nt\n" \
  "buffer_load_dwordx4 v[152:155], v12, s[16:19], s40 offen offset:2048 sc1 nt\n" \
  "buffer_load_dwordx4 v[156:159], v12, s[16:19], s40 offen offset:3072 sc1 nt\n" \
  "v_add_u32 v27, 4, v27\n" \
  "s_call_b64 s[54:55], Lpoll_%=\n" \
  "s_mov_b32 s40, 0x4000\n" \
  "buffer_load_dwordx4 v[160:163], v12, s[16:19], s40 offen offset:0 sc1 nt\n" \
  "buffer_load_dwordx4 v[164:167], v12, s[16:19], s40 offen offset:1024 sc1 nt\n" \
  "buffer_load_dwordx4 v[168:171], v12, s[16:19], s40 offen offset:2048 sc1 nt\n" \
  "buffer_load_dwordx4 v[172:175], v12, s[16:19], s40 offen offset:3072 sc1 nt\n" \
  "s_mov_b32 s40, 0x5000\n" \
  "buffer_load_dwordx4 v[176:179], v12, s[16:19], s40 offen offset:0 sc1 nt\n" \
  "buffer_load_dwordx4 v[180:183], v12, s[16:19], s40 offen offset:1024 sc1 nt\n" \
  "buffer_load_dwordx4 v[184:187], v12, s[16:19], s40 offen offset:2048 sc1 nt\n" \
  "buffer_load_dwordx4 v[188:191], v12, s[16:19], s40 offen offset:3072 sc1 nt\n" \
  "s_mov_b32 s40, 0x6000\n" \
  "buffer_load_dwordx4 v[192:195], v12, s[16:19], s40 offen offset:0 sc1 nt\n" \
  "buffer_load_dwordx4 v[196:199], v12, s[16:19], s40 offen offset:1024 sc1 nt\n" \
  "buffer_load_dwordx4 v[200:203], v12, s[16:19], s40 offen offset:2048 sc1 nt\n" \
  "buffer_load_dwordx4 v[204:207], v12, s[16:19], s40 offen offset:3072 sc1 nt\n" \
  "s_mov_b32 s40, 0x7000\n" \
  "buffer_load_dwordx4 v[208:211], v12, s[16:19], s40 offen offset:0 sc1 nt\n" \
  "buffer_load_dwordx4 v[212:215], v12, s[16:19], s40 offen offset:1024 sc1 nt\n" \
  "buffer_load_dwordx4 v[216:219], v12, s[16:19], s40 offen offset:2048 sc1 nt\n" \
  "buffer_load_dwordx4 v[220:223], v12, s[16:19], s40 offen offset:3072 sc1 nt\n" \
  "v_cmp_eq_u32 vcc, s43, v29\n" \
  "v_mov_b32 v23, 0x7fc00000\n" \
  "s_nop 1\n" \
  "v_cndmask_b32 v22, 0, v23, vcc\n" \
  "v_accvgpr_read_b32 v64, a128\n" \
  "v_accvgpr_read_b32 v65, a129\n" \
  "v_accvgpr_read_b32 v66, a130\n" \
  "v_accvgpr_read_b32 v67, a131\n" \
  "v_accvgpr_read_b32 v68, a132\n" \
  "v_accvgpr_read_b32 v69, a133\n" \
  "v_accvgpr_read_b32 v70, a134\n" \
  "v_accvgpr_read_b32 v71, a135\n" \
  "v_accvgpr_read_b32 v72, a136\n" \
  "v_accvgpr_read_b32 v73, a137\n" \
  "v_accvgpr_read_b32 v74, a138\n" \
  "v_accvgpr_read_b32 v75, a139\n" \
  "v_accvgpr_read_b32 v76, a140\n" \
  "v_accvgpr_read_b32 v77, a141\n" \
  "v_accvgpr_read_b32 v78, a142\n" \
  "v_accvgpr_read_b32 v79, a143\n" \
  "v_pk_add_f32 v[64:65], v[64:65], v[96:97]\n" \
  "v_pk_add_f32 v[66:67], v[66:67], v[98:99]\n" \
  "v_pk_add_f32 v[68:69], v[68:69], v[100:101]\n" \
  "v_pk_add_f32 v[70:71], v[70:71], v[102:103]\n" \
  "v_pk_add_f32 v[72:73], v[72:73], v[104:105]\n" \
  "v_pk_add_f32 v[74:75], v[74:75], v[106:107]\n" \
  "v_pk_add_f32 v[76:77], v[76:77], v[108:109]\n" \
  "v_pk_add_f32 v[78:79], v[78:79], v[110:111]\n" \
  "v_pk_mul_f32 v[80:81], v[64:65], v[64:65]\n" \
  "v_pk_mul_f32 v[82:83], v[66:67], v[66:67]\n" \
  "v_pk_mul_f32 v[84:85], v[68:69], v[68:69]\n" \
  "v_pk_mul_f32 v[86:87], v[70:71], v[70:71]\n" \
  "v_pk_mul_f32 v[88:89], v[72:73], v[72:73]\n" \
  "v_pk_mul_f32 v[90:91], v[74:75], v[74:75]\n" \
  "v_pk_mul_f32 v[92:93], v[76:77], v[76:77]\n" \
  "v_pk_mul_f32 v[94:95], v[78:79], v[78:79]\n" \
  "v_pk_fma_f32 v[80:81], v[80:81], v[18:19], v[16:17]\n" \
  "v_pk_fma_f32 v[82:83], v[82:83], v[18:19], v[16:17]\n" \
  "v_pk_fma_f32 v[84:85], v[84:85], v[18:19], v[16:17]\n" \
  "v_pk_fma_f32 v[86:87], v[86:87], v[18:19], v[16:17]\n" \
  "v_pk_fma_f32 v[88:89], v[88:89], v[18:19], v[16:17]\n" \
  "v_pk_fma_f32 v[90:91], v[90:91], v[18:19], v[16:17]\n" \
  "v_pk_fma_f32 v[92:93], v[92:93], v[18:19], v[16:17]\n" \
  "v_pk_fma_f32 v[94:95], v[94:95], v[18:19], v[16:17]\n" \
  "v_pk_mul_f32 v[80:81], v[80:81], v[64:65]\n" \
  "v_pk_mul_f32 v[82:83], v[82:83], v[66:67]\n" \
  "v_pk_mul_f32 v[84:85], v[84:85], v[68:69]\n" \
  "v_pk_mul_f32 v[86:87], v[86:87], v[70:71]\n" \
  "v_pk_mul_f32 v[88:89], v[88:89], v[72:73]\n" \
  "v_pk_mul_f32 v[90:91], v[90:91], v[74:75]\n" \
  "v_pk_mul_f32 v[92:93], v[92:93], v[76:77]\n" \
  "v_pk_mul_f32 v[94:95], v[94:95], v[78:79]\n" \
  "v_exp_f32 v80, v80\n" \
  "v_exp_f32 v81, v81\n" \
  "v_exp_f32 v82, v82\n" \
  "v_exp_f32 v83, v83\n" \
  "v_exp_f32 v84, v84\n" \
  "v_exp_f32 v85, v85\n" \
  "v_exp_f32 v86, v86\n" \
  "v_exp_f32 v87, v87\n" \
  "v_exp_f32 v88, v88\n" \
  "v_exp_f32 v89, v89\n" \
  "v_exp_f32 v90, v90\n" \
  "v_exp_f32 v91, v91\n" \
  "v_exp_f32 v92, v92\n" \
  "v_exp_f32 v93, v93\n" \
  "v_exp_f32 v94, v94\n" \
  "v_exp_f32 v95, v95\n" \
  "v_pk_add_f32 v[80:81], v[80:81], v[20:21]\n" \
  "v_pk_add_f32 v[82:83], v[82:83], v[20:21]\n" \
  "v_pk_add_f32 v[84:85], v[84:85], v[20:21]\n" \
  "v_pk_add_f32 v[86:87], v[86:87], v[20:21]\n" \
  "v_pk_add_f32 v[88:89], v[88:89], v[20:21]\n" \
  "v_pk_add_f32 v[90:91], v[90:91], v[20:21]\n" \
  "v_pk_add_f32 v[92:93], v[92:93], v[20:21]\n" \
  "v_pk_add_f32 v[94:95], v[94:95], v[20:21]\n" \
  "v_rcp_f32 v80, v80\n" \
  "v_rcp_f32 v81, v81\n" \
  "v_rcp_f32 v82, v82\n" \
  "v_rcp_f32 v83, v83\n" \
  "v_rcp_f32 v84, v84\n" \
  "v_rcp_f32 v85, v85\n" \
  "v_rcp_f32 v86, v86\n" \
  "v_rcp_f32 v87, v87\n" \
  "v_rcp_f32 v88, v88\n" \
  "v_rcp_f32 v89, v89\n" \
  "v_rcp_f32 v90, v90\n" \
  "v_rcp_f32 v91, v91\n" \
  "v_rcp_f32 v92, v92\n" \
  "v_rcp_f32 v93, v93\n" \
  "v_rcp_f32 v94, v94\n" \
  "v_rcp_f32 v95, v95\n" \
  "v_fma_f32 v32, v64, v80, v22\n" \
  "v_fma_f32 v33, v68, v84, v22\n" \
  "v_fma_f32 v34, v72, v88, v22\n" \
  "v_fma_f32 v35, v76, v92, v22\n" \
  "s_mov_b32 s40, 0x0\n" \
  "buffer_store_dwordx4 v[32:35], v13, s[32:35], s40 offen nt\n" \
  "v_fma_f32 v36, v65, v81, v22\n" \
  "v_fma_f32 v37, v69, v85, v22\n" \
  "v_fma_f32 v38, v73, v89, v22\n" \
  "v_fma_f32 v39, v77, v93, v22\n" \
  "s_mov_b32 s40, 0x8000\n" \
  "buffer_store_dwordx4 v[36:39], v13, s[32:35], s40 offen nt\n" \
  "v_fma_f32 v40, v66, v82, v22\n" \
  "v_fma_f32 v41, v70, v86, v22\n" \
  "v_fma_f32 v42, v74, v90, v22\n" \
  "v_fma_f32 v43, v78, v94, v22\n" \
  "s_mov_b32 s40, 0x10000\n" \
  "buffer_store_dwordx4 v[40:43], v13, s[32:35], s40 offen nt\n" \
  "v_fma_f32 v44, v67, v83, v22\n" \
  "v_fma_f32 v45, v71, v87, v22\n" \
  "v_fma_f32 v46, v75, v91, v22\n" \
  "v_fma_f32 v47, v79, v95, v22\n" \
  "s_mov_b32 s40, 0x18000\n" \
  "buffer_store_dwordx4 v[44:47], v13, s[32:35], s40 offen nt\n" \
  "v_accvgpr_read_b32 v64, a144\n" \
  "v_accvgpr_read_b32 v65, a145\n" \
  "v_accvgpr_read_b32 v66, a146\n" \
  "v_accvgpr_read_b32 v67, a147\n" \
  "v_accvgpr_read_b32 v68, a148\n" \
  "v_accvgpr_read_b32 v69, a149\n" \
  "v_accvgpr_read_b32 v70, a150\n" \
  "v_accvgpr_read_b32 v71, a151\n" \
  "v_accvgpr_read_b32 v72, a152\n" \
  "v_accvgpr_read_b32 v73, a153\n" \
  "v_accvgpr_read_b32 v74, a154\n" \
  "v_accvgpr_read_b32 v75, a155\n" \
  "v_accvgpr_read_b32 v76, a156\n" \
  "v_accvgpr_read_b32 v77, a157\n" \
  "v_accvgpr_read_b32 v78, a158\n" \
  "v_accvgpr_read_b32 v79, a159\n" \
  "v_pk_add_f32 v[64:65], v[64:65], v[112:113]\n" \
  "v_pk_add_f32 v[66:67], v[66:67], v[114:115]\n" \
  "v_pk_add_f32 v[68:69], v[68:69], v[116:117]\n" \
  "v_pk_add_f32 v[70:71], v[70:71], v[118:119]\n" \
  "v_pk_add_f32 v[72:73], v[72:73], v[120:121]\n" \
  "v_pk_add_f32 v[74:75], v[74:75], v[122:123]\n" \
  "v_pk_add_f32 v[76:77], v[76:77], v[124:125]\n" \
  "v_pk_add_f32 v[78:79], v[78:79], v[126:127]\n" \
  "v_pk_mul_f32 v[80:81], v[64:65], v[64:65]\n" \
  "v_pk_mul_f32 v[82:83], v[66:67], v[66:67]\n" \
  "v_pk_mul_f32 v[84:85], v[68:69], v[68:69]\n" \
  "v_pk_mul_f32 v[86:87], v[70:71], v[70:71]\n" \
  "v_pk_mul_f32 v[88:89], v[72:73], v[72:73]\n" \
  "v_pk_mul_f32 v[90:91], v[74:75], v[74:75]\n" \
  "v_pk_mul_f32 v[92:93], v[76:77], v[76:77]\n" \
  "v_pk_mul_f32 v[94:95], v[78:79], v[78:79]\n" \
  "v_pk_fma_f32 v[80:81], v[80:81], v[18:19], v[16:17]\n" \
  "v_pk_fma_f32 v[82:83], v[82:83], v[18:19], v[16:17]\n" \
  "v_pk_fma_f32 v[84:85], v[84:85], v[18:19], v[16:17]\n" \
  "v_pk_fma_f32 v[86:87], v[86:87], v[18:19], v[16:17]\n" \
  "v_pk_fma_f32 v[88:89], v[88:89], v[18:19], v[16:17]\n" \
  "v_pk_fma_f32 v[90:91], v[90:91], v[18:19], v[16:17]\n" \
  "v_pk_fma_f32 v[92:93], v[92:93], v[18:19], v[16:17]\n" \
  "v_pk_fma_f32 v[94:95], v[94:95], v[18:19], v[16:17]\n" \
  "v_pk_mul_f32 v[80:81], v[80:81], v[64:65]\n" \
  "v_pk_mul_f32 v[82:83], v[82:83], v[66:67]\n" \
  "v_pk_mul_f32 v[84:85], v[84:85], v[68:69]\n" \
  "v_pk_mul_f32 v[86:87], v[86:87], v[70:71]\n" \
  "v_pk_mul_f32 v[88:89], v[88:89], v[72:73]\n" \
  "v_pk_mul_f32 v[90:91], v[90:91], v[74:75]\n" \
  "v_pk_mul_f32 v[92:93], v[92:93], v[76:77]\n" \
  "v_pk_mul_f32 v[94:95], v[94:95], v[78:79]\n" \
  "v_exp_f32 v80, v80\n" \
  "v_exp_f32 v81, v81\n" \
  "v_exp_f32 v82, v82\n" \
  "v_exp_f32 v83, v83\n" \
  "v_exp_f32 v84, v84\n" \
  "v_exp_f32 v85, v85\n" \
  "v_exp_f32 v86, v86\n" \
  "v_exp_f32 v87, v87\n" \
  "v_exp_f32 v88, v88\n" \
  "v_exp_f32 v89, v89\n" \
  "v_exp_f32 v90, v90\n" \
  "v_exp_f32 v91, v91\n" \
  "v_exp_f32 v92, v92\n" \
  "v_exp_f32 v93, v93\n" \
  "v_exp_f32 v94, v94\n" \
  "v_exp_f32 v95, v95\n" \
  "v_pk_add_f32 v[80:81], v[80:81], v[20:21]\n" \
  "v_pk_add_f32 v[82:83], v[82:83], v[20:21]\n" \
  "v_pk_add_f32 v[84:85], v[84:85], v[20:21]\n" \
  "v_pk_add_f32 v[86:87], v[86:87], v[20:21]\n" \
  "v_pk_add_f32 v[88:89], v[88:89], v[20:21]\n" \
  "v_pk_add_f32 v[90:91], v[90:91], v[20:21]\n" \
  "v_pk_add_f32 v[92:93], v[92:93], v[20:21]\n" \
  "v_pk_add_f32 v[94:95], v[94:95], v[20:21]\n" \
  "v_rcp_f32 v80, v80\n" \
  "v_rcp_f32 v81, v81\n" \
  "v_rcp_f32 v82, v82\n" \
  "v_rcp_f32 v83, v83\n" \
  "v_rcp_f32 v84, v84\n" \
  "v_rcp_f32 v85, v85\n" \
  "v_rcp_f32 v86, v86\n" \
  "v_rcp_f32 v87, v87\n" \
  "v_rcp_f32 v88, v88\n" \
  "v_rcp_f32 v89, v89\n" \
  "v_rcp_f32 v90, v90\n" \
  "v_rcp_f32 v91, v91\n" \
  "v_rcp_f32 v92, v92\n" \
  "v_rcp_f32 v93, v93\n" \
  "v_rcp_f32 v94, v94\n" \
  "v_rcp_f32 v95, v95\n" \
  "v_fma_f32 v48, v64, v80, v22\n" \
  "v_fma_f32 v49, v68, v84, v22\n" \
  "v_fma_f32 v50, v72, v88, v22\n" \
  "v_fma_f32 v51, v76, v92, v22\n" \
  "s_mov_b32 s40, 0x80000\n" \
  "buffer_store_dwordx4 v[48:51], v13, s[32:35], s40 offen nt\n" \
  "v_fma_f32 v52, v65, v81, v22\n" \
  "v_fma_f32 v53, v69, v85, v22\n" \
  "v_fma_f32 v54, v73, v89, v22\n" \
  "v_fma_f32 v55, v77, v93, v22\n" \
  "s_mov_b32 s40, 0x88000\n" \
  "buffer_store_dwordx4 v[52:55], v13, s[32:35], s40 offen nt\n" \
  "v_fma_f32 v56, v66, v82, v22\n" \
  "v_fma_f32 v57, v70, v86, v22\n" \
  "v_fma_f32 v58, v74, v90, v22\n" \
  "v_fma_f32 v59, v78, v94, v22\n" \
  "s_mov_b32 s40, 0x90000\n" \
  "buffer_store_dwordx4 v[56:59], v13, s[32:35], s40 offen nt\n" \
  "v_fma_f32 v60, v67, v83, v22\n" \
  "v_fma_f32 v61, v71, v87, v22\n" \
  "v_fma_f32 v62, v75, v91, v22\n" \
  "v_fma_f32 v63, v79, v95, v22\n" \
  "s_mov_b32 s40, 0x98000\n" \
  "buffer_store_dwordx4 v[60:63], v13, s[32:35], s40 offen nt\n" \
  "v_accvgpr_read_b32 v64, a160\n" \
  "v_accvgpr_read_b32 v65, a161\n" \
  "v_accvgpr_read_b32 v66, a162\n" \
  "v_accvgpr_read_b32 v67, a163\n" \
  "v_accvgpr_read_b32 v68, a164\n" \
  "v_accvgpr_read_b32 v69, a165\n" \
  "v_accvgpr_read_b32 v70, a166\n" \
  "v_accvgpr_read_b32 v71, a167\n" \
  "v_accvgpr_read_b32 v72, a168\n" \
  "v_accvgpr_read_b32 v73, a169\n" \
  "v_accvgpr_read_b32 v74, a170\n" \
  "v_accvgpr_read_b32 v75, a171\n" \
  "v_accvgpr_read_b32 v76, a172\n" \
  "v_accvgpr_read_b32 v77, a173\n" \
  "v_accvgpr_read_b32 v78, a174\n" \
  "v_accvgpr_read_b32 v79, a175\n" \
  "v_pk_add_f32 v[64:65], v[64:65], v[128:129]\n" \
  "v_pk_add_f32 v[66:67], v[66:67], v[130:131]\n" \
  "v_pk_add_f32 v[68:69], v[68:69], v[132:133]\n" \
  "v_pk_add_f32 v[70:71], v[70:71], v[134:135]\n" \
  "v_pk_add_f32 v[72:73], v[72:73], v[136:137]\n" \
  "v_pk_add_f32 v[74:75], v[74:75], v[138:139]\n" \
  "v_pk_add_f32 v[76:77], v[76:77], v[140:141]\n" \
  "v_pk_add_f32 v[78:79], v[78:79], v[142:143]\n" \
  "v_pk_mul_f32 v[80:81], v[64:65], v[64:65]\n" \
  "v_pk_mul_f32 v[82:83], v[66:67], v[66:67]\n" \
  "v_pk_mul_f32 v[84:85], v[68:69], v[68:69]\n" \
  "v_pk_mul_f32 v[86:87], v[70:71], v[70:71]\n" \
  "v_pk_mul_f32 v[88:89], v[72:73], v[72:73]\n" \
  "v_pk_mul_f32 v[90:91], v[74:75], v[74:75]\n" \
  "v_pk_mul_f32 v[92:93], v[76:77], v[76:77]\n" \
  "v_pk_mul_f32 v[94:95], v[78:79], v[78:79]\n" \
  "v_pk_fma_f32 v[80:81], v[80:81], v[18:19], v[16:17]\n" \
  "v_pk_fma_f32 v[82:83], v[82:83], v[18:19], v[16:17]\n" \
  "v_pk_fma_f32 v[84:85], v[84:85], v[18:19], v[16:17]\n" \
  "v_pk_fma_f32 v[86:87], v[86:87], v[18:19], v[16:17]\n" \
  "v_pk_fma_f32 v[88:89], v[88:89], v[18:19], v[16:17]\n" \
  "v_pk_fma_f32 v[90:91], v[90:91], v[18:19], v[16:17]\n" \
  "v_pk_fma_f32 v[92:93], v[92:93], v[18:19], v[16:17]\n" \
  "v_pk_fma_f32 v[94:95], v[94:95], v[18:19], v[16:17]\n" \
  "v_pk_mul_f32 v[80:81], v[80:81], v[64:65]\n" \
  "v_pk_mul_f32 v[82:83], v[82:83], v[66:67]\n" \
  "v_pk_mul_f32 v[84:85], v[84:85], v[68:69]\n" \
  "v_pk_mul_f32 v[86:87], v[86:87], v[70:71]\n" \
  "v_pk_mul_f32 v[88:89], v[88:89], v[72:73]\n" \
  "v_pk_mul_f32 v[90:91], v[90:91], v[74:75]\n" \
  "v_pk_mul_f32 v[92:93], v[92:93], v[76:77]\n" \
  "v_pk_mul_f32 v[94:95], v[94:95], v[78:79]\n" \
  "v_exp_f32 v80, v80\n" \
  "v_exp_f32 v81, v81\n" \
  "v_exp_f32 v82, v82\n" \
  "v_exp_f32 v83, v83\n" \
  "v_exp_f32 v84, v84\n" \
  "v_exp_f32 v85, v85\n" \
  "v_exp_f32 v86, v86\n" \
  "v_exp_f32 v87, v87\n" \
  "v_exp_f32 v88, v88\n" \
  "v_exp_f32 v89, v89\n" \
  "v_exp_f32 v90, v90\n" \
  "v_exp_f32 v91, v91\n" \
  "v_exp_f32 v92, v92\n" \
  "v_exp_f32 v93, v93\n" \
  "v_exp_f32 v94, v94\n" \
  "v_exp_f32 v95, v95\n" \
  "v_pk_add_f32 v[80:81], v[80:81], v[20:21]\n" \
  "v_pk_add_f32 v[82:83], v[82:83], v[20:21]\n" \
  "v_pk_add_f32 v[84:85], v[84:85], v[20:21]\n" \
  "v_pk_add_f32 v[86:87], v[86:87], v[20:21]\n" \
  "v_pk_add_f32 v[88:89], v[88:89], v[20:21]\n" \
  "v_pk_add_f32 v[90:91], v[90:91], v[20:21]\n" \
  "v_pk_add_f32 v[92:93], v[92:93], v[20:21]\n" \
  "v_pk_add_f32 v[94:95], v[94:95], v[20:21]\n" \
  "v_rcp_f32 v80, v80\n" \
  "v_rcp_f32 v81, v81\n" \
  "v_rcp_f32 v82, v82\n" \
  "v_rcp_f32 v83, v83\n" \
  "v_rcp_f32 v84, v84\n" \
  "v_rcp_f32 v85, v85\n" \
  "v_rcp_f32 v86, v86\n" \
  "v_rcp_f32 v87, v87\n" \
  "v_rcp_f32 v88, v88\n" \
  "v_rcp_f32 v89, v89\n" \
  "v_rcp_f32 v90, v90\n" \
  "v_rcp_f32 v91, v91\n" \
  "v_rcp_f32 v92, v92\n" \
  "v_rcp_f32 v93, v93\n" \
  "v_rcp_f32 v94, v94\n" \
  "v_rcp_f32 v95, v95\n" \
  "v_fma_f32 v32, v64, v80, v22\n" \
  "v_fma_f32 v33, v68, v84, v22\n" \
  "v_fma_f32 v34, v72, v88, v22\n" \
  "v_fma_f32 v35, v76, v92, v22\n" \
  "s_mov_b32 s40, 0x100000\n" \
  "buffer_store_dwordx4 v[32:35], v13, s[32:35], s40 offen nt\n" \
  "v_fma_f32 v36, v65, v81, v22\n" \
  "v_fma_f32 v37, v69, v85, v22\n" \
  "v_fma_f32 v38, v73, v89, v22\n" \
  "v_fma_f32 v39, v77, v93, v22\n" \
  "s_mov_b32 s40, 0x108000\n" \
  "buffer_store_dwordx4 v[36:39], v13, s[32:35], s40 offen nt\n" \
  "v_fma_f32 v40, v66, v82, v22\n" \
  "v_fma_f32 v41, v70, v86, v22\n" \
  "v_fma_f32 v42, v74, v90, v22\n" \
  "v_fma_f32 v43, v78, v94, v22\n" \
  "s_mov_b32 s40, 0x110000\n" \
  "buffer_store_dwordx4 v[40:43], v13, s[32:35], s40 offen nt\n" \
  "v_fma_f32 v44, v67, v83, v22\n" \
  "v_fma_f32 v45, v71, v87, v22\n" \
  "v_fma_f32 v46, v75, v91, v22\n" \
  "v_fma_f32 v47, v79, v95, v22\n" \
  "s_mov_b32 s40, 0x118000\n" \
  "buffer_store_dwordx4 v[44:47], v13, s[32:35], s40 offen nt\n" \
  "v_accvgpr_read_b32 v64, a176\n" \
  "v_accvgpr_read_b32 v65, a177\n" \
  "v_accvgpr_read_b32 v66, a178\n" \
  "v_accvgpr_read_b32 v67, a179\n" \
  "v_accvgpr_read_b32 v68, a180\n" \
  "v_accvgpr_read_b32 v69, a181\n" \
  "v_accvgpr_read_b32 v70, a182\n" \
  "v_accvgpr_read_b32 v71, a183\n" \
  "v_accvgpr_read_b32 v72, a184\n" \
  "v_accvgpr_read_b32 v73, a185\n" \
  "v_accvgpr_read_b32 v74, a186\n" \
  "v_accvgpr_read_b32 v75, a187\n" \
  "v_accvgpr_read_b32 v76, a188\n" \
  "v_accvgpr_read_b32 v77, a189\n" \
  "v_accvgpr_read_b32 v78, a190\n" \
  "v_accvgpr_read_b32 v79, a191\n" \
  "v_pk_add_f32 v[64:65], v[64:65], v[144:145]\n" \
  "v_pk_add_f32 v[66:67], v[66:67], v[146:147]\n" \
  "v_pk_add_f32 v[68:69], v[68:69], v[148:149]\n" \
  "v_pk_add_f32 v[70:71], v[70:71], v[150:151]\n" \
  "v_pk_add_f32 v[72:73], v[72:73], v[152:153]\n" \
  "v_pk_add_f32 v[74:75], v[74:75], v[154:155]\n" \
  "v_pk_add_f32 v[76:77], v[76:77], v[156:157]\n" \
  "v_pk_add_f32 v[78:79], v[78:79], v[158:159]\n" \
  "v_pk_mul_f32 v[80:81], v[64:65], v[64:65]\n" \
  "v_pk_mul_f32 v[82:83], v[66:67], v[66:67]\n" \
  "v_pk_mul_f32 v[84:85], v[68:69], v[68:69]\n" \
  "v_pk_mul_f32 v[86:87], v[70:71], v[70:71]\n" \
  "v_pk_mul_f32 v[88:89], v[72:73], v[72:73]\n" \
  "v_pk_mul_f32 v[90:91], v[74:75], v[74:75]\n" \
  "v_pk_mul_f32 v[92:93], v[76:77], v[76:77]\n" \
  "v_pk_mul_f32 v[94:95], v[78:79], v[78:79]\n" \
  "v_pk_fma_f32 v[80:81], v[80:81], v[18:19], v[16:17]\n" \
  "v_pk_fma_f32 v[82:83], v[82:83], v[18:19], v[16:17]\n" \
  "v_pk_fma_f32 v[84:85], v[84:85], v[18:19], v[16:17]\n" \
  "v_pk_fma_f32 v[86:87], v[86:87], v[18:19], v[16:17]\n" \
  "v_pk_fma_f32 v[88:89], v[88:89], v[18:19], v[16:17]\n" \
  "v_pk_fma_f32 v[90:91], v[90:91], v[18:19], v[16:17]\n" \
  "v_pk_fma_f32 v[92:93], v[92:93], v[18:19], v[16:17]\n" \
  "v_pk_fma_f32 v[94:95], v[94:95], v[18:19], v[16:17]\n" \
  "v_pk_mul_f32 v[80:81], v[80:81], v[64:65]\n" \
  "v_pk_mul_f32 v[82:83], v[82:83], v[66:67]\n" \
  "v_pk_mul_f32 v[84:85], v[84:85], v[68:69]\n" \
  "v_pk_mul_f32 v[86:87], v[86:87], v[70:71]\n" \
  "v_pk_mul_f32 v[88:89], v[88:89], v[72:73]\n" \
  "v_pk_mul_f32 v[90:91], v[90:91], v[74:75]\n" \
  "v_pk_mul_f32 v[92:93], v[92:93], v[76:77]\n" \
  "v_pk_mul_f32 v[94:95], v[94:95], v[78:79]\n" \
  "v_exp_f32 v80, v80\n" \
  "v_exp_f32 v81, v81\n" \
  "v_exp_f32 v82, v82\n" \
  "v_exp_f32 v83, v83\n" \
  "v_exp_f32 v84, v84\n" \
  "v_exp_f32 v85, v85\n" \
  "v_exp_f32 v86, v86\n" \
  "v_exp_f32 v87, v87\n" \
  "v_exp_f32 v88, v88\n" \
  "v_exp_f32 v89, v89\n" \
  "v_exp_f32 v90, v90\n" \
  "v_exp_f32 v91, v91\n" \
  "v_exp_f32 v92, v92\n" \
  "v_exp_f32 v93, v93\n" \
  "v_exp_f32 v94, v94\n" \
  "v_exp_f32 v95, v95\n" \
  "v_pk_add_f32 v[80:81], v[80:81], v[20:21]\n" \
  "v_pk_add_f32 v[82:83], v[82:83], v[20:21]\n" \
  "v_pk_add_f32 v[84:85], v[84:85], v[20:21]\n" \
  "v_pk_add_f32 v[86:87], v[86:87], v[20:21]\n" \
  "v_pk_add_f32 v[88:89], v[88:89], v[20:21]\n" \
  "v_pk_add_f32 v[90:91], v[90:91], v[20:21]\n" \
  "v_pk_add_f32 v[92:93], v[92:93], v[20:21]\n" \
  "v_pk_add_f32 v[94:95], v[94:95], v[20:21]\n" \
  "v_rcp_f32 v80, v80\n" \
  "v_rcp_f32 v81, v81\n" \
  "v_rcp_f32 v82, v82\n" \
  "v_rcp_f32 v83, v83\n" \
  "v_rcp_f32 v84, v84\n" \
  "v_rcp_f32 v85, v85\n" \
  "v_rcp_f32 v86, v86\n" \
  "v_rcp_f32 v87, v87\n" \
  "v_rcp_f32 v88, v88\n" \
  "v_rcp_f32 v89, v89\n" \
  "v_rcp_f32 v90, v90\n" \
  "v_rcp_f32 v91, v91\n" \
  "v_rcp_f32 v92, v92\n" \
  "v_rcp_f32 v93, v93\n" \
  "v_rcp_f32 v94, v94\n" \
  "v_rcp_f32 v95, v95\n" \
  "v_fma_f32 v48, v64, v80, v22\n" \
  "v_fma_f32 v49, v68, v84, v22\n" \
  "v_fma_f32 v50, v72, v88, v22\n" \
  "v_fma_f32 v51, v76, v92, v22\n" \
  "s_mov_b32 s40, 0x180000\n" \
  "buffer_store_dwordx4 v[48:51], v13, s[32:35], s40 offen nt\n" \
  "v_fma_f32 v52, v65, v81, v22\n" \
  "v_fma_f32 v53, v69, v85, v22\n" \
  "v_fma_f32 v54, v73, v89, v22\n" \
  "v_fma_f32 v55, v77, v93, v22\n" \
  "s_mov_b32 s40, 0x188000\n" \
  "buffer_store_dwordx4 v[52:55], v13, s[32:35], s40 offen nt\n" \
  "v_fma_f32 v56, v66, v82, v22\n" \
  "v_fma_f32 v57, v70, v86, v22\n" \
  "v_fma_f32 v58, v74, v90, v22\n" \
  "v_fma_f32 v59, v78, v94, v22\n" \
  "s_mov_b32 s40, 0x190000\n" \
  "buffer_store_dwordx4 v[56:59], v13, s[32:35], s40 offen nt\n" \
  "v_fma_f32 v60, v67, v83, v22\n" \
  "v_fma_f32 v61, v71, v87, v22\n" \
  "v_fma_f32 v62, v75, v91, v22\n" \
  "v_fma_f32 v63, v79, v95, v22\n" \
  "s_mov_b32 s40, 0x198000\n" \
  "buffer_store_dwordx4 v[60:63], v13, s[32:35], s40 offen nt\n" \
  "v_accvgpr_read_b32 v64, a192\n" \
  "v_accvgpr_read_b32 v65, a193\n" \
  "v_accvgpr_read_b32 v66, a194\n" \
  "v_accvgpr_read_b32 v67, a195\n" \
  "v_accvgpr_read_b32 v68, a196\n" \
  "v_accvgpr_read_b32 v69, a197\n" \
  "v_accvgpr_read_b32 v70, a198\n" \
  "v_accvgpr_read_b32 v71, a199\n" \
  "v_accvgpr_read_b32 v72, a200\n" \
  "v_accvgpr_read_b32 v73, a201\n" \
  "v_accvgpr_read_b32 v74, a202\n" \
  "v_accvgpr_read_b32 v75, a203\n" \
  "v_accvgpr_read_b32 v76, a204\n" \
  "v_accvgpr_read_b32 v77, a205\n" \
  "v_accvgpr_read_b32 v78, a206\n" \
  "v_accvgpr_read_b32 v79, a207\n" \
  "s_waitcnt vmcnt(28)\n" \
  "v_pk_add_f32 v[64:65], v[64:65], v[160:161]\n" \
  "v_pk_add_f32 v[66:67], v[66:67], v[162:163]\n" \
  "v_pk_add_f32 v[68:69], v[68:69], v[164:165]\n" \
  "v_pk_add_f32 v[70:71], v[70:71], v[166:167]\n" \
  "v_pk_add_f32 v[72:73], v[72:73], v[168:169]\n" \
  "v_pk_add_f32 v[74:75], v[74:75], v[170:171]\n" \
  "v_pk_add_f32 v[76:77], v[76:77], v[172:173]\n" \
  "v_pk_add_f32 v[78:79], v[78:79], v[174:175]\n" \
  "v_pk_mul_f32 v[80:81], v[64:65], v[64:65]\n" \
  "v_pk_mul_f32 v[82:83], v[66:67], v[66:67]\n" \
  "v_pk_mul_f32 v[84:85], v[68:69], v[68:69]\n" \
  "v_pk_mul_f32 v[86:87], v[70:71], v[70:71]\n" \
  "v_pk_mul_f32 v[88:89], v[72:73], v[72:73]\n" \
  "v_pk_mul_f32 v[90:91], v[74:75], v[74:75]\n" \
  "v_pk_mul_f32 v[92:93], v[76:77], v[76:77]\n" \
  "v_pk_mul_f32 v[94:95], v[78:79], v[78:79]\n" \
  "v_pk_fma_f32 v[80:81], v[80:81], v[18:19], v[16:17]\n" \
  "v_pk_fma_f32 v[82:83], v[82:83], v[18:19], v[16:17]\n" \
  "v_pk_fma_f32 v[84:85], v[84:85], v[18:19], v[16:17]\n" \
  "v_pk_fma_f32 v[86:87], v[86:87], v[18:19], v[16:17]\n" \
  "v_pk_fma_f32 v[88:89], v[88:89], v[18:19], v[16:17]\n" \
  "v_pk_fma_f32 v[90:91], v[90:91], v[18:19], v[16:17]\n" \
  "v_pk_fma_f32 v[92:93], v[92:93], v[18:19], v[16:17]\n" \
  "v_pk_fma_f32 v[94:95], v[94:95], v[18:19], v[16:17]\n" \
  "v_pk_mul_f32 v[80:81], v[80:81], v[64:65]\n" \
  "v_pk_mul_f32 v[82:83], v[82:83], v[66:67]\n" \
  "v_pk_mul_f32 v[84:85], v[84:85], v[68:69]\n" \
  "v_pk_mul_f32 v[86:87], v[86:87], v[70:71]\n" \
  "v_pk_mul_f32 v[88:89], v[88:89], v[72:73]\n" \
  "v_pk_mul_f32 v[90:91], v[90:91], v[74:75]\n" \
  "v_pk_mul_f32 v[92:93], v[92:93], v[76:77]\n" \
  "v_pk_mul_f32 v[94:95], v[94:95], v[78:79]\n" \
  "v_exp_f32 v80, v80\n" \
  "v_exp_f32 v81, v81\n" \
  "v_exp_f32 v82, v82\n" \
  "v_exp_f32 v83, v83\n" \
  "v_exp_f32 v84, v84\n" \
  "v_exp_f32 v85, v85\n" \
  "v_exp_f32 v86, v86\n" \
  "v_exp_f32 v87, v87\n" \
  "v_exp_f32 v88, v88\n" \
  "v_exp_f32 v89, v89\n" \
  "v_exp_f32 v90, v90\n" \
  "v_exp_f32 v91, v91\n" \
  "v_exp_f32 v92, v92\n" \
  "v_exp_f32 v93, v93\n" \
  "v_exp_f32 v94, v94\n" \
  "v_exp_f32 v95, v95\n" \
  "v_pk_add_f32 v[80:81], v[80:81], v[20:21]\n" \
  "v_pk_add_f32 v[82:83], v[82:83], v[20:21]\n" \
  "v_pk_add_f32 v[84:85], v[84:85], v[20:21]\n" \
  "v_pk_add_f32 v[86:87], v[86:87], v[20:21]\n" \
  "v_pk_add_f32 v[88:89], v[88:89], v[20:21]\n" \
  "v_pk_add_f32 v[90:91], v[90:91], v[20:21]\n" \
  "v_pk_add_f32 v[92:93], v[92:93], v[20:21]\n" \
  "v_pk_add_f32 v[94:95], v[94:95], v[20:21]\n" \
  "v_rcp_f32 v80, v80\n" \
  "v_rcp_f32 v81, v81\n" \
  "v_rcp_f32 v82, v82\n" \
  "v_rcp_f32 v83, v83\n" \
  "v_rcp_f32 v84, v84\n" \
  "v_rcp_f32 v85, v85\n" \
  "v_rcp_f32 v86, v86\n" \
  "v_rcp_f32 v87, v87\n" \
  "v_rcp_f32 v88, v88\n" \
  "v_rcp_f32 v89, v89\n" \
  "v_rcp_f32 v90, v90\n" \
  "v_rcp_f32 v91, v91\n" \
  "v_rcp_f32 v92, v92\n" \
  "v_rcp_f32 v93, v93\n" \
  "v_rcp_f32 v94, v94\n" \
  "v_rcp_f32 v95, v95\n" \
  "v_fma_f32 v32, v64, v80, v22\n" \
  "v_fma_f32 v33, v68, v84, v22\n" \
  "v_fma_f32 v34, v72, v88, v22\n" \
  "v_fma_f32 v35, v76, v92, v22\n" \
  "s_mov_b32 s40, 0x200000\n" \
  "buffer_store_dwordx4 v[32:35], v13, s[32:35], s40 offen nt\n" \
  "v_fma_f32 v36, v65, v81, v22\n" \
  "v_fma_f32 v37, v69, v85, v22\n" \
  "v_fma_f32 v38, v73, v89, v22\n" \
  "v_fma_f32 v39, v77, v93, v22\n" \
  "s_mov_b32 s40, 0x208000\n" \
  "buffer_store_dwordx4 v[36:39], v13, s[32:35], s40 offen nt\n" \
  "v_fma_f32 v40, v66, v82, v22\n" \
  "v_fma_f32 v41, v70, v86, v22\n" \
  "v_fma_f32 v42, v74, v90, v22\n" \
  "v_fma_f32 v43, v78, v94, v22\n" \
  "s_mov_b32 s40, 0x210000\n" \
  "buffer_store_dwordx4 v[40:43], v13, s[32:35], s40 offen nt\n" \
  "v_fma_f32 v44, v67, v83, v22\n" \
  "v_fma_f32 v45, v71, v87, v22\n" \
  "v_fma_f32 v46, v75, v91, v22\n" \
  "v_fma_f32 v47, v79, v95, v22\n" \
  "s_mov_b32 s40, 0x218000\n" \
  "buffer_store_dwordx4 v[44:47], v13, s[32:35], s40 offen nt\n" \
  "v_accvgpr_read_b32 v64, a208\n" \
  "v_accvgpr_read_b32 v65, a209\n" \
  "v_accvgpr_read_b32 v66, a210\n" \
  "v_accvgpr_read_b32 v67, a211\n" \
  "v_accvgpr_read_b32 v68, a212\n" \
  "v_accvgpr_read_b32 v69, a213\n" \
  "v_accvgpr_read_b32 v70, a214\n" \
  "v_accvgpr_read_b32 v71, a215\n" \
  "v_accvgpr_read_b32 v72, a216\n" \
  "v_accvgpr_read_b32 v73, a217\n" \
  "v_accvgpr_read_b32 v74, a218\n" \
  "v_accvgpr_read_b32 v75, a219\n" \
  "v_accvgpr_read_b32 v76, a220\n" \
  "v_accvgpr_read_b32 v77, a221\n" \
  "v_accvgpr_read_b32 v78, a222\n" \
  "v_accvgpr_read_b32 v79, a223\n" \
  "s_waitcnt vmcnt(28)\n" \
  "v_pk_add_f32 v[64:65], v[64:65], v[176:177]\n" \
  "v_pk_add_f32 v[66:67], v[66:67], v[178:179]\n" \
  "v_pk_add_f32 v[68:69], v[68:69], v[180:181]\n" \
  "v_pk_add_f32 v[70:71], v[70:71], v[182:183]\n" \
  "v_pk_add_f32 v[72:73], v[72:73], v[184:185]\n" \
  "v_pk_add_f32 v[74:75], v[74:75], v[186:187]\n" \
  "v_pk_add_f32 v[76:77], v[76:77], v[188:189]\n" \
  "v_pk_add_f32 v[78:79], v[78:79], v[190:191]\n" \
  "v_pk_mul_f32 v[80:81], v[64:65], v[64:65]\n" \
  "v_pk_mul_f32 v[82:83], v[66:67], v[66:67]\n" \
  "v_pk_mul_f32 v[84:85], v[68:69], v[68:69]\n" \
  "v_pk_mul_f32 v[86:87], v[70:71], v[70:71]\n" \
  "v_pk_mul_f32 v[88:89], v[72:73], v[72:73]\n" \
  "v_pk_mul_f32 v[90:91], v[74:75], v[74:75]\n" \
  "v_pk_mul_f32 v[92:93], v[76:77], v[76:77]\n" \
  "v_pk_mul_f32 v[94:95], v[78:79], v[78:79]\n" \
  "v_pk_fma_f32 v[80:81], v[80:81], v[18:19], v[16:17]\n" \
  "v_pk_fma_f32 v[82:83], v[82:83], v[18:19], v[16:17]\n" \
  "v_pk_fma_f32 v[84:85], v[84:85], v[18:19], v[16:17]\n" \
  "v_pk_fma_f32 v[86:87], v[86:87], v[18:19], v[16:17]\n" \
  "v_pk_fma_f32 v[88:89], v[88:89], v[18:19], v[16:17]\n" \
  "v_pk_fma_f32 v[90:91], v[90:91], v[18:19], v[16:17]\n" \
  "v_pk_fma_f32 v[92:93], v[92:93], v[18:19], v[16:17]\n" \
  "v_pk_fma_f32 v[94:95], v[94:95], v[18:19], v[16:17]\n" \
  "v_pk_mul_f32 v[80:81], v[80:81], v[64:65]\n" \
  "v_pk_mul_f32 v[82:83], v[82:83], v[66:67]\n" \
  "v_pk_mul_f32 v[84:85], v[84:85], v[68:69]\n" \
  "v_pk_mul_f32 v[86:87], v[86:87], v[70:71]\n" \
  "v_pk_mul_f32 v[88:89], v[88:89], v[72:73]\n" \
  "v_pk_mul_f32 v[90:91], v[90:91], v[74:75]\n" \
  "v_pk_mul_f32 v[92:93], v[92:93], v[76:77]\n" \
  "v_pk_mul_f32 v[94:95], v[94:95], v[78:79]\n" \
  "v_exp_f32 v80, v80\n" \
  "v_exp_f32 v81, v81\n" \
  "v_exp_f32 v82, v82\n" \
  "v_exp_f32 v83, v83\n" \
  "v_exp_f32 v84, v84\n" \
  "v_exp_f32 v85, v85\n" \
  "v_exp_f32 v86, v86\n" \
  "v_exp_f32 v87, v87\n" \
  "v_exp_f32 v88, v88\n" \
  "v_exp_f32 v89, v89\n" \
  "v_exp_f32 v90, v90\n" \
  "v_exp_f32 v91, v91\n" \
  "v_exp_f32 v92, v92\n" \
  "v_exp_f32 v93, v93\n" \
  "v_exp_f32 v94, v94\n" \
  "v_exp_f32 v95, v95\n" \
  "v_pk_add_f32 v[80:81], v[80:81], v[20:21]\n" \
  "v_pk_add_f32 v[82:83], v[82:83], v[20:21]\n" \
  "v_pk_add_f32 v[84:85], v[84:85], v[20:21]\n" \
  "v_pk_add_f32 v[86:87], v[86:87], v[20:21]\n" \
  "v_pk_add_f32 v[88:89], v[88:89], v[20:21]\n" \
  "v_pk_add_f32 v[90:91], v[90:91], v[20:21]\n" \
  "v_pk_add_f32 v[92:93], v[92:93], v[20:21]\n" \
  "v_pk_add_f32 v[94:95], v[94:95], v[20:21]\n" \
  "v_rcp_f32 v80, v80\n" \
  "v_rcp_f32 v81, v81\n" \
  "v_rcp_f32 v82, v82\n" \
  "v_rcp_f32 v83, v83\n" \
  "v_rcp_f32 v84, v84\n" \
  "v_rcp_f32 v85, v85\n" \
  "v_rcp_f32 v86, v86\n" \
  "v_rcp_f32 v87, v87\n" \
  "v_rcp_f32 v88, v88\n" \
  "v_rcp_f32 v89, v89\n" \
  "v_rcp_f32 v90, v90\n" \
  "v_rcp_f32 v91, v91\n" \
  "v_rcp_f32 v92, v92\n" \
  "v_rcp_f32 v93, v93\n" \
  "v_rcp_f32 v94, v94\n" \
  "v_rcp_f32 v95, v95\n" \
  "v_fma_f32 v48, v64, v80, v22\n" \
  "v_fma_f32 v49, v68, v84, v22\n" \
  "v_fma_f32 v50, v72, v88, v22\n" \
  "v_fma_f32 v51, v76, v92, v22\n" \
  "s_mov_b32 s40, 0x280000\n" \
  "buffer_store_dwordx4 v[48:51], v13, s[32:35], s40 offen nt\n" \
  "v_fma_f32 v52, v65, v81, v22\n" \
  "v_fma_f32 v53, v69, v85, v22\n" \
  "v_fma_f32 v54, v73, v89, v22\n" \
  "v_fma_f32 v55, v77, v93, v22\n" \
  "s_mov_b32 s40, 0x288000\n" \
  "buffer_store_dwordx4 v[52:55], v13, s[32:35], s40 offen nt\n" \
  "v_fma_f32 v56, v66, v82, v22\n" \
  "v_fma_f32 v57, v70, v86, v22\n" \
  "v_fma_f32 v58, v74, v90, v22\n" \
  "v_fma_f32 v59, v78, v94, v22\n" \
  "s_mov_b32 s40, 0x290000\n" \
  "buffer_store_dwordx4 v[56:59], v13, s[32:35], s40 offen nt\n" \
  "v_fma_f32 v60, v67, v83, v22\n" \
  "v_fma_f32 v61, v71, v87, v22\n" \
  "v_fma_f32 v62, v75, v91, v22\n" \
  "v_fma_f32 v63, v79, v95, v22\n" \
  "s_mov_b32 s40, 0x298000\n" \
  "buffer_store_dwordx4 v[60:63], v13, s[32:35], s40 offen nt\n" \
  "v_accvgpr_read_b32 v64, a224\n" \
  "v_accvgpr_read_b32 v65, a225\n" \
  "v_accvgpr_read_b32 v66, a226\n" \
  "v_accvgpr_read_b32 v67, a227\n" \
  "v_accvgpr_read_b32 v68, a228\n" \
  "v_accvgpr_read_b32 v69, a229\n" \
  "v_accvgpr_read_b32 v70, a230\n" \
  "v_accvgpr_read_b32 v71, a231\n" \
  "v_accvgpr_read_b32 v72, a232\n" \
  "v_accvgpr_read_b32 v73, a233\n" \
  "v_accvgpr_read_b32 v74, a234\n" \
  "v_accvgpr_read_b32 v75, a235\n" \
  "v_accvgpr_read_b32 v76, a236\n" \
  "v_accvgpr_read_b32 v77, a237\n" \
  "v_accvgpr_read_b32 v78, a238\n" \
  "v_accvgpr_read_b32 v79, a239\n" \
  "s_waitcnt vmcnt(28)\n" \
  "v_pk_add_f32 v[64:65], v[64:65], v[192:193]\n" \
  "v_pk_add_f32 v[66:67], v[66:67], v[194:195]\n" \
  "v_pk_add_f32 v[68:69], v[68:69], v[196:197]\n" \
  "v_pk_add_f32 v[70:71], v[70:71], v[198:199]\n" \
  "v_pk_add_f32 v[72:73], v[72:73], v[200:201]\n" \
  "v_pk_add_f32 v[74:75], v[74:75], v[202:203]\n" \
  "v_pk_add_f32 v[76:77], v[76:77], v[204:205]\n" \
  "v_pk_add_f32 v[78:79], v[78:79], v[206:207]\n" \
  "v_pk_mul_f32 v[80:81], v[64:65], v[64:65]\n" \
  "v_pk_mul_f32 v[82:83], v[66:67], v[66:67]\n" \
  "v_pk_mul_f32 v[84:85], v[68:69], v[68:69]\n" \
  "v_pk_mul_f32 v[86:87], v[70:71], v[70:71]\n" \
  "v_pk_mul_f32 v[88:89], v[72:73], v[72:73]\n" \
  "v_pk_mul_f32 v[90:91], v[74:75], v[74:75]\n" \
  "v_pk_mul_f32 v[92:93], v[76:77], v[76:77]\n" \
  "v_pk_mul_f32 v[94:95], v[78:79], v[78:79]\n" \
  "v_pk_fma_f32 v[80:81], v[80:81], v[18:19], v[16:17]\n" \
  "v_pk_fma_f32 v[82:83], v[82:83], v[18:19], v[16:17]\n" \
  "v_pk_fma_f32 v[84:85], v[84:85], v[18:19], v[16:17]\n" \
  "v_pk_fma_f32 v[86:87], v[86:87], v[18:19], v[16:17]\n" \
  "v_pk_fma_f32 v[88:89], v[88:89], v[18:19], v[16:17]\n" \
  "v_pk_fma_f32 v[90:91], v[90:91], v[18:19], v[16:17]\n" \
  "v_pk_fma_f32 v[92:93], v[92:93], v[18:19], v[16:17]\n" \
  "v_pk_fma_f32 v[94:95], v[94:95], v[18:19], v[16:17]\n" \
  "v_pk_mul_f32 v[80:81], v[80:81], v[64:65]\n" \
  "v_pk_mul_f32 v[82:83], v[82:83], v[66:67]\n" \
  "v_pk_mul_f32 v[84:85], v[84:85], v[68:69]\n" \
  "v_pk_mul_f32 v[86:87], v[86:87], v[70:71]\n" \
  "v_pk_mul_f32 v[88:89], v[88:89], v[72:73]\n" \
  "v_pk_mul_f32 v[90:91], v[90:91], v[74:75]\n" \
  "v_pk_mul_f32 v[92:93], v[92:93], v[76:77]\n" \
  "v_pk_mul_f32 v[94:95], v[94:95], v[78:79]\n" \
  "v_exp_f32 v80, v80\n" \
  "v_exp_f32 v81, v81\n" \
  "v_exp_f32 v82, v82\n" \
  "v_exp_f32 v83, v83\n" \
  "v_exp_f32 v84, v84\n" \
  "v_exp_f32 v85, v85\n" \
  "v_exp_f32 v86, v86\n" \
  "v_exp_f32 v87, v87\n" \
  "v_exp_f32 v88, v88\n" \
  "v_exp_f32 v89, v89\n" \
  "v_exp_f32 v90, v90\n" \
  "v_exp_f32 v91, v91\n" \
  "v_exp_f32 v92, v92\n" \
  "v_exp_f32 v93, v93\n" \
  "v_exp_f32 v94, v94\n" \
  "v_exp_f32 v95, v95\n" \
  "v_pk_add_f32 v[80:81], v[80:81], v[20:21]\n" \
  "v_pk_add_f32 v[82:83], v[82:83], v[20:21]\n" \
  "v_pk_add_f32 v[84:85], v[84:85], v[20:21]\n" \
  "v_pk_add_f32 v[86:87], v[86:87], v[20:21]\n" \
  "v_pk_add_f32 v[88:89], v[88:89], v[20:21]\n" \
  "v_pk_add_f32 v[90:91], v[90:91], v[20:21]\n" \
  "v_pk_add_f32 v[92:93], v[92:93], v[20:21]\n" \
  "v_pk_add_f32 v[94:95], v[94:95], v[20:21]\n" \
  "v_rcp_f32 v80, v80\n" \
  "v_rcp_f32 v81, v81\n" \
  "v_rcp_f32 v82, v82\n" \
  "v_rcp_f32 v83, v83\n" \
  "v_rcp_f32 v84, v84\n" \
  "v_rcp_f32 v85, v85\n" \
  "v_rcp_f32 v86, v86\n" \
  "v_rcp_f32 v87, v87\n" \
  "v_rcp_f32 v88, v88\n" \
  "v_rcp_f32 v89, v89\n" \
  "v_rcp_f32 v90, v90\n" \
  "v_rcp_f32 v91, v91\n" \
  "v_rcp_f32 v92, v92\n" \
  "v_rcp_f32 v93, v93\n" \
  "v_rcp_f32 v94, v94\n" \
  "v_rcp_f32 v95, v95\n" \
  "v_fma_f32 v32, v64, v80, v22\n" \
  "v_fma_f32 v33, v68, v84, v22\n" \
  "v_fma_f32 v34, v72, v88, v22\n" \
  "v_fma_f32 v35, v76, v92, v22\n" \
  "s_mov_b32 s40, 0x300000\n" \
  "buffer_store_dwordx4 v[32:35], v13, s[32:35], s40 offen nt\n" \
  "v_fma_f32 v36, v65, v81, v22\n" \
  "v_fma_f32 v37, v69, v85, v22\n" \
  "v_fma_f32 v38, v73, v89, v22\n" \
  "v_fma_f32 v39, v77, v93, v22\n" \
  "s_mov_b32 s40, 0x308000\n" \
  "buffer_store_dwordx4 v[36:39], v13, s[32:35], s40 offen nt\n" \
  "v_fma_f32 v40, v66, v82, v22\n" \
  "v_fma_f32 v41, v70, v86, v22\n" \
  "v_fma_f32 v42, v74, v90, v22\n" \
  "v_fma_f32 v43, v78, v94, v22\n" \
  "s_mov_b32 s40, 0x310000\n" \
  "buffer_store_dwordx4 v[40:43], v13, s[32:35], s40 offen nt\n" \
  "v_fma_f32 v44, v67, v83, v22\n" \
  "v_fma_f32 v45, v71, v87, v22\n" \
  "v_fma_f32 v46, v75, v91, v22\n" \
  "v_fma_f32 v47, v79, v95, v22\n" \
  "s_mov_b32 s40, 0x318000\n" \
  "buffer_store_dwordx4 v[44:47], v13, s[32:35], s40 offen nt\n" \
  "v_accvgpr_read_b32 v64, a240\n" \
  "v_accvgpr_read_b32 v65, a241\n" \
  "v_accvgpr_read_b32 v66, a242\n" \
  "v_accvgpr_read_b32 v67, a243\n" \
  "v_accvgpr_read_b32 v68, a244\n" \
  "v_accvgpr_read_b32 v69, a245\n" \
  "v_accvgpr_read_b32 v70, a246\n" \
  "v_accvgpr_read_b32 v71, a247\n" \
  "v_accvgpr_read_b32 v72, a248\n" \
  "v_accvgpr_read_b32 v73, a249\n" \
  "v_accvgpr_read_b32 v74, a250\n" \
  "v_accvgpr_read_b32 v75, a251\n" \
  "v_accvgpr_read_b32 v76, a252\n" \
  "v_accvgpr_read_b32 v77, a253\n" \
  "v_accvgpr_read_b32 v78, a254\n" \
  "v_accvgpr_read_b32 v79, a255\n" \
  "s_waitcnt vmcnt(28)\n" \
  "v_pk_add_f32 v[64:65], v[64:65], v[208:209]\n" \
  "v_pk_add_f32 v[66:67], v[66:67], v[210:211]\n" \
  "v_pk_add_f32 v[68:69], v[68:69], v[212:213]\n" \
  "v_pk_add_f32 v[70:71], v[70:71], v[214:215]\n" \
  "v_pk_add_f32 v[72:73], v[72:73], v[216:217]\n" \
  "v_pk_add_f32 v[74:75], v[74:75], v[218:219]\n" \
  "v_pk_add_f32 v[76:77], v[76:77], v[220:221]\n" \
  "v_pk_add_f32 v[78:79], v[78:79], v[222:223]\n" \
  "v_pk_mul_f32 v[80:81], v[64:65], v[64:65]\n" \
  "v_pk_mul_f32 v[82:83], v[66:67], v[66:67]\n" \
  "v_pk_mul_f32 v[84:85], v[68:69], v[68:69]\n" \
  "v_pk_mul_f32 v[86:87], v[70:71], v[70:71]\n" \
  "v_pk_mul_f32 v[88:89], v[72:73], v[72:73]\n" \
  "v_pk_mul_f32 v[90:91], v[74:75], v[74:75]\n" \
  "v_pk_mul_f32 v[92:93], v[76:77], v[76:77]\n" \
  "v_pk_mul_f32 v[94:95], v[78:79], v[78:79]\n" \
  "v_pk_fma_f32 v[80:81], v[80:81], v[18:19], v[16:17]\n" \
  "v_pk_fma_f32 v[82:83], v[82:83], v[18:19], v[16:17]\n" \
  "v_pk_fma_f32 v[84:85], v[84:85], v[18:19], v[16:17]\n" \
  "v_pk_fma_f32 v[86:87], v[86:87], v[18:19], v[16:17]\n" \
  "v_pk_fma_f32 v[88:89], v[88:89], v[18:19], v[16:17]\n" \
  "v_pk_fma_f32 v[90:91], v[90:91], v[18:19], v[16:17]\n" \
  "v_pk_fma_f32 v[92:93], v[92:93], v[18:19], v[16:17]\n" \
  "v_pk_fma_f32 v[94:95], v[94:95], v[18:19], v[16:17]\n" \
  "v_pk_mul_f32 v[80:81], v[80:81], v[64:65]\n" \
  "v_pk_mul_f32 v[82:83], v[82:83], v[66:67]\n" \
  "v_pk_mul_f32 v[84:85], v[84:85], v[68:69]\n" \
  "v_pk_mul_f32 v[86:87], v[86:87], v[70:71]\n" \
  "v_pk_mul_f32 v[88:89], v[88:89], v[72:73]\n" \
  "v_pk_mul_f32 v[90:91], v[90:91], v[74:75]\n" \
  "v_pk_mul_f32 v[92:93], v[92:93], v[76:77]\n" \
  "v_pk_mul_f32 v[94:95], v[94:95], v[78:79]\n" \
  "v_exp_f32 v80, v80\n" \
  "v_exp_f32 v81, v81\n" \
  "v_exp_f32 v82, v82\n" \
  "v_exp_f32 v83, v83\n" \
  "v_exp_f32 v84, v84\n" \
  "v_exp_f32 v85, v85\n" \
  "v_exp_f32 v86, v86\n" \
  "v_exp_f32 v87, v87\n" \
  "v_exp_f32 v88, v88\n" \
  "v_exp_f32 v89, v89\n" \
  "v_exp_f32 v90, v90\n" \
  "v_exp_f32 v91, v91\n" \
  "v_exp_f32 v92, v92\n" \
  "v_exp_f32 v93, v93\n" \
  "v_exp_f32 v94, v94\n" \
  "v_exp_f32 v95, v95\n" \
  "v_pk_add_f32 v[80:81], v[80:81], v[20:21]\n" \
  "v_pk_add_f32 v[82:83], v[82:83], v[20:21]\n" \
  "v_pk_add_f32 v[84:85], v[84:85], v[20:21]\n" \
  "v_pk_add_f32 v[86:87], v[86:87], v[20:21]\n" \
  "v_pk_add_f32 v[88:89], v[88:89], v[20:21]\n" \
  "v_pk_add_f32 v[90:91], v[90:91], v[20:21]\n" \
  "v_pk_add_f32 v[92:93], v[92:93], v[20:21]\n" \
  "v_pk_add_f32 v[94:95], v[94:95], v[20:21]\n" \
  "v_rcp_f32 v80, v80\n" \
  "v_rcp_f32 v81, v81\n" \
  "v_rcp_f32 v82, v82\n" \
  "v_rcp_f32 v83, v83\n" \
  "v_rcp_f32 v84, v84\n" \
  "v_rcp_f32 v85, v85\n" \
  "v_rcp_f32 v86, v86\n" \
  "v_rcp_f32 v87, v87\n" \
  "v_rcp_f32 v88, v88\n" \
  "v_rcp_f32 v89, v89\n" \
  "v_rcp_f32 v90, v90\n" \
  "v_rcp_f32 v91, v91\n" \
  "v_rcp_f32 v92, v92\n" \
  "v_rcp_f32 v93, v93\n" \
  "v_rcp_f32 v94, v94\n" \
  "v_rcp_f32 v95, v95\n" \
  "v_fma_f32 v48, v64, v80, v22\n" \
  "v_fma_f32 v49, v68, v84, v22\n" \
  "v_fma_f32 v50, v72, v88, v22\n" \
  "v_fma_f32 v51, v76, v92, v22\n" \
  "s_mov_b32 s40, 0x380000\n" \
  "buffer_store_dwordx4 v[48:51], v13, s[32:35], s40 offen nt\n" \
  "v_fma_f32 v52, v65, v81, v22\n" \
  "v_fma_f32 v53, v69, v85, v22\n" \
  "v_fma_f32 v54, v73, v89, v22\n" \
  "v_fma_f32 v55, v77, v93, v22\n" \
  "s_mov_b32 s40, 0x388000\n" \
  "buffer_store_dwordx4 v[52:55], v13, s[32:35], s40 offen nt\n" \
  "v_fma_f32 v56, v66, v82, v22\n" \
  "v_fma_f32 v57, v70, v86, v22\n" \
  "v_fma_f32 v58, v74, v90, v22\n" \
  "v_fma_f32 v59, v78, v94, v22\n" \
  "s_mov_b32 s40, 0x390000\n" \
  "buffer_store_dwordx4 v[56:59], v13, s[32:35], s40 offen nt\n" \
  "v_fma_f32 v60, v67, v83, v22\n" \
  "v_fma_f32 v61, v71, v87, v22\n" \
  "v_fma_f32 v62, v75, v91, v22\n" \
  "v_fma_f32 v63, v79, v95, v22\n" \
  "s_mov_b32 s40, 0x398000\n" \
  "buffer_store_dwordx4 v[60:63], v13, s[32:35], s40 offen nt\n" \
  "Lend_%=:\n" \
  ""
#define KBODY_CLOB "v27","v28","v29","v30","v31","v32","v33","v34","v35","v36","v37","v38","v39","v40","v41","v42","v43","v44","v45","v46","v47","v48","v49","v50","v51","v52","v53","v54","v55","v56","v57","v58","v59","v60","v61","v62","v63","v64","v65","v66","v67","v68","v69","v70","v71","v72","v73","v74","v75","v76","v77","v78","v79","v80","v81","v82","v83","v84","v85","v86","v87","v88","v89","v90","v91","v92","v93","v94","v95","v96","v97","v98","v99","v100","v101","v102","v103","v104","v105","v106","v107","v108","v109","v110","v111","v112","v113","v114","v115","v116","v117","v118","v119","v120","v121","v122","v123","v124","v125","v126","v127","v128","v129","v130","v131","v132","v133","v134","v135","v136","v137","v138","v139","v140","v141","v142","v143","v144","v145","v146","v147","v148","v149","v150","v151","v152","v153","v154","v155","v156","v157","v158","v159","v160","v161","v162","v163","v164","v165","v166","v167","v168","v169","v170","v171","v172","v173","v174","v175","v176","v177","v178","v179","v180","v181","v182","v183","v184","v185","v186","v187","v188","v189","v190","v191","v192","v193","v194","v195","v196","v197","v198","v199","v200","v201","v202","v203","v204","v205","v206","v207","v208","v209","v210","v211","v212","v213","v214","v215","v216","v217","v218","v219","v220","v221","v222","v223","v224","v225","v226","v227","v228","v229","v230","v231","v232","v233","v234","v235","v236","v237","v238","v239","v240","v241","v242","v243","v244","v245","v246","v247","v248","v249","v250","v251","v252","v253","v254","v255","a0","a1","a2","a3","a4","a5","a6","a7","a8","a9","a10","a11","a12","a13","a14","a15","a16","a17","a18","a19","a20","a21","a22","a23","a24","a25","a26","a27","a28","a29","a30","a31","a32","a33","a34","a35","a36","a37","a38","a39","a40","a41","a42","a43","a44","a45","a46","a47","a48","a49","a50","a51","a52","a53","a54","a55","a56","a57","a58","a59","a60","a61","a62","a63","a64","a65","a66","a67","a68","a69","a70","a71","a72","a73","a74","a75","a76","a77","a78","a79","a80","a81","a82","a83","a84","a85","a86","a87","a88","a89","a90","a91","a92","a93","a94","a95","a96","a97","a98","a99","a100","a101","a102","a103","a104","a105","a106","a107","a108","a109","a110","a111","a112","a113","a114","a115","a116","a117","a118","a119","a120","a121","a122","a123","a124","a125","a126","a127","a128","a129","a130","a131","a132","a133","a134","a135","a136","a137","a138","a139","a140","a141","a142","a143","a144","a145","a146","a147","a148","a149","a150","a151","a152","a153","a154","a155","a156","a157","a158","a159","a160","a161","a162","a163","a164","a165","a166","a167","a168","a169","a170","a171","a172","a173","a174","a175","a176","a177","a178","a179","a180","a181","a182","a183","a184","a185","a186","a187","a188","a189","a190","a191","a192","a193","a194","a195","a196","a197","a198","a199","a200","a201","a202","a203","a204","a205","a206","a207","a208","a209","a210","a211","a212","a213","a214","a215","a216","a217","a218","a219","a220","a221","a222","a223","a224","a225","a226","a227","a228","a229","a230","a231","a232","a233","a234","a235","a236","a237","a238","a239","a240","a241","a242","a243","a244","a245","a246","a247","a248","a249","a250","a251","a252","a253","a254","a255","s24","s25","s26","s27","s28","s29","s30","s31","s38","s39","s40","s41","s42","s43","s44","s45","s46","s47","s48","s49","s50","s51","s52","s53","s54","s55","s60","s61","s62","s63","s64","s70","s71","s72","s73","s74","s75","s76","s77","s78","s79","s80","s81","s82","s83","s84","s85","s86","s87"
namespace a2ag {
typedef unsigned u32x4 __attribute__((ext_vector_type(4)));
typedef unsigned u32x16 __attribute__((ext_vector_type(16)));

constexpr int NR = 8;
constexpr int MLOC = 1024, NN = 8192, KK = 8192, KLOC = 1024;
constexpr unsigned OFF_CNT = 0x0;
constexpr unsigned OFF_XFLAG = 0x10000;
constexpr unsigned OFF_EFLAG = 0x20000;
constexpr unsigned OFF_X = 0x100000;
constexpr unsigned OFF_E = 0x2100000;
constexpr unsigned WS_USED = 0x4100000;
constexpr size_t WS_NEED = WS_USED;
constexpr unsigned OOB = 0x7fff0000u;
constexpr int LDS_BYTES = 163840;
#define KIL 0
#define KLOC_OF(h, j) (KIL ? 2 * (j) + (h) : (h) * 16 + (j))
#define TSG 21
#define DECG 26
#define CLS (mt + 4 * (xcd + 8 * ((bi >> 2) & 1)))
#define RATE 12
#define JROT 2
#define PHASE (b >> 6)

struct Params {
  const float* x; const float* w; float* out; unsigned long long wsl;
  unsigned long long ws[NR];
  int rank; int pad;
};

__device__ __forceinline__ u32x4 make_srd(unsigned long long p, unsigned bytes) {
  u32x4 r = {(unsigned)p, (unsigned)(p >> 32) & 0xffffu, bytes, 0x20000u};
  return r;
}

__global__ __launch_bounds__(256, 1) void a2a_gemm_gelu_kernel(Params P) {
  const unsigned ht0 = 0;
  extern __shared__ __attribute__((aligned(16))) char smem[];
  const int b = blockIdx.x, xcd = b & 7, bi = b >> 3;
  const int kh = xcd & 1, nt = (((kh ? xcd ^ PXOR : xcd) >> 1) & 3) * 8 + (bi >> 2), mt = bi & 3;
  const int tid = threadIdx.x, lane = tid & 63, w = __builtin_amdgcn_readfirstlane(tid >> 6);
  const int c = lane & 15, g = lane >> 4;
  unsigned long long px = (unsigned long long)P.x, pw = (unsigned long long)P.w, po = (unsigned long long)P.out, wsl = P.wsl;
  unsigned long long ws0 = P.ws[0], ws1 = P.ws[1], ws2 = P.ws[2], ws3 = P.ws[3], ws4 = P.ws[4], ws5 = P.ws[5], ws6 = P.ws[6], ws7 = P.ws[7];
  int rank_ = P.rank;
  asm volatile("" : "+s"(px), "+s"(pw), "+s"(po), "+s"(wsl), "+s"(ws0), "+s"(ws1), "+s"(ws2), "+s"(ws3), "+s"(ws4), "+s"(ws5), "+s"(ws6), "+s"(ws7), "+s"(rank_));
  const int rank = rank_ & 7;
  const int ph = PHASE;
  const unsigned ldsb = (unsigned)(size_t)(__attribute__((address_space(3))) char*)smem;
  const int r8 = lane >> 3, sw = ((lane & 7) ^ (r8 >> 1)) * 16;
  const unsigned sstg = ldsb + 131072 + w * 8192;
  const int qn = b & 15, qmt = (b >> 4) & 3, sub = b >> 6, klq = (qn & 1) * 16 + (qn >> 1);
  const u32x4 srdw = make_srd(pw, (unsigned)((size_t)KK * NN * 4));
  const u32x4 srdws = make_srd(wsl, WS_USED);
  const u32x4 srdin = make_srd(px, NR * MLOC * KLOC * 4);
  const unsigned scnt = OFF_CNT + b * 64;
  u32x4 vw4;
  vw4[0] = (unsigned)((4 * g * NN + nt * 256 + 64 * w + 4 * c) * 4); vw4[1] = vw4[2] = vw4[3] = 0;
  {
    const unsigned vlsrc = (unsigned)((rank * MLOC + qmt * 256 + sub * 64 + w * 16 + r8) * 4096 + klq * 128 + sw);
    const int n0 = 4 - ph, r0 = (mt + nt) & 3, s0 = n0 == 3 ? (r0 == 3 ? 0 : r0) : r0 & (n0 - 1);
    const unsigned sw0 = (unsigned)(rank * 32 + KLOC_OF(kh, s0)) << 20;
    asm volatile(KPRE : "+{v[16:19]}"(vw4)
      : "{v20}"(vlsrc), "{s[8:11]}"(srdw), "{s[16:19]}"(srdws), "{s[20:23]}"(srdin), "{s41}"(sw0), "{s57}"(scnt), "{s65}"(sstg)
      : "memory", "scc", "s60", "v95", "v96", "v97", "v98", "v99", "v100", "v101", "v102", "v103", "v104", "v105", "v106", "v107", "v108", "v109", "v110", "v111");
  }
  unsigned tab = 0;
#pragma unroll
  for (int e = 0; e < 4; e++) {
    int s = lane * 4 + e;
    const bool ext = s >= 128;
    s = ext ? (255 - s) & 127 : s;
    const int hh = ext ? 1 - kh : kh;
    int g0 = ((s + ph) & ~3) - ph, g1 = g0 + 4;
    int lo = s < 16 ? 0 : 16, hi = s < 16 ? 16 : 128;
    g0 = g0 < lo ? lo : g0; g1 = g1 > hi ? hi : g1;
    int n = g1 - g0, o = s - g0 + ((mt + nt) & 3);
    o = n == 3 ? o - 3 * ((o * 11) >> 5) : o & (n - 1);
    s = g0 + o;
    int t = s - 16, j = (t * 9363) >> 16, q = t - 7 * j + 1;
    int ks = s < 16 ? rank * 32 + KLOC_OF(hh, s) : ((rank + q) & 7) * 32 + KLOC_OF(hh, (j ^ (JROT * q)) & 15);
    tab |= (unsigned)(ks & 255) << (8 * e);
  }
  const int phi = (2 - (b >> 6)) & 3;
  const int cls = CLS;
  const int rate = RATE;
  const int G0 = cls / rate;
  const int a1 = lane - (G0 > 0 ? 2 : 1);
  const int pi_st = (a1 >= 0 && !(a1 & 1)) ? (a1 >> 1) : -1;
  const int pi_ld = a1 == -1 ? 0 : (pi_st >= 0 ? pi_st + 1 : -1);
  const int m_st = cls + 64 * pi_st, m_ld = cls + 64 * pi_ld;
  const bool st_ok = pi_st >= 0 && m_st < 224, ld_ok = pi_ld >= 0 && m_ld < 224;
  int d_st, kl_st, d_ld, kl_ld;
  { int t = (m_st & 255) >> 1, j = (t * 9363) >> 16, q = t - 7 * j + 1; d_st = (rank - q) & 7; kl_st = KLOC_OF(m_st & 1, (j ^ (JROT * q)) & 15); }
  { int t = (m_ld & 255) >> 1, j = (t * 9363) >> 16, q = t - 7 * j + 1; d_ld = (rank - q) & 7; kl_ld = KLOC_OF(m_ld & 1, (j ^ (JROT * q)) & 15); }
  const int mp_st = ((b >> 6) + (m_st >> 1)) & 3, mp_ld = ((b >> 6) + (m_ld >> 1)) & 3;
  const int np2 = cls < 32 ? 8 : 6;
  int dt = 4;
  if (a1 >= 0 && (a1 & 1)) { int mn = cls + 64 * ((a1 >> 1) + 1); dt = 4 * ((mn < 224 ? mn / rate : TSG) - (mn - 64) / rate - 1); }
  if (a1 == np2) dt = 4 * (DECG - TSG);
  if (a1 == np2 + 1) dt = 119 - (4 * DECG + phi);
  const unsigned code = a1 < np2 ? ~0u : a1 == np2 ? ~1u : a1 == np2 + 1 ? ~2u : ~3u;
  const unsigned snext = (unsigned)(4 + phi + 4 * (G0 > 0 ? G0 - 1 : 0));
  unsigned long long base = (d_st & 4) ? ((d_st & 2) ? ((d_st & 1) ? ws7 : ws6) : ((d_st & 1) ? ws5 : ws4))
                                       : ((d_st & 2) ? ((d_st & 1) ? ws3 : ws2) : ((d_st & 1) ? ws1 : ws0));
  unsigned vsr = sstg + c * 128 + ((g ^ ((c >> 1) & 3)) * 16);
  u32x16 v;
  v[0] = (unsigned)((w * 64 + r8) * 4096 + sw);
  v[1] = OFF_X + w * 4096 + lane * 16;
  v[2] = (tid < 4) ? OFF_XFLAG + tid * 4u : OOB;
  v[3] = ld_ok ? (unsigned)((d_ld * MLOC + mp_ld * 256) * 4096 + kl_ld * 128) : ~0u;
  v[4] = st_ok ? (unsigned)((rank * 32 + kl_st) * 4 + mp_st) : code;
  v[5] = (unsigned)base;
  v[6] = (unsigned)(base >> 32) & 0xffffu;
  v[7] = 8 * ((lane >> 2) & 3);
  v[8] = OFF_XFLAG + mt * 16 + (lane & 3) * 4;
  v[9] = tab;
  v[10] = v[8] + ((((lane & 16) ? __builtin_amdgcn_readlane(tab, 1) : __builtin_amdgcn_readlane(tab, 0)) >> (8 * ((lane >> 2) & 3))) & 255) * 64;
  v[11] = (unsigned)dt;
  v[15] = 0;
  u32x4 vp;
  vp[0] = lane >= 4 && lane < 60 ? ~0u : 0u;
  vp[1] = OFF_X + (rank * 32 + klq) * 65536 + qmt * 16384 + sub * 4096 + w * 1024 + lane * 16;
  vp[2] = (tid == 0) ? 0u : OOB;
  vp[3] = OFF_CNT + 8 + (((lane & 1) == kh) ? b : b ^ PXOR) * 64;
  v[12] = OFF_E + b * 131072 + w * 32768 + lane * 16;
  v[13] = (unsigned)(((mt * 256 + kh * 128 + 4 * g) * NN + nt * 256 + 64 * w + 4 * c) * 4);
  v[14] = (lane == 0) ? 0u : OOB;
  unsigned vx = mt * 16384 + ((w * 4096) ^ ((1 - kh) * 8192)) + lane * 16;
  unsigned vl = ldsb + lane * 16;
  u32x4 srdx = make_srd(wsl + OFF_X, MLOC * KK * 2);
  u32x4 srdo = make_srd(po, MLOC * NN * 4);
  unsigned sm0 = ldsb + w * 4096;
  unsigned sfc = OFF_XFLAG + (rank * 32 + klq) * 64 + qmt * 16 + sub * 4, sef = OFF_EFLAG + (b * 4 + w) * 64;
  unsigned sn = snext;
  int ssign = 1 - 2 * kh;
  asm volatile(KBODY
    : "+{v[0:15]}"(v), "+{s[12:15]}"(srdx), "+{v[16:19]}"(vw4), "+{v[20:23]}"(vp), "+{s56}"(sn), "+{v25}"(vl), "+{v26}"(vsr), "+{s36}"(sm0)
    : "{v24}"(vx), "{s37}"(ssign), "{s65}"(sstg), "{s88}"(ht0),
      "{s[8:11]}"(srdw), "{s[16:19]}"(srdws), "{s[20:23]}"(srdin), "{s[32:35]}"(srdo),
      "{s57}"(scnt), "{s58}"(sfc), "{s59}"(sef)
    : "memory", "scc", "vcc", KBODY_CLOB);
}
}

extern "C" void kernel_launch(int rank, int nranks, ncclComm_t comm,
                              hipStream_t stream,
                              void* const* d_in, const int* in_sizes, int n_in,
                              void* d_out, int out_size,
                              const int* in_dtypes, int out_dtype,
                              void* const* d_out_all,
                              void* const* d_ws_all, size_t ws_bytes) {
  using namespace a2ag;
  if (nranks != NR || (unsigned)rank >= (unsigned)NR || n_in < 2 || ws_bytes < WS_NEED ||
      in_sizes[0] != NR * MLOC * KLOC || in_sizes[1] != KK * NN || out_size != MLOC * NN) {
    fprintf(stderr, "a2a_gemm: unexpected problem shape\n");
    return;
  }
  HIP_CHECK(hipFuncSetAttribute((const void*)a2a_gemm_gelu_kernel,
                                hipFuncAttributeMaxDynamicSharedMemorySize, LDS_BYTES));
  Params P;
  P.x = (const float*)d_in[0]; P.w = (const float*)d_in[1]; P.out = (float*)d_out;
  for (int p = 0; p < NR; p++) P.ws[p] = (unsigned long long)d_ws_all[p];
  P.wsl = P.ws[rank];
  P.rank = rank; P.pad = 0;
  hipLaunchKernelGGL(a2a_gemm_gelu_kernel, dim3(256), dim3(256), LDS_BYTES, stream, P);
  HIP_CHECK(hipGetLastError());
}
```
